```python
import numpy as np

kernel_name = "gpu_a2a_gemm_m4096_k4096_n4096_e5m2e5m2_dqrelu"


ABSMAX_RTOL = 0.05

COMPUTE_DTYPE = "float32"
EPILOGUE = "dequant_relu"
IN_QUANT = None
OUT_QUANT = None
IN_FP8 = "e5m2e5m2"
IN_DTYPES = {"x": "e5m2", "w_mat": "e5m2",
             "scale_x": "float32", "scale_w": "float32"}
WORLD_SIZE = 8

def _snap_fp8(x, mant_bits, max_abs, min_normal):
    a = np.abs(x.astype(np.float64))
    _m, _e = np.frexp(a)
    q = np.where(a >= min_normal, np.ldexp(1.0, _e - 1 - mant_bits),
                 min_normal * 2.0 ** -mant_bits)
    snapped = np.minimum(np.round(a / q) * q, max_abs)
    return (np.sign(x) * snapped).astype(np.float32)

def setup_inputs():

    x = _snap_fp8(np.random.randn(4096, 4096).astype(np.float32), 2, 57344.0, 2.0 ** -14)
    w_mat = _snap_fp8(np.random.randn(4096, 4096).astype(np.float32), 2, 57344.0, 2.0 ** -14)
    scale_x = (0.02 * (1.0 + np.random.rand(1))).astype(np.float32)
    scale_w = (0.02 * (1.0 + np.random.rand(1))).astype(np.float32)
    return {"x": x, "w_mat": w_mat, "scale_x": scale_x, "scale_w": scale_w}

def reference(x, w_mat, scale_x, scale_w):
    acc = (x @ w_mat).astype(np.float32)
    y = (acc * (scale_x * scale_w)).astype(np.float32)
    return np.maximum(y, 0.0).astype(np.float32)
```

```cpp
#include <hip/hip_runtime.h>
#include <rccl/rccl.h>
#include <cstdio>
#include <utility>

#define HIP_CHECK(expr) do { hipError_t _e = (expr); if (_e != hipSuccess) { \
    fprintf(stderr, "HIP error %s at %s:%d\n", hipGetErrorString(_e), __FILE__, __LINE__); \
    return; } } while (0)

namespace {

typedef unsigned u32;
typedef unsigned long long u64;
typedef u32   u32x4 __attribute__((ext_vector_type(4)));
typedef u32   u32x8 __attribute__((ext_vector_type(8)));
typedef float f32x4 __attribute__((ext_vector_type(4)));
template <int... Is> using iseq = std::integer_sequence<int, Is...>;
template <int N> using make_iseq = std::make_integer_sequence<int, N>;

constexpr int NR = 8;
constexpr int NBLK = 256;
constexpr u32 SLOT_B   = 2u << 20;
constexpr u32 PART_OFF = 3u * SLOT_B;
constexpr u32 FLAG_OFF = PART_OFF + (8u << 20);
constexpr u32 CNT_OFF  = FLAG_OFF + (128u << 10);
constexpr u32 TMO_OFF  = CNT_OFF + (128u << 10);
constexpr u32 DBG_OFF  = TMO_OFF + 4096;
constexpr u32 WS_NEED  = DBG_OFF + (1u << 20);
constexpr u64 DEADLINE = 40000000ull;
constexpr u32 TMO_CHECK_AFTER = 20000u;

#define PACE_TICKS_X16 240

constexpr u32 LDS_RED0 = 128u << 10;
constexpr u32 LDS_RED1 = 0;
constexpr u32 LDS_BYTES = LDS_RED0 + (24u << 10);

struct Params {
  unsigned char* ws[NR];
  const unsigned char* x;
  const unsigned char* w;
  const float* sx;
  const float* sw;
  float* out;
  int rank;
};

#define DI __device__ __forceinline__
#define ACLOB \
 "a0","a1","a2","a3","a4","a5","a6","a7","a8","a9","a10","a11","a12","a13","a14","a15","a16","a17","a18","a19","a20","a21","a22","a23","a24","a25","a26","a27","a28","a29","a30","a31", \
 "a32","a33","a34","a35","a36","a37","a38","a39","a40","a41","a42","a43","a44","a45","a46","a47","a48","a49","a50","a51","a52","a53","a54","a55","a56","a57","a58","a59","a60","a61","a62","a63", \
 "a64","a65","a66","a67","a68","a69","a70","a71","a72","a73","a74","a75","a76","a77","a78","a79","a80","a81","a82","a83","a84","a85","a86","a87","a88","a89","a90","a91","a92","a93","a94","a95", \
 "a96","a97","a98","a99","a100","a101","a102","a103","a104","a105","a106","a107","a108","a109","a110","a111","a112","a113","a114","a115","a116","a117","a118","a119","a120","a121","a122","a123","a124","a125","a126","a127", \
 "a128","a129","a130","a131","a132","a133","a134","a135","a136","a137","a138","a139","a140","a141","a142","a143","a144","a145","a146","a147","a148","a149","a150","a151","a152","a153","a154","a155","a156","a157","a158","a159", \
 "a160","a161","a162","a163","a164","a165","a166","a167","a168","a169","a170","a171","a172","a173","a174","a175","a176","a177","a178","a179","a180","a181","a182","a183","a184","a185","a186","a187","a188","a189","a190","a191", \
 "a192","a193","a194","a195","a196","a197","a198","a199","a200","a201","a202","a203","a204","a205","a206","a207","a208","a209","a210","a211","a212","a213","a214","a215","a216","a217","a218","a219","a220","a221","a222","a223", \
 "a224","a225","a226","a227","a228","a229","a230","a231","a232","a233","a234","a235","a236","a237","a238","a239","a240","a241","a242","a243","a244","a245","a246","a247","a248","a249","a250","a251","a252","a253","a254","a255"

DI u32 rfl(u32 x) { return __builtin_amdgcn_readfirstlane(x); }
DI u64 now() { return __builtin_amdgcn_s_memrealtime(); }
template <int N> DI void waitv() { asm volatile("s_waitcnt vmcnt(%0)" :: "n"(N) : "memory"); }
DI void waitl() { asm volatile("s_waitcnt lgkmcnt(0)" ::: "memory"); }
DI void bar() { asm volatile("s_barrier" ::: "memory"); }
#define GAP() __builtin_amdgcn_sched_barrier(0)

template <int OFF> DI void ld_sys(u32x4& d, u32 voff, const void* base) {
  asm volatile("global_load_dwordx4 %0, %1, %2 offset:%3 sc0 sc1" : "=v"(d) : "v"(voff), "s"(base), "n"(OFF) : "memory");
}
template <int OFF> DI void ld_plain(u32x4& d, u32 voff, const void* base) {
  asm volatile("global_load_dwordx4 %0, %1, %2 offset:%3" : "=v"(d) : "v"(voff), "s"(base), "n"(OFF) : "memory");
}
template <int OFF, typename T> DI void st_sys(u32 voff, const T& v, void* base) {
  asm volatile("global_store_dwordx4 %0, %1, %2 offset:%3 sc0 sc1\n\ts_nop 1" :: "v"(voff), "v"(v), "s"(base), "n"(OFF) : "memory");
}
DI void ld1_sys(u32& d, const void* base) {
  asm volatile("global_load_dword %0, %1, %2 sc0 sc1" : "=v"(d) : "v"(0u), "s"(base) : "memory");
}
DI void st1_sys(u32 v, void* base) {
  asm volatile("global_store_dword %0, %1, %2 sc0 sc1" :: "v"(0u), "v"(v), "s"(base) : "memory");
}

struct Ctx {
  u32x4 pd0, pd1; unsigned char* pdst; u64 tgo; bool pushed;
  u32 lane16;
  u64 t0; u32 seq; unsigned char* myws;
};

DI void do_push(Ctx& c) {
  st_sys<0>(c.lane16, c.pd0, c.pdst);
  st_sys<1024>(c.lane16, c.pd1, c.pdst);
  c.pushed = true;
}
DI void maybe_push(Ctx& c) {
  if (!c.pushed) { if (now() >= c.tgo) do_push(c); }
}
DI bool give_up(Ctx& c) {
  const u64 el = now() - c.t0;
  if (el < TMO_CHECK_AFTER) return false;
  u32 t; ld1_sys(t, c.myws + TMO_OFF); waitv<0>(); asm volatile("" : "+v"(t));
  if (rfl(t) == c.seq || el > DEADLINE) { st1_sys(c.seq, c.myws + TMO_OFF); return true; }
  return false;
}

template <int I> DI void w_dma1(u32 m0base, u32 voff0, u32 voff1, __amdgpu_buffer_rsrc_t srd, u32 soff) {
  asm volatile("s_mov_b32 m0, %0\n\ts_nop 0\n\tbuffer_load_dwordx4 %1, %2, %3 offen lds"
               :: "s"(m0base + 1024u * I), "v"((I & 2) ? voff1 : voff0), "s"(srd), "s"(soff + 32768u * I) : "m0", "memory");
}
template <int... Is> DI void w_dma(u32 m0base, u32 voff0, u32 voff1, __amdgpu_buffer_rsrc_t srd, u32 soff, iseq<Is...>) {
  (w_dma1<Is>(m0base, voff0, voff1, srd, soff), ...);
}
template <int KS, int J> DI void w_tr1(const u32 (&va)[8]) {
  constexpr int t = J >> 2, i = J & 3, a = (KS * 8 + t) * 8 + 2 * i;
  asm volatile("ds_read_b64_tr_b8 a[%c1:%c2], %0 offset:%c3"
               :: "v"(va[t]), "i"(a), "i"(a + 1), "i"((KS & 1) * 16384 + (64 * (i >> 1) + 8 * (i & 1)) * 128) : "memory", ACLOB);
}
template <int KS, int... Js> DI void w_tr(const u32 (&va)[8], iseq<Js...>) { (w_tr1<KS, Js>(va), ...); }

template <int KS, int T> DI void mfma1(f32x4& acc, const u32x8& xf) {
  constexpr int a = (KS * 8 + T) * 8;
  if constexpr (KS == 0)
    asm volatile("v_mfma_f32_16x16x128_f8f6f4 %0, a[%c2:%c3], %1, 0 cbsz:1 blgp:1" : "=&v"(acc) : "v"(xf), "i"(a), "i"(a + 7) : ACLOB);
  else
    asm volatile("v_mfma_f32_16x16x128_f8f6f4 %0, a[%c2:%c3], %1, %0 cbsz:1 blgp:1" : "+v"(acc) : "v"(xf), "i"(a), "i"(a + 7) : ACLOB);
}
template <int KS, int... Ts> DI void mfma_ks(f32x4 (&acc)[8], const u32x8& xf, iseq<Ts...>) { (mfma1<KS, Ts>(acc[Ts], xf), ...); }

DI void x_issue(u32x4 (&xb)[8], u32 lane16, const unsigned char* base4k) {
  ld_sys<-4096>(xb[0], lane16, base4k); ld_sys<-3072>(xb[1], lane16, base4k);
  ld_sys<-2048>(xb[2], lane16, base4k); ld_sys<-1024>(xb[3], lane16, base4k);
  ld_sys<0>(xb[4], lane16, base4k);     ld_sys<1024>(xb[5], lane16, base4k);
  ld_sys<2048>(xb[6], lane16, base4k);  ld_sys<3072>(xb[7], lane16, base4k);
}
#define XPIN(xb) "+v"(xb[0]), "+v"(xb[1]), "+v"(xb[2]), "+v"(xb[3]), "+v"(xb[4]), "+v"(xb[5]), "+v"(xb[6]), "+v"(xb[7])
DI u32 min3u(u32 a, u32 b, u32 c) { u32 r; asm("v_min3_u32 %0, %1, %2, %3" : "=v"(r) : "v"(a), "v"(b), "v"(c)); return r; }
DI bool x_valid(const u32x4 (&xb)[8]) {
  u32 m0 = min3u(xb[0][0], xb[0][1], xb[0][2]), m1 = min3u(xb[0][3], xb[1][0], xb[1][1]);
  m0 = min3u(m0, xb[1][2], xb[1][3]);
#pragma unroll
  for (int i = 2; i < 8; ++i) { m1 = min3u(m1, xb[i][0], xb[i][1]); m0 = min3u(m0, xb[i][2], xb[i][3]); }
  return __all((m0 != 0u) & (m1 != 0u));
}
DI u32 nz(u32 v) { return v == 0u ? 0x80808080u : v; }

#define STAMP(i) do {} while (0)

template <int PAR>
DI void phase(Ctx& c, const int j, u32x4 (&xc)[8], u32x4 (&xn)[8], const unsigned char* xbase4k, const u32 wr_addr, const u32 rd_addr,
              unsigned char* part_mine, const unsigned char* part_peer, unsigned char* flag_mine, const unsigned char* flag_peer,
              float* outp, const u32 out_voff, const float scale, const int b, const int w, const int lane) {
  asm volatile("s_waitcnt vmcnt(0)" : XPIN(xc) :: "memory");
  if (j >= 1 && j <= 4) st1_sys(c.seq * 4u + (u32)j, flag_mine);
  while (!x_valid(xc)) {
    maybe_push(c);
    if (give_up(c)) {
#pragma unroll
      for (int i = 0; i < 8; ++i) xc[i] = u32x4{0x7f7f7f7fu, 0x7f7f7f7fu, 0x7f7f7f7fu, 0x7f7f7f7fu};
      break;
    }
    x_issue(xc, c.lane16, xbase4k + (u32)j * 32768u);
    asm volatile("s_waitcnt vmcnt(0)" : XPIN(xc) :: "memory");
  }
  STAMP(8 + j);
  if (j < 7) x_issue(xn, c.lane16, xbase4k + (u32)(j + 1) * 32768u);
  maybe_push(c);
  f32x4 acc[8];
  {
    u32x8 f0 = __builtin_shufflevector(xc[0], xc[1], 0, 1, 2, 3, 4, 5, 6, 7);
    u32x8 f1 = __builtin_shufflevector(xc[2], xc[3], 0, 1, 2, 3, 4, 5, 6, 7);
    u32x8 f2 = __builtin_shufflevector(xc[4], xc[5], 0, 1, 2, 3, 4, 5, 6, 7);
    u32x8 f3 = __builtin_shufflevector(xc[6], xc[7], 0, 1, 2, 3, 4, 5, 6, 7);
    asm volatile("s_nop 1" : "+v"(f0), "+v"(f1), "+v"(f2), "+v"(f3));
    mfma_ks<0>(acc, f0, make_iseq<8>{});
    mfma_ks<1>(acc, f1, make_iseq<8>{});
    maybe_push(c);
    mfma_ks<2>(acc, f2, make_iseq<8>{});
    mfma_ks<3>(acc, f3, make_iseq<8>{});
  }
  asm volatile("s_nop 11" : "+v"(acc[0]), "+v"(acc[1]), "+v"(acc[2]), "+v"(acc[3]), "+v"(acc[4]), "+v"(acc[5]), "+v"(acc[6]), "+v"(acc[7]));
  constexpr u32 RB = PAR ? LDS_RED1 : LDS_RED0;
  {
    const u32 a1 = ((wr_addr + 1u * 6144u) % 24576u) + RB, a2 = ((wr_addr + 2u * 6144u) % 24576u) + RB, a3 = ((wr_addr + 3u * 6144u) % 24576u) + RB;
    asm volatile("ds_write_b128 %0, %1 offset:0"    :: "v"(a1), "v"(acc[2]) : "memory");
    asm volatile("ds_write_b128 %0, %1 offset:1024" :: "v"(a1), "v"(acc[3]) : "memory");
    asm volatile("ds_write_b128 %0, %1 offset:2048" :: "v"(a2), "v"(acc[4]) : "memory");
    asm volatile("ds_write_b128 %0, %1 offset:3072" :: "v"(a2), "v"(acc[5]) : "memory");
    asm volatile("ds_write_b128 %0, %1 offset:4096" :: "v"(a3), "v"(acc[6]) : "memory");
    asm volatile("ds_write_b128 %0, %1 offset:5120" :: "v"(a3), "v"(acc[7]) : "memory");
  }
  waitl();
  bar();
  f32x4 r[6];
  {
    const u32 ra = rd_addr + RB;
    asm volatile("ds_read_b128 %0, %1 offset:0"    : "=v"(r[0]) : "v"(ra) : "memory");
    asm volatile("ds_read_b128 %0, %1 offset:1024" : "=v"(r[1]) : "v"(ra) : "memory");
    asm volatile("ds_read_b128 %0, %1 offset:2048" : "=v"(r[2]) : "v"(ra) : "memory");
    asm volatile("ds_read_b128 %0, %1 offset:3072" : "=v"(r[3]) : "v"(ra) : "memory");
    asm volatile("ds_read_b128 %0, %1 offset:4096" : "=v"(r[4]) : "v"(ra) : "memory");
    asm volatile("ds_read_b128 %0, %1 offset:5120" : "=v"(r[5]) : "v"(ra) : "memory");
  }
  asm volatile("s_waitcnt lgkmcnt(0)" : "+v"(r[0]), "+v"(r[1]), "+v"(r[2]), "+v"(r[3]), "+v"(r[4]), "+v"(r[5]) :: "memory");
  f32x4 s0 = (acc[0] + r[0]) + (r[2] + r[4]);
  f32x4 s1 = (acc[1] + r[1]) + (r[3] + r[5]);
  if (j < 4) {
    st_sys<0>(c.lane16, s0, part_mine + (u32)j * 8192u);
    st_sys<1024>(c.lane16, s1, part_mine + (u32)j * 8192u);
  } else {
    const u32 want = c.seq * 4u + (u32)(j - 3);
    bool bad = false;
    for (;;) {
      u32 f; ld1_sys(f, flag_peer); waitv<0>(); asm volatile("" : "+v"(f));
      if ((int)(rfl(f) - want) >= 0) break;
      maybe_push(c);
      if (give_up(c)) { bad = true; break; }
    }
    u32x4 p0, p1;
    ld_sys<0>(p0, c.lane16, part_peer + (u32)(j - 4) * 8192u);
    ld_sys<1024>(p1, c.lane16, part_peer + (u32)(j - 4) * 8192u);
    asm volatile("s_waitcnt vmcnt(0)" : "+v"(p0), "+v"(p1) :: "memory");
    const float sc = bad ? __builtin_nanf("") : scale;
    s0 = (s0 + __builtin_bit_cast(f32x4, p0)) * sc;
    s1 = (s1 + __builtin_bit_cast(f32x4, p1)) * sc;
#pragma unroll
    for (int e = 0; e < 4; ++e) { s0[e] = s0[e] < 0.f ? 0.f : s0[e]; s1[e] = s1[e] < 0.f ? 0.f : s1[e]; }
    float* o = outp + (size_t)(j - 4) * 16 * 4096;
    st_sys<0>(out_voff, s0, o);
    st_sys<64>(out_voff, s1, o);
  }
  STAMP(16 + j);
}

__global__ __launch_bounds__(256, 1) void a2a_gemm_kernel(const Params p) {
  __shared__ __attribute__((aligned(16))) unsigned char smem[LDS_BYTES];
  Ctx c;
  c.t0 = now();
  const int b = blockIdx.x;
  const int lane = threadIdx.x & 63;
  const int w = (int)rfl(threadIdx.x >> 6);
  const u32 rank = (u32)p.rank & 7u;
  c.lane16 = (u32)lane * 16u;
  c.myws = p.ws[rank];
  const u32 g = (u32)lane >> 4, m = (u32)lane & 15u;

  unsigned char* cntp = c.myws + CNT_OFF + (u32)(b * 4 + w) * 128u;
  u32 cnt; ld1_sys(cnt, cntp);
  const u32 dst = ((u32)b + ((u32)b >> 3)) & 7u, u = (u32)b >> 3;
  {
    const u32 hs = rank >> 2, uj = u >> 2;
    const u32 row = dst * 512u + 128u * (u & 3u) + 64u * (uj < 4u ? 1u - hs : hs) + 16u * (uj & 3u);
    const unsigned char* src = p.x + (size_t)row * 512u + 128u * (u32)w;
    const u32 voff = m * 512u + g * 16u;
    ld_plain<0>(c.pd0, voff, src);
    ld_plain<64>(c.pd1, voff, src);
  }
  const u32 bi = (u32)b >> 3;
  const u32 cg = ((u32)b & 7u) * 4u + (bi & 3u), pt = (bi >> 2) & 3u, H = bi >> 4;
  const u32 src_rank = 4u * H + (u32)w;
  const u32 ldsb = (u32)(size_t)(__attribute__((address_space(3))) unsigned char*)smem;
  const __amdgpu_buffer_rsrc_t srdw = __builtin_amdgcn_make_buffer_rsrc((void*)p.w, 0, 4096u * 4096u, 0x00020000);
  const u32 wv0 = ((u32)lane >> 3) * 4096u + ((((u32)lane & 7u) ^ (g & 3u)) * 16u), wv1 = wv0 ^ 64u;
  const u32 wso = src_rank * 512u * 4096u + cg * 128u;
  const u32 m0b = ldsb + (u32)w * 32768u;
  const bool early = (dst == rank) || (u < 2u);
  if (!early) {
    w_dma(m0b, wv0, wv1, srdw, wso, make_iseq<16>{});
    asm volatile("s_waitcnt vmcnt(16)" : "+v"(cnt), "+v"(c.pd0), "+v"(c.pd1) :: "memory");
  } else {
    asm volatile("s_waitcnt vmcnt(0)" : "+v"(cnt), "+v"(c.pd0), "+v"(c.pd1) :: "memory");
  }
  c.seq = rfl(cnt) + 1u;
  const u32 slot = c.seq % 3u;
#pragma unroll
  for (int e = 0; e < 4; ++e) { c.pd0[e] = nz(c.pd0[e]); c.pd1[e] = nz(c.pd1[e]); }
  c.pdst = p.ws[dst] + slot * SLOT_B + rank * 262144u + u * 8192u + (u32)w * 2048u;
  c.tgo = c.t0 + ((u * PACE_TICKS_X16) >> 4);
  c.pushed = false;
  if (early) { do_push(c); w_dma(m0b, wv0, wv1, srdw, wso, make_iseq<16>{}); }
  else maybe_push(c);
  STAMP(0);
  w_dma(m0b + 16384u, wv0, wv1, srdw, wso + 128u * 4096u, make_iseq<16>{});

  u32 va[8];
  {
    const u32 q = m >> 1, f = ((q >> 1) & 3u) | ((g & 1u) << 2);
#pragma unroll
    for (int t = 0; t < 8; ++t)
      va[t] = m0b + (16u * g + q) * 128u + (((((u32)t + 2u * (u32)w) & 7u) ^ f) * 16u) + 8u * (m & 1u);
  }
  maybe_push(c);
  waitv<16>(); maybe_push(c);
  w_tr<0>(va, make_iseq<32>{}); waitl();
  w_dma(m0b, wv0, wv1, srdw, wso + 256u * 4096u, make_iseq<16>{});
  maybe_push(c);
  waitv<16>(); maybe_push(c);
  w_tr<1>(va, make_iseq<32>{}); waitl();
  w_dma(m0b + 16384u, wv0, wv1, srdw, wso + 384u * 4096u, make_iseq<16>{});
  maybe_push(c);
  waitv<16>(); maybe_push(c);
  w_tr<2>(va, make_iseq<32>{});
  const unsigned char* xbase4k = c.myws + slot * SLOT_B + src_rank * 262144u + pt * 8192u + 4096u;
  u32x4 xa[8], xb[8];
  x_issue(xa, c.lane16, xbase4k);
  {
    unsigned char* z = c.myws + ((slot + 2u) % 3u) * SLOT_B + (u32)b * 8192u + (u32)w * 2048u;
    const u32x4 zero = {0u, 0u, 0u, 0u};
    st_sys<0>(c.lane16, zero, z);
    st_sys<1024>(c.lane16, zero, z);
  }
  maybe_push(c);
  waitv<10>(); maybe_push(c);
  w_tr<3>(va, make_iseq<32>{}); waitl();
  STAMP(1);

  const float scale = p.sx[0] * p.sw[0];
  const u32 wr_addr = ldsb + (u32)w * 6144u + c.lane16, rd_addr = wr_addr;
  const u32 pb = (u32)b ^ 128u;
  unsigned char* part_mine = c.myws + PART_OFF + (u32)b * 32768u + (u32)w * 2048u;
  const unsigned char* part_peer = c.myws + PART_OFF + pb * 32768u + (u32)w * 2048u;
  unsigned char* flag_mine = c.myws + FLAG_OFF + (u32)(b * 4 + w) * 128u;
  const unsigned char* flag_peer = c.myws + FLAG_OFF + (pb * 4u + (u32)w) * 128u;
  float* outp = p.out + (size_t)(128u * pt + 64u * H) * 4096u + cg * 128u + 32u * (u32)w;
  const u32 out_voff = m * 16384u + g * 16u;

#pragma unroll 1
  for (int j = 0; j < 8; j += 2) {
    phase<0>(c, j, xa, xb, xbase4k, wr_addr, rd_addr, part_mine, part_peer, flag_mine, flag_peer, outp, out_voff, scale, b, w, lane);
    phase<1>(c, j + 1, xb, xa, xbase4k, wr_addr, rd_addr, part_mine, part_peer, flag_mine, flag_peer, outp, out_voff, scale, b, w, lane);
  }
  if (!c.pushed) do_push(c);
  st1_sys(c.seq, cntp);
  STAMP(2);
}

}

extern "C" void kernel_launch(int rank, int nranks, ncclComm_t comm,
                              hipStream_t stream,
                              void* const* d_in, const int* in_sizes, int n_in,
                              void* d_out, int out_size,
                              const int* in_dtypes, int out_dtype,
                              void* const* d_out_all,
                              void* const* d_ws_all, size_t ws_bytes) {
  (void)comm; (void)in_sizes; (void)in_dtypes; (void)out_dtype; (void)d_out_all;
  if (nranks != NR || n_in != 4 || out_size != 512 * 4096 || ws_bytes < WS_NEED || (unsigned)rank >= (unsigned)NR) {
    fprintf(stderr, "unexpected problem shape\n");
    return;
  }
  Params p;
  for (int i = 0; i < NR; ++i) p.ws[i] = (unsigned char*)d_ws_all[i];
  p.x = (const unsigned char*)d_in[0];
  p.w = (const unsigned char*)d_in[1];
  p.sx = (const float*)d_in[2];
  p.sw = (const float*)d_in[3];
  p.out = (float*)d_out;
  p.rank = rank;
  a2a_gemm_kernel<<<dim3(NBLK), dim3(256), 0, stream>>>(p);
  HIP_CHECK(hipGetLastError());
}
```

```cpp
#include <hip/hip_runtime.h>
#include <rccl/rccl.h>
#include <cstdio>

#define HIP_CHECK(expr) do { hipError_t _e = (expr); if (_e != hipSuccess) { \
    fprintf(stderr, "HIP error %s at %s:%d\n", hipGetErrorString(_e), __FILE__, __LINE__); \
    return; } } while (0)

namespace {

typedef unsigned u32;

constexpr int NR = 8;
constexpr int NBLK = 256;
constexpr u32 WS_NEED = 16777216;
constexpr u32 LDS_BYTES = 156416;

struct Params {
  unsigned char* myws;
  const unsigned char* x;
  const unsigned char* w;
  float* out;
  const float* sx;
  const float* sw;
  int rank;
  int pad[3];
  struct { const unsigned char* src; unsigned char* dst; } rel[NR];
  u32 cls[32][16];
};
static_assert(sizeof(Params) == 64 + 16 * NR + 2048, "kernarg layout");

__global__ __launch_bounds__(256, 1) void a2a_gemm_kernel(const Params p) {
  __shared__ __attribute__((aligned(16))) unsigned char smem[LDS_BYTES];
  const u32 ldsb = (u32)(size_t)(__attribute__((address_space(3))) unsigned char*)smem;
  asm volatile(
      "s_load_dwordx16 s[16:31], %[karg], 0x0\n\t"
      "s_lshr_b32 s69, %[bid], 3\n\t"
      "s_and_b32 s80, %[bid], 31\n\t"
      "s_add_u32 s81, s80, 24\n\t"
      "s_lshr_b32 s81, s81, 1\n\t"
      "s_min_u32 s81, s80, s81\n\t"
      "s_mul_i32 s82, s81, 37\n\t"
      "s_lshr_b32 s82, s82, 8\n\t"
      "s_mul_i32 s83, s82, 7\n\t"
      "s_sub_u32 s79, s81, s83\n\t"
      "s_add_u32 s79, s79, 1\n\t"
      "s_and_b32 s80, s80, 25\n\t"
      "s_cmp_eq_u32 s80, 24\n\t"
      "s_cselect_b32 s79, 0, s79\n\t"
      "s_and_b32 s85, s69, 28\n\t"
      "s_add_u32 s85, s85, s82\n\t"
      "s_lshl_b32 s80, s79, 4\n\t"
      "s_load_dwordx4 s[96:99], %[karg], s80 offset:0x40\n\t"
      "s_lshl_b32 s80, s69, 6\n\t"
      "s_load_dwordx16 s[48:63], %[karg], s80 offset:0xc0\n\t"
      "s_memrealtime s[86:87]\n\t"
      "v_lshrrev_b32 v205, 6, %[tid]\n\t"
      "v_and_b32 v255, 63, %[tid]\n\t"
      "v_readfirstlane_b32 s68, v205\n\t"
      "v_lshrrev_b32 v209, 3, v255\n\t"
      "v_lshrrev_b32 v218, 4, v255\n\t"
      "v_and_b32 v210, 7, v255\n\t"
      "v_xor_b32 v211, v210, v218\n\t"
      "v_lshlrev_b32 v211, 4, v211\n\t"
      "v_lshl_or_b32 v201, v209, 9, v211\n\t"
      "v_xor_b32 v202, 64, v201\n\t"
      "v_add_u32 v202, 3072, v202\n\t"
      "v_lshlrev_b32 v200, 4, v255\n\t"
      "s_lshr_b32 s71, s69, 4\n\t"
      "s_bfe_u32 s72, s69, 0x20002\n\t"
      "s_lshl_b32 s70, s72, 1\n\t"
      "s_or_b32 s70, s70, s71\n\t"
      "s_and_b32 s73, %[bid], 7\n\t"
      "s_lshl_b32 s73, s73, 2\n\t"
      "s_and_b32 s80, s69, 3\n\t"
      "s_or_b32 s73, s73, s80\n\t"
      "s_lshl_b32 s76, %[bid], 2\n\t"
      "s_add_u32 s76, s76, s68\n\t"
      "s_lshl_b32 s82, s76, 9\n\t"
      "s_add_u32 s82, s82, 6291456\n\t"
      "v_min_u32 v208, 2, v255\n\t"
      "v_lshl_add_u32 v208, v208, 7, s82\n\t"
      "s_cmp_lt_u32 s68, 2\n\t"
      "s_cbranch_scc1 LOADER_%=\n\t"
      "v_mov_b32 v217, 0\n\t"
      "v_lshlrev_b32 v203, 2, v255\n\t"
      "v_add_u32 v203, 155648, v203\n\t"
      "ds_write2st64_b32 v203, v217, v217 offset1:1\n\t"
      "s_lshr_b32 s81, s85, 3\n\t"
      "s_sub_u32 s80, s85, s79\n\t"
      "s_and_b32 s80, s80, 7\n\t"
      "s_xor_b32 s82, s80, s71\n\t"
      "s_xor_b32 s82, s82, 1\n\t"
      "s_lshl_b32 s82, s82, 15\n\t"
      "s_and_b32 s83, s80, 1\n\t"
      "s_lshl_b32 s83, s83, 8\n\t"
      "s_or_b32 s82, s82, s83\n\t"
      "s_and_b32 s83, s81, 1\n\t"
      "s_lshl_b32 s83, s83, 14\n\t"
      "s_or_b32 s82, s82, s83\n\t"
      "s_and_b32 s74, s68, 1\n\t"
      "s_lshl_b32 s83, s74, 13\n\t"
      "s_or_b32 s82, s82, s83\n\t"
      "v_add_u32 v205, s82, v201\n\t"
      "v_add_u32 v206, s82, v202\n\t"
      "s_lshl_b32 s80, s80, 2\n\t"
      "s_add_u32 s80, s80, s81\n\t"
      "s_lshl_b32 s80, s80, 13\n\t"
      "s_lshl_b32 s77, s74, 12\n\t"
      "s_add_u32 s66, s80, s77\n\t"
      "s_min_u32 s65, s79, 1\n\t"
      "s_cmp_eq_u32 s65, 0\n\t"
      "s_waitcnt lgkmcnt(0)\n\t"
      "s_cbranch_scc1 Ln173_%=\n\t"
      "global_load_dwordx4 v[136:139], v205, s[96:97] offset:0\n\t"
      "global_load_dwordx4 v[140:143], v206, s[96:97] offset:1024\n\t"
      "global_load_dwordx4 v[144:147], v205, s[96:97] offset:128\n\t"
      "global_load_dwordx4 v[148:151], v206, s[96:97] offset:1152\n\t"
      "Ln173_%=:\n\t"
      "global_load_dword v207, v208, s[16:17] sc0 sc1\n\t"
      "s_mov_b32 s67, s86\n\t"
      "s_barrier\n\t"
      "s_load_dword s94, s[24:25], 0x0\n\t"
      "s_load_dword s95, s[26:27], 0x0\n\t"
      "v_lshlrev_b32 v203, 2, v255\n\t"
      "v_add_u32 v203, 155648, v203\n\t"
      "v_mov_b32 v204, 1\n\t"
      "s_mov_b32 s84, 0\n\t"
      "v_and_b32 v216, 15, v255\n\t"
      "v_lshrrev_b32 v211, 1, v216\n\t"
      "v_lshl_add_u32 v244, v218, 4, v211\n\t"
      "v_lshlrev_b32 v244, 7, v244\n\t"
      "v_xor_b32 v238, v211, v218\n\t"
      "v_lshlrev_b32 v238, 4, v238\n\t"
      "v_lshl_or_b32 v238, v216, 7, v238\n\t"
      "v_xor_b32 v239, 64, v238\n\t"
      "v_lshrrev_b32 v211, 2, v216\n\t"
      "v_and_b32 v205, 1, v218\n\t"
      "v_lshl_or_b32 v211, v205, 2, v211\n\t"
      "s_lshl_b32 s81, s68, 1\n\t"
      "v_xor_b32 v211, s81, v211\n\t"
      "v_lshl_or_b32 v244, v211, 4, v244\n\t"
      "v_and_b32 v211, 1, v216\n\t"
      "v_lshl_or_b32 v244, v211, 3, v244\n\t"
      "v_xor_b32 v245, 16, v244\n\t"
      "s_lshl_b32 s52, s68, 11\n\t"
      "v_lshl_add_u32 v250, v255, 4, s52\n\t"
      "v_add_u32 v246, 65536, v244\n\t"
      "v_add_u32 v240, 65536, v238\n\t"
      "v_add_u32 v247, 65536, v245\n\t"
      "v_add_u32 v241, 65536, v239\n\t"
      "v_add_u32 v251, 65536, v250\n\t"
      "v_add_u32 v248, 131072, v244\n\t"
      "v_add_u32 v242, 131072, v238\n\t"
      "v_add_u32 v249, 131072, v245\n\t"
      "v_add_u32 v243, 131072, v239\n\t"
      "v_add_u32 v252, 131072, v250\n\t"
      "s_lshl_b32 s28, s68, 12\n\t"
      "v_and_b32 v211, 7, v216\n\t"
      "v_xor_b32 v211, v211, v218\n\t"
      "v_lshlrev_b32 v211, 4, v211\n\t"
      "v_lshl_or_b32 v211, v216, 7, v211\n\t"
      "v_add_u32 v236, s28, v211\n\t"
      "v_xor_b32 v237, 64, v236\n\t"
      "v_xor_b32 v211, v210, v209\n\t"
      "v_lshlrev_b32 v211, 4, v211\n\t"
      "v_lshl_or_b32 v211, v209, 7, v211\n\t"
      "v_mov_b32 v232, 0\n\t"
      "v_mov_b32 v233, 0\n\t"
      "v_mov_b32 v234, 0\n\t"
      "v_mov_b32 v235, 0\n\t"
      "v_lshlrev_b32 v210, 4, v210\n\t"
      "v_lshl_or_b32 v210, v209, 14, v210\n\t"
      "v_add_u32 v209, s28, v211\n\t"
      "v_add_u32 v211, 0x20000, v210\n\t"
      "s_lshl_b32 s28, s76, 9\n\t"
      "s_add_u32 s81, s28, 6291584\n\t"
      "v_mov_b32 v212, s81\n\t"
      "s_xor_b32 s28, s28, 262144\n\t"
      "s_add_u32 s81, s28, 6291584\n\t"
      "v_mov_b32 v213, s81\n\t"
      "s_lshl_b32 s28, s70, 20\n\t"
      "s_lshl_b32 s81, s73, 9\n\t"
      "s_or_b32 s28, s28, s81\n\t"
      "s_lshl_b32 s81, s68, 7\n\t"
      "s_or_b32 s28, s28, s81\n\t"
      "s_xor_b32 s81, s28, 0x100000\n\t"
      "s_add_u32 s30, s22, s81\n\t"
      "s_addc_u32 s31, s23, 0\n\t"
      "s_add_u32 s22, s22, s28\n\t"
      "s_addc_u32 s23, s23, 0\n\t"
      "v_mov_b32 v206, 0x80\n\t"
      "s_max_u32 s83, s85, 5\n\t"
      "s_sub_u32 s83, s83, 5\n\t"
      "s_mul_i32 s83, s83, 270\n\t"
      "s_lshr_b32 s83, s83, 4\n\t"
      "s_waitcnt vmcnt(0)\n\t"
      "s_memrealtime s[86:87]\n\t"
      "v_readfirstlane_b32 s64, v207\n\t"
      "s_add_u32 s64, s64, 1\n\t"
      "s_mul_hi_u32 s74, s64, 0xaaaaaaab\n\t"
      "s_lshr_b32 s74, s74, 1\n\t"
      "s_mul_i32 s74, s74, 3\n\t"
      "s_sub_u32 s74, s64, s74\n\t"
      "s_lshl_b32 s82, s74, 21\n\t"
      "s_add_u32 s81, s82, s66\n\t"
      "s_add_u32 s98, s98, s81\n\t"
      "s_addc_u32 s99, s99, 0\n\t"
      "v_cmp_eq_u32 s[36:37], 0, v136\n\t"
      "v_cmp_eq_u32 s[38:39], 0, v140\n\t"
      "v_cmp_eq_u32 s[40:41], 0, v144\n\t"
      "v_cmp_eq_u32 s[42:43], 0, v148\n\t"
      "v_cndmask_b32 v136, v136, v206, s[36:37]\n\t"
      "v_cndmask_b32 v140, v140, v206, s[38:39]\n\t"
      "v_cndmask_b32 v144, v144, v206, s[40:41]\n\t"
      "v_cndmask_b32 v148, v148, v206, s[42:43]\n\t"
      "s_cmp_eq_u32 s65, 0\n\t"
      "s_cbranch_scc1 Lq174_%=\n\t"
      "s_cmp_lg_u32 s83, 0\n\t"
      "s_cbranch_scc1 Lq174_%=\n\t"
      "s_call_b64 s[88:89], PUSH_%=\n\t"
      "Lq174_%=:\n\t"
      "s_barrier\n\t"
      "s_waitcnt lgkmcnt(0)\n\t"
      "s_add_u32 s66, s86, s83\n\t"
      "s_add_u32 s80, s74, 2\n\t"
      "s_cmp_ge_u32 s80, 3\n\t"
      "s_cselect_b32 s81, 3, 0\n\t"
      "s_sub_u32 s80, s80, s81\n\t"
      "s_lshl_b32 s80, s80, 21\n\t"
      "s_lshl_b32 s81, %[bid], 13\n\t"
      "s_add_u32 s80, s80, s81\n\t"
      "s_add_u32 s80, s80, s77\n\t"
      "v_add_u32 v214, s80, v200\n\t"
      "s_lshl_b32 s81, s70, 15\n\t"
      "s_add_u32 s82, s82, s81\n\t"
      "s_add_u32 s82, s82, s52\n\t"
      "s_add_u32 s44, s16, s82\n\t"
      "s_addc_u32 s45, s17, 0\n\t"
      "s_bfe_u32 s18, s48, 0x30000\n\t"
      "s_lshl_b32 s18, s18, 18\n\t"
      "s_bfe_u32 s19, s48, 0x30004\n\t"
      "s_lshl_b32 s19, s19, 18\n\t"
      "s_bfe_u32 s20, s48, 0x30008\n\t"
      "s_lshl_b32 s20, s20, 18\n\t"
      "s_bfe_u32 s21, s48, 0x3000c\n\t"
      "s_lshl_b32 s21, s21, 18\n\t"
      "s_bfe_u32 s24, s48, 0x30010\n\t"
      "s_lshl_b32 s24, s24, 18\n\t"
      "s_bfe_u32 s25, s48, 0x30014\n\t"
      "s_lshl_b32 s25, s25, 18\n\t"
      "s_bfe_u32 s26, s48, 0x30018\n\t"
      "s_lshl_b32 s26, s26, 18\n\t"
      "s_mov_b32 s46, 2097152\n\t"
      "s_mov_b32 s47, 0x20000\n\t"
      "v_mov_b32 v205, s94\n\t"
      "v_mul_f32 v205, s95, v205\n\t"
      "s_mov_b64 s[94:95], s[22:23]\n\t"
      "v_readfirstlane_b32 s75, v205\n\t"
      "s_cmp_lg_u32 s65, 0\n\t"
      "s_cbranch_scc1 Lp175_%=\n\t"
      "Lr176_%=:\n\t"
      "ds_add_u32 v203, v204 offset:256\n\t"
      "s_cmp_lg_u32 s65, 0\n\t"
      "s_cbranch_scc1 Lp177_%=\n\t"
      "Lr178_%=:\n\t"
      "ds_read_b32 v253, v203 offset:256\n\t"
      "s_waitcnt lgkmcnt(0)\n\t"
      "v_cmp_gt_u32 vcc, 4, v253\n\t"
      "s_cbranch_vccnz AQ179_%=\n\t"
      "AR180_%=:\n\t"
      "ds_read_b128 v[32:35], v242 offset:16384\n\t"
      "ds_read_b64_tr_b8 a[0:1], v244 offset:0\n\t"
      "ds_read_b64_tr_b8 a[2:3], v244 offset:1024\n\t"
      "ds_read_b64_tr_b8 a[4:5], v244 offset:8192\n\t"
      "ds_read_b128 v[36:39], v243 offset:16384\n\t"
      "s_cmp_lg_u32 s65, 0\n\t"
      "s_cbranch_scc1 Lp181_%=\n\t"
      "Lr182_%=:\n\t"
      "ds_read_b64_tr_b8 a[6:7], v244 offset:9216\n\t"
      "ds_read_b128 v[40:43], v242 offset:18432\n\t"
      "ds_read_b64_tr_b8 a[8:9], v245 offset:0\n\t"
      "ds_read_b64_tr_b8 a[10:11], v245 offset:1024\n\t"
      "ds_read_b64_tr_b8 a[12:13], v245 offset:8192\n\t"
      "ds_read_b64_tr_b8 a[14:15], v245 offset:9216\n\t"
      "ds_read_b128 v[44:47], v243 offset:18432\n\t"
      "s_waitcnt lgkmcnt(1)\n\t"
      "ds_add_u32 v203, v204 offset:0\n\t"
      "ds_read_b128 v[48:51], v242 offset:20480\n\t"
      "ds_read_b128 v[52:55], v243 offset:20480\n\t"
      "s_cmp_lg_u32 s65, 0\n\t"
      "s_cbranch_scc1 Lp183_%=\n\t"
      "Lr184_%=:\n\t"
      "v_mfma_f32_16x16x128_f8f6f4 v[72:75], a[0:7], v[32:39], 0 cbsz:1 blgp:1\n\t"
      "ds_read_b32 v253, v203 offset:0\n\t"
      "ds_read_b128 v[56:59], v242 offset:22528\n\t"
      "v_mfma_f32_16x16x128_f8f6f4 v[76:79], a[8:15], v[32:39], 0 cbsz:1 blgp:1\n\t"
      "s_waitcnt lgkmcnt(1)\n\t"
      "v_cmp_gt_u32 vcc, 4, v253\n\t"
      "ds_read_b128 v[60:63], v243 offset:22528\n\t"
      "v_mfma_f32_16x16x128_f8f6f4 v[80:83], a[0:7], v[40:47], 0 cbsz:1 blgp:1\n\t"
      "s_cbranch_vccnz AQ185_%=\n\t"
      "AR186_%=:\n\t"
      "ds_read_b128 v[64:67], v242 offset:8192\n\t"
      "ds_read_b64_tr_b8 a[16:17], v244 offset:32768\n\t"
      "ds_read_b64_tr_b8 a[18:19], v244 offset:33792\n\t"
      "ds_read_b64_tr_b8 a[20:21], v244 offset:40960\n\t"
      "global_store_dwordx4 v214, v[232:235], s[16:17] offset:0 sc0 sc1\n\t"
      "v_mfma_f32_16x16x128_f8f6f4 v[84:87], a[8:15], v[40:47], 0 cbsz:1 blgp:1\n\t"
      "ds_read_b128 v[68:71], v243 offset:8192\n\t"
      "s_cmp_lg_u32 s65, 0\n\t"
      "s_cbranch_scc1 Lp187_%=\n\t"
      "Lr188_%=:\n\t"
      "ds_read_b64_tr_b8 a[22:23], v244 offset:41984\n\t"
      "global_store_dwordx4 v214, v[232:235], s[16:17] offset:1024 sc0 sc1\n\t"
      "v_mfma_f32_16x16x128_f8f6f4 v[88:91], a[0:7], v[48:55], 0 cbsz:1 blgp:1\n\t"
      "ds_read_b128 v[32:35], v242 offset:10240\n\t"
      "ds_read_b64_tr_b8 a[24:25], v245 offset:32768\n\t"
      "ds_read_b64_tr_b8 a[26:27], v245 offset:33792\n\t"
      "ds_read_b64_tr_b8 a[28:29], v245 offset:40960\n\t"
      "ds_read_b64_tr_b8 a[30:31], v245 offset:41984\n\t"
      "ds_read_b64_tr_b8 a[32:33], v244 offset:16384\n\t"
      "ds_read_b64_tr_b8 a[34:35], v244 offset:17408\n\t"
      "v_mfma_f32_16x16x128_f8f6f4 v[92:95], a[8:15], v[48:55], 0 cbsz:1 blgp:1\n\t"
      "ds_read_b128 v[36:39], v243 offset:10240\n\t"
      "ds_read_b64_tr_b8 a[36:37], v244 offset:24576\n\t"
      "ds_read_b64_tr_b8 a[38:39], v244 offset:25600\n\t"
      "ds_read_b64_tr_b8 a[40:41], v245 offset:16384\n\t"
      "ds_read_b64_tr_b8 a[42:43], v245 offset:17408\n\t"
      "ds_read_b64_tr_b8 a[44:45], v245 offset:24576\n\t"
      "ds_read_b64_tr_b8 a[46:47], v245 offset:25600\n\t"
      "s_waitcnt lgkmcnt(15)\n\t"
      "v_mfma_f32_16x16x128_f8f6f4 v[96:99], a[0:7], v[56:63], 0 cbsz:1 blgp:1\n\t"
      "s_waitcnt lgkmcnt(0)\n\t"
      "ds_add_u32 v203, v204 offset:256\n\t"
      "ds_read_b128 v[40:43], v242 offset:12288\n\t"
      "global_store_dwordx4 v214, v[232:235], s[16:17] offset:2048 sc0 sc1\n\t"
      "v_mfma_f32_16x16x128_f8f6f4 v[100:103], a[8:15], v[56:63], 0 cbsz:1 blgp:1\n\t"
      "ds_read_b128 v[44:47], v243 offset:12288\n\t"
      "s_cmp_lg_u32 s65, 0\n\t"
      "s_cbranch_scc1 Lp189_%=\n\t"
      "Lr190_%=:\n\t"
      "global_store_dwordx4 v214, v[232:235], s[16:17] offset:3072 sc0 sc1\n\t"
      "v_mfma_f32_16x16x128_f8f6f4 v[104:107], a[0:7], v[64:71], 0 cbsz:1 blgp:1\n\t"
      "ds_read_b32 v253, v203 offset:256\n\t"
      "ds_read_b128 v[48:51], v242 offset:14336\n\t"
      "v_mfma_f32_16x16x128_f8f6f4 v[108:111], a[8:15], v[64:71], 0 cbsz:1 blgp:1\n\t"
      "s_waitcnt lgkmcnt(1)\n\t"
      "v_cmp_gt_u32 vcc, 8, v253\n\t"
      "ds_read_b128 v[52:55], v243 offset:14336\n\t"
      "v_mfma_f32_16x16x128_f8f6f4 v[112:115], a[0:7], v[32:39], 0 cbsz:1 blgp:1\n\t"
      "s_cbranch_vccnz AQ191_%=\n\t"
      "AR192_%=:\n\t"
      "ds_read_b128 v[56:59], v242 offset:0\n\t"
      "ds_read_b64_tr_b8 a[48:49], v244 offset:49152\n\t"
      "ds_read_b64_tr_b8 a[50:51], v244 offset:50176\n\t"
      "ds_read_b64_tr_b8 a[52:53], v244 offset:57344\n\t"
      "v_mfma_f32_16x16x128_f8f6f4 v[116:119], a[8:15], v[32:39], 0 cbsz:1 blgp:1\n\t"
      "ds_read_b128 v[60:63], v243 offset:0\n\t"
      "s_cmp_lg_u32 s65, 0\n\t"
      "s_cbranch_scc1 Lp193_%=\n\t"
      "Lr194_%=:\n\t"
      "ds_read_b64_tr_b8 a[54:55], v244 offset:58368\n\t"
      "v_mfma_f32_16x16x128_f8f6f4 v[120:123], a[0:7], v[40:47], 0 cbsz:1 blgp:1\n\t"
      "ds_read_b128 v[64:67], v242 offset:2048\n\t"
      "ds_read_b64_tr_b8 a[56:57], v245 offset:49152\n\t"
      "ds_read_b64_tr_b8 a[58:59], v245 offset:50176\n\t"
      "ds_read_b64_tr_b8 a[60:61], v245 offset:57344\n\t"
      "ds_read_b64_tr_b8 a[62:63], v245 offset:58368\n\t"
      "v_mfma_f32_16x16x128_f8f6f4 v[124:127], a[8:15], v[40:47], 0 cbsz:1 blgp:1\n\t"
      "ds_read_b128 v[68:71], v243 offset:2048\n\t"
      "s_waitcnt lgkmcnt(12)\n\t"
      "v_mfma_f32_16x16x128_f8f6f4 v[128:131], a[0:7], v[48:55], 0 cbsz:1 blgp:1\n\t"
      "s_waitcnt lgkmcnt(1)\n\t"
      "ds_add_u32 v203, v204 offset:0\n\t"
      "ds_read_b128 v[32:35], v242 offset:4096\n\t"
      "v_mfma_f32_16x16x128_f8f6f4 v[132:135], a[8:15], v[48:55], 0 cbsz:1 blgp:1\n\t"
      "ds_read_b128 v[36:39], v243 offset:4096\n\t"
      "s_cmp_lg_u32 s65, 0\n\t"
      "s_cbranch_scc1 Lp195_%=\n\t"
      "Lr196_%=:\n\t"
      "v_mfma_f32_16x16x128_f8f6f4 v[72:75], a[16:23], v[56:63], v[72:75] cbsz:1 blgp:1\n\t"
      "ds_read_b32 v253, v203 offset:0\n\t"
      "ds_read_b128 v[40:43], v242 offset:6144\n\t"
      "v_mfma_f32_16x16x128_f8f6f4 v[76:79], a[24:31], v[56:63], v[76:79] cbsz:1 blgp:1\n\t"
      "s_waitcnt lgkmcnt(1)\n\t"
      "v_cmp_gt_u32 vcc, 8, v253\n\t"
      "ds_read_b128 v[44:47], v243 offset:6144\n\t"
      "v_mfma_f32_16x16x128_f8f6f4 v[80:83], a[16:23], v[64:71], v[80:83] cbsz:1 blgp:1\n\t"
      "s_cbranch_vccnz AQ197_%=\n\t"
      "AR198_%=:\n\t"
      "ds_read_b128 v[48:51], v240 offset:57344\n\t"
      "ds_read_b64_tr_b8 a[64:65], v246 offset:0\n\t"
      "ds_read_b64_tr_b8 a[66:67], v246 offset:1024\n\t"
      "ds_read_b64_tr_b8 a[68:69], v246 offset:8192\n\t"
      "v_mfma_f32_16x16x128_f8f6f4 v[84:87], a[24:31], v[64:71], v[84:87] cbsz:1 blgp:1\n\t"
      "ds_read_b128 v[52:55], v241 offset:57344\n\t"
      "s_cmp_lg_u32 s65, 0\n\t"
      "s_cbranch_scc1 Lp199_%=\n\t"
      "Lr200_%=:\n\t"
      "ds_read_b64_tr_b8 a[70:71], v246 offset:9216\n\t"
      "v_mfma_f32_16x16x128_f8f6f4 v[88:91], a[16:23], v[32:39], v[88:91] cbsz:1 blgp:1\n\t"
      "ds_read_b128 v[56:59], v240 offset:59392\n\t"
      "ds_read_b64_tr_b8 a[72:73], v247 offset:0\n\t"
      "ds_read_b64_tr_b8 a[74:75], v247 offset:1024\n\t"
      "ds_read_b64_tr_b8 a[76:77], v247 offset:8192\n\t"
      "ds_read_b64_tr_b8 a[78:79], v247 offset:9216\n\t"
      "ds_read_b64_tr_b8 a[80:81], v246 offset:16384\n\t"
      "ds_read_b64_tr_b8 a[82:83], v246 offset:17408\n\t"
      "v_mfma_f32_16x16x128_f8f6f4 v[92:95], a[24:31], v[32:39], v[92:95] cbsz:1 blgp:1\n\t"
      "ds_read_b128 v[60:63], v241 offset:59392\n\t"
      "ds_read_b64_tr_b8 a[84:85], v246 offset:24576\n\t"
      "ds_read_b64_tr_b8 a[86:87], v246 offset:25600\n\t"
      "ds_read_b64_tr_b8 a[88:89], v247 offset:16384\n\t"
      "ds_read_b64_tr_b8 a[90:91], v247 offset:17408\n\t"
      "ds_read_b64_tr_b8 a[92:93], v247 offset:24576\n\t"
      "ds_read_b64_tr_b8 a[94:95], v247 offset:25600\n\t"
      "s_waitcnt lgkmcnt(15)\n\t"
      "v_mfma_f32_16x16x128_f8f6f4 v[96:99], a[16:23], v[40:47], v[96:99] cbsz:1 blgp:1\n\t"
      "s_waitcnt lgkmcnt(0)\n\t"
      "ds_add_u32 v203, v204 offset:256\n\t"
      "ds_read_b128 v[64:67], v240 offset:61440\n\t"
      "v_mfma_f32_16x16x128_f8f6f4 v[100:103], a[24:31], v[40:47], v[100:103] cbsz:1 blgp:1\n\t"
      "ds_read_b128 v[68:71], v241 offset:61440\n\t"
      "s_cmp_lg_u32 s65, 0\n\t"
      "s_cbranch_scc1 Lp201_%=\n\t"
      "Lr202_%=:\n\t"
      "v_mfma_f32_16x16x128_f8f6f4 v[104:107], a[16:23], v[48:55], v[104:107] cbsz:1 blgp:1\n\t"
      "ds_read_b32 v253, v203 offset:256\n\t"
      "ds_read_b128 v[32:35], v240 offset:63488\n\t"
      "v_mfma_f32_16x16x128_f8f6f4 v[108:111], a[24:31], v[48:55], v[108:111] cbsz:1 blgp:1\n\t"
      "s_waitcnt lgkmcnt(1)\n\t"
      "v_cmp_gt_u32 vcc, 12, v253\n\t"
      "ds_read_b128 v[36:39], v241 offset:63488\n\t"
      "v_mfma_f32_16x16x128_f8f6f4 v[112:115], a[16:23], v[56:63], v[112:115] cbsz:1 blgp:1\n\t"
      "s_cbranch_vccnz AQ203_%=\n\t"
      "AR204_%=:\n\t"
      "ds_read_b64_tr_b8 a[96:97], v244 offset:0\n\t"
      "ds_read_b64_tr_b8 a[98:99], v244 offset:1024\n\t"
      "ds_read_b64_tr_b8 a[100:101], v244 offset:8192\n\t"
      "ds_read_b64_tr_b8 a[102:103], v244 offset:9216\n\t"
      "v_mfma_f32_16x16x128_f8f6f4 v[116:119], a[24:31], v[56:63], v[116:119] cbsz:1 blgp:1\n\t"
      "s_cmp_lg_u32 s65, 0\n\t"
      "s_cbranch_scc1 Lp205_%=\n\t"
      "Lr206_%=:\n\t"
      "ds_read_b64_tr_b8 a[104:105], v245 offset:0\n\t"
      "ds_read_b64_tr_b8 a[106:107], v245 offset:1024\n\t"
      "v_mfma_f32_16x16x128_f8f6f4 v[120:123], a[16:23], v[64:71], v[120:123] cbsz:1 blgp:1\n\t"
      "ds_read_b64_tr_b8 a[108:109], v245 offset:8192\n\t"
      "ds_read_b64_tr_b8 a[110:111], v245 offset:9216\n\t"
      "ds_read_b64_tr_b8 a[112:113], v246 offset:32768\n\t"
      "ds_read_b64_tr_b8 a[114:115], v246 offset:33792\n\t"
      "ds_read_b64_tr_b8 a[116:117], v246 offset:40960\n\t"
      "v_mfma_f32_16x16x128_f8f6f4 v[124:127], a[24:31], v[64:71], v[124:127] cbsz:1 blgp:1\n\t"
      "ds_read_b64_tr_b8 a[118:119], v246 offset:41984\n\t"
      "ds_read_b64_tr_b8 a[120:121], v247 offset:32768\n\t"
      "ds_read_b64_tr_b8 a[122:123], v247 offset:33792\n\t"
      "ds_read_b64_tr_b8 a[124:125], v247 offset:40960\n\t"
      "ds_read_b64_tr_b8 a[126:127], v247 offset:41984\n\t"
      "s_waitcnt lgkmcnt(15)\n\t"
      "v_mfma_f32_16x16x128_f8f6f4 v[128:131], a[16:23], v[32:39], v[128:131] cbsz:1 blgp:1\n\t"
      "s_waitcnt lgkmcnt(0)\n\t"
      "ds_add_u32 v203, v204 offset:0\n\t"
      "v_mfma_f32_16x16x128_f8f6f4 v[132:135], a[24:31], v[32:39], v[132:135] cbsz:1 blgp:1\n\t"
      "s_cmp_lg_u32 s65, 0\n\t"
      "s_cbranch_scc1 Lp207_%=\n\t"
      "Lr208_%=:\n\t"
      "ds_read_b32 v253, v203 offset:0\n\t"
      "s_add_u32 m0, s52, 147456\n\t"
      "s_add_u32 s80, s18, 0\n\t"
      "buffer_load_dwordx4 v200, s[44:47], s80 offen offset:0 sc0 sc1 lds\n\t"
      "buffer_load_dwordx4 v200, s[44:47], s80 offen offset:1024 sc0 sc1 lds\n\t"
      "s_waitcnt lgkmcnt(0)\n\t"
      "v_cmp_gt_u32 vcc, 12, v253\n\t"
      "s_cbranch_vccnz AQ209_%=\n\t"
      "AR210_%=:\n\t"
      "ds_read_b64_tr_b8 a[128:129], v244 offset:16384\n\t"
      "ds_read_b64_tr_b8 a[130:131], v244 offset:17408\n\t"
      "ds_read_b64_tr_b8 a[132:133], v244 offset:24576\n\t"
      "ds_read_b64_tr_b8 a[134:135], v244 offset:25600\n\t"
      "s_cmp_lg_u32 s65, 0\n\t"
      "s_cbranch_scc1 Lp211_%=\n\t"
      "Lr212_%=:\n\t"
      "ds_read_b64_tr_b8 a[136:137], v245 offset:16384\n\t"
      "ds_read_b64_tr_b8 a[138:139], v245 offset:17408\n\t"
      "ds_read_b64_tr_b8 a[140:141], v245 offset:24576\n\t"
      "ds_read_b64_tr_b8 a[142:143], v245 offset:25600\n\t"
      "ds_read_b64_tr_b8 a[144:145], v244 offset:32768\n\t"
      "ds_read_b64_tr_b8 a[146:147], v244 offset:33792\n\t"
      "ds_read_b64_tr_b8 a[148:149], v244 offset:40960\n\t"
      "s_add_u32 m0, s52, 122880\n\t"
      "s_add_u32 s80, s19, 0\n\t"
      "buffer_load_dwordx4 v200, s[44:47], s80 offen offset:0 sc0 sc1 lds\n\t"
      "buffer_load_dwordx4 v200, s[44:47], s80 offen offset:1024 sc0 sc1 lds\n\t"
      "ds_read_b64_tr_b8 a[150:151], v244 offset:41984\n\t"
      "ds_read_b64_tr_b8 a[152:153], v245 offset:32768\n\t"
      "ds_read_b64_tr_b8 a[154:155], v245 offset:33792\n\t"
      "ds_read_b64_tr_b8 a[156:157], v245 offset:40960\n\t"
      "ds_read_b64_tr_b8 a[158:159], v245 offset:41984\n\t"
      "s_waitcnt lgkmcnt(0)\n\t"
      "ds_add_u32 v203, v204 offset:256\n\t"
      "s_cmp_lg_u32 s65, 0\n\t"
      "s_cbranch_scc1 Lp213_%=\n\t"
      "Lr214_%=:\n\t"
      "ds_read_b32 v253, v203 offset:256\n\t"
      "s_waitcnt lgkmcnt(0)\n\t"
      "v_cmp_gt_u32 vcc, 16, v253\n\t"
      "s_cbranch_vccnz AQ215_%=\n\t"
      "AR216_%=:\n\t"
      "s_add_u32 m0, s52, 114688\n\t"
      "s_add_u32 s80, s20, 0\n\t"
      "buffer_load_dwordx4 v200, s[44:47], s80 offen offset:0 sc0 sc1 lds\n\t"
      "buffer_load_dwordx4 v200, s[44:47], s80 offen offset:1024 sc0 sc1 lds\n\t"
      "s_cmp_lg_u32 s65, 0\n\t"
      "s_cbranch_scc1 Lp217_%=\n\t"
      "Lr218_%=:\n\t"
      "ds_read_b64_tr_b8 a[160:161], v244 offset:49152\n\t"
      "ds_read_b64_tr_b8 a[162:163], v244 offset:50176\n\t"
      "ds_read_b64_tr_b8 a[164:165], v244 offset:57344\n\t"
      "ds_read_b64_tr_b8 a[166:167], v244 offset:58368\n\t"
      "s_waitcnt vmcnt(2)\n\t"
      "ds_read2st64_b32 v[228:229], v252 offset0:64 offset1:68\n\t"
      "ds_read2st64_b32 v[230:231], v251 offset0:224 offset1:228\n\t"
      "ds_read_b64_tr_b8 a[168:169], v245 offset:49152\n\t"
      "ds_read_b64_tr_b8 a[170:171], v245 offset:50176\n\t"
      "ds_read_b64_tr_b8 a[172:173], v245 offset:57344\n\t"
      "ds_read_b64_tr_b8 a[174:175], v245 offset:58368\n\t"
      "ds_read_b64_tr_b8 a[176:177], v246 offset:0\n\t"
      "ds_read_b64_tr_b8 a[178:179], v246 offset:1024\n\t"
      "ds_read_b64_tr_b8 a[180:181], v246 offset:8192\n\t"
      "ds_read_b64_tr_b8 a[182:183], v246 offset:9216\n\t"
      "ds_read_b64_tr_b8 a[184:185], v247 offset:0\n\t"
      "ds_read_b64_tr_b8 a[186:187], v247 offset:1024\n\t"
      "ds_read_b64_tr_b8 a[188:189], v247 offset:8192\n\t"
      "ds_read_b64_tr_b8 a[190:191], v247 offset:9216\n\t"
      "s_waitcnt lgkmcnt(0)\n\t"
      "v_min3_u32 v228, v228, v229, v230\n\t"
      "v_min_u32 v228, v228, v231\n\t"
      "v_cmp_eq_u32 vcc, 0, v228\n\t"
      "s_cbranch_vccnz XS219_%=\n\t"
      "XR220_%=:\n\t"
      "ds_add_u32 v203, v204 offset:0\n\t"
      "s_cmp_lg_u32 s65, 0\n\t"
      "s_cbranch_scc1 Lp225_%=\n\t"
      "Lr226_%=:\n\t"
      "s_add_u32 m0, s52, 106496\n\t"
      "s_add_u32 s80, s21, 0\n\t"
      "buffer_load_dwordx4 v200, s[44:47], s80 offen offset:0 sc0 sc1 lds\n\t"
      "buffer_load_dwordx4 v200, s[44:47], s80 offen offset:1024 sc0 sc1 lds\n\t"
      "ds_read_b32 v253, v203 offset:0\n\t"
      "s_add_u32 m0, s52, 98304\n\t"
      "s_add_u32 s80, s24, 0\n\t"
      "buffer_load_dwordx4 v200, s[44:47], s80 offen offset:0 sc0 sc1 lds\n\t"
      "buffer_load_dwordx4 v200, s[44:47], s80 offen offset:1024 sc0 sc1 lds\n\t"
      "s_waitcnt lgkmcnt(0)\n\t"
      "v_cmp_gt_u32 vcc, 16, v253\n\t"
      "s_add_u32 m0, s52, 40960\n\t"
      "s_add_u32 s80, s25, 0\n\t"
      "buffer_load_dwordx4 v200, s[44:47], s80 offen offset:0 sc0 sc1 lds\n\t"
      "buffer_load_dwordx4 v200, s[44:47], s80 offen offset:1024 sc0 sc1 lds\n\t"
      "s_cbranch_vccnz AQ227_%=\n\t"
      "AR228_%=:\n\t"
      "ds_read_b128 v[40:43], v242 offset:16384\n\t"
      "ds_read_b64_tr_b8 a[192:193], v246 offset:16384\n\t"
      "ds_read_b64_tr_b8 a[194:195], v246 offset:17408\n\t"
      "ds_read_b64_tr_b8 a[196:197], v246 offset:24576\n\t"
      "ds_read_b128 v[44:47], v243 offset:16384\n\t"
      "s_cmp_lg_u32 s65, 0\n\t"
      "s_cbranch_scc1 Lp229_%=\n\t"
      "Lr230_%=:\n\t"
      "ds_read_b64_tr_b8 a[198:199], v246 offset:25600\n\t"
      "ds_read_b128 v[48:51], v242 offset:18432\n\t"
      "ds_read_b64_tr_b8 a[200:201], v247 offset:16384\n\t"
      "ds_read_b64_tr_b8 a[202:203], v247 offset:17408\n\t"
      "ds_read_b64_tr_b8 a[204:205], v247 offset:24576\n\t"
      "ds_read_b64_tr_b8 a[206:207], v247 offset:25600\n\t"
      "ds_read_b64_tr_b8 a[208:209], v248 offset:0\n\t"
      "ds_read_b64_tr_b8 a[210:211], v248 offset:1024\n\t"
      "ds_read_b128 v[52:55], v243 offset:18432\n\t"
      "ds_read_b64_tr_b8 a[212:213], v248 offset:8192\n\t"
      "ds_read_b64_tr_b8 a[214:215], v248 offset:9216\n\t"
      "ds_read_b64_tr_b8 a[216:217], v249 offset:0\n\t"
      "ds_read_b64_tr_b8 a[218:219], v249 offset:1024\n\t"
      "ds_read_b64_tr_b8 a[220:221], v249 offset:8192\n\t"
      "ds_read_b64_tr_b8 a[222:223], v249 offset:9216\n\t"
      "s_waitcnt lgkmcnt(0)\n\t"
      "ds_add_u32 v203, v204 offset:256\n\t"
      "ds_read_b128 v[56:59], v242 offset:20480\n\t"
      "ds_read_b128 v[60:63], v243 offset:20480\n\t"
      "s_cmp_lg_u32 s65, 0\n\t"
      "s_cbranch_scc1 Lp231_%=\n\t"
      "Lr232_%=:\n\t"
      "v_mfma_f32_16x16x128_f8f6f4 v[72:75], a[32:39], v[40:47], v[72:75] cbsz:1 blgp:1\n\t"
      "ds_read_b128 v[64:67], v242 offset:22528\n\t"
      "s_add_u32 m0, s52, 32768\n\t"
      "s_add_u32 s80, s26, 0\n\t"
      "buffer_load_dwordx4 v200, s[44:47], s80 offen offset:0 sc0 sc1 lds\n\t"
      "v_mfma_f32_16x16x128_f8f6f4 v[76:79], a[40:47], v[40:47], v[76:79] cbsz:1 blgp:1\n\t"
      "ds_read_b128 v[68:71], v243 offset:22528\n\t"
      "s_cmp_lg_u32 s65, 0\n\t"
      "s_cbranch_scc1 Lp233_%=\n\t"
      "Lr234_%=:\n\t"
      "buffer_load_dwordx4 v200, s[44:47], s80 offen offset:1024 sc0 sc1 lds\n\t"
      "v_mfma_f32_16x16x128_f8f6f4 v[72:75], a[144:151], v[48:55], v[72:75] cbsz:1 blgp:1\n\t"
      "s_waitcnt vmcnt(6)\n\t"
      "ds_read2st64_b32 v[228:229], v251 offset0:192 offset1:196\n\t"
      "ds_read2st64_b32 v[230:231], v251 offset0:160 offset1:164\n\t"
      "ds_read_b128 v[32:35], v240 offset:57344\n\t"
      "v_mfma_f32_16x16x128_f8f6f4 v[76:79], a[152:159], v[48:55], v[76:79] cbsz:1 blgp:1\n\t"
      "ds_read_b128 v[36:39], v241 offset:57344\n\t"
      "s_waitcnt lgkmcnt(6)\n\t"
      "v_mfma_f32_16x16x128_f8f6f4 v[80:83], a[32:39], v[56:63], v[80:83] cbsz:1 blgp:1\n\t"
      "ds_read_b128 v[40:43], v240 offset:59392\n\t"
      "v_mfma_f32_16x16x128_f8f6f4 v[84:87], a[40:47], v[56:63], v[84:87] cbsz:1 blgp:1\n\t"
      "s_waitcnt lgkmcnt(3)\n\t"
      "v_min3_u32 v228, v228, v229, v230\n\t"
      "v_min_u32 v228, v228, v231\n\t"
      "v_cmp_eq_u32 vcc, 0, v228\n\t"
      "ds_read_b128 v[44:47], v241 offset:59392\n\t"
      "v_mfma_f32_16x16x128_f8f6f4 v[80:83], a[144:151], v[64:71], v[80:83] cbsz:1 blgp:1\n\t"
      "s_cbranch_vccnz XS235_%=\n\t"
      "XR236_%=:\n\t"
      "ds_add_u32 v203, v204 offset:0\n\t"
      "ds_read_b128 v[48:51], v240 offset:61440\n\t"
      "v_mfma_f32_16x16x128_f8f6f4 v[84:87], a[152:159], v[64:71], v[84:87] cbsz:1 blgp:1\n\t"
      "ds_read_b128 v[52:55], v241 offset:61440\n\t"
      "s_waitcnt lgkmcnt(5)\n\t"
      "v_mfma_f32_16x16x128_f8f6f4 v[72:75], a[48:55], v[32:39], v[72:75] cbsz:1 blgp:1\n\t"
      "ds_read_b32 v253, v203 offset:0\n\t"
      "ds_read_b128 v[56:59], v240 offset:63488\n\t"
      "s_add_u32 m0, s52, 73728\n\t"
      "s_add_u32 s80, s18, 8192\n\t"
      "buffer_load_dwordx4 v200, s[44:47], s80 offen offset:0 sc0 sc1 lds\n\t"
      "v_mfma_f32_16x16x128_f8f6f4 v[76:79], a[56:63], v[32:39], v[76:79] cbsz:1 blgp:1\n\t"
      "s_waitcnt lgkmcnt(1)\n\t"
      "v_cmp_gt_u32 vcc, 20, v253\n\t"
      "ds_read_b128 v[60:63], v241 offset:63488\n\t"
      "s_cmp_lg_u32 s65, 0\n\t"
      "s_cbranch_scc1 Lp241_%=\n\t"
      "Lr242_%=:\n\t"
      "v_mfma_f32_16x16x128_f8f6f4 v[72:75], a[160:167], v[40:47], v[72:75] cbsz:1 blgp:1\n\t"
      "s_cbranch_vccnz AQ243_%=\n\t"
      "AR244_%=:\n\t"
      "s_waitcnt vmcnt(5)\n\t"
      "ds_read2st64_b32 v[228:229], v251 offset0:128 offset1:132\n\t"
      "ds_read_b128 v[64:67], v240 offset:49152\n\t"
      "buffer_load_dwordx4 v200, s[44:47], s80 offen offset:1024 sc0 sc1 lds\n\t"
      "ds_read_b64_tr_b8 a[224:225], v244 offset:0\n\t"
      "v_mfma_f32_16x16x128_f8f6f4 v[76:79], a[168:175], v[40:47], v[76:79] cbsz:1 blgp:1\n\t"
      "ds_read_b128 v[68:71], v241 offset:49152\n\t"
      "ds_read_b64_tr_b8 a[226:227], v244 offset:1024\n\t"
      "ds_read_b64_tr_b8 a[228:229], v244 offset:8192\n\t"
      "ds_read_b64_tr_b8 a[230:231], v244 offset:9216\n\t"
      "v_mfma_f32_16x16x128_f8f6f4 v[80:83], a[48:55], v[48:55], v[80:83] cbsz:1 blgp:1\n\t"
      "ds_read_b128 v[32:35], v240 offset:51200\n\t"
      "ds_read_b64_tr_b8 a[232:233], v245 offset:0\n\t"
      "ds_read_b64_tr_b8 a[234:235], v245 offset:1024\n\t"
      "ds_read_b64_tr_b8 a[236:237], v245 offset:8192\n\t"
      "ds_read_b64_tr_b8 a[238:239], v245 offset:9216\n\t"
      "v_mfma_f32_16x16x128_f8f6f4 v[84:87], a[56:63], v[48:55], v[84:87] cbsz:1 blgp:1\n\t"
      "s_waitcnt lgkmcnt(11)\n\t"
      "v_min_u32 v228, v228, v229\n\t"
      "v_cmp_eq_u32 vcc, 0, v228\n\t"
      "ds_read_b128 v[36:39], v241 offset:51200\n\t"
      "ds_read_b64_tr_b8 a[240:241], v244 offset:16384\n\t"
      "ds_read_b64_tr_b8 a[242:243], v244 offset:17408\n\t"
      "v_mfma_f32_16x16x128_f8f6f4 v[80:83], a[160:167], v[56:63], v[80:83] cbsz:1 blgp:1\n\t"
      "s_cbranch_vccnz XS245_%=\n\t"
      "XR246_%=:\n\t"
      "ds_add_u32 v203, v204 offset:256\n\t"
      "ds_read_b128 v[40:43], v240 offset:53248\n\t"
      "ds_read_b64_tr_b8 a[244:245], v244 offset:24576\n\t"
      "ds_read_b64_tr_b8 a[246:247], v244 offset:25600\n\t"
      "ds_read_b64_tr_b8 a[248:249], v245 offset:16384\n\t"
      "v_mfma_f32_16x16x128_f8f6f4 v[84:87], a[168:175], v[56:63], v[84:87] cbsz:1 blgp:1\n\t"
      "ds_read_b128 v[44:47], v241 offset:53248\n\t"
      "ds_read_b64_tr_b8 a[250:251], v245 offset:17408\n\t"
      "ds_read_b64_tr_b8 a[252:253], v245 offset:24576\n\t"
      "ds_read_b64_tr_b8 a[254:255], v245 offset:25600\n\t"
      "s_waitcnt lgkmcnt(15)\n\t"
      "v_mfma_f32_16x16x128_f8f6f4 v[72:75], a[64:71], v[64:71], v[72:75] cbsz:1 blgp:1\n\t"
      "ds_read_b128 v[48:51], v240 offset:55296\n\t"
      "s_add_u32 m0, s52, 147456\n\t"
      "s_add_u32 s80, s19, 8192\n\t"
      "buffer_load_dwordx4 v200, s[44:47], s80 offen offset:0 sc0 sc1 lds\n\t"
      "v_mfma_f32_16x16x128_f8f6f4 v[76:79], a[72:79], v[64:71], v[76:79] cbsz:1 blgp:1\n\t"
      "ds_read_b128 v[52:55], v241 offset:55296\n\t"
      "s_cmp_lg_u32 s65, 0\n\t"
      "s_cbranch_scc1 Lp251_%=\n\t"
      "Lr252_%=:\n\t"
      "buffer_load_dwordx4 v200, s[44:47], s80 offen offset:1024 sc0 sc1 lds\n\t"
      "s_waitcnt lgkmcnt(13)\n\t"
      "v_mfma_f32_16x16x128_f8f6f4 v[72:75], a[176:183], v[32:39], v[72:75] cbsz:1 blgp:1\n\t"
      "s_waitcnt vmcnt(6)\n\t"
      "ds_read2st64_b32 v[228:229], v250 offset0:160 offset1:164\n\t"
      "ds_read_b128 v[56:59], v240 offset:40960\n\t"
      "v_mfma_f32_16x16x128_f8f6f4 v[76:79], a[184:191], v[32:39], v[76:79] cbsz:1 blgp:1\n\t"
      "ds_read_b128 v[60:63], v241 offset:40960\n\t"
      "s_waitcnt lgkmcnt(8)\n\t"
      "v_mfma_f32_16x16x128_f8f6f4 v[80:83], a[64:71], v[40:47], v[80:83] cbsz:1 blgp:1\n\t"
      "ds_read_b128 v[64:67], v240 offset:43008\n\t"
      "v_mfma_f32_16x16x128_f8f6f4 v[84:87], a[72:79], v[40:47], v[84:87] cbsz:1 blgp:1\n\t"
      "s_waitcnt lgkmcnt(3)\n\t"
      "v_min_u32 v228, v228, v229\n\t"
      "v_cmp_eq_u32 vcc, 0, v228\n\t"
      "ds_read_b128 v[68:71], v241 offset:43008\n\t"
      "v_mfma_f32_16x16x128_f8f6f4 v[80:83], a[176:183], v[48:55], v[80:83] cbsz:1 blgp:1\n\t"
      "s_cbranch_vccnz XS253_%=\n\t"
      "XR254_%=:\n\t"
      "ds_add_u32 v203, v204 offset:0\n\t"
      "ds_read_b32 v253, v203 offset:256\n\t"
      "ds_read_b128 v[32:35], v240 offset:45056\n\t"
      "v_mfma_f32_16x16x128_f8f6f4 v[84:87], a[184:191], v[48:55], v[84:87] cbsz:1 blgp:1\n\t"
      "ds_read_b128 v[36:39], v241 offset:45056\n\t"
      "s_waitcnt lgkmcnt(6)\n\t"
      "v_mfma_f32_16x16x128_f8f6f4 v[72:75], a[80:87], v[56:63], v[72:75] cbsz:1 blgp:1\n\t"
      "ds_read_b128 v[40:43], v240 offset:47104\n\t"
      "s_add_u32 m0, s52, 65536\n\t"
      "s_add_u32 s80, s20, 8192\n\t"
      "buffer_load_dwordx4 v200, s[44:47], s80 offen offset:0 sc0 sc1 lds\n\t"
      "v_mfma_f32_16x16x128_f8f6f4 v[76:79], a[88:95], v[56:63], v[76:79] cbsz:1 blgp:1\n\t"
      "s_waitcnt lgkmcnt(3)\n\t"
      "v_cmp_gt_u32 vcc, 24, v253\n\t"
      "ds_read_b128 v[44:47], v241 offset:47104\n\t"
      "s_cmp_lg_u32 s65, 0\n\t"
      "s_cbranch_scc1 Lp259_%=\n\t"
      "Lr260_%=:\n\t"
      "v_mfma_f32_16x16x128_f8f6f4 v[72:75], a[192:199], v[64:71], v[72:75] cbsz:1 blgp:1\n\t"
      "s_cbranch_vccnz AQ261_%=\n\t"
      "AR262_%=:\n\t"
      "s_waitcnt vmcnt(5)\n\t"
      "ds_read2st64_b32 v[228:229], v250 offset0:128 offset1:132\n\t"
      "ds_read_b128 v[48:51], v240 offset:32768\n\t"
      "buffer_load_dwordx4 v200, s[44:47], s80 offen offset:1024 sc0 sc1 lds\n\t"
      "v_mfma_f32_16x16x128_f8f6f4 v[76:79], a[200:207], v[64:71], v[76:79] cbsz:1 blgp:1\n\t"
      "ds_read_b128 v[52:55], v241 offset:32768\n\t"
      "s_waitcnt lgkmcnt(5)\n\t"
      "v_mfma_f32_16x16x128_f8f6f4 v[80:83], a[80:87], v[32:39], v[80:83] cbsz:1 blgp:1\n\t"
      "ds_read_b128 v[56:59], v240 offset:34816\n\t"
      "v_mfma_f32_16x16x128_f8f6f4 v[84:87], a[88:95], v[32:39], v[84:87] cbsz:1 blgp:1\n\t"
      "s_waitcnt lgkmcnt(3)\n\t"
      "v_min_u32 v228, v228, v229\n\t"
      "v_cmp_eq_u32 vcc, 0, v228\n\t"
      "ds_read_b128 v[60:63], v241 offset:34816\n\t"
      "v_mfma_f32_16x16x128_f8f6f4 v[80:83], a[192:199], v[40:47], v[80:83] cbsz:1 blgp:1\n\t"
      "s_cbranch_vccnz XS263_%=\n\t"
      "XR264_%=:\n\t"
      "ds_add_u32 v203, v204 offset:256\n\t"
      "ds_read_b32 v253, v203 offset:0\n\t"
      "ds_read_b128 v[64:67], v240 offset:36864\n\t"
      "v_mfma_f32_16x16x128_f8f6f4 v[84:87], a[200:207], v[40:47], v[84:87] cbsz:1 blgp:1\n\t"
      "ds_read_b128 v[68:71], v241 offset:36864\n\t"
      "s_waitcnt lgkmcnt(6)\n\t"
      "v_mfma_f32_16x16x128_f8f6f4 v[72:75], a[96:103], v[48:55], v[72:75] cbsz:1 blgp:1\n\t"
      "ds_read_b128 v[32:35], v240 offset:38912\n\t"
      "s_add_u32 m0, s52, 122880\n\t"
      "s_add_u32 s80, s21, 8192\n\t"
      "buffer_load_dwordx4 v200, s[44:47], s80 offen offset:0 sc0 sc1 lds\n\t"
      "v_mfma_f32_16x16x128_f8f6f4 v[76:79], a[104:111], v[48:55], v[76:79] cbsz:1 blgp:1\n\t"
      "s_waitcnt lgkmcnt(3)\n\t"
      "v_cmp_gt_u32 vcc, 24, v253\n\t"
      "ds_read_b128 v[36:39], v241 offset:38912\n\t"
      "s_cmp_lg_u32 s65, 0\n\t"
      "s_cbranch_scc1 Lp269_%=\n\t"
      "Lr270_%=:\n\t"
      "v_mfma_f32_16x16x128_f8f6f4 v[72:75], a[208:215], v[56:63], v[72:75] cbsz:1 blgp:1\n\t"
      "s_cbranch_vccnz AQ271_%=\n\t"
      "AR272_%=:\n\t"
      "s_waitcnt vmcnt(5)\n\t"
      "ds_read2st64_b32 v[228:229], v251 offset0:32 offset1:36\n\t"
      "ds_read_b128 v[40:43], v238 offset:40960\n\t"
      "buffer_load_dwordx4 v200, s[44:47], s80 offen offset:1024 sc0 sc1 lds\n\t"
      "v_mfma_f32_16x16x128_f8f6f4 v[76:79], a[216:223], v[56:63], v[76:79] cbsz:1 blgp:1\n\t"
      "ds_read_b128 v[44:47], v239 offset:40960\n\t"
      "s_waitcnt lgkmcnt(5)\n\t"
      "v_mfma_f32_16x16x128_f8f6f4 v[80:83], a[96:103], v[64:71], v[80:83] cbsz:1 blgp:1\n\t"
      "ds_read_b128 v[48:51], v238 offset:43008\n\t"
      "v_mfma_f32_16x16x128_f8f6f4 v[84:87], a[104:111], v[64:71], v[84:87] cbsz:1 blgp:1\n\t"
      "s_waitcnt lgkmcnt(3)\n\t"
      "v_min_u32 v228, v228, v229\n\t"
      "v_cmp_eq_u32 vcc, 0, v228\n\t"
      "ds_read_b128 v[52:55], v239 offset:43008\n\t"
      "v_mfma_f32_16x16x128_f8f6f4 v[80:83], a[208:215], v[32:39], v[80:83] cbsz:1 blgp:1\n\t"
      "s_cbranch_vccnz XS273_%=\n\t"
      "XR274_%=:\n\t"
      "ds_add_u32 v203, v204 offset:0\n\t"
      "ds_read_b32 v253, v203 offset:256\n\t"
      "ds_read_b128 v[56:59], v238 offset:45056\n\t"
      "v_mfma_f32_16x16x128_f8f6f4 v[84:87], a[216:223], v[32:39], v[84:87] cbsz:1 blgp:1\n\t"
      "ds_read_b128 v[60:63], v239 offset:45056\n\t"
      "s_waitcnt lgkmcnt(6)\n\t"
      "v_mfma_f32_16x16x128_f8f6f4 v[72:75], a[112:119], v[40:47], v[72:75] cbsz:1 blgp:1\n\t"
      "ds_read_b128 v[64:67], v238 offset:47104\n\t"
      "s_add_u32 m0, s52, 114688\n\t"
      "s_add_u32 s80, s24, 8192\n\t"
      "buffer_load_dwordx4 v200, s[44:47], s80 offen offset:0 sc0 sc1 lds\n\t"
      "v_mfma_f32_16x16x128_f8f6f4 v[76:79], a[120:127], v[40:47], v[76:79] cbsz:1 blgp:1\n\t"
      "s_waitcnt lgkmcnt(3)\n\t"
      "v_cmp_gt_u32 vcc, 28, v253\n\t"
      "ds_read_b128 v[68:71], v239 offset:47104\n\t"
      "s_cmp_lg_u32 s65, 0\n\t"
      "s_cbranch_scc1 Lp279_%=\n\t"
      "Lr280_%=:\n\t"
      "v_mfma_f32_16x16x128_f8f6f4 v[72:75], a[224:231], v[48:55], v[72:75] cbsz:1 blgp:1\n\t"
      "s_cbranch_vccnz AQ281_%=\n\t"
      "AR282_%=:\n\t"
      "s_waitcnt vmcnt(5)\n\t"
      "ds_read2st64_b32 v[228:229], v252 offset0:64 offset1:68\n\t"
      "ds_read_b128 v[32:35], v238 offset:32768\n\t"
      "buffer_load_dwordx4 v200, s[44:47], s80 offen offset:1024 sc0 sc1 lds\n\t"
      "v_mfma_f32_16x16x128_f8f6f4 v[76:79], a[232:239], v[48:55], v[76:79] cbsz:1 blgp:1\n\t"
      "ds_read_b128 v[36:39], v239 offset:32768\n\t"
      "s_waitcnt lgkmcnt(5)\n\t"
      "v_mfma_f32_16x16x128_f8f6f4 v[80:83], a[112:119], v[56:63], v[80:83] cbsz:1 blgp:1\n\t"
      "ds_read_b128 v[40:43], v238 offset:34816\n\t"
      "v_mfma_f32_16x16x128_f8f6f4 v[84:87], a[120:127], v[56:63], v[84:87] cbsz:1 blgp:1\n\t"
      "s_waitcnt lgkmcnt(3)\n\t"
      "v_min_u32 v228, v228, v229\n\t"
      "v_cmp_eq_u32 vcc, 0, v228\n\t"
      "ds_read_b128 v[44:47], v239 offset:34816\n\t"
      "v_mfma_f32_16x16x128_f8f6f4 v[80:83], a[224:231], v[64:71], v[80:83] cbsz:1 blgp:1\n\t"
      "s_cbranch_vccnz XS283_%=\n\t"
      "XR284_%=:\n\t"
      "ds_add_u32 v203, v204 offset:256\n\t"
      "ds_read_b32 v253, v203 offset:0\n\t"
      "ds_read_b128 v[48:51], v238 offset:36864\n\t"
      "v_mfma_f32_16x16x128_f8f6f4 v[84:87], a[232:239], v[64:71], v[84:87] cbsz:1 blgp:1\n\t"
      "ds_read_b128 v[52:55], v239 offset:36864\n\t"
      "s_waitcnt lgkmcnt(6)\n\t"
      "v_mfma_f32_16x16x128_f8f6f4 v[72:75], a[128:135], v[32:39], v[72:75] cbsz:1 blgp:1\n\t"
      "ds_read_b128 v[56:59], v238 offset:38912\n\t"
      "s_add_u32 m0, s52, 106496\n\t"
      "s_add_u32 s80, s25, 8192\n\t"
      "buffer_load_dwordx4 v200, s[44:47], s80 offen offset:0 sc0 sc1 lds\n\t"
      "v_mfma_f32_16x16x128_f8f6f4 v[76:79], a[136:143], v[32:39], v[76:79] cbsz:1 blgp:1\n\t"
      "s_waitcnt lgkmcnt(3)\n\t"
      "v_cmp_gt_u32 vcc, 28, v253\n\t"
      "ds_read_b128 v[60:63], v239 offset:38912\n\t"
      "s_cmp_lg_u32 s65, 0\n\t"
      "s_cbranch_scc1 Lp289_%=\n\t"
      "Lr290_%=:\n\t"
      "v_mfma_f32_16x16x128_f8f6f4 v[72:75], a[240:247], v[40:47], v[72:75] cbsz:1 blgp:1\n\t"
      "s_cbranch_vccnz AQ291_%=\n\t"
      "AR292_%=:\n\t"
      "s_waitcnt vmcnt(5)\n\t"
      "ds_read2st64_b32 v[228:229], v251 offset0:0 offset1:4\n\t"
      "ds_read_b128 v[64:67], v240 offset:8192\n\t"
      "buffer_load_dwordx4 v200, s[44:47], s80 offen offset:1024 sc0 sc1 lds\n\t"
      "v_mfma_f32_16x16x128_f8f6f4 v[76:79], a[248:255], v[40:47], v[76:79] cbsz:1 blgp:1\n\t"
      "ds_read_b128 v[68:71], v241 offset:8192\n\t"
      "s_waitcnt lgkmcnt(5)\n\t"
      "v_mfma_f32_16x16x128_f8f6f4 v[80:83], a[128:135], v[48:55], v[80:83] cbsz:1 blgp:1\n\t"
      "ds_read_b128 v[32:35], v240 offset:10240\n\t"
      "v_mfma_f32_16x16x128_f8f6f4 v[84:87], a[136:143], v[48:55], v[84:87] cbsz:1 blgp:1\n\t"
      "s_waitcnt lgkmcnt(3)\n\t"
      "v_min_u32 v228, v228, v229\n\t"
      "v_cmp_eq_u32 vcc, 0, v228\n\t"
      "ds_read_b128 v[36:39], v241 offset:10240\n\t"
      "v_mfma_f32_16x16x128_f8f6f4 v[80:83], a[240:247], v[56:63], v[80:83] cbsz:1 blgp:1\n\t"
      "s_cbranch_vccnz XS293_%=\n\t"
      "XR294_%=:\n\t"
      "ds_add_u32 v203, v204 offset:0\n\t"
      "ds_read_b32 v253, v203 offset:256\n\t"
      "ds_read_b128 v[40:43], v240 offset:12288\n\t"
      "v_mfma_f32_16x16x128_f8f6f4 v[84:87], a[248:255], v[56:63], v[84:87] cbsz:1 blgp:1\n\t"
      "ds_read_b128 v[44:47], v241 offset:12288\n\t"
      "s_waitcnt lgkmcnt(6)\n\t"
      "v_mfma_f32_16x16x128_f8f6f4 v[88:91], a[32:39], v[64:71], v[88:91] cbsz:1 blgp:1\n\t"
      "ds_read_b128 v[48:51], v240 offset:14336\n\t"
      "s_add_u32 m0, s52, 98304\n\t"
      "s_add_u32 s80, s26, 8192\n\t"
      "buffer_load_dwordx4 v200, s[44:47], s80 offen offset:0 sc0 sc1 lds\n\t"
      "v_mfma_f32_16x16x128_f8f6f4 v[92:95], a[40:47], v[64:71], v[92:95] cbsz:1 blgp:1\n\t"
      "s_waitcnt lgkmcnt(3)\n\t"
      "v_cmp_gt_u32 vcc, 32, v253\n\t"
      "ds_read_b128 v[52:55], v241 offset:14336\n\t"
      "s_cmp_lg_u32 s65, 0\n\t"
      "s_cbranch_scc1 Lp299_%=\n\t"
      "Lr300_%=:\n\t"
      "v_mfma_f32_16x16x128_f8f6f4 v[88:91], a[144:151], v[32:39], v[88:91] cbsz:1 blgp:1\n\t"
      "s_cbranch_vccnz AQ301_%=\n\t"
      "AR302_%=:\n\t"
      "s_waitcnt vmcnt(5)\n\t"
      "ds_read2st64_b32 v[228:229], v251 offset0:224 offset1:228\n\t"
      "ds_read_b128 v[56:59], v242 offset:16384\n\t"
      "buffer_load_dwordx4 v200, s[44:47], s80 offen offset:1024 sc0 sc1 lds\n\t"
      "v_mfma_f32_16x16x128_f8f6f4 v[92:95], a[152:159], v[32:39], v[92:95] cbsz:1 blgp:1\n\t"
      "ds_read_b128 v[60:63], v243 offset:16384\n\t"
      "global_store_dwordx4 v210, v[72:75], s[30:31] sc0 sc1\n\t"
      "s_waitcnt lgkmcnt(5)\n\t"
      "v_mfma_f32_16x16x128_f8f6f4 v[96:99], a[32:39], v[40:47], v[96:99] cbsz:1 blgp:1\n\t"
      "ds_read_b128 v[64:67], v242 offset:18432\n\t"
      "v_mfma_f32_16x16x128_f8f6f4 v[100:103], a[40:47], v[40:47], v[100:103] cbsz:1 blgp:1\n\t"
      "s_waitcnt lgkmcnt(3)\n\t"
      "v_min_u32 v228, v228, v229\n\t"
      "v_cmp_eq_u32 vcc, 0, v228\n\t"
      "ds_read_b128 v[68:71], v243 offset:18432\n\t"
      "v_mfma_f32_16x16x128_f8f6f4 v[96:99], a[144:151], v[48:55], v[96:99] cbsz:1 blgp:1\n\t"
      "s_cbranch_vccnz XS303_%=\n\t"
      "XR304_%=:\n\t"
      "ds_add_u32 v203, v204 offset:256\n\t"
      "ds_read_b32 v253, v203 offset:0\n\t"
      "ds_read_b128 v[32:35], v242 offset:20480\n\t"
      "v_mfma_f32_16x16x128_f8f6f4 v[100:103], a[152:159], v[48:55], v[100:103] cbsz:1 blgp:1\n\t"
      "ds_read_b128 v[36:39], v243 offset:20480\n\t"
      "s_waitcnt lgkmcnt(6)\n\t"
      "v_mfma_f32_16x16x128_f8f6f4 v[88:91], a[48:55], v[56:63], v[88:91] cbsz:1 blgp:1\n\t"
      "ds_read_b128 v[40:43], v242 offset:22528\n\t"
      "s_add_u32 m0, s52, 40960\n\t"
      "s_add_u32 s80, s18, 16384\n\t"
      "buffer_load_dwordx4 v200, s[44:47], s80 offen offset:0 sc0 sc1 lds\n\t"
      "v_mfma_f32_16x16x128_f8f6f4 v[92:95], a[56:63], v[56:63], v[92:95] cbsz:1 blgp:1\n\t"
      "s_waitcnt lgkmcnt(3)\n\t"
      "v_cmp_gt_u32 vcc, 32, v253\n\t"
      "ds_read_b128 v[44:47], v243 offset:22528\n\t"
      "s_cmp_lg_u32 s65, 0\n\t"
      "s_cbranch_scc1 Lp309_%=\n\t"
      "Lr310_%=:\n\t"
      "v_mfma_f32_16x16x128_f8f6f4 v[88:91], a[160:167], v[64:71], v[88:91] cbsz:1 blgp:1\n\t"
      "s_cbranch_vccnz AQ311_%=\n\t"
      "AR312_%=:\n\t"
      "s_waitcnt vmcnt(6)\n\t"
      "ds_read2st64_b32 v[228:229], v251 offset0:192 offset1:196\n\t"
      "ds_read_b128 v[48:51], v240 offset:0\n\t"
      "buffer_load_dwordx4 v200, s[44:47], s80 offen offset:1024 sc0 sc1 lds\n\t"
      "v_mfma_f32_16x16x128_f8f6f4 v[92:95], a[168:175], v[64:71], v[92:95] cbsz:1 blgp:1\n\t"
      "ds_read_b128 v[52:55], v241 offset:0\n\t"
      "global_store_dwordx4 v211, v[76:79], s[30:31] sc0 sc1\n\t"
      "s_add_u32 s30, s30, 0x40000\n\t"
      "s_addc_u32 s31, s31, 0\n\t"
      "s_waitcnt lgkmcnt(5)\n\t"
      "v_mfma_f32_16x16x128_f8f6f4 v[96:99], a[48:55], v[32:39], v[96:99] cbsz:1 blgp:1\n\t"
      "ds_read_b128 v[56:59], v240 offset:2048\n\t"
      "v_mfma_f32_16x16x128_f8f6f4 v[100:103], a[56:63], v[32:39], v[100:103] cbsz:1 blgp:1\n\t"
      "s_waitcnt lgkmcnt(3)\n\t"
      "v_min_u32 v228, v228, v229\n\t"
      "v_cmp_eq_u32 vcc, 0, v228\n\t"
      "ds_read_b128 v[60:63], v241 offset:2048\n\t"
      "v_mfma_f32_16x16x128_f8f6f4 v[96:99], a[160:167], v[40:47], v[96:99] cbsz:1 blgp:1\n\t"
      "s_cbranch_vccnz XS313_%=\n\t"
      "XR314_%=:\n\t"
      "ds_add_u32 v203, v204 offset:0\n\t"
      "ds_read_b32 v253, v203 offset:256\n\t"
      "ds_read_b128 v[64:67], v240 offset:4096\n\t"
      "v_mfma_f32_16x16x128_f8f6f4 v[100:103], a[168:175], v[40:47], v[100:103] cbsz:1 blgp:1\n\t"
      "ds_read_b128 v[68:71], v241 offset:4096\n\t"
      "s_waitcnt lgkmcnt(6)\n\t"
      "v_mfma_f32_16x16x128_f8f6f4 v[88:91], a[64:71], v[48:55], v[88:91] cbsz:1 blgp:1\n\t"
      "ds_read_b128 v[32:35], v240 offset:6144\n\t"
      "s_add_u32 m0, s52, 32768\n\t"
      "s_add_u32 s80, s19, 16384\n\t"
      "buffer_load_dwordx4 v200, s[44:47], s80 offen offset:0 sc0 sc1 lds\n\t"
      "v_mfma_f32_16x16x128_f8f6f4 v[92:95], a[72:79], v[48:55], v[92:95] cbsz:1 blgp:1\n\t"
      "s_waitcnt lgkmcnt(3)\n\t"
      "v_cmp_gt_u32 vcc, 36, v253\n\t"
      "ds_read_b128 v[36:39], v241 offset:6144\n\t"
      "s_cmp_lg_u32 s65, 0\n\t"
      "s_cbranch_scc1 Lp319_%=\n\t"
      "Lr320_%=:\n\t"
      "v_mfma_f32_16x16x128_f8f6f4 v[88:91], a[176:183], v[56:63], v[88:91] cbsz:1 blgp:1\n\t"
      "s_cbranch_vccnz AQ321_%=\n\t"
      "AR322_%=:\n\t"
      "s_waitcnt vmcnt(7)\n\t"
      "ds_read2st64_b32 v[228:229], v251 offset0:160 offset1:164\n\t"
      "ds_read_b128 v[40:43], v240 offset:57344\n\t"
      "buffer_load_dwordx4 v200, s[44:47], s80 offen offset:1024 sc0 sc1 lds\n\t"
      "v_mfma_f32_16x16x128_f8f6f4 v[92:95], a[184:191], v[56:63], v[92:95] cbsz:1 blgp:1\n\t"
      "ds_read_b128 v[44:47], v241 offset:57344\n\t"
      "global_store_dwordx4 v210, v[80:83], s[30:31] sc0 sc1\n\t"
      "s_waitcnt lgkmcnt(5)\n\t"
      "v_mfma_f32_16x16x128_f8f6f4 v[96:99], a[64:71], v[64:71], v[96:99] cbsz:1 blgp:1\n\t"
      "ds_read_b128 v[48:51], v240 offset:59392\n\t"
      "v_mfma_f32_16x16x128_f8f6f4 v[100:103], a[72:79], v[64:71], v[100:103] cbsz:1 blgp:1\n\t"
      "s_waitcnt lgkmcnt(3)\n\t"
      "v_min_u32 v228, v228, v229\n\t"
      "v_cmp_eq_u32 vcc, 0, v228\n\t"
      "ds_read_b128 v[52:55], v241 offset:59392\n\t"
      "v_mfma_f32_16x16x128_f8f6f4 v[96:99], a[176:183], v[32:39], v[96:99] cbsz:1 blgp:1\n\t"
      "s_cbranch_vccnz XS323_%=\n\t"
      "XR324_%=:\n\t"
      "ds_add_u32 v203, v204 offset:256\n\t"
      "ds_read_b32 v253, v203 offset:0\n\t"
      "ds_read_b128 v[56:59], v240 offset:61440\n\t"
      "v_mfma_f32_16x16x128_f8f6f4 v[100:103], a[184:191], v[32:39], v[100:103] cbsz:1 blgp:1\n\t"
      "ds_read_b128 v[60:63], v241 offset:61440\n\t"
      "s_waitcnt lgkmcnt(6)\n\t"
      "v_mfma_f32_16x16x128_f8f6f4 v[88:91], a[80:87], v[40:47], v[88:91] cbsz:1 blgp:1\n\t"
      "ds_read_b128 v[64:67], v240 offset:63488\n\t"
      "s_add_u32 m0, s52, 73728\n\t"
      "s_add_u32 s80, s20, 16384\n\t"
      "buffer_load_dwordx4 v200, s[44:47], s80 offen offset:0 sc0 sc1 lds\n\t"
      "v_mfma_f32_16x16x128_f8f6f4 v[92:95], a[88:95], v[40:47], v[92:95] cbsz:1 blgp:1\n\t"
      "s_waitcnt lgkmcnt(3)\n\t"
      "v_cmp_gt_u32 vcc, 36, v253\n\t"
      "ds_read_b128 v[68:71], v241 offset:63488\n\t"
      "s_cmp_lg_u32 s65, 0\n\t"
      "s_cbranch_scc1 Lp329_%=\n\t"
      "Lr330_%=:\n\t"
      "v_mfma_f32_16x16x128_f8f6f4 v[88:91], a[192:199], v[48:55], v[88:91] cbsz:1 blgp:1\n\t"
      "s_cbranch_vccnz AQ331_%=\n\t"
      "AR332_%=:\n\t"
      "s_waitcnt vmcnt(8)\n\t"
      "ds_read2st64_b32 v[228:229], v251 offset0:128 offset1:132\n\t"
      "ds_read_b128 v[32:35], v240 offset:49152\n\t"
      "buffer_load_dwordx4 v200, s[44:47], s80 offen offset:1024 sc0 sc1 lds\n\t"
      "v_mfma_f32_16x16x128_f8f6f4 v[92:95], a[200:207], v[48:55], v[92:95] cbsz:1 blgp:1\n\t"
      "ds_read_b128 v[36:39], v241 offset:49152\n\t"
      "global_store_dwordx4 v211, v[84:87], s[30:31] sc0 sc1\n\t"
      "s_add_u32 s30, s30, 0x40000\n\t"
      "s_addc_u32 s31, s31, 0\n\t"
      "s_waitcnt lgkmcnt(5)\n\t"
      "v_mfma_f32_16x16x128_f8f6f4 v[96:99], a[80:87], v[56:63], v[96:99] cbsz:1 blgp:1\n\t"
      "ds_read_b128 v[40:43], v240 offset:51200\n\t"
      "v_mfma_f32_16x16x128_f8f6f4 v[100:103], a[88:95], v[56:63], v[100:103] cbsz:1 blgp:1\n\t"
      "s_waitcnt lgkmcnt(3)\n\t"
      "v_min_u32 v228, v228, v229\n\t"
      "v_cmp_eq_u32 vcc, 0, v228\n\t"
      "ds_read_b128 v[44:47], v241 offset:51200\n\t"
      "v_mfma_f32_16x16x128_f8f6f4 v[96:99], a[192:199], v[64:71], v[96:99] cbsz:1 blgp:1\n\t"
      "s_cbranch_vccnz XS333_%=\n\t"
      "XR334_%=:\n\t"
      "ds_add_u32 v203, v204 offset:0\n\t"
      "ds_read_b32 v253, v203 offset:256\n\t"
      "ds_read_b128 v[48:51], v240 offset:53248\n\t"
      "v_mfma_f32_16x16x128_f8f6f4 v[100:103], a[200:207], v[64:71], v[100:103] cbsz:1 blgp:1\n\t"
      "ds_read_b128 v[52:55], v241 offset:53248\n\t"
      "s_waitcnt lgkmcnt(6)\n\t"
      "v_mfma_f32_16x16x128_f8f6f4 v[88:91], a[96:103], v[32:39], v[88:91] cbsz:1 blgp:1\n\t"
      "ds_read_b128 v[56:59], v240 offset:55296\n\t"
      "s_add_u32 m0, s52, 147456\n\t"
      "s_add_u32 s80, s21, 16384\n\t"
      "buffer_load_dwordx4 v200, s[44:47], s80 offen offset:0 sc0 sc1 lds\n\t"
      "v_mfma_f32_16x16x128_f8f6f4 v[92:95], a[104:111], v[32:39], v[92:95] cbsz:1 blgp:1\n\t"
      "s_waitcnt lgkmcnt(3)\n\t"
      "v_cmp_gt_u32 vcc, 40, v253\n\t"
      "ds_read_b128 v[60:63], v241 offset:55296\n\t"
      "s_cmp_lg_u32 s65, 0\n\t"
      "s_cbranch_scc1 Lp339_%=\n\t"
      "Lr340_%=:\n\t"
      "v_mfma_f32_16x16x128_f8f6f4 v[88:91], a[208:215], v[40:47], v[88:91] cbsz:1 blgp:1\n\t"
      "s_cbranch_vccnz AQ341_%=\n\t"
      "AR342_%=:\n\t"
      "s_waitcnt vmcnt(8)\n\t"
      "ds_read2st64_b32 v[228:229], v250 offset0:160 offset1:164\n\t"
      "ds_read_b128 v[64:67], v240 offset:40960\n\t"
      "buffer_load_dwordx4 v200, s[44:47], s80 offen offset:1024 sc0 sc1 lds\n\t"
      "v_mfma_f32_16x16x128_f8f6f4 v[92:95], a[216:223], v[40:47], v[92:95] cbsz:1 blgp:1\n\t"
      "ds_read_b128 v[68:71], v241 offset:40960\n\t"
      "s_waitcnt lgkmcnt(5)\n\t"
      "v_mfma_f32_16x16x128_f8f6f4 v[96:99], a[96:103], v[48:55], v[96:99] cbsz:1 blgp:1\n\t"
      "ds_read_b128 v[32:35], v240 offset:43008\n\t"
      "v_mfma_f32_16x16x128_f8f6f4 v[100:103], a[104:111], v[48:55], v[100:103] cbsz:1 blgp:1\n\t"
      "s_waitcnt lgkmcnt(3)\n\t"
      "v_min_u32 v228, v228, v229\n\t"
      "v_cmp_eq_u32 vcc, 0, v228\n\t"
      "ds_read_b128 v[36:39], v241 offset:43008\n\t"
      "v_mfma_f32_16x16x128_f8f6f4 v[96:99], a[208:215], v[56:63], v[96:99] cbsz:1 blgp:1\n\t"
      "s_cbranch_vccnz XS343_%=\n\t"
      "XR344_%=:\n\t"
      "ds_add_u32 v203, v204 offset:256\n\t"
      "ds_read_b32 v253, v203 offset:0\n\t"
      "ds_read_b128 v[40:43], v240 offset:45056\n\t"
      "v_mfma_f32_16x16x128_f8f6f4 v[100:103], a[216:223], v[56:63], v[100:103] cbsz:1 blgp:1\n\t"
      "ds_read_b128 v[44:47], v241 offset:45056\n\t"
      "s_waitcnt lgkmcnt(6)\n\t"
      "v_mfma_f32_16x16x128_f8f6f4 v[88:91], a[112:119], v[64:71], v[88:91] cbsz:1 blgp:1\n\t"
      "ds_read_b128 v[48:51], v240 offset:47104\n\t"
      "s_add_u32 m0, s52, 65536\n\t"
      "s_add_u32 s80, s24, 16384\n\t"
      "buffer_load_dwordx4 v200, s[44:47], s80 offen offset:0 sc0 sc1 lds\n\t"
      "v_mfma_f32_16x16x128_f8f6f4 v[92:95], a[120:127], v[64:71], v[92:95] cbsz:1 blgp:1\n\t"
      "s_waitcnt lgkmcnt(3)\n\t"
      "v_cmp_gt_u32 vcc, 40, v253\n\t"
      "ds_read_b128 v[52:55], v241 offset:47104\n\t"
      "s_cmp_lg_u32 s65, 0\n\t"
      "s_cbranch_scc1 Lp349_%=\n\t"
      "Lr350_%=:\n\t"
      "v_mfma_f32_16x16x128_f8f6f4 v[88:91], a[224:231], v[32:39], v[88:91] cbsz:1 blgp:1\n\t"
      "s_cbranch_vccnz AQ351_%=\n\t"
      "AR352_%=:\n\t"
      "s_waitcnt vmcnt(7)\n\t"
      "ds_read2st64_b32 v[228:229], v250 offset0:128 offset1:132\n\t"
      "ds_read_b128 v[56:59], v240 offset:32768\n\t"
      "buffer_load_dwordx4 v200, s[44:47], s80 offen offset:1024 sc0 sc1 lds\n\t"
      "v_mfma_f32_16x16x128_f8f6f4 v[92:95], a[232:239], v[32:39], v[92:95] cbsz:1 blgp:1\n\t"
      "ds_read_b128 v[60:63], v241 offset:32768\n\t"
      "s_waitcnt lgkmcnt(5)\n\t"
      "v_mfma_f32_16x16x128_f8f6f4 v[96:99], a[112:119], v[40:47], v[96:99] cbsz:1 blgp:1\n\t"
      "ds_read_b128 v[64:67], v240 offset:34816\n\t"
      "v_mfma_f32_16x16x128_f8f6f4 v[100:103], a[120:127], v[40:47], v[100:103] cbsz:1 blgp:1\n\t"
      "s_waitcnt lgkmcnt(3)\n\t"
      "v_min_u32 v228, v228, v229\n\t"
      "v_cmp_eq_u32 vcc, 0, v228\n\t"
      "ds_read_b128 v[68:71], v241 offset:34816\n\t"
      "v_mfma_f32_16x16x128_f8f6f4 v[96:99], a[224:231], v[48:55], v[96:99] cbsz:1 blgp:1\n\t"
      "s_cbranch_vccnz XS353_%=\n\t"
      "XR354_%=:\n\t"
      "ds_add_u32 v203, v204 offset:0\n\t"
      "ds_read_b32 v253, v203 offset:256\n\t"
      "ds_read_b128 v[32:35], v240 offset:36864\n\t"
      "v_mfma_f32_16x16x128_f8f6f4 v[100:103], a[232:239], v[48:55], v[100:103] cbsz:1 blgp:1\n\t"
      "ds_read_b128 v[36:39], v241 offset:36864\n\t"
      "s_waitcnt lgkmcnt(6)\n\t"
      "v_mfma_f32_16x16x128_f8f6f4 v[88:91], a[128:135], v[56:63], v[88:91] cbsz:1 blgp:1\n\t"
      "ds_read_b128 v[40:43], v240 offset:38912\n\t"
      "s_add_u32 m0, s52, 122880\n\t"
      "s_add_u32 s80, s25, 16384\n\t"
      "buffer_load_dwordx4 v200, s[44:47], s80 offen offset:0 sc0 sc1 lds\n\t"
      "v_mfma_f32_16x16x128_f8f6f4 v[92:95], a[136:143], v[56:63], v[92:95] cbsz:1 blgp:1\n\t"
      "s_waitcnt lgkmcnt(3)\n\t"
      "v_cmp_gt_u32 vcc, 44, v253\n\t"
      "ds_read_b128 v[44:47], v241 offset:38912\n\t"
      "s_cmp_lg_u32 s65, 0\n\t"
      "s_cbranch_scc1 Lp359_%=\n\t"
      "Lr360_%=:\n\t"
      "v_mfma_f32_16x16x128_f8f6f4 v[88:91], a[240:247], v[64:71], v[88:91] cbsz:1 blgp:1\n\t"
      "s_cbranch_vccnz AQ361_%=\n\t"
      "AR362_%=:\n\t"
      "s_waitcnt vmcnt(6)\n\t"
      "ds_read2st64_b32 v[228:229], v251 offset0:32 offset1:36\n\t"
      "ds_read_b128 v[48:51], v238 offset:40960\n\t"
      "buffer_load_dwordx4 v200, s[44:47], s80 offen offset:1024 sc0 sc1 lds\n\t"
      "v_mfma_f32_16x16x128_f8f6f4 v[92:95], a[248:255], v[64:71], v[92:95] cbsz:1 blgp:1\n\t"
      "ds_read_b128 v[52:55], v239 offset:40960\n\t"
      "s_waitcnt lgkmcnt(5)\n\t"
      "v_mfma_f32_16x16x128_f8f6f4 v[96:99], a[128:135], v[32:39], v[96:99] cbsz:1 blgp:1\n\t"
      "ds_read_b128 v[56:59], v238 offset:43008\n\t"
      "v_mfma_f32_16x16x128_f8f6f4 v[100:103], a[136:143], v[32:39], v[100:103] cbsz:1 blgp:1\n\t"
      "s_waitcnt lgkmcnt(3)\n\t"
      "v_min_u32 v228, v228, v229\n\t"
      "v_cmp_eq_u32 vcc, 0, v228\n\t"
      "ds_read_b128 v[60:63], v239 offset:43008\n\t"
      "v_mfma_f32_16x16x128_f8f6f4 v[96:99], a[240:247], v[40:47], v[96:99] cbsz:1 blgp:1\n\t"
      "s_cbranch_vccnz XS363_%=\n\t"
      "XR364_%=:\n\t"
      "ds_add_u32 v203, v204 offset:256\n\t"
      "ds_read_b32 v253, v203 offset:0\n\t"
      "ds_read_b128 v[64:67], v238 offset:45056\n\t"
      "v_mfma_f32_16x16x128_f8f6f4 v[100:103], a[248:255], v[40:47], v[100:103] cbsz:1 blgp:1\n\t"
      "ds_read_b128 v[68:71], v239 offset:45056\n\t"
      "s_waitcnt lgkmcnt(6)\n\t"
      "v_mfma_f32_16x16x128_f8f6f4 v[104:107], a[32:39], v[48:55], v[104:107] cbsz:1 blgp:1\n\t"
      "ds_read_b128 v[32:35], v238 offset:47104\n\t"
      "s_add_u32 m0, s52, 114688\n\t"
      "s_add_u32 s80, s26, 16384\n\t"
      "buffer_load_dwordx4 v200, s[44:47], s80 offen offset:0 sc0 sc1 lds\n\t"
      "v_mfma_f32_16x16x128_f8f6f4 v[108:111], a[40:47], v[48:55], v[108:111] cbsz:1 blgp:1\n\t"
      "s_waitcnt lgkmcnt(3)\n\t"
      "v_cmp_gt_u32 vcc, 44, v253\n\t"
      "ds_read_b128 v[36:39], v239 offset:47104\n\t"
      "s_cmp_lg_u32 s65, 0\n\t"
      "s_cbranch_scc1 Lp369_%=\n\t"
      "Lr370_%=:\n\t"
      "v_mfma_f32_16x16x128_f8f6f4 v[104:107], a[144:151], v[56:63], v[104:107] cbsz:1 blgp:1\n\t"
      "s_cbranch_vccnz AQ371_%=\n\t"
      "AR372_%=:\n\t"
      "s_waitcnt vmcnt(5)\n\t"
      "ds_read2st64_b32 v[228:229], v252 offset0:64 offset1:68\n\t"
      "ds_read_b128 v[40:43], v238 offset:32768\n\t"
      "buffer_load_dwordx4 v200, s[44:47], s80 offen offset:1024 sc0 sc1 lds\n\t"
      "v_mfma_f32_16x16x128_f8f6f4 v[108:111], a[152:159], v[56:63], v[108:111] cbsz:1 blgp:1\n\t"
      "ds_read_b128 v[44:47], v239 offset:32768\n\t"
      "global_store_dwordx4 v210, v[88:91], s[30:31] sc0 sc1\n\t"
      "s_waitcnt lgkmcnt(5)\n\t"
      "v_mfma_f32_16x16x128_f8f6f4 v[112:115], a[32:39], v[64:71], v[112:115] cbsz:1 blgp:1\n\t"
      "ds_read_b128 v[48:51], v238 offset:34816\n\t"
      "global_store_dwordx4 v211, v[92:95], s[30:31] sc0 sc1\n\t"
      "s_add_u32 s30, s30, 0x40000\n\t"
      "s_addc_u32 s31, s31, 0\n\t"
      "v_mfma_f32_16x16x128_f8f6f4 v[116:119], a[40:47], v[64:71], v[116:119] cbsz:1 blgp:1\n\t"
      "s_waitcnt lgkmcnt(3)\n\t"
      "v_min_u32 v228, v228, v229\n\t"
      "v_cmp_eq_u32 vcc, 0, v228\n\t"
      "ds_read_b128 v[52:55], v239 offset:34816\n\t"
      "v_mfma_f32_16x16x128_f8f6f4 v[112:115], a[144:151], v[32:39], v[112:115] cbsz:1 blgp:1\n\t"
      "s_cbranch_vccnz XS373_%=\n\t"
      "XR374_%=:\n\t"
      "ds_add_u32 v203, v204 offset:0\n\t"
      "ds_read_b32 v253, v203 offset:256\n\t"
      "ds_read_b128 v[56:59], v238 offset:36864\n\t"
      "v_mfma_f32_16x16x128_f8f6f4 v[116:119], a[152:159], v[32:39], v[116:119] cbsz:1 blgp:1\n\t"
      "ds_read_b128 v[60:63], v239 offset:36864\n\t"
      "s_waitcnt lgkmcnt(6)\n\t"
      "v_mfma_f32_16x16x128_f8f6f4 v[104:107], a[48:55], v[40:47], v[104:107] cbsz:1 blgp:1\n\t"
      "ds_read_b128 v[64:67], v238 offset:38912\n\t"
      "s_add_u32 m0, s52, 106496\n\t"
      "s_add_u32 s80, s18, 24576\n\t"
      "buffer_load_dwordx4 v200, s[44:47], s80 offen offset:0 sc0 sc1 lds\n\t"
      "v_mfma_f32_16x16x128_f8f6f4 v[108:111], a[56:63], v[40:47], v[108:111] cbsz:1 blgp:1\n\t"
      "s_waitcnt lgkmcnt(3)\n\t"
      "v_cmp_gt_u32 vcc, 48, v253\n\t"
      "ds_read_b128 v[68:71], v239 offset:38912\n\t"
      "s_cmp_lg_u32 s65, 0\n\t"
      "s_cbranch_scc1 Lp379_%=\n\t"
      "Lr380_%=:\n\t"
      "v_mfma_f32_16x16x128_f8f6f4 v[104:107], a[160:167], v[48:55], v[104:107] cbsz:1 blgp:1\n\t"
      "s_cbranch_vccnz AQ381_%=\n\t"
      "AR382_%=:\n\t"
      "s_waitcnt vmcnt(7)\n\t"
      "ds_read2st64_b32 v[228:229], v251 offset0:0 offset1:4\n\t"
      "ds_read_b128 v[32:35], v240 offset:8192\n\t"
      "buffer_load_dwordx4 v200, s[44:47], s80 offen offset:1024 sc0 sc1 lds\n\t"
      "v_mfma_f32_16x16x128_f8f6f4 v[108:111], a[168:175], v[48:55], v[108:111] cbsz:1 blgp:1\n\t"
      "ds_read_b128 v[36:39], v241 offset:8192\n\t"
      "global_store_dwordx4 v210, v[96:99], s[30:31] sc0 sc1\n\t"
      "s_waitcnt lgkmcnt(5)\n\t"
      "v_mfma_f32_16x16x128_f8f6f4 v[112:115], a[48:55], v[56:63], v[112:115] cbsz:1 blgp:1\n\t"
      "ds_read_b128 v[40:43], v240 offset:10240\n\t"
      "global_store_dwordx4 v211, v[100:103], s[30:31] sc0 sc1\n\t"
      "s_add_u32 s30, s30, 0x40000\n\t"
      "s_addc_u32 s31, s31, 0\n\t"
      "v_mfma_f32_16x16x128_f8f6f4 v[116:119], a[56:63], v[56:63], v[116:119] cbsz:1 blgp:1\n\t"
      "s_waitcnt lgkmcnt(3)\n\t"
      "v_min_u32 v228, v228, v229\n\t"
      "v_cmp_eq_u32 vcc, 0, v228\n\t"
      "ds_read_b128 v[44:47], v241 offset:10240\n\t"
      "v_mfma_f32_16x16x128_f8f6f4 v[112:115], a[160:167], v[64:71], v[112:115] cbsz:1 blgp:1\n\t"
      "s_cbranch_vccnz XS383_%=\n\t"
      "XR384_%=:\n\t"
      "ds_add_u32 v203, v204 offset:256\n\t"
      "ds_read_b32 v253, v203 offset:0\n\t"
      "ds_read_b128 v[48:51], v240 offset:12288\n\t"
      "v_mfma_f32_16x16x128_f8f6f4 v[116:119], a[168:175], v[64:71], v[116:119] cbsz:1 blgp:1\n\t"
      "ds_read_b128 v[52:55], v241 offset:12288\n\t"
      "s_waitcnt lgkmcnt(6)\n\t"
      "v_mfma_f32_16x16x128_f8f6f4 v[104:107], a[64:71], v[32:39], v[104:107] cbsz:1 blgp:1\n\t"
      "ds_read_b128 v[56:59], v240 offset:14336\n\t"
      "s_add_u32 m0, s52, 98304\n\t"
      "s_add_u32 s80, s19, 24576\n\t"
      "buffer_load_dwordx4 v200, s[44:47], s80 offen offset:0 sc0 sc1 lds\n\t"
      "v_mfma_f32_16x16x128_f8f6f4 v[108:111], a[72:79], v[32:39], v[108:111] cbsz:1 blgp:1\n\t"
      "s_waitcnt lgkmcnt(3)\n\t"
      "v_cmp_gt_u32 vcc, 48, v253\n\t"
      "ds_read_b128 v[60:63], v241 offset:14336\n\t"
      "s_cmp_lg_u32 s65, 0\n\t"
      "s_cbranch_scc1 Lp389_%=\n\t"
      "Lr390_%=:\n\t"
      "v_mfma_f32_16x16x128_f8f6f4 v[104:107], a[176:183], v[40:47], v[104:107] cbsz:1 blgp:1\n\t"
      "s_cbranch_vccnz AQ391_%=\n\t"
      "AR392_%=:\n\t"
      "s_waitcnt vmcnt(9)\n\t"
      "ds_read2st64_b32 v[228:229], v251 offset0:224 offset1:228\n\t"
      "ds_read_b128 v[64:67], v242 offset:16384\n\t"
      "buffer_load_dwordx4 v200, s[44:47], s80 offen offset:1024 sc0 sc1 lds\n\t"
      "v_mfma_f32_16x16x128_f8f6f4 v[108:111], a[184:191], v[40:47], v[108:111] cbsz:1 blgp:1\n\t"
      "ds_read_b128 v[68:71], v243 offset:16384\n\t"
      "s_waitcnt lgkmcnt(5)\n\t"
      "v_mfma_f32_16x16x128_f8f6f4 v[112:115], a[64:71], v[48:55], v[112:115] cbsz:1 blgp:1\n\t"
      "ds_read_b128 v[32:35], v242 offset:18432\n\t"
      "v_mfma_f32_16x16x128_f8f6f4 v[116:119], a[72:79], v[48:55], v[116:119] cbsz:1 blgp:1\n\t"
      "s_waitcnt lgkmcnt(3)\n\t"
      "v_min_u32 v228, v228, v229\n\t"
      "v_cmp_eq_u32 vcc, 0, v228\n\t"
      "ds_read_b128 v[36:39], v243 offset:18432\n\t"
      "v_mfma_f32_16x16x128_f8f6f4 v[112:115], a[176:183], v[56:63], v[112:115] cbsz:1 blgp:1\n\t"
      "s_cbranch_vccnz XS393_%=\n\t"
      "XR394_%=:\n\t"
      "ds_add_u32 v203, v204 offset:0\n\t"
      "ds_read_b32 v253, v203 offset:256\n\t"
      "ds_read_b128 v[40:43], v242 offset:20480\n\t"
      "v_mfma_f32_16x16x128_f8f6f4 v[116:119], a[184:191], v[56:63], v[116:119] cbsz:1 blgp:1\n\t"
      "ds_read_b128 v[44:47], v243 offset:20480\n\t"
      "s_waitcnt lgkmcnt(6)\n\t"
      "v_mfma_f32_16x16x128_f8f6f4 v[104:107], a[80:87], v[64:71], v[104:107] cbsz:1 blgp:1\n\t"
      "ds_read_b128 v[48:51], v242 offset:22528\n\t"
      "s_add_u32 m0, s52, 40960\n\t"
      "s_add_u32 s80, s20, 24576\n\t"
      "buffer_load_dwordx4 v200, s[44:47], s80 offen offset:0 sc0 sc1 lds\n\t"
      "v_mfma_f32_16x16x128_f8f6f4 v[108:111], a[88:95], v[64:71], v[108:111] cbsz:1 blgp:1\n\t"
      "s_waitcnt lgkmcnt(3)\n\t"
      "v_cmp_gt_u32 vcc, 52, v253\n\t"
      "ds_read_b128 v[52:55], v243 offset:22528\n\t"
      "s_cmp_lg_u32 s65, 0\n\t"
      "s_cbranch_scc1 Lp399_%=\n\t"
      "Lr400_%=:\n\t"
      "v_mfma_f32_16x16x128_f8f6f4 v[104:107], a[192:199], v[32:39], v[104:107] cbsz:1 blgp:1\n\t"
      "s_cbranch_vccnz AQ401_%=\n\t"
      "AR402_%=:\n\t"
      "s_waitcnt vmcnt(9)\n\t"
      "ds_read2st64_b32 v[228:229], v251 offset0:192 offset1:196\n\t"
      "ds_read_b128 v[56:59], v240 offset:0\n\t"
      "buffer_load_dwordx4 v200, s[44:47], s80 offen offset:1024 sc0 sc1 lds\n\t"
      "v_mfma_f32_16x16x128_f8f6f4 v[108:111], a[200:207], v[32:39], v[108:111] cbsz:1 blgp:1\n\t"
      "ds_read_b128 v[60:63], v241 offset:0\n\t"
      "s_waitcnt lgkmcnt(5)\n\t"
      "v_mfma_f32_16x16x128_f8f6f4 v[112:115], a[80:87], v[40:47], v[112:115] cbsz:1 blgp:1\n\t"
      "ds_read_b128 v[64:67], v240 offset:2048\n\t"
      "v_mfma_f32_16x16x128_f8f6f4 v[116:119], a[88:95], v[40:47], v[116:119] cbsz:1 blgp:1\n\t"
      "s_waitcnt lgkmcnt(3)\n\t"
      "v_min_u32 v228, v228, v229\n\t"
      "v_cmp_eq_u32 vcc, 0, v228\n\t"
      "ds_read_b128 v[68:71], v241 offset:2048\n\t"
      "v_mfma_f32_16x16x128_f8f6f4 v[112:115], a[192:199], v[48:55], v[112:115] cbsz:1 blgp:1\n\t"
      "s_cbranch_vccnz XS403_%=\n\t"
      "XR404_%=:\n\t"
      "ds_add_u32 v203, v204 offset:256\n\t"
      "ds_read_b32 v253, v203 offset:0\n\t"
      "ds_read_b128 v[32:35], v240 offset:4096\n\t"
      "v_mfma_f32_16x16x128_f8f6f4 v[116:119], a[200:207], v[48:55], v[116:119] cbsz:1 blgp:1\n\t"
      "ds_read_b128 v[36:39], v241 offset:4096\n\t"
      "s_waitcnt lgkmcnt(6)\n\t"
      "v_mfma_f32_16x16x128_f8f6f4 v[104:107], a[96:103], v[56:63], v[104:107] cbsz:1 blgp:1\n\t"
      "ds_read_b128 v[40:43], v240 offset:6144\n\t"
      "s_add_u32 m0, s52, 32768\n\t"
      "s_add_u32 s80, s21, 24576\n\t"
      "buffer_load_dwordx4 v200, s[44:47], s80 offen offset:0 sc0 sc1 lds\n\t"
      "v_mfma_f32_16x16x128_f8f6f4 v[108:111], a[104:111], v[56:63], v[108:111] cbsz:1 blgp:1\n\t"
      "s_waitcnt lgkmcnt(3)\n\t"
      "v_cmp_gt_u32 vcc, 52, v253\n\t"
      "ds_read_b128 v[44:47], v241 offset:6144\n\t"
      "s_cmp_lg_u32 s65, 0\n\t"
      "s_cbranch_scc1 Lp409_%=\n\t"
      "Lr410_%=:\n\t"
      "v_mfma_f32_16x16x128_f8f6f4 v[104:107], a[208:215], v[64:71], v[104:107] cbsz:1 blgp:1\n\t"
      "s_cbranch_vccnz AQ411_%=\n\t"
      "AR412_%=:\n\t"
      "s_waitcnt vmcnt(7)\n\t"
      "ds_read2st64_b32 v[228:229], v251 offset0:160 offset1:164\n\t"
      "ds_read_b128 v[48:51], v240 offset:57344\n\t"
      "buffer_load_dwordx4 v200, s[44:47], s80 offen offset:1024 sc0 sc1 lds\n\t"
      "s_waitcnt vmcnt(6)\n\t"
      "v_mfma_f32_16x16x128_f8f6f4 v[108:111], a[216:223], v[64:71], v[108:111] cbsz:1 blgp:1\n\t"
      "ds_read_b128 v[52:55], v241 offset:57344\n\t"
      "v_mov_b32 v218, s64\n\t"
      "global_store_dword v212, v218, s[16:17] offset:128 sc0 sc1\n\t"
      "s_waitcnt lgkmcnt(5)\n\t"
      "v_mfma_f32_16x16x128_f8f6f4 v[112:115], a[96:103], v[32:39], v[112:115] cbsz:1 blgp:1\n\t"
      "ds_read_b128 v[56:59], v240 offset:59392\n\t"
      "v_mfma_f32_16x16x128_f8f6f4 v[116:119], a[104:111], v[32:39], v[116:119] cbsz:1 blgp:1\n\t"
      "s_waitcnt lgkmcnt(3)\n\t"
      "v_min_u32 v228, v228, v229\n\t"
      "v_cmp_eq_u32 vcc, 0, v228\n\t"
      "ds_read_b128 v[60:63], v241 offset:59392\n\t"
      "v_mfma_f32_16x16x128_f8f6f4 v[112:115], a[208:215], v[40:47], v[112:115] cbsz:1 blgp:1\n\t"
      "s_cbranch_vccnz XS413_%=\n\t"
      "XR414_%=:\n\t"
      "ds_add_u32 v203, v204 offset:0\n\t"
      "ds_read_b32 v253, v203 offset:256\n\t"
      "ds_read_b128 v[64:67], v240 offset:61440\n\t"
      "v_mfma_f32_16x16x128_f8f6f4 v[116:119], a[216:223], v[40:47], v[116:119] cbsz:1 blgp:1\n\t"
      "ds_read_b128 v[68:71], v241 offset:61440\n\t"
      "s_waitcnt lgkmcnt(6)\n\t"
      "v_mfma_f32_16x16x128_f8f6f4 v[104:107], a[112:119], v[48:55], v[104:107] cbsz:1 blgp:1\n\t"
      "ds_read_b128 v[32:35], v240 offset:63488\n\t"
      "s_add_u32 m0, s52, 73728\n\t"
      "s_add_u32 s80, s24, 24576\n\t"
      "buffer_load_dwordx4 v200, s[44:47], s80 offen offset:0 sc0 sc1 lds\n\t"
      "v_mfma_f32_16x16x128_f8f6f4 v[108:111], a[120:127], v[48:55], v[108:111] cbsz:1 blgp:1\n\t"
      "s_waitcnt lgkmcnt(3)\n\t"
      "v_cmp_gt_u32 vcc, 56, v253\n\t"
      "ds_read_b128 v[36:39], v241 offset:63488\n\t"
      "s_cmp_lg_u32 s65, 0\n\t"
      "s_cbranch_scc1 Lp419_%=\n\t"
      "Lr420_%=:\n\t"
      "v_mfma_f32_16x16x128_f8f6f4 v[104:107], a[224:231], v[56:63], v[104:107] cbsz:1 blgp:1\n\t"
      "s_cbranch_vccnz AQ421_%=\n\t"
      "AR422_%=:\n\t"
      "s_waitcnt vmcnt(6)\n\t"
      "ds_read2st64_b32 v[228:229], v251 offset0:128 offset1:132\n\t"
      "ds_read_b128 v[40:43], v240 offset:49152\n\t"
      "buffer_load_dwordx4 v200, s[44:47], s80 offen offset:1024 sc0 sc1 lds\n\t"
      "v_mfma_f32_16x16x128_f8f6f4 v[108:111], a[232:239], v[56:63], v[108:111] cbsz:1 blgp:1\n\t"
      "ds_read_b128 v[44:47], v241 offset:49152\n\t"
      "s_waitcnt lgkmcnt(5)\n\t"
      "v_mfma_f32_16x16x128_f8f6f4 v[112:115], a[112:119], v[64:71], v[112:115] cbsz:1 blgp:1\n\t"
      "ds_read_b128 v[48:51], v240 offset:51200\n\t"
      "v_mfma_f32_16x16x128_f8f6f4 v[116:119], a[120:127], v[64:71], v[116:119] cbsz:1 blgp:1\n\t"
      "s_waitcnt lgkmcnt(3)\n\t"
      "v_min_u32 v228, v228, v229\n\t"
      "v_cmp_eq_u32 vcc, 0, v228\n\t"
      "ds_read_b128 v[52:55], v241 offset:51200\n\t"
      "v_mfma_f32_16x16x128_f8f6f4 v[112:115], a[224:231], v[32:39], v[112:115] cbsz:1 blgp:1\n\t"
      "s_cbranch_vccnz XS423_%=\n\t"
      "XR424_%=:\n\t"
      "ds_add_u32 v203, v204 offset:256\n\t"
      "ds_read_b32 v253, v203 offset:0\n\t"
      "ds_read_b128 v[56:59], v240 offset:53248\n\t"
      "v_mfma_f32_16x16x128_f8f6f4 v[116:119], a[232:239], v[32:39], v[116:119] cbsz:1 blgp:1\n\t"
      "ds_read_b128 v[60:63], v241 offset:53248\n\t"
      "s_waitcnt lgkmcnt(6)\n\t"
      "v_mfma_f32_16x16x128_f8f6f4 v[104:107], a[128:135], v[40:47], v[104:107] cbsz:1 blgp:1\n\t"
      "ds_read_b128 v[64:67], v240 offset:55296\n\t"
      "s_add_u32 m0, s52, 147456\n\t"
      "s_add_u32 s80, s25, 24576\n\t"
      "buffer_load_dwordx4 v200, s[44:47], s80 offen offset:0 sc0 sc1 lds\n\t"
      "v_mfma_f32_16x16x128_f8f6f4 v[108:111], a[136:143], v[40:47], v[108:111] cbsz:1 blgp:1\n\t"
      "s_waitcnt lgkmcnt(3)\n\t"
      "v_cmp_gt_u32 vcc, 56, v253\n\t"
      "ds_read_b128 v[68:71], v241 offset:55296\n\t"
      "s_cmp_lg_u32 s65, 0\n\t"
      "s_cbranch_scc1 Lp429_%=\n\t"
      "Lr430_%=:\n\t"
      "v_mfma_f32_16x16x128_f8f6f4 v[104:107], a[240:247], v[48:55], v[104:107] cbsz:1 blgp:1\n\t"
      "s_cbranch_vccnz AQ431_%=\n\t"
      "AR432_%=:\n\t"
      "s_waitcnt vmcnt(6)\n\t"
      "ds_read2st64_b32 v[228:229], v250 offset0:160 offset1:164\n\t"
      "ds_read_b128 v[32:35], v240 offset:40960\n\t"
      "buffer_load_dwordx4 v200, s[44:47], s80 offen offset:1024 sc0 sc1 lds\n\t"
      "v_mfma_f32_16x16x128_f8f6f4 v[108:111], a[248:255], v[48:55], v[108:111] cbsz:1 blgp:1\n\t"
      "ds_read_b128 v[36:39], v241 offset:40960\n\t"
      "global_load_dword v185, v213, s[16:17] offset:128 sc0 sc1\n\t"
      "s_waitcnt lgkmcnt(5)\n\t"
      "v_mfma_f32_16x16x128_f8f6f4 v[112:115], a[128:135], v[56:63], v[112:115] cbsz:1 blgp:1\n\t"
      "ds_read_b128 v[40:43], v240 offset:43008\n\t"
      "v_mfma_f32_16x16x128_f8f6f4 v[116:119], a[136:143], v[56:63], v[116:119] cbsz:1 blgp:1\n\t"
      "s_waitcnt lgkmcnt(3)\n\t"
      "v_min_u32 v228, v228, v229\n\t"
      "v_cmp_eq_u32 vcc, 0, v228\n\t"
      "ds_read_b128 v[44:47], v241 offset:43008\n\t"
      "v_mfma_f32_16x16x128_f8f6f4 v[112:115], a[240:247], v[64:71], v[112:115] cbsz:1 blgp:1\n\t"
      "s_cbranch_vccnz XS433_%=\n\t"
      "XR434_%=:\n\t"
      "ds_add_u32 v203, v204 offset:0\n\t"
      "ds_read_b32 v253, v203 offset:256\n\t"
      "ds_read_b128 v[48:51], v240 offset:45056\n\t"
      "v_mfma_f32_16x16x128_f8f6f4 v[116:119], a[248:255], v[64:71], v[116:119] cbsz:1 blgp:1\n\t"
      "ds_read_b128 v[52:55], v241 offset:45056\n\t"
      "s_waitcnt lgkmcnt(6)\n\t"
      "v_mfma_f32_16x16x128_f8f6f4 v[120:123], a[32:39], v[32:39], v[120:123] cbsz:1 blgp:1\n\t"
      "ds_read_b128 v[56:59], v240 offset:47104\n\t"
      "s_add_u32 m0, s52, 65536\n\t"
      "s_add_u32 s80, s26, 24576\n\t"
      "buffer_load_dwordx4 v200, s[44:47], s80 offen offset:0 sc0 sc1 lds\n\t"
      "v_mfma_f32_16x16x128_f8f6f4 v[124:127], a[40:47], v[32:39], v[124:127] cbsz:1 blgp:1\n\t"
      "s_waitcnt lgkmcnt(3)\n\t"
      "v_cmp_gt_u32 vcc, 60, v253\n\t"
      "ds_read_b128 v[60:63], v241 offset:47104\n\t"
      "s_cmp_lg_u32 s65, 0\n\t"
      "s_cbranch_scc1 Lp439_%=\n\t"
      "Lr440_%=:\n\t"
      "v_mfma_f32_16x16x128_f8f6f4 v[120:123], a[144:151], v[40:47], v[120:123] cbsz:1 blgp:1\n\t"
      "s_cbranch_vccnz AQ441_%=\n\t"
      "AR442_%=:\n\t"
      "s_waitcnt vmcnt(7)\n\t"
      "ds_read2st64_b32 v[228:229], v250 offset0:128 offset1:132\n\t"
      "ds_read_b128 v[64:67], v240 offset:32768\n\t"
      "buffer_load_dwordx4 v200, s[44:47], s80 offen offset:1024 sc0 sc1 lds\n\t"
      "v_mfma_f32_16x16x128_f8f6f4 v[124:127], a[152:159], v[40:47], v[124:127] cbsz:1 blgp:1\n\t"
      "ds_read_b128 v[68:71], v241 offset:32768\n\t"
      "s_waitcnt lgkmcnt(5)\n\t"
      "v_mfma_f32_16x16x128_f8f6f4 v[128:131], a[32:39], v[48:55], v[128:131] cbsz:1 blgp:1\n\t"
      "ds_read_b128 v[32:35], v240 offset:34816\n\t"
      "v_mfma_f32_16x16x128_f8f6f4 v[132:135], a[40:47], v[48:55], v[132:135] cbsz:1 blgp:1\n\t"
      "s_waitcnt lgkmcnt(3)\n\t"
      "v_min_u32 v228, v228, v229\n\t"
      "v_cmp_eq_u32 vcc, 0, v228\n\t"
      "ds_read_b128 v[36:39], v241 offset:34816\n\t"
      "v_mfma_f32_16x16x128_f8f6f4 v[128:131], a[144:151], v[56:63], v[128:131] cbsz:1 blgp:1\n\t"
      "s_cbranch_vccnz XS443_%=\n\t"
      "XR444_%=:\n\t"
      "ds_add_u32 v203, v204 offset:256\n\t"
      "ds_read_b32 v253, v203 offset:0\n\t"
      "ds_read_b128 v[40:43], v240 offset:36864\n\t"
      "v_mfma_f32_16x16x128_f8f6f4 v[132:135], a[152:159], v[56:63], v[132:135] cbsz:1 blgp:1\n\t"
      "ds_read_b128 v[44:47], v241 offset:36864\n\t"
      "s_waitcnt lgkmcnt(6)\n\t"
      "v_mfma_f32_16x16x128_f8f6f4 v[120:123], a[48:55], v[64:71], v[120:123] cbsz:1 blgp:1\n\t"
      "ds_read_b128 v[48:51], v240 offset:38912\n\t"
      "v_mfma_f32_16x16x128_f8f6f4 v[124:127], a[56:63], v[64:71], v[124:127] cbsz:1 blgp:1\n\t"
      "s_waitcnt lgkmcnt(3)\n\t"
      "v_cmp_gt_u32 vcc, 60, v253\n\t"
      "ds_read_b128 v[52:55], v241 offset:38912\n\t"
      "s_cmp_lg_u32 s65, 0\n\t"
      "s_cbranch_scc1 Lp449_%=\n\t"
      "Lr450_%=:\n\t"
      "v_mfma_f32_16x16x128_f8f6f4 v[120:123], a[160:167], v[32:39], v[120:123] cbsz:1 blgp:1\n\t"
      "s_cbranch_vccnz AQ451_%=\n\t"
      "AR452_%=:\n\t"
      "s_waitcnt vmcnt(5)\n\t"
      "ds_read2st64_b32 v[228:229], v251 offset0:32 offset1:36\n\t"
      "ds_read_b128 v[56:59], v238 offset:40960\n\t"
      "v_mfma_f32_16x16x128_f8f6f4 v[124:127], a[168:175], v[32:39], v[124:127] cbsz:1 blgp:1\n\t"
      "ds_read_b128 v[60:63], v239 offset:40960\n\t"
      "s_waitcnt lgkmcnt(5)\n\t"
      "v_mfma_f32_16x16x128_f8f6f4 v[128:131], a[48:55], v[40:47], v[128:131] cbsz:1 blgp:1\n\t"
      "ds_read_b128 v[64:67], v238 offset:43008\n\t"
      "v_mfma_f32_16x16x128_f8f6f4 v[132:135], a[56:63], v[40:47], v[132:135] cbsz:1 blgp:1\n\t"
      "s_waitcnt lgkmcnt(3)\n\t"
      "v_min_u32 v228, v228, v229\n\t"
      "v_cmp_eq_u32 vcc, 0, v228\n\t"
      "ds_read_b128 v[68:71], v239 offset:43008\n\t"
      "v_mfma_f32_16x16x128_f8f6f4 v[128:131], a[160:167], v[48:55], v[128:131] cbsz:1 blgp:1\n\t"
      "s_cbranch_vccnz XS453_%=\n\t"
      "XR454_%=:\n\t"
      "ds_add_u32 v203, v204 offset:0\n\t"
      "ds_read_b32 v253, v203 offset:256\n\t"
      "ds_read_b128 v[32:35], v238 offset:45056\n\t"
      "v_mfma_f32_16x16x128_f8f6f4 v[132:135], a[168:175], v[48:55], v[132:135] cbsz:1 blgp:1\n\t"
      "ds_read_b128 v[36:39], v239 offset:45056\n\t"
      "s_waitcnt lgkmcnt(6)\n\t"
      "v_mfma_f32_16x16x128_f8f6f4 v[120:123], a[64:71], v[56:63], v[120:123] cbsz:1 blgp:1\n\t"
      "ds_read_b128 v[40:43], v238 offset:47104\n\t"
      "v_mfma_f32_16x16x128_f8f6f4 v[124:127], a[72:79], v[56:63], v[124:127] cbsz:1 blgp:1\n\t"
      "s_waitcnt lgkmcnt(3)\n\t"
      "v_cmp_gt_u32 vcc, 64, v253\n\t"
      "ds_read_b128 v[44:47], v239 offset:47104\n\t"
      "s_cmp_lg_u32 s65, 0\n\t"
      "s_cbranch_scc1 Lp459_%=\n\t"
      "Lr460_%=:\n\t"
      "v_mfma_f32_16x16x128_f8f6f4 v[120:123], a[176:183], v[64:71], v[120:123] cbsz:1 blgp:1\n\t"
      "s_cbranch_vccnz AQ461_%=\n\t"
      "AR462_%=:\n\t"
      "s_waitcnt vmcnt(3)\n\t"
      "ds_read2st64_b32 v[228:229], v252 offset0:64 offset1:68\n\t"
      "ds_read_b128 v[48:51], v238 offset:32768\n\t"
      "s_cmp_lg_u32 s65, 0\n\t"
      "s_cbranch_scc1 Lf463_%=\n\t"
      "Lg464_%=:\n\t"
      "v_mfma_f32_16x16x128_f8f6f4 v[124:127], a[184:191], v[64:71], v[124:127] cbsz:1 blgp:1\n\t"
      "ds_read_b128 v[52:55], v239 offset:32768\n\t"
      "s_waitcnt vmcnt(2)\n\t"
      "v_cmp_ne_u32 vcc, s64, v185\n\t"
      "s_cbranch_vccnz FS465_%=\n\t"
      "FR466_%=:\n\t"
      "global_load_dwordx4 v[136:139], v210, s[94:95] sc0 sc1\n\t"
      "s_waitcnt lgkmcnt(5)\n\t"
      "v_mfma_f32_16x16x128_f8f6f4 v[128:131], a[64:71], v[32:39], v[128:131] cbsz:1 blgp:1\n\t"
      "ds_read_b128 v[56:59], v238 offset:34816\n\t"
      "global_load_dwordx4 v[140:143], v211, s[94:95] sc0 sc1\n\t"
      "s_add_u32 s94, s94, 0x40000\n\t"
      "s_addc_u32 s95, s95, 0\n\t"
      "v_mfma_f32_16x16x128_f8f6f4 v[132:135], a[72:79], v[32:39], v[132:135] cbsz:1 blgp:1\n\t"
      "s_waitcnt lgkmcnt(3)\n\t"
      "v_min_u32 v228, v228, v229\n\t"
      "v_cmp_eq_u32 vcc, 0, v228\n\t"
      "ds_read_b128 v[60:63], v239 offset:34816\n\t"
      "v_mfma_f32_16x16x128_f8f6f4 v[128:131], a[176:183], v[40:47], v[128:131] cbsz:1 blgp:1\n\t"
      "s_cbranch_vccnz XS467_%=\n\t"
      "XR468_%=:\n\t"
      "ds_add_u32 v203, v204 offset:256\n\t"
      "ds_read_b32 v253, v203 offset:0\n\t"
      "ds_read_b128 v[64:67], v238 offset:36864\n\t"
      "global_load_dwordx4 v[144:147], v210, s[94:95] sc0 sc1\n\t"
      "v_mfma_f32_16x16x128_f8f6f4 v[132:135], a[184:191], v[40:47], v[132:135] cbsz:1 blgp:1\n\t"
      "ds_read_b128 v[68:71], v239 offset:36864\n\t"
      "global_load_dwordx4 v[148:151], v211, s[94:95] sc0 sc1\n\t"
      "s_add_u32 s94, s94, 0x40000\n\t"
      "s_addc_u32 s95, s95, 0\n\t"
      "s_waitcnt lgkmcnt(6)\n\t"
      "v_mfma_f32_16x16x128_f8f6f4 v[120:123], a[80:87], v[48:55], v[120:123] cbsz:1 blgp:1\n\t"
      "ds_read_b128 v[32:35], v238 offset:38912\n\t"
      "global_load_dwordx4 v[152:155], v210, s[94:95] sc0 sc1\n\t"
      "v_mfma_f32_16x16x128_f8f6f4 v[124:127], a[88:95], v[48:55], v[124:127] cbsz:1 blgp:1\n\t"
      "s_waitcnt lgkmcnt(3)\n\t"
      "v_cmp_gt_u32 vcc, 64, v253\n\t"
      "ds_read_b128 v[36:39], v239 offset:38912\n\t"
      "s_cmp_lg_u32 s65, 0\n\t"
      "s_cbranch_scc1 Lp473_%=\n\t"
      "Lr474_%=:\n\t"
      "v_mfma_f32_16x16x128_f8f6f4 v[120:123], a[192:199], v[56:63], v[120:123] cbsz:1 blgp:1\n\t"
      "s_cbranch_vccnz AQ475_%=\n\t"
      "AR476_%=:\n\t"
      "s_waitcnt vmcnt(5)\n\t"
      "ds_read2st64_b32 v[228:229], v251 offset0:0 offset1:4\n\t"
      "ds_read_b128 v[40:43], v240 offset:8192\n\t"
      "global_load_dwordx4 v[156:159], v211, s[94:95] sc0 sc1\n\t"
      "v_mfma_f32_16x16x128_f8f6f4 v[124:127], a[200:207], v[56:63], v[124:127] cbsz:1 blgp:1\n\t"
      "ds_read_b128 v[44:47], v241 offset:8192\n\t"
      "s_add_u32 s94, s94, 0x40000\n\t"
      "s_addc_u32 s95, s95, 0\n\t"
      "global_load_dwordx4 v[160:163], v210, s[94:95] sc0 sc1\n\t"
      "s_waitcnt lgkmcnt(5)\n\t"
      "v_mfma_f32_16x16x128_f8f6f4 v[128:131], a[80:87], v[64:71], v[128:131] cbsz:1 blgp:1\n\t"
      "ds_read_b128 v[48:51], v240 offset:10240\n\t"
      "global_load_dwordx4 v[164:167], v211, s[94:95] sc0 sc1\n\t"
      "s_add_u32 s94, s94, 0x40000\n\t"
      "s_addc_u32 s95, s95, 0\n\t"
      "v_mfma_f32_16x16x128_f8f6f4 v[132:135], a[88:95], v[64:71], v[132:135] cbsz:1 blgp:1\n\t"
      "s_waitcnt lgkmcnt(3)\n\t"
      "v_min_u32 v228, v228, v229\n\t"
      "v_cmp_eq_u32 vcc, 0, v228\n\t"
      "ds_read_b128 v[52:55], v241 offset:10240\n\t"
      "v_mfma_f32_16x16x128_f8f6f4 v[128:131], a[192:199], v[32:39], v[128:131] cbsz:1 blgp:1\n\t"
      "s_cbranch_vccnz XS477_%=\n\t"
      "XR478_%=:\n\t"
      "ds_add_u32 v203, v204 offset:0\n\t"
      "ds_read_b32 v253, v203 offset:256\n\t"
      "ds_read_b128 v[56:59], v240 offset:12288\n\t"
      "v_mfma_f32_16x16x128_f8f6f4 v[132:135], a[200:207], v[32:39], v[132:135] cbsz:1 blgp:1\n\t"
      "ds_read_b128 v[60:63], v241 offset:12288\n\t"
      "s_waitcnt lgkmcnt(6)\n\t"
      "v_mfma_f32_16x16x128_f8f6f4 v[120:123], a[96:103], v[40:47], v[120:123] cbsz:1 blgp:1\n\t"
      "ds_read_b128 v[64:67], v240 offset:14336\n\t"
      "v_mfma_f32_16x16x128_f8f6f4 v[124:127], a[104:111], v[40:47], v[124:127] cbsz:1 blgp:1\n\t"
      "s_waitcnt lgkmcnt(3)\n\t"
      "v_cmp_gt_u32 vcc, 68, v253\n\t"
      "ds_read_b128 v[68:71], v241 offset:14336\n\t"
      "s_cmp_lg_u32 s65, 0\n\t"
      "s_cbranch_scc1 Lp483_%=\n\t"
      "Lr484_%=:\n\t"
      "v_mfma_f32_16x16x128_f8f6f4 v[120:123], a[208:215], v[48:55], v[120:123] cbsz:1 blgp:1\n\t"
      "s_cbranch_vccnz AQ485_%=\n\t"
      "AR486_%=:\n\t"
      "ds_read_b128 v[32:35], v242 offset:16384\n\t"
      "v_mfma_f32_16x16x128_f8f6f4 v[124:127], a[216:223], v[48:55], v[124:127] cbsz:1 blgp:1\n\t"
      "ds_read_b128 v[36:39], v243 offset:16384\n\t"
      "s_waitcnt lgkmcnt(4)\n\t"
      "v_mfma_f32_16x16x128_f8f6f4 v[128:131], a[96:103], v[56:63], v[128:131] cbsz:1 blgp:1\n\t"
      "ds_read_b128 v[40:43], v242 offset:18432\n\t"
      "v_mfma_f32_16x16x128_f8f6f4 v[132:135], a[104:111], v[56:63], v[132:135] cbsz:1 blgp:1\n\t"
      "ds_read_b128 v[44:47], v243 offset:18432\n\t"
      "s_waitcnt lgkmcnt(4)\n\t"
      "v_mfma_f32_16x16x128_f8f6f4 v[128:131], a[208:215], v[64:71], v[128:131] cbsz:1 blgp:1\n\t"
      "ds_read_b32 v253, v203 offset:0\n\t"
      "ds_read_b128 v[48:51], v242 offset:20480\n\t"
      "v_mfma_f32_16x16x128_f8f6f4 v[132:135], a[216:223], v[64:71], v[132:135] cbsz:1 blgp:1\n\t"
      "ds_read_b128 v[52:55], v243 offset:20480\n\t"
      "s_waitcnt lgkmcnt(5)\n\t"
      "v_mfma_f32_16x16x128_f8f6f4 v[120:123], a[112:119], v[32:39], v[120:123] cbsz:1 blgp:1\n\t"
      "ds_read_b128 v[56:59], v242 offset:22528\n\t"
      "v_mfma_f32_16x16x128_f8f6f4 v[124:127], a[120:127], v[32:39], v[124:127] cbsz:1 blgp:1\n\t"
      "s_waitcnt lgkmcnt(3)\n\t"
      "v_cmp_gt_u32 vcc, 68, v253\n\t"
      "ds_read_b128 v[60:63], v243 offset:22528\n\t"
      "s_cmp_lg_u32 s65, 0\n\t"
      "s_cbranch_scc1 Lp487_%=\n\t"
      "Lr488_%=:\n\t"
      "v_mfma_f32_16x16x128_f8f6f4 v[120:123], a[224:231], v[40:47], v[120:123] cbsz:1 blgp:1\n\t"
      "s_cbranch_vccnz AQ489_%=\n\t"
      "AR490_%=:\n\t"
      "ds_read_b128 v[64:67], v240 offset:0\n\t"
      "s_waitcnt vmcnt(6)\n\t"
      "v_mul_f32 v136, s75, v136\n\t"
      "v_mul_f32 v137, s75, v137\n\t"
      "v_mul_f32 v138, s75, v138\n\t"
      "v_mfma_f32_16x16x128_f8f6f4 v[124:127], a[232:239], v[40:47], v[124:127] cbsz:1 blgp:1\n\t"
      "ds_read_b128 v[68:71], v241 offset:0\n\t"
      "v_mul_f32 v139, s75, v139\n\t"
      "v_mul_f32 v140, s75, v140\n\t"
      "v_mul_f32 v141, s75, v141\n\t"
      "v_mul_f32 v142, s75, v142\n\t"
      "s_waitcnt lgkmcnt(4)\n\t"
      "v_mfma_f32_16x16x128_f8f6f4 v[128:131], a[112:119], v[48:55], v[128:131] cbsz:1 blgp:1\n\t"
      "ds_read_b128 v[32:35], v240 offset:2048\n\t"
      "v_mul_f32 v143, s75, v143\n\t"
      "s_waitcnt vmcnt(4)\n\t"
      "v_mul_f32 v144, s75, v144\n\t"
      "v_mul_f32 v145, s75, v145\n\t"
      "v_mul_f32 v146, s75, v146\n\t"
      "v_mfma_f32_16x16x128_f8f6f4 v[132:135], a[120:127], v[48:55], v[132:135] cbsz:1 blgp:1\n\t"
      "ds_read_b128 v[36:39], v241 offset:2048\n\t"
      "v_mul_f32 v147, s75, v147\n\t"
      "v_mul_f32 v148, s75, v148\n\t"
      "v_mul_f32 v149, s75, v149\n\t"
      "v_mul_f32 v150, s75, v150\n\t"
      "s_waitcnt lgkmcnt(4)\n\t"
      "v_mfma_f32_16x16x128_f8f6f4 v[128:131], a[224:231], v[56:63], v[128:131] cbsz:1 blgp:1\n\t"
      "ds_read_b128 v[40:43], v240 offset:4096\n\t"
      "v_mul_f32 v151, s75, v151\n\t"
      "v_fma_f32 v104, s75, v104, v136\n\t"
      "v_fma_f32 v105, s75, v105, v137\n\t"
      "v_fma_f32 v106, s75, v106, v138\n\t"
      "v_fma_f32 v107, s75, v107, v139\n\t"
      "v_mfma_f32_16x16x128_f8f6f4 v[132:135], a[232:239], v[56:63], v[132:135] cbsz:1 blgp:1\n\t"
      "ds_read_b128 v[44:47], v241 offset:4096\n\t"
      "v_maximum3_f32 v104, 0, v104, 0\n\t"
      "v_maximum3_f32 v105, 0, v105, 0\n\t"
      "v_maximum3_f32 v106, 0, v106, 0\n\t"
      "v_maximum3_f32 v107, 0, v107, 0\n\t"
      "s_waitcnt lgkmcnt(4)\n\t"
      "v_mfma_f32_16x16x128_f8f6f4 v[120:123], a[128:135], v[64:71], v[120:123] cbsz:1 blgp:1\n\t"
      "ds_read_b128 v[48:51], v240 offset:6144\n\t"
      "ds_write_b128 v236, v[104:107] offset:0\n\t"
      "v_fma_f32 v108, s75, v108, v140\n\t"
      "v_fma_f32 v109, s75, v109, v141\n\t"
      "v_fma_f32 v110, s75, v110, v142\n\t"
      "v_fma_f32 v111, s75, v111, v143\n\t"
      "v_mfma_f32_16x16x128_f8f6f4 v[124:127], a[136:143], v[64:71], v[124:127] cbsz:1 blgp:1\n\t"
      "ds_read_b128 v[52:55], v241 offset:6144\n\t"
      "s_cmp_lg_u32 s65, 0\n\t"
      "s_cbranch_scc1 Lp491_%=\n\t"
      "Lr492_%=:\n\t"
      "v_maximum3_f32 v108, 0, v108, 0\n\t"
      "v_maximum3_f32 v109, 0, v109, 0\n\t"
      "s_waitcnt lgkmcnt(5)\n\t"
      "v_mfma_f32_16x16x128_f8f6f4 v[120:123], a[240:247], v[32:39], v[120:123] cbsz:1 blgp:1\n\t"
      "v_maximum3_f32 v110, 0, v110, 0\n\t"
      "v_maximum3_f32 v111, 0, v111, 0\n\t"
      "ds_write_b128 v237, v[108:111] offset:0\n\t"
      "ds_read_b128 v[168:171], v209 offset:0\n\t"
      "ds_read_b128 v[172:175], v209 offset:1024\n\t"
      "v_fma_f32 v112, s75, v112, v144\n\t"
      "v_mfma_f32_16x16x128_f8f6f4 v[124:127], a[248:255], v[32:39], v[124:127] cbsz:1 blgp:1\n\t"
      "v_fma_f32 v113, s75, v113, v145\n\t"
      "v_fma_f32 v114, s75, v114, v146\n\t"
      "v_fma_f32 v115, s75, v115, v147\n\t"
      "v_maximum3_f32 v112, 0, v112, 0\n\t"
      "v_maximum3_f32 v113, 0, v113, 0\n\t"
      "s_waitcnt lgkmcnt(6)\n\t"
      "v_mfma_f32_16x16x128_f8f6f4 v[128:131], a[128:135], v[40:47], v[128:131] cbsz:1 blgp:1\n\t"
      "v_maximum3_f32 v114, 0, v114, 0\n\t"
      "v_maximum3_f32 v115, 0, v115, 0\n\t"
      "ds_write_b128 v236, v[112:115] offset:2048\n\t"
      "v_fma_f32 v116, s75, v116, v148\n\t"
      "v_fma_f32 v117, s75, v117, v149\n\t"
      "v_fma_f32 v118, s75, v118, v150\n\t"
      "v_mfma_f32_16x16x128_f8f6f4 v[132:135], a[136:143], v[40:47], v[132:135] cbsz:1 blgp:1\n\t"
      "v_fma_f32 v119, s75, v119, v151\n\t"
      "v_maximum3_f32 v116, 0, v116, 0\n\t"
      "v_maximum3_f32 v117, 0, v117, 0\n\t"
      "v_maximum3_f32 v118, 0, v118, 0\n\t"
      "v_maximum3_f32 v119, 0, v119, 0\n\t"
      "s_waitcnt lgkmcnt(4)\n\t"
      "v_mfma_f32_16x16x128_f8f6f4 v[128:131], a[240:247], v[48:55], v[128:131] cbsz:1 blgp:1\n\t"
      "ds_write_b128 v237, v[116:119] offset:2048\n\t"
      "ds_read_b128 v[176:179], v209 offset:2048\n\t"
      "ds_read_b128 v[180:183], v209 offset:3072\n\t"
      "s_waitcnt lgkmcnt(4)\n\t"
      "global_store_dwordx4 v210, v[168:171], s[22:23] sc0 sc1\n\t"
      "v_mfma_f32_16x16x128_f8f6f4 v[132:135], a[248:255], v[48:55], v[132:135] cbsz:1 blgp:1\n\t"
      "global_store_dwordx4 v211, v[172:175], s[22:23] sc0 sc1\n\t"
      "s_add_u32 s22, s22, 0x40000\n\t"
      "s_addc_u32 s23, s23, 0\n\t"
      "s_nop 7\n\t"
      "s_waitcnt lgkmcnt(0)\n\t"
      "global_store_dwordx4 v210, v[176:179], s[22:23] sc0 sc1\n\t"
      "global_store_dwordx4 v211, v[180:183], s[22:23] sc0 sc1\n\t"
      "s_add_u32 s22, s22, 0x40000\n\t"
      "s_addc_u32 s23, s23, 0\n\t"
      "s_waitcnt vmcnt(6)\n\t"
      "v_mul_f32 v152, s75, v152\n\t"
      "v_mul_f32 v153, s75, v153\n\t"
      "v_mul_f32 v154, s75, v154\n\t"
      "v_mul_f32 v155, s75, v155\n\t"
      "v_mul_f32 v156, s75, v156\n\t"
      "v_mul_f32 v157, s75, v157\n\t"
      "v_mul_f32 v158, s75, v158\n\t"
      "v_mul_f32 v159, s75, v159\n\t"
      "s_waitcnt vmcnt(4)\n\t"
      "v_mul_f32 v160, s75, v160\n\t"
      "v_mul_f32 v161, s75, v161\n\t"
      "v_mul_f32 v162, s75, v162\n\t"
      "v_mul_f32 v163, s75, v163\n\t"
      "v_mul_f32 v164, s75, v164\n\t"
      "v_mul_f32 v165, s75, v165\n\t"
      "v_mul_f32 v166, s75, v166\n\t"
      "v_mul_f32 v167, s75, v167\n\t"
      "v_fma_f32 v120, s75, v120, v152\n\t"
      "v_fma_f32 v121, s75, v121, v153\n\t"
      "v_fma_f32 v122, s75, v122, v154\n\t"
      "v_fma_f32 v123, s75, v123, v155\n\t"
      "v_maximum3_f32 v120, 0, v120, 0\n\t"
      "v_maximum3_f32 v121, 0, v121, 0\n\t"
      "v_maximum3_f32 v122, 0, v122, 0\n\t"
      "v_maximum3_f32 v123, 0, v123, 0\n\t"
      "ds_write_b128 v236, v[120:123] offset:0\n\t"
      "v_fma_f32 v124, s75, v124, v156\n\t"
      "v_fma_f32 v125, s75, v125, v157\n\t"
      "v_fma_f32 v126, s75, v126, v158\n\t"
      "v_fma_f32 v127, s75, v127, v159\n\t"
      "v_maximum3_f32 v124, 0, v124, 0\n\t"
      "v_maximum3_f32 v125, 0, v125, 0\n\t"
      "v_maximum3_f32 v126, 0, v126, 0\n\t"
      "v_maximum3_f32 v127, 0, v127, 0\n\t"
      "ds_write_b128 v237, v[124:127] offset:0\n\t"
      "ds_read_b128 v[168:171], v209 offset:0\n\t"
      "ds_read_b128 v[172:175], v209 offset:1024\n\t"
      "v_fma_f32 v128, s75, v128, v160\n\t"
      "v_fma_f32 v129, s75, v129, v161\n\t"
      "v_fma_f32 v130, s75, v130, v162\n\t"
      "v_fma_f32 v131, s75, v131, v163\n\t"
      "v_maximum3_f32 v128, 0, v128, 0\n\t"
      "v_maximum3_f32 v129, 0, v129, 0\n\t"
      "v_maximum3_f32 v130, 0, v130, 0\n\t"
      "v_maximum3_f32 v131, 0, v131, 0\n\t"
      "ds_write_b128 v236, v[128:131] offset:2048\n\t"
      "v_fma_f32 v132, s75, v132, v164\n\t"
      "v_fma_f32 v133, s75, v133, v165\n\t"
      "v_fma_f32 v134, s75, v134, v166\n\t"
      "v_fma_f32 v135, s75, v135, v167\n\t"
      "v_maximum3_f32 v132, 0, v132, 0\n\t"
      "v_maximum3_f32 v133, 0, v133, 0\n\t"
      "v_maximum3_f32 v134, 0, v134, 0\n\t"
      "v_maximum3_f32 v135, 0, v135, 0\n\t"
      "ds_write_b128 v237, v[132:135] offset:2048\n\t"
      "ds_read_b128 v[176:179], v209 offset:2048\n\t"
      "ds_read_b128 v[180:183], v209 offset:3072\n\t"
      "s_waitcnt lgkmcnt(4)\n\t"
      "global_store_dwordx4 v210, v[168:171], s[22:23] sc0 sc1\n\t"
      "global_store_dwordx4 v211, v[172:175], s[22:23] sc0 sc1\n\t"
      "s_add_u32 s22, s22, 0x40000\n\t"
      "s_addc_u32 s23, s23, 0\n\t"
      "s_waitcnt lgkmcnt(0)\n\t"
      "global_store_dwordx4 v210, v[176:179], s[22:23] sc0 sc1\n\t"
      "global_store_dwordx4 v211, v[180:183], s[22:23] sc0 sc1\n\t"
      "s_branch FINISH_%=\n\t"
      "FINISH_%=:\n\t"
      "s_cmp_eq_u32 s65, 0\n\t"
      "s_cbranch_scc1 Lq493_%=\n\t"
      "s_call_b64 s[88:89], PUSH_%=\n\t"
      "Lq493_%=:\n\t"
      "v_mov_b32 v205, s64\n\t"
      "global_store_dword v212, v205, s[16:17] offset:-128 sc0 sc1\n\t"
      "s_branch END_%=\n\t"
      "LOADER_%=:\n\t"
      "s_lshl_b32 s80, s73, 7\n\t"
      "s_lshl_b32 s81, s68, 18\n\t"
      "s_or_b32 s80, s80, s81\n\t"
      "v_lshl_or_b32 v224, v209, 12, v211\n\t"
      "v_or_b32 v224, s80, v224\n\t"
      "v_add_u32 v225, 31744, v224\n\t"
      "v_xor_b32 v227, 64, v224\n\t"
      "v_add_u32 v226, 63488, v227\n\t"
      "v_add_u32 v227, 95232, v227\n\t"
      "s_lshl_b32 s78, s68, 13\n\t"
      "s_lshl_b32 s77, s68, 12\n\t"
      "s_lshl_b32 s81, s68, 14\n\t"
      "s_mov_b32 s38, 16777216\n\t"
      "s_mov_b32 s39, 0x20000\n\t"
      "s_mov_b32 s42, 2097152\n\t"
      "s_mov_b32 s43, 0x20000\n\t"
      "s_mov_b32 s65, 0\n\t"
      "s_waitcnt lgkmcnt(0)\n\t"
      "s_mov_b64 s[36:37], s[20:21]\n\t"
      "s_mov_b64 s[40:41], s[18:19]\n\t"
      "s_or_b32 s35, s49, s81\n\t"
      "s_xor_b32 s34, s35, 32768\n\t"
      "s_mov_b32 s67, s86\n\t"
      "global_load_dword v207, v208, s[16:17] sc0 sc1\n\t"
      "s_barrier\n\t"
      "s_add_u32 m0, s78, 0\n\t"
      "s_add_u32 s80, s56, 0\n\t"
      "buffer_load_dwordx4 v224, s[36:39], s80 offen offset:0 lds\n\t"
      "buffer_load_dwordx4 v225, s[36:39], s80 offen offset:1024 lds\n\t"
      "buffer_load_dwordx4 v226, s[36:39], s80 offen offset:2048 lds\n\t"
      "buffer_load_dwordx4 v227, s[36:39], s80 offen offset:3072 lds\n\t"
      "s_add_u32 m0, s78, 4096\n\t"
      "s_add_u32 s80, s56, 131072\n\t"
      "buffer_load_dwordx4 v224, s[36:39], s80 offen offset:0 lds\n\t"
      "buffer_load_dwordx4 v225, s[36:39], s80 offen offset:1024 lds\n\t"
      "buffer_load_dwordx4 v226, s[36:39], s80 offen offset:2048 lds\n\t"
      "buffer_load_dwordx4 v227, s[36:39], s80 offen offset:3072 lds\n\t"
      "s_add_u32 m0, s77, 147456\n\t"
      "s_add_u32 s80, s34, 0\n\t"
      "buffer_load_dwordx4 v201, s[40:43], s80 offen lds\n\t"
      "buffer_load_dwordx4 v202, s[40:43], s80 offen offset:1024 lds\n\t"
      "s_add_u32 m0, s77, 149504\n\t"
      "s_add_u32 s80, s34, 8192\n\t"
      "buffer_load_dwordx4 v201, s[40:43], s80 offen lds\n\t"
      "buffer_load_dwordx4 v202, s[40:43], s80 offen offset:1024 lds\n\t"
      "s_load_dword s94, s[24:25], 0x0\n\t"
      "s_load_dword s95, s[26:27], 0x0\n\t"
      "v_lshlrev_b32 v203, 2, v255\n\t"
      "v_add_u32 v203, 155648, v203\n\t"
      "v_mov_b32 v204, 1\n\t"
      "s_mov_b32 s84, 0\n\t"
      "v_and_b32 v216, 15, v255\n\t"
      "v_lshrrev_b32 v211, 1, v216\n\t"
      "v_lshl_add_u32 v244, v218, 4, v211\n\t"
      "v_lshlrev_b32 v244, 7, v244\n\t"
      "v_xor_b32 v238, v211, v218\n\t"
      "v_lshlrev_b32 v238, 4, v238\n\t"
      "v_lshl_or_b32 v238, v216, 7, v238\n\t"
      "v_xor_b32 v239, 64, v238\n\t"
      "v_lshrrev_b32 v211, 2, v216\n\t"
      "v_and_b32 v205, 1, v218\n\t"
      "v_lshl_or_b32 v211, v205, 2, v211\n\t"
      "s_lshl_b32 s81, s68, 1\n\t"
      "v_xor_b32 v211, s81, v211\n\t"
      "v_lshl_or_b32 v244, v211, 4, v244\n\t"
      "v_and_b32 v211, 1, v216\n\t"
      "v_lshl_or_b32 v244, v211, 3, v244\n\t"
      "v_xor_b32 v245, 16, v244\n\t"
      "s_lshl_b32 s52, s68, 11\n\t"
      "v_lshl_add_u32 v250, v255, 4, s52\n\t"
      "v_add_u32 v246, 65536, v244\n\t"
      "v_add_u32 v240, 65536, v238\n\t"
      "v_add_u32 v247, 65536, v245\n\t"
      "v_add_u32 v241, 65536, v239\n\t"
      "v_add_u32 v251, 65536, v250\n\t"
      "v_add_u32 v248, 131072, v244\n\t"
      "v_add_u32 v242, 131072, v238\n\t"
      "v_add_u32 v249, 131072, v245\n\t"
      "v_add_u32 v243, 131072, v239\n\t"
      "v_add_u32 v252, 131072, v250\n\t"
      "s_lshl_b32 s28, s68, 12\n\t"
      "v_and_b32 v211, 7, v216\n\t"
      "v_xor_b32 v211, v211, v218\n\t"
      "v_lshlrev_b32 v211, 4, v211\n\t"
      "v_lshl_or_b32 v211, v216, 7, v211\n\t"
      "v_add_u32 v236, s28, v211\n\t"
      "v_xor_b32 v237, 64, v236\n\t"
      "v_xor_b32 v211, v210, v209\n\t"
      "v_lshlrev_b32 v211, 4, v211\n\t"
      "v_lshl_or_b32 v211, v209, 7, v211\n\t"
      "v_lshlrev_b32 v210, 4, v210\n\t"
      "v_lshl_or_b32 v210, v209, 14, v210\n\t"
      "v_add_u32 v209, s28, v211\n\t"
      "v_add_u32 v211, 0x20000, v210\n\t"
      "s_lshl_b32 s28, s76, 9\n\t"
      "s_add_u32 s81, s28, 6291584\n\t"
      "v_mov_b32 v212, s81\n\t"
      "s_xor_b32 s28, s28, 262144\n\t"
      "s_add_u32 s81, s28, 6291584\n\t"
      "v_mov_b32 v213, s81\n\t"
      "s_lshl_b32 s28, s70, 20\n\t"
      "s_lshl_b32 s81, s73, 9\n\t"
      "s_or_b32 s28, s28, s81\n\t"
      "s_lshl_b32 s81, s68, 7\n\t"
      "s_or_b32 s28, s28, s81\n\t"
      "s_xor_b32 s81, s28, 0x100000\n\t"
      "s_add_u32 s30, s22, s81\n\t"
      "s_addc_u32 s31, s23, 0\n\t"
      "s_add_u32 s22, s22, s28\n\t"
      "s_addc_u32 s23, s23, 0\n\t"
      "s_barrier\n\t"
      "s_add_u32 m0, s77, 139264\n\t"
      "s_add_u32 s80, s35, 0\n\t"
      "buffer_load_dwordx4 v201, s[40:43], s80 offen lds\n\t"
      "buffer_load_dwordx4 v202, s[40:43], s80 offen offset:1024 lds\n\t"
      "s_add_u32 m0, s77, 141312\n\t"
      "s_add_u32 s80, s35, 8192\n\t"
      "buffer_load_dwordx4 v201, s[40:43], s80 offen lds\n\t"
      "buffer_load_dwordx4 v202, s[40:43], s80 offen offset:1024 lds\n\t"
      "s_add_u32 m0, s78, 16384\n\t"
      "s_add_u32 s80, s57, 0\n\t"
      "buffer_load_dwordx4 v224, s[36:39], s80 offen offset:0 lds\n\t"
      "buffer_load_dwordx4 v225, s[36:39], s80 offen offset:1024 lds\n\t"
      "buffer_load_dwordx4 v226, s[36:39], s80 offen offset:2048 lds\n\t"
      "buffer_load_dwordx4 v227, s[36:39], s80 offen offset:3072 lds\n\t"
      "s_add_u32 m0, s78, 20480\n\t"
      "s_add_u32 s80, s57, 131072\n\t"
      "buffer_load_dwordx4 v224, s[36:39], s80 offen offset:0 lds\n\t"
      "buffer_load_dwordx4 v225, s[36:39], s80 offen offset:1024 lds\n\t"
      "buffer_load_dwordx4 v226, s[36:39], s80 offen offset:2048 lds\n\t"
      "buffer_load_dwordx4 v227, s[36:39], s80 offen offset:3072 lds\n\t"
      "s_waitcnt vmcnt(24)\n\t"
      "v_readfirstlane_b32 s64, v207\n\t"
      "s_add_u32 s64, s64, 1\n\t"
      "s_mul_hi_u32 s74, s64, 0xaaaaaaab\n\t"
      "s_lshr_b32 s74, s74, 1\n\t"
      "s_mul_i32 s74, s74, 3\n\t"
      "s_sub_u32 s74, s64, s74\n\t"
      "s_lshl_b32 s82, s74, 21\n\t"
      "s_waitcnt lgkmcnt(0)\n\t"
      "s_lshl_b32 s81, s70, 15\n\t"
      "s_add_u32 s82, s82, s81\n\t"
      "s_add_u32 s82, s82, s52\n\t"
      "s_add_u32 s44, s16, s82\n\t"
      "s_addc_u32 s45, s17, 0\n\t"
      "s_bfe_u32 s18, s48, 0x30000\n\t"
      "s_lshl_b32 s18, s18, 18\n\t"
      "s_bfe_u32 s19, s48, 0x30004\n\t"
      "s_lshl_b32 s19, s19, 18\n\t"
      "s_bfe_u32 s20, s48, 0x30008\n\t"
      "s_lshl_b32 s20, s20, 18\n\t"
      "s_bfe_u32 s21, s48, 0x3000c\n\t"
      "s_lshl_b32 s21, s21, 18\n\t"
      "s_bfe_u32 s24, s48, 0x30010\n\t"
      "s_lshl_b32 s24, s24, 18\n\t"
      "s_bfe_u32 s25, s48, 0x30014\n\t"
      "s_lshl_b32 s25, s25, 18\n\t"
      "s_bfe_u32 s26, s48, 0x30018\n\t"
      "s_lshl_b32 s26, s26, 18\n\t"
      "s_mov_b32 s46, 2097152\n\t"
      "s_mov_b32 s47, 0x20000\n\t"
      "v_mov_b32 v205, s94\n\t"
      "v_mul_f32 v205, s95, v205\n\t"
      "s_mov_b64 s[94:95], s[22:23]\n\t"
      "v_readfirstlane_b32 s75, v205\n\t"
      "s_add_u32 m0, s78, 32768\n\t"
      "s_add_u32 s80, s56, 524288\n\t"
      "buffer_load_dwordx4 v224, s[36:39], s80 offen offset:0 lds\n\t"
      "buffer_load_dwordx4 v225, s[36:39], s80 offen offset:1024 lds\n\t"
      "buffer_load_dwordx4 v226, s[36:39], s80 offen offset:2048 lds\n\t"
      "buffer_load_dwordx4 v227, s[36:39], s80 offen offset:3072 lds\n\t"
      "s_add_u32 m0, s78, 36864\n\t"
      "s_add_u32 s80, s56, 655360\n\t"
      "buffer_load_dwordx4 v224, s[36:39], s80 offen offset:0 lds\n\t"
      "buffer_load_dwordx4 v225, s[36:39], s80 offen offset:1024 lds\n\t"
      "buffer_load_dwordx4 v226, s[36:39], s80 offen offset:2048 lds\n\t"
      "buffer_load_dwordx4 v227, s[36:39], s80 offen offset:3072 lds\n\t"
      "s_waitcnt vmcnt(16)\n\t"
      "s_add_u32 m0, s77, 131072\n\t"
      "s_add_u32 s80, s34, 128\n\t"
      "buffer_load_dwordx4 v201, s[40:43], s80 offen lds\n\t"
      "buffer_load_dwordx4 v202, s[40:43], s80 offen offset:1024 lds\n\t"
      "s_add_u32 m0, s77, 133120\n\t"
      "s_add_u32 s80, s34, 8320\n\t"
      "buffer_load_dwordx4 v201, s[40:43], s80 offen lds\n\t"
      "buffer_load_dwordx4 v202, s[40:43], s80 offen offset:1024 lds\n\t"
      "ds_add_u32 v203, v204 offset:256\n\t"
      "s_add_u32 m0, s77, 122880\n\t"
      "s_add_u32 s80, s35, 128\n\t"
      "buffer_load_dwordx4 v201, s[40:43], s80 offen lds\n\t"
      "buffer_load_dwordx4 v202, s[40:43], s80 offen offset:1024 lds\n\t"
      "s_add_u32 m0, s77, 124928\n\t"
      "s_add_u32 s80, s35, 8320\n\t"
      "buffer_load_dwordx4 v201, s[40:43], s80 offen lds\n\t"
      "buffer_load_dwordx4 v202, s[40:43], s80 offen offset:1024 lds\n\t"
      "ds_read_b32 v253, v203 offset:256\n\t"
      "s_add_u32 m0, s78, 49152\n\t"
      "s_add_u32 s80, s58, 0\n\t"
      "buffer_load_dwordx4 v224, s[36:39], s80 offen offset:0 lds\n\t"
      "buffer_load_dwordx4 v225, s[36:39], s80 offen offset:1024 lds\n\t"
      "s_waitcnt lgkmcnt(0)\n\t"
      "v_cmp_gt_u32 vcc, 4, v253\n\t"
      "buffer_load_dwordx4 v226, s[36:39], s80 offen offset:2048 lds\n\t"
      "buffer_load_dwordx4 v227, s[36:39], s80 offen offset:3072 lds\n\t"
      "s_cbranch_vccnz AQ1_%=\n\t"
      "AR2_%=:\n\t"
      "ds_read_b128 v[32:35], v242 offset:16384\n\t"
      "s_add_u32 m0, s78, 53248\n\t"
      "s_add_u32 s80, s58, 131072\n\t"
      "buffer_load_dwordx4 v224, s[36:39], s80 offen offset:0 lds\n\t"
      "buffer_load_dwordx4 v225, s[36:39], s80 offen offset:1024 lds\n\t"
      "ds_read_b128 v[36:39], v243 offset:16384\n\t"
      "buffer_load_dwordx4 v226, s[36:39], s80 offen offset:2048 lds\n\t"
      "buffer_load_dwordx4 v227, s[36:39], s80 offen offset:3072 lds\n\t"
      "ds_read_b64_tr_b8 a[0:1], v244 offset:0\n\t"
      "s_waitcnt vmcnt(12)\n\t"
      "ds_read_b128 v[40:43], v242 offset:18432\n\t"
      "s_add_u32 m0, s78, 65536\n\t"
      "s_add_u32 s80, s59, 0\n\t"
      "buffer_load_dwordx4 v224, s[36:39], s80 offen offset:0 lds\n\t"
      "buffer_load_dwordx4 v225, s[36:39], s80 offen offset:1024 lds\n\t"
      "ds_read_b64_tr_b8 a[2:3], v244 offset:1024\n\t"
      "ds_read_b128 v[44:47], v243 offset:18432\n\t"
      "buffer_load_dwordx4 v226, s[36:39], s80 offen offset:2048 lds\n\t"
      "buffer_load_dwordx4 v227, s[36:39], s80 offen offset:3072 lds\n\t"
      "ds_read_b64_tr_b8 a[4:5], v244 offset:8192\n\t"
      "ds_read_b64_tr_b8 a[6:7], v244 offset:9216\n\t"
      "ds_read_b64_tr_b8 a[8:9], v245 offset:0\n\t"
      "ds_read_b64_tr_b8 a[10:11], v245 offset:1024\n\t"
      "ds_read_b64_tr_b8 a[12:13], v245 offset:8192\n\t"
      "ds_read_b64_tr_b8 a[14:15], v245 offset:9216\n\t"
      "s_waitcnt lgkmcnt(0)\n\t"
      "ds_add_u32 v203, v204 offset:0\n\t"
      "ds_read_b128 v[48:51], v242 offset:20480\n\t"
      "s_add_u32 m0, s78, 69632\n\t"
      "s_add_u32 s80, s59, 131072\n\t"
      "buffer_load_dwordx4 v224, s[36:39], s80 offen offset:0 lds\n\t"
      "buffer_load_dwordx4 v225, s[36:39], s80 offen offset:1024 lds\n\t"
      "ds_read_b128 v[52:55], v243 offset:20480\n\t"
      "buffer_load_dwordx4 v226, s[36:39], s80 offen offset:2048 lds\n\t"
      "buffer_load_dwordx4 v227, s[36:39], s80 offen offset:3072 lds\n\t"
      "v_mfma_f32_16x16x128_f8f6f4 v[72:75], a[0:7], v[32:39], 0 cbsz:1 blgp:1\n\t"
      "ds_read_b32 v253, v203 offset:0\n\t"
      "ds_read_b128 v[56:59], v242 offset:22528\n\t"
      "s_add_u32 m0, s78, 81920\n\t"
      "s_add_u32 s80, s60, 0\n\t"
      "buffer_load_dwordx4 v224, s[36:39], s80 offen offset:0 lds\n\t"
      "buffer_load_dwordx4 v225, s[36:39], s80 offen offset:1024 lds\n\t"
      "v_mfma_f32_16x16x128_f8f6f4 v[76:79], a[8:15], v[32:39], 0 cbsz:1 blgp:1\n\t"
      "s_waitcnt lgkmcnt(1)\n\t"
      "v_cmp_gt_u32 vcc, 4, v253\n\t"
      "ds_read_b128 v[60:63], v243 offset:22528\n\t"
      "buffer_load_dwordx4 v226, s[36:39], s80 offen offset:2048 lds\n\t"
      "buffer_load_dwordx4 v227, s[36:39], s80 offen offset:3072 lds\n\t"
      "v_mfma_f32_16x16x128_f8f6f4 v[80:83], a[0:7], v[40:47], 0 cbsz:1 blgp:1\n\t"
      "s_cbranch_vccnz AQ3_%=\n\t"
      "AR4_%=:\n\t"
      "ds_read_b128 v[64:67], v242 offset:8192\n\t"
      "s_add_u32 m0, s78, 86016\n\t"
      "s_add_u32 s80, s60, 131072\n\t"
      "buffer_load_dwordx4 v224, s[36:39], s80 offen offset:0 lds\n\t"
      "buffer_load_dwordx4 v225, s[36:39], s80 offen offset:1024 lds\n\t"
      "v_mfma_f32_16x16x128_f8f6f4 v[84:87], a[8:15], v[40:47], 0 cbsz:1 blgp:1\n\t"
      "ds_read_b128 v[68:71], v243 offset:8192\n\t"
      "buffer_load_dwordx4 v226, s[36:39], s80 offen offset:2048 lds\n\t"
      "buffer_load_dwordx4 v227, s[36:39], s80 offen offset:3072 lds\n\t"
      "ds_read_b64_tr_b8 a[16:17], v244 offset:32768\n\t"
      "ds_read_b64_tr_b8 a[18:19], v244 offset:33792\n\t"
      "ds_read_b64_tr_b8 a[20:21], v244 offset:40960\n\t"
      "ds_read_b64_tr_b8 a[22:23], v244 offset:41984\n\t"
      "v_mfma_f32_16x16x128_f8f6f4 v[88:91], a[0:7], v[48:55], 0 cbsz:1 blgp:1\n\t"
      "s_waitcnt vmcnt(16)\n\t"
      "ds_read_b128 v[32:35], v242 offset:10240\n\t"
      "s_add_u32 m0, s78, 0\n\t"
      "s_add_u32 s80, s61, 0\n\t"
      "buffer_load_dwordx4 v224, s[36:39], s80 offen offset:0 lds\n\t"
      "buffer_load_dwordx4 v225, s[36:39], s80 offen offset:1024 lds\n\t"
      "ds_read_b64_tr_b8 a[24:25], v245 offset:32768\n\t"
      "ds_read_b64_tr_b8 a[26:27], v245 offset:33792\n\t"
      "ds_read_b64_tr_b8 a[28:29], v245 offset:40960\n\t"
      "ds_read_b64_tr_b8 a[30:31], v245 offset:41984\n\t"
      "ds_read_b64_tr_b8 a[32:33], v244 offset:16384\n\t"
      "ds_read_b64_tr_b8 a[34:35], v244 offset:17408\n\t"
      "v_mfma_f32_16x16x128_f8f6f4 v[92:95], a[8:15], v[48:55], 0 cbsz:1 blgp:1\n\t"
      "ds_read_b128 v[36:39], v243 offset:10240\n\t"
      "buffer_load_dwordx4 v226, s[36:39], s80 offen offset:2048 lds\n\t"
      "buffer_load_dwordx4 v227, s[36:39], s80 offen offset:3072 lds\n\t"
      "ds_read_b64_tr_b8 a[36:37], v244 offset:24576\n\t"
      "ds_read_b64_tr_b8 a[38:39], v244 offset:25600\n\t"
      "ds_read_b64_tr_b8 a[40:41], v245 offset:16384\n\t"
      "ds_read_b64_tr_b8 a[42:43], v245 offset:17408\n\t"
      "ds_read_b64_tr_b8 a[44:45], v245 offset:24576\n\t"
      "ds_read_b64_tr_b8 a[46:47], v245 offset:25600\n\t"
      "s_waitcnt lgkmcnt(15)\n\t"
      "v_mfma_f32_16x16x128_f8f6f4 v[96:99], a[0:7], v[56:63], 0 cbsz:1 blgp:1\n\t"
      "s_waitcnt lgkmcnt(0)\n\t"
      "ds_add_u32 v203, v204 offset:256\n\t"
      "ds_read_b128 v[40:43], v242 offset:12288\n\t"
      "s_add_u32 m0, s78, 4096\n\t"
      "s_add_u32 s80, s61, 131072\n\t"
      "buffer_load_dwordx4 v224, s[36:39], s80 offen offset:0 lds\n\t"
      "buffer_load_dwordx4 v225, s[36:39], s80 offen offset:1024 lds\n\t"
      "v_mfma_f32_16x16x128_f8f6f4 v[100:103], a[8:15], v[56:63], 0 cbsz:1 blgp:1\n\t"
      "ds_read_b128 v[44:47], v243 offset:12288\n\t"
      "buffer_load_dwordx4 v226, s[36:39], s80 offen offset:2048 lds\n\t"
      "buffer_load_dwordx4 v227, s[36:39], s80 offen offset:3072 lds\n\t"
      "v_mfma_f32_16x16x128_f8f6f4 v[104:107], a[0:7], v[64:71], 0 cbsz:1 blgp:1\n\t"
      "ds_read_b32 v253, v203 offset:256\n\t"
      "ds_read_b128 v[48:51], v242 offset:14336\n\t"
      "s_add_u32 m0, s78, 98304\n\t"
      "s_add_u32 s80, s62, 0\n\t"
      "buffer_load_dwordx4 v224, s[36:39], s80 offen offset:0 lds\n\t"
      "buffer_load_dwordx4 v225, s[36:39], s80 offen offset:1024 lds\n\t"
      "v_mfma_f32_16x16x128_f8f6f4 v[108:111], a[8:15], v[64:71], 0 cbsz:1 blgp:1\n\t"
      "s_waitcnt lgkmcnt(1)\n\t"
      "v_cmp_gt_u32 vcc, 8, v253\n\t"
      "ds_read_b128 v[52:55], v243 offset:14336\n\t"
      "buffer_load_dwordx4 v226, s[36:39], s80 offen offset:2048 lds\n\t"
      "buffer_load_dwordx4 v227, s[36:39], s80 offen offset:3072 lds\n\t"
      "v_mfma_f32_16x16x128_f8f6f4 v[112:115], a[0:7], v[32:39], 0 cbsz:1 blgp:1\n\t"
      "s_cbranch_vccnz AQ5_%=\n\t"
      "AR6_%=:\n\t"
      "ds_read_b128 v[56:59], v242 offset:0\n\t"
      "s_add_u32 m0, s78, 102400\n\t"
      "s_add_u32 s80, s62, 131072\n\t"
      "buffer_load_dwordx4 v224, s[36:39], s80 offen offset:0 lds\n\t"
      "buffer_load_dwordx4 v225, s[36:39], s80 offen offset:1024 lds\n\t"
      "v_mfma_f32_16x16x128_f8f6f4 v[116:119], a[8:15], v[32:39], 0 cbsz:1 blgp:1\n\t"
      "ds_read_b128 v[60:63], v243 offset:0\n\t"
      "buffer_load_dwordx4 v226, s[36:39], s80 offen offset:2048 lds\n\t"
      "buffer_load_dwordx4 v227, s[36:39], s80 offen offset:3072 lds\n\t"
      "ds_read_b64_tr_b8 a[48:49], v244 offset:49152\n\t"
      "v_mfma_f32_16x16x128_f8f6f4 v[120:123], a[0:7], v[40:47], 0 cbsz:1 blgp:1\n\t"
      "s_waitcnt vmcnt(16)\n\t"
      "ds_read_b128 v[64:67], v242 offset:2048\n\t"
      "s_add_u32 m0, s78, 16384\n\t"
      "s_add_u32 s80, s63, 0\n\t"
      "buffer_load_dwordx4 v224, s[36:39], s80 offen offset:0 lds\n\t"
      "buffer_load_dwordx4 v225, s[36:39], s80 offen offset:1024 lds\n\t"
      "ds_read_b64_tr_b8 a[50:51], v244 offset:50176\n\t"
      "v_mfma_f32_16x16x128_f8f6f4 v[124:127], a[8:15], v[40:47], 0 cbsz:1 blgp:1\n\t"
      "ds_read_b128 v[68:71], v243 offset:2048\n\t"
      "buffer_load_dwordx4 v226, s[36:39], s80 offen offset:2048 lds\n\t"
      "buffer_load_dwordx4 v227, s[36:39], s80 offen offset:3072 lds\n\t"
      "ds_read_b64_tr_b8 a[52:53], v244 offset:57344\n\t"
      "ds_read_b64_tr_b8 a[54:55], v244 offset:58368\n\t"
      "ds_read_b64_tr_b8 a[56:57], v245 offset:49152\n\t"
      "ds_read_b64_tr_b8 a[58:59], v245 offset:50176\n\t"
      "ds_read_b64_tr_b8 a[60:61], v245 offset:57344\n\t"
      "ds_read_b64_tr_b8 a[62:63], v245 offset:58368\n\t"
      "s_waitcnt lgkmcnt(12)\n\t"
      "v_mfma_f32_16x16x128_f8f6f4 v[128:131], a[0:7], v[48:55], 0 cbsz:1 blgp:1\n\t"
      "s_waitcnt lgkmcnt(0)\n\t"
      "ds_add_u32 v203, v204 offset:0\n\t"
      "ds_read_b128 v[32:35], v242 offset:4096\n\t"
      "s_add_u32 m0, s78, 20480\n\t"
      "s_add_u32 s80, s63, 131072\n\t"
      "buffer_load_dwordx4 v224, s[36:39], s80 offen offset:0 lds\n\t"
      "buffer_load_dwordx4 v225, s[36:39], s80 offen offset:1024 lds\n\t"
      "v_mfma_f32_16x16x128_f8f6f4 v[132:135], a[8:15], v[48:55], 0 cbsz:1 blgp:1\n\t"
      "ds_read_b128 v[36:39], v243 offset:4096\n\t"
      "buffer_load_dwordx4 v226, s[36:39], s80 offen offset:2048 lds\n\t"
      "buffer_load_dwordx4 v227, s[36:39], s80 offen offset:3072 lds\n\t"
      "v_mfma_f32_16x16x128_f8f6f4 v[72:75], a[16:23], v[56:63], v[72:75] cbsz:1 blgp:1\n\t"
      "ds_read_b32 v253, v203 offset:0\n\t"
      "ds_read_b128 v[40:43], v242 offset:6144\n\t"
      "s_add_u32 m0, s78, 32768\n\t"
      "s_add_u32 s80, s57, 524288\n\t"
      "buffer_load_dwordx4 v224, s[36:39], s80 offen offset:0 lds\n\t"
      "buffer_load_dwordx4 v225, s[36:39], s80 offen offset:1024 lds\n\t"
      "v_mfma_f32_16x16x128_f8f6f4 v[76:79], a[24:31], v[56:63], v[76:79] cbsz:1 blgp:1\n\t"
      "s_waitcnt lgkmcnt(1)\n\t"
      "v_cmp_gt_u32 vcc, 8, v253\n\t"
      "ds_read_b128 v[44:47], v243 offset:6144\n\t"
      "buffer_load_dwordx4 v226, s[36:39], s80 offen offset:2048 lds\n\t"
      "buffer_load_dwordx4 v227, s[36:39], s80 offen offset:3072 lds\n\t"
      "v_mfma_f32_16x16x128_f8f6f4 v[80:83], a[16:23], v[64:71], v[80:83] cbsz:1 blgp:1\n\t"
      "s_cbranch_vccnz AQ7_%=\n\t"
      "AR8_%=:\n\t"
      "ds_read_b128 v[48:51], v240 offset:57344\n\t"
      "s_add_u32 m0, s78, 36864\n\t"
      "s_add_u32 s80, s57, 655360\n\t"
      "buffer_load_dwordx4 v224, s[36:39], s80 offen offset:0 lds\n\t"
      "buffer_load_dwordx4 v225, s[36:39], s80 offen offset:1024 lds\n\t"
      "v_mfma_f32_16x16x128_f8f6f4 v[84:87], a[24:31], v[64:71], v[84:87] cbsz:1 blgp:1\n\t"
      "ds_read_b128 v[52:55], v241 offset:57344\n\t"
      "buffer_load_dwordx4 v226, s[36:39], s80 offen offset:2048 lds\n\t"
      "buffer_load_dwordx4 v227, s[36:39], s80 offen offset:3072 lds\n\t"
      "ds_read_b64_tr_b8 a[64:65], v246 offset:0\n\t"
      "ds_read_b64_tr_b8 a[66:67], v246 offset:1024\n\t"
      "ds_read_b64_tr_b8 a[68:69], v246 offset:8192\n\t"
      "ds_read_b64_tr_b8 a[70:71], v246 offset:9216\n\t"
      "v_mfma_f32_16x16x128_f8f6f4 v[88:91], a[16:23], v[32:39], v[88:91] cbsz:1 blgp:1\n\t"
      "s_waitcnt vmcnt(16)\n\t"
      "ds_read_b128 v[56:59], v240 offset:59392\n\t"
      "s_add_u32 m0, s78, 49152\n\t"
      "s_add_u32 s80, s58, 524288\n\t"
      "buffer_load_dwordx4 v224, s[36:39], s80 offen offset:0 lds\n\t"
      "buffer_load_dwordx4 v225, s[36:39], s80 offen offset:1024 lds\n\t"
      "ds_read_b64_tr_b8 a[72:73], v247 offset:0\n\t"
      "ds_read_b64_tr_b8 a[74:75], v247 offset:1024\n\t"
      "ds_read_b64_tr_b8 a[76:77], v247 offset:8192\n\t"
      "ds_read_b64_tr_b8 a[78:79], v247 offset:9216\n\t"
      "ds_read_b64_tr_b8 a[80:81], v246 offset:16384\n\t"
      "ds_read_b64_tr_b8 a[82:83], v246 offset:17408\n\t"
      "v_mfma_f32_16x16x128_f8f6f4 v[92:95], a[24:31], v[32:39], v[92:95] cbsz:1 blgp:1\n\t"
      "ds_read_b128 v[60:63], v241 offset:59392\n\t"
      "buffer_load_dwordx4 v226, s[36:39], s80 offen offset:2048 lds\n\t"
      "buffer_load_dwordx4 v227, s[36:39], s80 offen offset:3072 lds\n\t"
      "ds_read_b64_tr_b8 a[84:85], v246 offset:24576\n\t"
      "ds_read_b64_tr_b8 a[86:87], v246 offset:25600\n\t"
      "ds_read_b64_tr_b8 a[88:89], v247 offset:16384\n\t"
      "ds_read_b64_tr_b8 a[90:91], v247 offset:17408\n\t"
      "ds_read_b64_tr_b8 a[92:93], v247 offset:24576\n\t"
      "ds_read_b64_tr_b8 a[94:95], v247 offset:25600\n\t"
      "s_waitcnt lgkmcnt(15)\n\t"
      "v_mfma_f32_16x16x128_f8f6f4 v[96:99], a[16:23], v[40:47], v[96:99] cbsz:1 blgp:1\n\t"
      "s_waitcnt lgkmcnt(0)\n\t"
      "ds_add_u32 v203, v204 offset:256\n\t"
      "ds_read_b128 v[64:67], v240 offset:61440\n\t"
      "s_add_u32 m0, s78, 53248\n\t"
      "s_add_u32 s80, s58, 655360\n\t"
      "buffer_load_dwordx4 v224, s[36:39], s80 offen offset:0 lds\n\t"
      "buffer_load_dwordx4 v225, s[36:39], s80 offen offset:1024 lds\n\t"
      "v_mfma_f32_16x16x128_f8f6f4 v[100:103], a[24:31], v[40:47], v[100:103] cbsz:1 blgp:1\n\t"
      "ds_read_b128 v[68:71], v241 offset:61440\n\t"
      "buffer_load_dwordx4 v226, s[36:39], s80 offen offset:2048 lds\n\t"
      "buffer_load_dwordx4 v227, s[36:39], s80 offen offset:3072 lds\n\t"
      "v_mfma_f32_16x16x128_f8f6f4 v[104:107], a[16:23], v[48:55], v[104:107] cbsz:1 blgp:1\n\t"
      "ds_read_b32 v253, v203 offset:256\n\t"
      "ds_read_b128 v[32:35], v240 offset:63488\n\t"
      "v_mfma_f32_16x16x128_f8f6f4 v[108:111], a[24:31], v[48:55], v[108:111] cbsz:1 blgp:1\n\t"
      "s_waitcnt lgkmcnt(1)\n\t"
      "v_cmp_gt_u32 vcc, 12, v253\n\t"
      "ds_read_b128 v[36:39], v241 offset:63488\n\t"
      "v_mfma_f32_16x16x128_f8f6f4 v[112:115], a[16:23], v[56:63], v[112:115] cbsz:1 blgp:1\n\t"
      "s_cbranch_vccnz AQ9_%=\n\t"
      "AR10_%=:\n\t"
      "s_add_u32 m0, s78, 65536\n\t"
      "s_add_u32 s80, s59, 524288\n\t"
      "buffer_load_dwordx4 v224, s[36:39], s80 offen offset:0 lds\n\t"
      "buffer_load_dwordx4 v225, s[36:39], s80 offen offset:1024 lds\n\t"
      "buffer_load_dwordx4 v226, s[36:39], s80 offen offset:2048 lds\n\t"
      "v_mfma_f32_16x16x128_f8f6f4 v[116:119], a[24:31], v[56:63], v[116:119] cbsz:1 blgp:1\n\t"
      "buffer_load_dwordx4 v227, s[36:39], s80 offen offset:3072 lds\n\t"
      "s_add_u32 m0, s78, 69632\n\t"
      "s_add_u32 s80, s59, 655360\n\t"
      "buffer_load_dwordx4 v224, s[36:39], s80 offen offset:0 lds\n\t"
      "buffer_load_dwordx4 v225, s[36:39], s80 offen offset:1024 lds\n\t"
      "ds_read_b64_tr_b8 a[96:97], v244 offset:0\n\t"
      "ds_read_b64_tr_b8 a[98:99], v244 offset:1024\n\t"
      "ds_read_b64_tr_b8 a[100:101], v244 offset:8192\n\t"
      "ds_read_b64_tr_b8 a[102:103], v244 offset:9216\n\t"
      "v_mfma_f32_16x16x128_f8f6f4 v[120:123], a[16:23], v[64:71], v[120:123] cbsz:1 blgp:1\n\t"
      "s_waitcnt vmcnt(14)\n\t"
      "buffer_load_dwordx4 v226, s[36:39], s80 offen offset:2048 lds\n\t"
      "buffer_load_dwordx4 v227, s[36:39], s80 offen offset:3072 lds\n\t"
      "s_add_u32 m0, s78, 81920\n\t"
      "s_add_u32 s80, s60, 524288\n\t"
      "buffer_load_dwordx4 v224, s[36:39], s80 offen offset:0 lds\n\t"
      "ds_read_b64_tr_b8 a[104:105], v245 offset:0\n\t"
      "ds_read_b64_tr_b8 a[106:107], v245 offset:1024\n\t"
      "ds_read_b64_tr_b8 a[108:109], v245 offset:8192\n\t"
      "ds_read_b64_tr_b8 a[110:111], v245 offset:9216\n\t"
      "ds_read_b64_tr_b8 a[112:113], v246 offset:32768\n\t"
      "ds_read_b64_tr_b8 a[114:115], v246 offset:33792\n\t"
      "v_mfma_f32_16x16x128_f8f6f4 v[124:127], a[24:31], v[64:71], v[124:127] cbsz:1 blgp:1\n\t"
      "buffer_load_dwordx4 v225, s[36:39], s80 offen offset:1024 lds\n\t"
      "buffer_load_dwordx4 v226, s[36:39], s80 offen offset:2048 lds\n\t"
      "buffer_load_dwordx4 v227, s[36:39], s80 offen offset:3072 lds\n\t"
      "ds_read_b64_tr_b8 a[116:117], v246 offset:40960\n\t"
      "ds_read_b64_tr_b8 a[118:119], v246 offset:41984\n\t"
      "ds_read_b64_tr_b8 a[120:121], v247 offset:32768\n\t"
      "ds_read_b64_tr_b8 a[122:123], v247 offset:33792\n\t"
      "ds_read_b64_tr_b8 a[124:125], v247 offset:40960\n\t"
      "ds_read_b64_tr_b8 a[126:127], v247 offset:41984\n\t"
      "s_waitcnt lgkmcnt(15)\n\t"
      "v_mfma_f32_16x16x128_f8f6f4 v[128:131], a[16:23], v[32:39], v[128:131] cbsz:1 blgp:1\n\t"
      "s_waitcnt lgkmcnt(0)\n\t"
      "ds_add_u32 v203, v204 offset:0\n\t"
      "s_add_u32 m0, s78, 86016\n\t"
      "s_add_u32 s80, s60, 655360\n\t"
      "buffer_load_dwordx4 v224, s[36:39], s80 offen offset:0 lds\n\t"
      "buffer_load_dwordx4 v225, s[36:39], s80 offen offset:1024 lds\n\t"
      "v_mfma_f32_16x16x128_f8f6f4 v[132:135], a[24:31], v[32:39], v[132:135] cbsz:1 blgp:1\n\t"
      "buffer_load_dwordx4 v226, s[36:39], s80 offen offset:2048 lds\n\t"
      "buffer_load_dwordx4 v227, s[36:39], s80 offen offset:3072 lds\n\t"
      "ds_read_b32 v253, v203 offset:0\n\t"
      "s_add_u32 m0, s52, 147456\n\t"
      "s_add_u32 s80, s18, 0\n\t"
      "buffer_load_dwordx4 v200, s[44:47], s80 offen offset:0 sc0 sc1 lds\n\t"
      "buffer_load_dwordx4 v200, s[44:47], s80 offen offset:1024 sc0 sc1 lds\n\t"
      "s_waitcnt lgkmcnt(0)\n\t"
      "v_cmp_gt_u32 vcc, 12, v253\n\t"
      "s_add_u32 m0, s78, 131072\n\t"
      "s_add_u32 s80, s61, 524288\n\t"
      "buffer_load_dwordx4 v224, s[36:39], s80 offen offset:0 lds\n\t"
      "buffer_load_dwordx4 v225, s[36:39], s80 offen offset:1024 lds\n\t"
      "s_cbranch_vccnz AQ11_%=\n\t"
      "AR12_%=:\n\t"
      "buffer_load_dwordx4 v226, s[36:39], s80 offen offset:2048 lds\n\t"
      "buffer_load_dwordx4 v227, s[36:39], s80 offen offset:3072 lds\n\t"
      "ds_read_b64_tr_b8 a[128:129], v244 offset:16384\n\t"
      "ds_read_b64_tr_b8 a[130:131], v244 offset:17408\n\t"
      "s_add_u32 m0, s78, 135168\n\t"
      "s_add_u32 s80, s61, 655360\n\t"
      "buffer_load_dwordx4 v224, s[36:39], s80 offen offset:0 lds\n\t"
      "buffer_load_dwordx4 v225, s[36:39], s80 offen offset:1024 lds\n\t"
      "ds_read_b64_tr_b8 a[132:133], v244 offset:24576\n\t"
      "ds_read_b64_tr_b8 a[134:135], v244 offset:25600\n\t"
      "s_waitcnt vmcnt(16)\n\t"
      "buffer_load_dwordx4 v226, s[36:39], s80 offen offset:2048 lds\n\t"
      "buffer_load_dwordx4 v227, s[36:39], s80 offen offset:3072 lds\n\t"
      "ds_read_b64_tr_b8 a[136:137], v245 offset:16384\n\t"
      "ds_read_b64_tr_b8 a[138:139], v245 offset:17408\n\t"
      "ds_read_b64_tr_b8 a[140:141], v245 offset:24576\n\t"
      "ds_read_b64_tr_b8 a[142:143], v245 offset:25600\n\t"
      "ds_read_b64_tr_b8 a[144:145], v244 offset:32768\n\t"
      "ds_read_b64_tr_b8 a[146:147], v244 offset:33792\n\t"
      "s_add_u32 m0, s52, 122880\n\t"
      "s_add_u32 s80, s19, 0\n\t"
      "buffer_load_dwordx4 v200, s[44:47], s80 offen offset:0 sc0 sc1 lds\n\t"
      "buffer_load_dwordx4 v200, s[44:47], s80 offen offset:1024 sc0 sc1 lds\n\t"
      "ds_read_b64_tr_b8 a[148:149], v244 offset:40960\n\t"
      "ds_read_b64_tr_b8 a[150:151], v244 offset:41984\n\t"
      "ds_read_b64_tr_b8 a[152:153], v245 offset:32768\n\t"
      "ds_read_b64_tr_b8 a[154:155], v245 offset:33792\n\t"
      "ds_read_b64_tr_b8 a[156:157], v245 offset:40960\n\t"
      "ds_read_b64_tr_b8 a[158:159], v245 offset:41984\n\t"
      "s_waitcnt lgkmcnt(0)\n\t"
      "ds_add_u32 v203, v204 offset:256\n\t"
      "s_add_u32 m0, s78, 0\n\t"
      "s_add_u32 s80, s62, 524288\n\t"
      "buffer_load_dwordx4 v224, s[36:39], s80 offen offset:0 lds\n\t"
      "buffer_load_dwordx4 v225, s[36:39], s80 offen offset:1024 lds\n\t"
      "buffer_load_dwordx4 v226, s[36:39], s80 offen offset:2048 lds\n\t"
      "buffer_load_dwordx4 v227, s[36:39], s80 offen offset:3072 lds\n\t"
      "ds_read_b32 v253, v203 offset:256\n\t"
      "s_add_u32 m0, s78, 4096\n\t"
      "s_add_u32 s80, s62, 655360\n\t"
      "buffer_load_dwordx4 v224, s[36:39], s80 offen offset:0 lds\n\t"
      "buffer_load_dwordx4 v225, s[36:39], s80 offen offset:1024 lds\n\t"
      "s_waitcnt lgkmcnt(0)\n\t"
      "v_cmp_gt_u32 vcc, 16, v253\n\t"
      "buffer_load_dwordx4 v226, s[36:39], s80 offen offset:2048 lds\n\t"
      "buffer_load_dwordx4 v227, s[36:39], s80 offen offset:3072 lds\n\t"
      "s_cbranch_vccnz AQ13_%=\n\t"
      "AR14_%=:\n\t"
      "s_add_u32 m0, s52, 114688\n\t"
      "s_add_u32 s80, s20, 0\n\t"
      "buffer_load_dwordx4 v200, s[44:47], s80 offen offset:0 sc0 sc1 lds\n\t"
      "buffer_load_dwordx4 v200, s[44:47], s80 offen offset:1024 sc0 sc1 lds\n\t"
      "s_add_u32 m0, s78, 16384\n\t"
      "s_add_u32 s80, s63, 524288\n\t"
      "buffer_load_dwordx4 v224, s[36:39], s80 offen offset:0 lds\n\t"
      "buffer_load_dwordx4 v225, s[36:39], s80 offen offset:1024 lds\n\t"
      "ds_read_b64_tr_b8 a[160:161], v244 offset:49152\n\t"
      "ds_read_b64_tr_b8 a[162:163], v244 offset:50176\n\t"
      "ds_read_b64_tr_b8 a[164:165], v244 offset:57344\n\t"
      "ds_read_b64_tr_b8 a[166:167], v244 offset:58368\n\t"
      "s_waitcnt vmcnt(12)\n\t"
      "ds_read2st64_b32 v[228:229], v252 offset0:64 offset1:68\n\t"
      "ds_read2st64_b32 v[230:231], v251 offset0:224 offset1:228\n\t"
      "buffer_load_dwordx4 v226, s[36:39], s80 offen offset:2048 lds\n\t"
      "buffer_load_dwordx4 v227, s[36:39], s80 offen offset:3072 lds\n\t"
      "ds_read_b64_tr_b8 a[168:169], v245 offset:49152\n\t"
      "ds_read_b64_tr_b8 a[170:171], v245 offset:50176\n\t"
      "ds_read_b64_tr_b8 a[172:173], v245 offset:57344\n\t"
      "ds_read_b64_tr_b8 a[174:175], v245 offset:58368\n\t"
      "ds_read_b64_tr_b8 a[176:177], v246 offset:0\n\t"
      "ds_read_b64_tr_b8 a[178:179], v246 offset:1024\n\t"
      "s_add_u32 m0, s78, 20480\n\t"
      "s_add_u32 s80, s63, 655360\n\t"
      "buffer_load_dwordx4 v224, s[36:39], s80 offen offset:0 lds\n\t"
      "buffer_load_dwordx4 v225, s[36:39], s80 offen offset:1024 lds\n\t"
      "ds_read_b64_tr_b8 a[180:181], v246 offset:8192\n\t"
      "ds_read_b64_tr_b8 a[182:183], v246 offset:9216\n\t"
      "ds_read_b64_tr_b8 a[184:185], v247 offset:0\n\t"
      "ds_read_b64_tr_b8 a[186:187], v247 offset:1024\n\t"
      "ds_read_b64_tr_b8 a[188:189], v247 offset:8192\n\t"
      "ds_read_b64_tr_b8 a[190:191], v247 offset:9216\n\t"
      "s_waitcnt lgkmcnt(0)\n\t"
      "v_min3_u32 v228, v228, v229, v230\n\t"
      "v_min_u32 v228, v228, v231\n\t"
      "v_cmp_eq_u32 vcc, 0, v228\n\t"
      "s_cbranch_vccnz XS15_%=\n\t"
      "XR16_%=:\n\t"
      "ds_add_u32 v203, v204 offset:0\n\t"
      "buffer_load_dwordx4 v226, s[36:39], s80 offen offset:2048 lds\n\t"
      "buffer_load_dwordx4 v227, s[36:39], s80 offen offset:3072 lds\n\t"
      "s_add_u32 m0, s52, 106496\n\t"
      "s_add_u32 s80, s21, 0\n\t"
      "buffer_load_dwordx4 v200, s[44:47], s80 offen offset:0 sc0 sc1 lds\n\t"
      "buffer_load_dwordx4 v200, s[44:47], s80 offen offset:1024 sc0 sc1 lds\n\t"
      "ds_read_b32 v253, v203 offset:0\n\t"
      "s_add_u32 m0, s52, 98304\n\t"
      "s_add_u32 s80, s24, 0\n\t"
      "buffer_load_dwordx4 v200, s[44:47], s80 offen offset:0 sc0 sc1 lds\n\t"
      "buffer_load_dwordx4 v200, s[44:47], s80 offen offset:1024 sc0 sc1 lds\n\t"
      "s_waitcnt lgkmcnt(0)\n\t"
      "v_cmp_gt_u32 vcc, 16, v253\n\t"
      "s_add_u32 m0, s52, 40960\n\t"
      "s_add_u32 s80, s25, 0\n\t"
      "buffer_load_dwordx4 v200, s[44:47], s80 offen offset:0 sc0 sc1 lds\n\t"
      "buffer_load_dwordx4 v200, s[44:47], s80 offen offset:1024 sc0 sc1 lds\n\t"
      "s_cbranch_vccnz AQ19_%=\n\t"
      "AR20_%=:\n\t"
      "ds_read_b128 v[40:43], v242 offset:16384\n\t"
      "ds_read_b64_tr_b8 a[192:193], v246 offset:16384\n\t"
      "ds_read_b64_tr_b8 a[194:195], v246 offset:17408\n\t"
      "ds_read_b64_tr_b8 a[196:197], v246 offset:24576\n\t"
      "ds_read_b128 v[44:47], v243 offset:16384\n\t"
      "ds_read_b64_tr_b8 a[198:199], v246 offset:25600\n\t"
      "ds_read_b64_tr_b8 a[200:201], v247 offset:16384\n\t"
      "ds_read_b64_tr_b8 a[202:203], v247 offset:17408\n\t"
      "s_waitcnt vmcnt(16)\n\t"
      "ds_read_b128 v[48:51], v242 offset:18432\n\t"
      "ds_read_b64_tr_b8 a[204:205], v247 offset:24576\n\t"
      "ds_read_b64_tr_b8 a[206:207], v247 offset:25600\n\t"
      "ds_read_b64_tr_b8 a[208:209], v248 offset:0\n\t"
      "ds_read_b64_tr_b8 a[210:211], v248 offset:1024\n\t"
      "ds_read_b128 v[52:55], v243 offset:18432\n\t"
      "ds_read_b64_tr_b8 a[212:213], v248 offset:8192\n\t"
      "ds_read_b64_tr_b8 a[214:215], v248 offset:9216\n\t"
      "ds_read_b64_tr_b8 a[216:217], v249 offset:0\n\t"
      "ds_read_b64_tr_b8 a[218:219], v249 offset:1024\n\t"
      "ds_read_b64_tr_b8 a[220:221], v249 offset:8192\n\t"
      "ds_read_b64_tr_b8 a[222:223], v249 offset:9216\n\t"
      "s_waitcnt lgkmcnt(0)\n\t"
      "ds_add_u32 v203, v204 offset:256\n\t"
      "ds_read_b128 v[56:59], v242 offset:20480\n\t"
      "ds_read_b128 v[60:63], v243 offset:20480\n\t"
      "v_mfma_f32_16x16x128_f8f6f4 v[72:75], a[32:39], v[40:47], v[72:75] cbsz:1 blgp:1\n\t"
      "ds_read_b128 v[64:67], v242 offset:22528\n\t"
      "s_add_u32 m0, s52, 32768\n\t"
      "s_add_u32 s80, s26, 0\n\t"
      "buffer_load_dwordx4 v200, s[44:47], s80 offen offset:0 sc0 sc1 lds\n\t"
      "v_mfma_f32_16x16x128_f8f6f4 v[76:79], a[40:47], v[40:47], v[76:79] cbsz:1 blgp:1\n\t"
      "ds_read_b128 v[68:71], v243 offset:22528\n\t"
      "buffer_load_dwordx4 v200, s[44:47], s80 offen offset:1024 sc0 sc1 lds\n\t"
      "v_mfma_f32_16x16x128_f8f6f4 v[72:75], a[144:151], v[48:55], v[72:75] cbsz:1 blgp:1\n\t"
      "s_waitcnt vmcnt(6)\n\t"
      "ds_read2st64_b32 v[228:229], v251 offset0:192 offset1:196\n\t"
      "ds_read2st64_b32 v[230:231], v251 offset0:160 offset1:164\n\t"
      "ds_read_b128 v[32:35], v240 offset:57344\n\t"
      "v_mfma_f32_16x16x128_f8f6f4 v[76:79], a[152:159], v[48:55], v[76:79] cbsz:1 blgp:1\n\t"
      "ds_read_b128 v[36:39], v241 offset:57344\n\t"
      "s_waitcnt lgkmcnt(6)\n\t"
      "v_mfma_f32_16x16x128_f8f6f4 v[80:83], a[32:39], v[56:63], v[80:83] cbsz:1 blgp:1\n\t"
      "ds_read_b128 v[40:43], v240 offset:59392\n\t"
      "v_mfma_f32_16x16x128_f8f6f4 v[84:87], a[40:47], v[56:63], v[84:87] cbsz:1 blgp:1\n\t"
      "s_waitcnt lgkmcnt(3)\n\t"
      "v_min3_u32 v228, v228, v229, v230\n\t"
      "v_min_u32 v228, v228, v231\n\t"
      "v_cmp_eq_u32 vcc, 0, v228\n\t"
      "ds_read_b128 v[44:47], v241 offset:59392\n\t"
      "v_mfma_f32_16x16x128_f8f6f4 v[80:83], a[144:151], v[64:71], v[80:83] cbsz:1 blgp:1\n\t"
      "s_cbranch_vccnz XS21_%=\n\t"
      "XR22_%=:\n\t"
      "ds_add_u32 v203, v204 offset:0\n\t"
      "ds_read_b128 v[48:51], v240 offset:61440\n\t"
      "v_mfma_f32_16x16x128_f8f6f4 v[84:87], a[152:159], v[64:71], v[84:87] cbsz:1 blgp:1\n\t"
      "ds_read_b128 v[52:55], v241 offset:61440\n\t"
      "s_waitcnt lgkmcnt(5)\n\t"
      "v_mfma_f32_16x16x128_f8f6f4 v[72:75], a[48:55], v[32:39], v[72:75] cbsz:1 blgp:1\n\t"
      "ds_read_b32 v253, v203 offset:0\n\t"
      "ds_read_b128 v[56:59], v240 offset:63488\n\t"
      "s_add_u32 m0, s52, 73728\n\t"
      "s_add_u32 s80, s18, 8192\n\t"
      "buffer_load_dwordx4 v200, s[44:47], s80 offen offset:0 sc0 sc1 lds\n\t"
      "v_mfma_f32_16x16x128_f8f6f4 v[76:79], a[56:63], v[32:39], v[76:79] cbsz:1 blgp:1\n\t"
      "s_waitcnt lgkmcnt(1)\n\t"
      "v_cmp_gt_u32 vcc, 20, v253\n\t"
      "ds_read_b128 v[60:63], v241 offset:63488\n\t"
      "v_mfma_f32_16x16x128_f8f6f4 v[72:75], a[160:167], v[40:47], v[72:75] cbsz:1 blgp:1\n\t"
      "s_cbranch_vccnz AQ25_%=\n\t"
      "AR26_%=:\n\t"
      "s_waitcnt vmcnt(5)\n\t"
      "ds_read2st64_b32 v[228:229], v251 offset0:128 offset1:132\n\t"
      "ds_read_b128 v[64:67], v240 offset:49152\n\t"
      "buffer_load_dwordx4 v200, s[44:47], s80 offen offset:1024 sc0 sc1 lds\n\t"
      "ds_read_b64_tr_b8 a[224:225], v244 offset:0\n\t"
      "v_mfma_f32_16x16x128_f8f6f4 v[76:79], a[168:175], v[40:47], v[76:79] cbsz:1 blgp:1\n\t"
      "ds_read_b128 v[68:71], v241 offset:49152\n\t"
      "ds_read_b64_tr_b8 a[226:227], v244 offset:1024\n\t"
      "ds_read_b64_tr_b8 a[228:229], v244 offset:8192\n\t"
      "ds_read_b64_tr_b8 a[230:231], v244 offset:9216\n\t"
      "v_mfma_f32_16x16x128_f8f6f4 v[80:83], a[48:55], v[48:55], v[80:83] cbsz:1 blgp:1\n\t"
      "ds_read_b128 v[32:35], v240 offset:51200\n\t"
      "ds_read_b64_tr_b8 a[232:233], v245 offset:0\n\t"
      "ds_read_b64_tr_b8 a[234:235], v245 offset:1024\n\t"
      "ds_read_b64_tr_b8 a[236:237], v245 offset:8192\n\t"
      "ds_read_b64_tr_b8 a[238:239], v245 offset:9216\n\t"
      "v_mfma_f32_16x16x128_f8f6f4 v[84:87], a[56:63], v[48:55], v[84:87] cbsz:1 blgp:1\n\t"
      "s_waitcnt lgkmcnt(11)\n\t"
      "v_min_u32 v228, v228, v229\n\t"
      "v_cmp_eq_u32 vcc, 0, v228\n\t"
      "ds_read_b128 v[36:39], v241 offset:51200\n\t"
      "ds_read_b64_tr_b8 a[240:241], v244 offset:16384\n\t"
      "ds_read_b64_tr_b8 a[242:243], v244 offset:17408\n\t"
      "v_mfma_f32_16x16x128_f8f6f4 v[80:83], a[160:167], v[56:63], v[80:83] cbsz:1 blgp:1\n\t"
      "s_cbranch_vccnz XS27_%=\n\t"
      "XR28_%=:\n\t"
      "ds_add_u32 v203, v204 offset:256\n\t"
      "ds_read_b128 v[40:43], v240 offset:53248\n\t"
      "ds_read_b64_tr_b8 a[244:245], v244 offset:24576\n\t"
      "ds_read_b64_tr_b8 a[246:247], v244 offset:25600\n\t"
      "ds_read_b64_tr_b8 a[248:249], v245 offset:16384\n\t"
      "v_mfma_f32_16x16x128_f8f6f4 v[84:87], a[168:175], v[56:63], v[84:87] cbsz:1 blgp:1\n\t"
      "ds_read_b128 v[44:47], v241 offset:53248\n\t"
      "ds_read_b64_tr_b8 a[250:251], v245 offset:17408\n\t"
      "ds_read_b64_tr_b8 a[252:253], v245 offset:24576\n\t"
      "ds_read_b64_tr_b8 a[254:255], v245 offset:25600\n\t"
      "s_waitcnt lgkmcnt(15)\n\t"
      "v_mfma_f32_16x16x128_f8f6f4 v[72:75], a[64:71], v[64:71], v[72:75] cbsz:1 blgp:1\n\t"
      "ds_read_b128 v[48:51], v240 offset:55296\n\t"
      "s_add_u32 m0, s52, 147456\n\t"
      "s_add_u32 s80, s19, 8192\n\t"
      "buffer_load_dwordx4 v200, s[44:47], s80 offen offset:0 sc0 sc1 lds\n\t"
      "v_mfma_f32_16x16x128_f8f6f4 v[76:79], a[72:79], v[64:71], v[76:79] cbsz:1 blgp:1\n\t"
      "ds_read_b128 v[52:55], v241 offset:55296\n\t"
      "buffer_load_dwordx4 v200, s[44:47], s80 offen offset:1024 sc0 sc1 lds\n\t"
      "s_waitcnt lgkmcnt(13)\n\t"
      "v_mfma_f32_16x16x128_f8f6f4 v[72:75], a[176:183], v[32:39], v[72:75] cbsz:1 blgp:1\n\t"
      "s_waitcnt vmcnt(6)\n\t"
      "ds_read2st64_b32 v[228:229], v250 offset0:160 offset1:164\n\t"
      "ds_read_b128 v[56:59], v240 offset:40960\n\t"
      "v_mfma_f32_16x16x128_f8f6f4 v[76:79], a[184:191], v[32:39], v[76:79] cbsz:1 blgp:1\n\t"
      "ds_read_b128 v[60:63], v241 offset:40960\n\t"
      "s_waitcnt lgkmcnt(8)\n\t"
      "v_mfma_f32_16x16x128_f8f6f4 v[80:83], a[64:71], v[40:47], v[80:83] cbsz:1 blgp:1\n\t"
      "ds_read_b128 v[64:67], v240 offset:43008\n\t"
      "v_mfma_f32_16x16x128_f8f6f4 v[84:87], a[72:79], v[40:47], v[84:87] cbsz:1 blgp:1\n\t"
      "s_waitcnt lgkmcnt(3)\n\t"
      "v_min_u32 v228, v228, v229\n\t"
      "v_cmp_eq_u32 vcc, 0, v228\n\t"
      "ds_read_b128 v[68:71], v241 offset:43008\n\t"
      "v_mfma_f32_16x16x128_f8f6f4 v[80:83], a[176:183], v[48:55], v[80:83] cbsz:1 blgp:1\n\t"
      "s_cbranch_vccnz XS31_%=\n\t"
      "XR32_%=:\n\t"
      "ds_add_u32 v203, v204 offset:0\n\t"
      "ds_read_b32 v253, v203 offset:256\n\t"
      "ds_read_b128 v[32:35], v240 offset:45056\n\t"
      "v_mfma_f32_16x16x128_f8f6f4 v[84:87], a[184:191], v[48:55], v[84:87] cbsz:1 blgp:1\n\t"
      "ds_read_b128 v[36:39], v241 offset:45056\n\t"
      "s_waitcnt lgkmcnt(6)\n\t"
      "v_mfma_f32_16x16x128_f8f6f4 v[72:75], a[80:87], v[56:63], v[72:75] cbsz:1 blgp:1\n\t"
      "ds_read_b128 v[40:43], v240 offset:47104\n\t"
      "s_add_u32 m0, s52, 65536\n\t"
      "s_add_u32 s80, s20, 8192\n\t"
      "buffer_load_dwordx4 v200, s[44:47], s80 offen offset:0 sc0 sc1 lds\n\t"
      "v_mfma_f32_16x16x128_f8f6f4 v[76:79], a[88:95], v[56:63], v[76:79] cbsz:1 blgp:1\n\t"
      "s_waitcnt lgkmcnt(3)\n\t"
      "v_cmp_gt_u32 vcc, 24, v253\n\t"
      "ds_read_b128 v[44:47], v241 offset:47104\n\t"
      "v_mfma_f32_16x16x128_f8f6f4 v[72:75], a[192:199], v[64:71], v[72:75] cbsz:1 blgp:1\n\t"
      "s_cbranch_vccnz AQ35_%=\n\t"
      "AR36_%=:\n\t"
      "s_waitcnt vmcnt(5)\n\t"
      "ds_read2st64_b32 v[228:229], v250 offset0:128 offset1:132\n\t"
      "ds_read_b128 v[48:51], v240 offset:32768\n\t"
      "buffer_load_dwordx4 v200, s[44:47], s80 offen offset:1024 sc0 sc1 lds\n\t"
      "v_mfma_f32_16x16x128_f8f6f4 v[76:79], a[200:207], v[64:71], v[76:79] cbsz:1 blgp:1\n\t"
      "ds_read_b128 v[52:55], v241 offset:32768\n\t"
      "s_waitcnt lgkmcnt(5)\n\t"
      "v_mfma_f32_16x16x128_f8f6f4 v[80:83], a[80:87], v[32:39], v[80:83] cbsz:1 blgp:1\n\t"
      "ds_read_b128 v[56:59], v240 offset:34816\n\t"
      "v_mfma_f32_16x16x128_f8f6f4 v[84:87], a[88:95], v[32:39], v[84:87] cbsz:1 blgp:1\n\t"
      "s_waitcnt lgkmcnt(3)\n\t"
      "v_min_u32 v228, v228, v229\n\t"
      "v_cmp_eq_u32 vcc, 0, v228\n\t"
      "ds_read_b128 v[60:63], v241 offset:34816\n\t"
      "v_mfma_f32_16x16x128_f8f6f4 v[80:83], a[192:199], v[40:47], v[80:83] cbsz:1 blgp:1\n\t"
      "s_cbranch_vccnz XS37_%=\n\t"
      "XR38_%=:\n\t"
      "ds_add_u32 v203, v204 offset:256\n\t"
      "ds_read_b32 v253, v203 offset:0\n\t"
      "ds_read_b128 v[64:67], v240 offset:36864\n\t"
      "v_mfma_f32_16x16x128_f8f6f4 v[84:87], a[200:207], v[40:47], v[84:87] cbsz:1 blgp:1\n\t"
      "ds_read_b128 v[68:71], v241 offset:36864\n\t"
      "s_waitcnt lgkmcnt(6)\n\t"
      "v_mfma_f32_16x16x128_f8f6f4 v[72:75], a[96:103], v[48:55], v[72:75] cbsz:1 blgp:1\n\t"
      "ds_read_b128 v[32:35], v240 offset:38912\n\t"
      "s_add_u32 m0, s52, 122880\n\t"
      "s_add_u32 s80, s21, 8192\n\t"
      "buffer_load_dwordx4 v200, s[44:47], s80 offen offset:0 sc0 sc1 lds\n\t"
      "v_mfma_f32_16x16x128_f8f6f4 v[76:79], a[104:111], v[48:55], v[76:79] cbsz:1 blgp:1\n\t"
      "s_waitcnt lgkmcnt(3)\n\t"
      "v_cmp_gt_u32 vcc, 24, v253\n\t"
      "ds_read_b128 v[36:39], v241 offset:38912\n\t"
      "v_mfma_f32_16x16x128_f8f6f4 v[72:75], a[208:215], v[56:63], v[72:75] cbsz:1 blgp:1\n\t"
      "s_cbranch_vccnz AQ41_%=\n\t"
      "AR42_%=:\n\t"
      "s_waitcnt vmcnt(5)\n\t"
      "ds_read2st64_b32 v[228:229], v251 offset0:32 offset1:36\n\t"
      "ds_read_b128 v[40:43], v238 offset:40960\n\t"
      "buffer_load_dwordx4 v200, s[44:47], s80 offen offset:1024 sc0 sc1 lds\n\t"
      "v_mfma_f32_16x16x128_f8f6f4 v[76:79], a[216:223], v[56:63], v[76:79] cbsz:1 blgp:1\n\t"
      "ds_read_b128 v[44:47], v239 offset:40960\n\t"
      "s_waitcnt lgkmcnt(5)\n\t"
      "v_mfma_f32_16x16x128_f8f6f4 v[80:83], a[96:103], v[64:71], v[80:83] cbsz:1 blgp:1\n\t"
      "ds_read_b128 v[48:51], v238 offset:43008\n\t"
      "v_mfma_f32_16x16x128_f8f6f4 v[84:87], a[104:111], v[64:71], v[84:87] cbsz:1 blgp:1\n\t"
      "s_waitcnt lgkmcnt(3)\n\t"
      "v_min_u32 v228, v228, v229\n\t"
      "v_cmp_eq_u32 vcc, 0, v228\n\t"
      "ds_read_b128 v[52:55], v239 offset:43008\n\t"
      "v_mfma_f32_16x16x128_f8f6f4 v[80:83], a[208:215], v[32:39], v[80:83] cbsz:1 blgp:1\n\t"
      "s_cbranch_vccnz XS43_%=\n\t"
      "XR44_%=:\n\t"
      "ds_add_u32 v203, v204 offset:0\n\t"
      "ds_read_b32 v253, v203 offset:256\n\t"
      "ds_read_b128 v[56:59], v238 offset:45056\n\t"
      "v_mfma_f32_16x16x128_f8f6f4 v[84:87], a[216:223], v[32:39], v[84:87] cbsz:1 blgp:1\n\t"
      "ds_read_b128 v[60:63], v239 offset:45056\n\t"
      "s_waitcnt lgkmcnt(6)\n\t"
      "v_mfma_f32_16x16x128_f8f6f4 v[72:75], a[112:119], v[40:47], v[72:75] cbsz:1 blgp:1\n\t"
      "ds_read_b128 v[64:67], v238 offset:47104\n\t"
      "s_add_u32 m0, s52, 114688\n\t"
      "s_add_u32 s80, s24, 8192\n\t"
      "buffer_load_dwordx4 v200, s[44:47], s80 offen offset:0 sc0 sc1 lds\n\t"
      "v_mfma_f32_16x16x128_f8f6f4 v[76:79], a[120:127], v[40:47], v[76:79] cbsz:1 blgp:1\n\t"
      "s_waitcnt lgkmcnt(3)\n\t"
      "v_cmp_gt_u32 vcc, 28, v253\n\t"
      "ds_read_b128 v[68:71], v239 offset:47104\n\t"
      "v_mfma_f32_16x16x128_f8f6f4 v[72:75], a[224:231], v[48:55], v[72:75] cbsz:1 blgp:1\n\t"
      "s_cbranch_vccnz AQ47_%=\n\t"
      "AR48_%=:\n\t"
      "s_waitcnt vmcnt(5)\n\t"
      "ds_read2st64_b32 v[228:229], v252 offset0:64 offset1:68\n\t"
      "ds_read_b128 v[32:35], v238 offset:32768\n\t"
      "buffer_load_dwordx4 v200, s[44:47], s80 offen offset:1024 sc0 sc1 lds\n\t"
      "v_mfma_f32_16x16x128_f8f6f4 v[76:79], a[232:239], v[48:55], v[76:79] cbsz:1 blgp:1\n\t"
      "ds_read_b128 v[36:39], v239 offset:32768\n\t"
      "s_waitcnt lgkmcnt(5)\n\t"
      "v_mfma_f32_16x16x128_f8f6f4 v[80:83], a[112:119], v[56:63], v[80:83] cbsz:1 blgp:1\n\t"
      "ds_read_b128 v[40:43], v238 offset:34816\n\t"
      "v_mfma_f32_16x16x128_f8f6f4 v[84:87], a[120:127], v[56:63], v[84:87] cbsz:1 blgp:1\n\t"
      "s_waitcnt lgkmcnt(3)\n\t"
      "v_min_u32 v228, v228, v229\n\t"
      "v_cmp_eq_u32 vcc, 0, v228\n\t"
      "ds_read_b128 v[44:47], v239 offset:34816\n\t"
      "v_mfma_f32_16x16x128_f8f6f4 v[80:83], a[224:231], v[64:71], v[80:83] cbsz:1 blgp:1\n\t"
      "s_cbranch_vccnz XS49_%=\n\t"
      "XR50_%=:\n\t"
      "ds_add_u32 v203, v204 offset:256\n\t"
      "ds_read_b32 v253, v203 offset:0\n\t"
      "ds_read_b128 v[48:51], v238 offset:36864\n\t"
      "v_mfma_f32_16x16x128_f8f6f4 v[84:87], a[232:239], v[64:71], v[84:87] cbsz:1 blgp:1\n\t"
      "ds_read_b128 v[52:55], v239 offset:36864\n\t"
      "s_waitcnt lgkmcnt(6)\n\t"
      "v_mfma_f32_16x16x128_f8f6f4 v[72:75], a[128:135], v[32:39], v[72:75] cbsz:1 blgp:1\n\t"
      "ds_read_b128 v[56:59], v238 offset:38912\n\t"
      "s_add_u32 m0, s52, 106496\n\t"
      "s_add_u32 s80, s25, 8192\n\t"
      "buffer_load_dwordx4 v200, s[44:47], s80 offen offset:0 sc0 sc1 lds\n\t"
      "v_mfma_f32_16x16x128_f8f6f4 v[76:79], a[136:143], v[32:39], v[76:79] cbsz:1 blgp:1\n\t"
      "s_waitcnt lgkmcnt(3)\n\t"
      "v_cmp_gt_u32 vcc, 28, v253\n\t"
      "ds_read_b128 v[60:63], v239 offset:38912\n\t"
      "v_mfma_f32_16x16x128_f8f6f4 v[72:75], a[240:247], v[40:47], v[72:75] cbsz:1 blgp:1\n\t"
      "s_cbranch_vccnz AQ53_%=\n\t"
      "AR54_%=:\n\t"
      "s_waitcnt vmcnt(5)\n\t"
      "ds_read2st64_b32 v[228:229], v251 offset0:0 offset1:4\n\t"
      "ds_read_b128 v[64:67], v240 offset:8192\n\t"
      "buffer_load_dwordx4 v200, s[44:47], s80 offen offset:1024 sc0 sc1 lds\n\t"
      "v_mfma_f32_16x16x128_f8f6f4 v[76:79], a[248:255], v[40:47], v[76:79] cbsz:1 blgp:1\n\t"
      "ds_read_b128 v[68:71], v241 offset:8192\n\t"
      "s_waitcnt lgkmcnt(5)\n\t"
      "v_mfma_f32_16x16x128_f8f6f4 v[80:83], a[128:135], v[48:55], v[80:83] cbsz:1 blgp:1\n\t"
      "ds_read_b128 v[32:35], v240 offset:10240\n\t"
      "v_mfma_f32_16x16x128_f8f6f4 v[84:87], a[136:143], v[48:55], v[84:87] cbsz:1 blgp:1\n\t"
      "s_waitcnt lgkmcnt(3)\n\t"
      "v_min_u32 v228, v228, v229\n\t"
      "v_cmp_eq_u32 vcc, 0, v228\n\t"
      "ds_read_b128 v[36:39], v241 offset:10240\n\t"
      "v_mfma_f32_16x16x128_f8f6f4 v[80:83], a[240:247], v[56:63], v[80:83] cbsz:1 blgp:1\n\t"
      "s_cbranch_vccnz XS55_%=\n\t"
      "XR56_%=:\n\t"
      "ds_add_u32 v203, v204 offset:0\n\t"
      "ds_read_b32 v253, v203 offset:256\n\t"
      "ds_read_b128 v[40:43], v240 offset:12288\n\t"
      "v_mfma_f32_16x16x128_f8f6f4 v[84:87], a[248:255], v[56:63], v[84:87] cbsz:1 blgp:1\n\t"
      "ds_read_b128 v[44:47], v241 offset:12288\n\t"
      "s_waitcnt lgkmcnt(6)\n\t"
      "v_mfma_f32_16x16x128_f8f6f4 v[88:91], a[32:39], v[64:71], v[88:91] cbsz:1 blgp:1\n\t"
      "ds_read_b128 v[48:51], v240 offset:14336\n\t"
      "s_add_u32 m0, s52, 98304\n\t"
      "s_add_u32 s80, s26, 8192\n\t"
      "buffer_load_dwordx4 v200, s[44:47], s80 offen offset:0 sc0 sc1 lds\n\t"
      "v_mfma_f32_16x16x128_f8f6f4 v[92:95], a[40:47], v[64:71], v[92:95] cbsz:1 blgp:1\n\t"
      "s_waitcnt lgkmcnt(3)\n\t"
      "v_cmp_gt_u32 vcc, 32, v253\n\t"
      "ds_read_b128 v[52:55], v241 offset:14336\n\t"
      "v_mfma_f32_16x16x128_f8f6f4 v[88:91], a[144:151], v[32:39], v[88:91] cbsz:1 blgp:1\n\t"
      "s_cbranch_vccnz AQ59_%=\n\t"
      "AR60_%=:\n\t"
      "s_waitcnt vmcnt(5)\n\t"
      "ds_read2st64_b32 v[228:229], v251 offset0:224 offset1:228\n\t"
      "ds_read_b128 v[56:59], v242 offset:16384\n\t"
      "buffer_load_dwordx4 v200, s[44:47], s80 offen offset:1024 sc0 sc1 lds\n\t"
      "v_mfma_f32_16x16x128_f8f6f4 v[92:95], a[152:159], v[32:39], v[92:95] cbsz:1 blgp:1\n\t"
      "ds_read_b128 v[60:63], v243 offset:16384\n\t"
      "global_store_dwordx4 v210, v[72:75], s[30:31] sc0 sc1\n\t"
      "s_waitcnt lgkmcnt(5)\n\t"
      "v_mfma_f32_16x16x128_f8f6f4 v[96:99], a[32:39], v[40:47], v[96:99] cbsz:1 blgp:1\n\t"
      "ds_read_b128 v[64:67], v242 offset:18432\n\t"
      "v_mfma_f32_16x16x128_f8f6f4 v[100:103], a[40:47], v[40:47], v[100:103] cbsz:1 blgp:1\n\t"
      "s_waitcnt lgkmcnt(3)\n\t"
      "v_min_u32 v228, v228, v229\n\t"
      "v_cmp_eq_u32 vcc, 0, v228\n\t"
      "ds_read_b128 v[68:71], v243 offset:18432\n\t"
      "v_mfma_f32_16x16x128_f8f6f4 v[96:99], a[144:151], v[48:55], v[96:99] cbsz:1 blgp:1\n\t"
      "s_cbranch_vccnz XS61_%=\n\t"
      "XR62_%=:\n\t"
      "ds_add_u32 v203, v204 offset:256\n\t"
      "ds_read_b32 v253, v203 offset:0\n\t"
      "ds_read_b128 v[32:35], v242 offset:20480\n\t"
      "v_mfma_f32_16x16x128_f8f6f4 v[100:103], a[152:159], v[48:55], v[100:103] cbsz:1 blgp:1\n\t"
      "ds_read_b128 v[36:39], v243 offset:20480\n\t"
      "s_waitcnt lgkmcnt(6)\n\t"
      "v_mfma_f32_16x16x128_f8f6f4 v[88:91], a[48:55], v[56:63], v[88:91] cbsz:1 blgp:1\n\t"
      "ds_read_b128 v[40:43], v242 offset:22528\n\t"
      "s_add_u32 m0, s52, 40960\n\t"
      "s_add_u32 s80, s18, 16384\n\t"
      "buffer_load_dwordx4 v200, s[44:47], s80 offen offset:0 sc0 sc1 lds\n\t"
      "v_mfma_f32_16x16x128_f8f6f4 v[92:95], a[56:63], v[56:63], v[92:95] cbsz:1 blgp:1\n\t"
      "s_waitcnt lgkmcnt(3)\n\t"
      "v_cmp_gt_u32 vcc, 32, v253\n\t"
      "ds_read_b128 v[44:47], v243 offset:22528\n\t"
      "v_mfma_f32_16x16x128_f8f6f4 v[88:91], a[160:167], v[64:71], v[88:91] cbsz:1 blgp:1\n\t"
      "s_cbranch_vccnz AQ65_%=\n\t"
      "AR66_%=:\n\t"
      "s_waitcnt vmcnt(6)\n\t"
      "ds_read2st64_b32 v[228:229], v251 offset0:192 offset1:196\n\t"
      "ds_read_b128 v[48:51], v240 offset:0\n\t"
      "buffer_load_dwordx4 v200, s[44:47], s80 offen offset:1024 sc0 sc1 lds\n\t"
      "v_mfma_f32_16x16x128_f8f6f4 v[92:95], a[168:175], v[64:71], v[92:95] cbsz:1 blgp:1\n\t"
      "ds_read_b128 v[52:55], v241 offset:0\n\t"
      "global_store_dwordx4 v211, v[76:79], s[30:31] sc0 sc1\n\t"
      "s_add_u32 s30, s30, 0x40000\n\t"
      "s_addc_u32 s31, s31, 0\n\t"
      "s_waitcnt lgkmcnt(5)\n\t"
      "v_mfma_f32_16x16x128_f8f6f4 v[96:99], a[48:55], v[32:39], v[96:99] cbsz:1 blgp:1\n\t"
      "ds_read_b128 v[56:59], v240 offset:2048\n\t"
      "v_mfma_f32_16x16x128_f8f6f4 v[100:103], a[56:63], v[32:39], v[100:103] cbsz:1 blgp:1\n\t"
      "s_waitcnt lgkmcnt(3)\n\t"
      "v_min_u32 v228, v228, v229\n\t"
      "v_cmp_eq_u32 vcc, 0, v228\n\t"
      "ds_read_b128 v[60:63], v241 offset:2048\n\t"
      "v_mfma_f32_16x16x128_f8f6f4 v[96:99], a[160:167], v[40:47], v[96:99] cbsz:1 blgp:1\n\t"
      "s_cbranch_vccnz XS67_%=\n\t"
      "XR68_%=:\n\t"
      "ds_add_u32 v203, v204 offset:0\n\t"
      "ds_read_b32 v253, v203 offset:256\n\t"
      "ds_read_b128 v[64:67], v240 offset:4096\n\t"
      "v_mfma_f32_16x16x128_f8f6f4 v[100:103], a[168:175], v[40:47], v[100:103] cbsz:1 blgp:1\n\t"
      "ds_read_b128 v[68:71], v241 offset:4096\n\t"
      "s_waitcnt lgkmcnt(6)\n\t"
      "v_mfma_f32_16x16x128_f8f6f4 v[88:91], a[64:71], v[48:55], v[88:91] cbsz:1 blgp:1\n\t"
      "ds_read_b128 v[32:35], v240 offset:6144\n\t"
      "s_add_u32 m0, s52, 32768\n\t"
      "s_add_u32 s80, s19, 16384\n\t"
      "buffer_load_dwordx4 v200, s[44:47], s80 offen offset:0 sc0 sc1 lds\n\t"
      "v_mfma_f32_16x16x128_f8f6f4 v[92:95], a[72:79], v[48:55], v[92:95] cbsz:1 blgp:1\n\t"
      "s_waitcnt lgkmcnt(3)\n\t"
      "v_cmp_gt_u32 vcc, 36, v253\n\t"
      "ds_read_b128 v[36:39], v241 offset:6144\n\t"
      "v_mfma_f32_16x16x128_f8f6f4 v[88:91], a[176:183], v[56:63], v[88:91] cbsz:1 blgp:1\n\t"
      "s_cbranch_vccnz AQ71_%=\n\t"
      "AR72_%=:\n\t"
      "s_waitcnt vmcnt(7)\n\t"
      "ds_read2st64_b32 v[228:229], v251 offset0:160 offset1:164\n\t"
      "ds_read_b128 v[40:43], v240 offset:57344\n\t"
      "buffer_load_dwordx4 v200, s[44:47], s80 offen offset:1024 sc0 sc1 lds\n\t"
      "v_mfma_f32_16x16x128_f8f6f4 v[92:95], a[184:191], v[56:63], v[92:95] cbsz:1 blgp:1\n\t"
      "ds_read_b128 v[44:47], v241 offset:57344\n\t"
      "global_store_dwordx4 v210, v[80:83], s[30:31] sc0 sc1\n\t"
      "s_waitcnt lgkmcnt(5)\n\t"
      "v_mfma_f32_16x16x128_f8f6f4 v[96:99], a[64:71], v[64:71], v[96:99] cbsz:1 blgp:1\n\t"
      "ds_read_b128 v[48:51], v240 offset:59392\n\t"
      "v_mfma_f32_16x16x128_f8f6f4 v[100:103], a[72:79], v[64:71], v[100:103] cbsz:1 blgp:1\n\t"
      "s_waitcnt lgkmcnt(3)\n\t"
      "v_min_u32 v228, v228, v229\n\t"
      "v_cmp_eq_u32 vcc, 0, v228\n\t"
      "ds_read_b128 v[52:55], v241 offset:59392\n\t"
      "v_mfma_f32_16x16x128_f8f6f4 v[96:99], a[176:183], v[32:39], v[96:99] cbsz:1 blgp:1\n\t"
      "s_cbranch_vccnz XS73_%=\n\t"
      "XR74_%=:\n\t"
      "ds_add_u32 v203, v204 offset:256\n\t"
      "ds_read_b32 v253, v203 offset:0\n\t"
      "ds_read_b128 v[56:59], v240 offset:61440\n\t"
      "v_mfma_f32_16x16x128_f8f6f4 v[100:103], a[184:191], v[32:39], v[100:103] cbsz:1 blgp:1\n\t"
      "ds_read_b128 v[60:63], v241 offset:61440\n\t"
      "s_waitcnt lgkmcnt(6)\n\t"
      "v_mfma_f32_16x16x128_f8f6f4 v[88:91], a[80:87], v[40:47], v[88:91] cbsz:1 blgp:1\n\t"
      "ds_read_b128 v[64:67], v240 offset:63488\n\t"
      "s_add_u32 m0, s52, 73728\n\t"
      "s_add_u32 s80, s20, 16384\n\t"
      "buffer_load_dwordx4 v200, s[44:47], s80 offen offset:0 sc0 sc1 lds\n\t"
      "v_mfma_f32_16x16x128_f8f6f4 v[92:95], a[88:95], v[40:47], v[92:95] cbsz:1 blgp:1\n\t"
      "s_waitcnt lgkmcnt(3)\n\t"
      "v_cmp_gt_u32 vcc, 36, v253\n\t"
      "ds_read_b128 v[68:71], v241 offset:63488\n\t"
      "v_mfma_f32_16x16x128_f8f6f4 v[88:91], a[192:199], v[48:55], v[88:91] cbsz:1 blgp:1\n\t"
      "s_cbranch_vccnz AQ77_%=\n\t"
      "AR78_%=:\n\t"
      "s_waitcnt vmcnt(8)\n\t"
      "ds_read2st64_b32 v[228:229], v251 offset0:128 offset1:132\n\t"
      "ds_read_b128 v[32:35], v240 offset:49152\n\t"
      "buffer_load_dwordx4 v200, s[44:47], s80 offen offset:1024 sc0 sc1 lds\n\t"
      "v_mfma_f32_16x16x128_f8f6f4 v[92:95], a[200:207], v[48:55], v[92:95] cbsz:1 blgp:1\n\t"
      "ds_read_b128 v[36:39], v241 offset:49152\n\t"
      "global_store_dwordx4 v211, v[84:87], s[30:31] sc0 sc1\n\t"
      "s_add_u32 s30, s30, 0x40000\n\t"
      "s_addc_u32 s31, s31, 0\n\t"
      "s_waitcnt lgkmcnt(5)\n\t"
      "v_mfma_f32_16x16x128_f8f6f4 v[96:99], a[80:87], v[56:63], v[96:99] cbsz:1 blgp:1\n\t"
      "ds_read_b128 v[40:43], v240 offset:51200\n\t"
      "v_mfma_f32_16x16x128_f8f6f4 v[100:103], a[88:95], v[56:63], v[100:103] cbsz:1 blgp:1\n\t"
      "s_waitcnt lgkmcnt(3)\n\t"
      "v_min_u32 v228, v228, v229\n\t"
      "v_cmp_eq_u32 vcc, 0, v228\n\t"
      "ds_read_b128 v[44:47], v241 offset:51200\n\t"
      "v_mfma_f32_16x16x128_f8f6f4 v[96:99], a[192:199], v[64:71], v[96:99] cbsz:1 blgp:1\n\t"
      "s_cbranch_vccnz XS79_%=\n\t"
      "XR80_%=:\n\t"
      "ds_add_u32 v203, v204 offset:0\n\t"
      "ds_read_b32 v253, v203 offset:256\n\t"
      "ds_read_b128 v[48:51], v240 offset:53248\n\t"
      "v_mfma_f32_16x16x128_f8f6f4 v[100:103], a[200:207], v[64:71], v[100:103] cbsz:1 blgp:1\n\t"
      "ds_read_b128 v[52:55], v241 offset:53248\n\t"
      "s_waitcnt lgkmcnt(6)\n\t"
      "v_mfma_f32_16x16x128_f8f6f4 v[88:91], a[96:103], v[32:39], v[88:91] cbsz:1 blgp:1\n\t"
      "ds_read_b128 v[56:59], v240 offset:55296\n\t"
      "s_add_u32 m0, s52, 147456\n\t"
      "s_add_u32 s80, s21, 16384\n\t"
      "buffer_load_dwordx4 v200, s[44:47], s80 offen offset:0 sc0 sc1 lds\n\t"
      "v_mfma_f32_16x16x128_f8f6f4 v[92:95], a[104:111], v[32:39], v[92:95] cbsz:1 blgp:1\n\t"
      "s_waitcnt lgkmcnt(3)\n\t"
      "v_cmp_gt_u32 vcc, 40, v253\n\t"
      "ds_read_b128 v[60:63], v241 offset:55296\n\t"
      "v_mfma_f32_16x16x128_f8f6f4 v[88:91], a[208:215], v[40:47], v[88:91] cbsz:1 blgp:1\n\t"
      "s_cbranch_vccnz AQ83_%=\n\t"
      "AR84_%=:\n\t"
      "s_waitcnt vmcnt(8)\n\t"
      "ds_read2st64_b32 v[228:229], v250 offset0:160 offset1:164\n\t"
      "ds_read_b128 v[64:67], v240 offset:40960\n\t"
      "buffer_load_dwordx4 v200, s[44:47], s80 offen offset:1024 sc0 sc1 lds\n\t"
      "v_mfma_f32_16x16x128_f8f6f4 v[92:95], a[216:223], v[40:47], v[92:95] cbsz:1 blgp:1\n\t"
      "ds_read_b128 v[68:71], v241 offset:40960\n\t"
      "s_waitcnt lgkmcnt(5)\n\t"
      "v_mfma_f32_16x16x128_f8f6f4 v[96:99], a[96:103], v[48:55], v[96:99] cbsz:1 blgp:1\n\t"
      "ds_read_b128 v[32:35], v240 offset:43008\n\t"
      "v_mfma_f32_16x16x128_f8f6f4 v[100:103], a[104:111], v[48:55], v[100:103] cbsz:1 blgp:1\n\t"
      "s_waitcnt lgkmcnt(3)\n\t"
      "v_min_u32 v228, v228, v229\n\t"
      "v_cmp_eq_u32 vcc, 0, v228\n\t"
      "ds_read_b128 v[36:39], v241 offset:43008\n\t"
      "v_mfma_f32_16x16x128_f8f6f4 v[96:99], a[208:215], v[56:63], v[96:99] cbsz:1 blgp:1\n\t"
      "s_cbranch_vccnz XS85_%=\n\t"
      "XR86_%=:\n\t"
      "ds_add_u32 v203, v204 offset:256\n\t"
      "ds_read_b32 v253, v203 offset:0\n\t"
      "ds_read_b128 v[40:43], v240 offset:45056\n\t"
      "v_mfma_f32_16x16x128_f8f6f4 v[100:103], a[216:223], v[56:63], v[100:103] cbsz:1 blgp:1\n\t"
      "ds_read_b128 v[44:47], v241 offset:45056\n\t"
      "s_waitcnt lgkmcnt(6)\n\t"
      "v_mfma_f32_16x16x128_f8f6f4 v[88:91], a[112:119], v[64:71], v[88:91] cbsz:1 blgp:1\n\t"
      "ds_read_b128 v[48:51], v240 offset:47104\n\t"
      "s_add_u32 m0, s52, 65536\n\t"
      "s_add_u32 s80, s24, 16384\n\t"
      "buffer_load_dwordx4 v200, s[44:47], s80 offen offset:0 sc0 sc1 lds\n\t"
      "v_mfma_f32_16x16x128_f8f6f4 v[92:95], a[120:127], v[64:71], v[92:95] cbsz:1 blgp:1\n\t"
      "s_waitcnt lgkmcnt(3)\n\t"
      "v_cmp_gt_u32 vcc, 40, v253\n\t"
      "ds_read_b128 v[52:55], v241 offset:47104\n\t"
      "v_mfma_f32_16x16x128_f8f6f4 v[88:91], a[224:231], v[32:39], v[88:91] cbsz:1 blgp:1\n\t"
      "s_cbranch_vccnz AQ89_%=\n\t"
      "AR90_%=:\n\t"
      "s_waitcnt vmcnt(7)\n\t"
      "ds_read2st64_b32 v[228:229], v250 offset0:128 offset1:132\n\t"
      "ds_read_b128 v[56:59], v240 offset:32768\n\t"
      "buffer_load_dwordx4 v200, s[44:47], s80 offen offset:1024 sc0 sc1 lds\n\t"
      "v_mfma_f32_16x16x128_f8f6f4 v[92:95], a[232:239], v[32:39], v[92:95] cbsz:1 blgp:1\n\t"
      "ds_read_b128 v[60:63], v241 offset:32768\n\t"
      "s_waitcnt lgkmcnt(5)\n\t"
      "v_mfma_f32_16x16x128_f8f6f4 v[96:99], a[112:119], v[40:47], v[96:99] cbsz:1 blgp:1\n\t"
      "ds_read_b128 v[64:67], v240 offset:34816\n\t"
      "v_mfma_f32_16x16x128_f8f6f4 v[100:103], a[120:127], v[40:47], v[100:103] cbsz:1 blgp:1\n\t"
      "s_waitcnt lgkmcnt(3)\n\t"
      "v_min_u32 v228, v228, v229\n\t"
      "v_cmp_eq_u32 vcc, 0, v228\n\t"
      "ds_read_b128 v[68:71], v241 offset:34816\n\t"
      "v_mfma_f32_16x16x128_f8f6f4 v[96:99], a[224:231], v[48:55], v[96:99] cbsz:1 blgp:1\n\t"
      "s_cbranch_vccnz XS91_%=\n\t"
      "XR92_%=:\n\t"
      "ds_add_u32 v203, v204 offset:0\n\t"
      "ds_read_b32 v253, v203 offset:256\n\t"
      "ds_read_b128 v[32:35], v240 offset:36864\n\t"
      "v_mfma_f32_16x16x128_f8f6f4 v[100:103], a[232:239], v[48:55], v[100:103] cbsz:1 blgp:1\n\t"
      "ds_read_b128 v[36:39], v241 offset:36864\n\t"
      "s_waitcnt lgkmcnt(6)\n\t"
      "v_mfma_f32_16x16x128_f8f6f4 v[88:91], a[128:135], v[56:63], v[88:91] cbsz:1 blgp:1\n\t"
      "ds_read_b128 v[40:43], v240 offset:38912\n\t"
      "s_add_u32 m0, s52, 122880\n\t"
      "s_add_u32 s80, s25, 16384\n\t"
      "buffer_load_dwordx4 v200, s[44:47], s80 offen offset:0 sc0 sc1 lds\n\t"
      "v_mfma_f32_16x16x128_f8f6f4 v[92:95], a[136:143], v[56:63], v[92:95] cbsz:1 blgp:1\n\t"
      "s_waitcnt lgkmcnt(3)\n\t"
      "v_cmp_gt_u32 vcc, 44, v253\n\t"
      "ds_read_b128 v[44:47], v241 offset:38912\n\t"
      "v_mfma_f32_16x16x128_f8f6f4 v[88:91], a[240:247], v[64:71], v[88:91] cbsz:1 blgp:1\n\t"
      "s_cbranch_vccnz AQ95_%=\n\t"
      "AR96_%=:\n\t"
      "s_waitcnt vmcnt(6)\n\t"
      "ds_read2st64_b32 v[228:229], v251 offset0:32 offset1:36\n\t"
      "ds_read_b128 v[48:51], v238 offset:40960\n\t"
      "buffer_load_dwordx4 v200, s[44:47], s80 offen offset:1024 sc0 sc1 lds\n\t"
      "v_mfma_f32_16x16x128_f8f6f4 v[92:95], a[248:255], v[64:71], v[92:95] cbsz:1 blgp:1\n\t"
      "ds_read_b128 v[52:55], v239 offset:40960\n\t"
      "s_waitcnt lgkmcnt(5)\n\t"
      "v_mfma_f32_16x16x128_f8f6f4 v[96:99], a[128:135], v[32:39], v[96:99] cbsz:1 blgp:1\n\t"
      "ds_read_b128 v[56:59], v238 offset:43008\n\t"
      "v_mfma_f32_16x16x128_f8f6f4 v[100:103], a[136:143], v[32:39], v[100:103] cbsz:1 blgp:1\n\t"
      "s_waitcnt lgkmcnt(3)\n\t"
      "v_min_u32 v228, v228, v229\n\t"
      "v_cmp_eq_u32 vcc, 0, v228\n\t"
      "ds_read_b128 v[60:63], v239 offset:43008\n\t"
      "v_mfma_f32_16x16x128_f8f6f4 v[96:99], a[240:247], v[40:47], v[96:99] cbsz:1 blgp:1\n\t"
      "s_cbranch_vccnz XS97_%=\n\t"
      "XR98_%=:\n\t"
      "ds_add_u32 v203, v204 offset:256\n\t"
      "ds_read_b32 v253, v203 offset:0\n\t"
      "ds_read_b128 v[64:67], v238 offset:45056\n\t"
      "v_mfma_f32_16x16x128_f8f6f4 v[100:103], a[248:255], v[40:47], v[100:103] cbsz:1 blgp:1\n\t"
      "ds_read_b128 v[68:71], v239 offset:45056\n\t"
      "s_waitcnt lgkmcnt(6)\n\t"
      "v_mfma_f32_16x16x128_f8f6f4 v[104:107], a[32:39], v[48:55], v[104:107] cbsz:1 blgp:1\n\t"
      "ds_read_b128 v[32:35], v238 offset:47104\n\t"
      "s_add_u32 m0, s52, 114688\n\t"
      "s_add_u32 s80, s26, 16384\n\t"
      "buffer_load_dwordx4 v200, s[44:47], s80 offen offset:0 sc0 sc1 lds\n\t"
      "v_mfma_f32_16x16x128_f8f6f4 v[108:111], a[40:47], v[48:55], v[108:111] cbsz:1 blgp:1\n\t"
      "s_waitcnt lgkmcnt(3)\n\t"
      "v_cmp_gt_u32 vcc, 44, v253\n\t"
      "ds_read_b128 v[36:39], v239 offset:47104\n\t"
      "v_mfma_f32_16x16x128_f8f6f4 v[104:107], a[144:151], v[56:63], v[104:107] cbsz:1 blgp:1\n\t"
      "s_cbranch_vccnz AQ101_%=\n\t"
      "AR102_%=:\n\t"
      "s_waitcnt vmcnt(5)\n\t"
      "ds_read2st64_b32 v[228:229], v252 offset0:64 offset1:68\n\t"
      "ds_read_b128 v[40:43], v238 offset:32768\n\t"
      "buffer_load_dwordx4 v200, s[44:47], s80 offen offset:1024 sc0 sc1 lds\n\t"
      "v_mfma_f32_16x16x128_f8f6f4 v[108:111], a[152:159], v[56:63], v[108:111] cbsz:1 blgp:1\n\t"
      "ds_read_b128 v[44:47], v239 offset:32768\n\t"
      "global_store_dwordx4 v210, v[88:91], s[30:31] sc0 sc1\n\t"
      "s_waitcnt lgkmcnt(5)\n\t"
      "v_mfma_f32_16x16x128_f8f6f4 v[112:115], a[32:39], v[64:71], v[112:115] cbsz:1 blgp:1\n\t"
      "ds_read_b128 v[48:51], v238 offset:34816\n\t"
      "global_store_dwordx4 v211, v[92:95], s[30:31] sc0 sc1\n\t"
      "s_add_u32 s30, s30, 0x40000\n\t"
      "s_addc_u32 s31, s31, 0\n\t"
      "v_mfma_f32_16x16x128_f8f6f4 v[116:119], a[40:47], v[64:71], v[116:119] cbsz:1 blgp:1\n\t"
      "s_waitcnt lgkmcnt(3)\n\t"
      "v_min_u32 v228, v228, v229\n\t"
      "v_cmp_eq_u32 vcc, 0, v228\n\t"
      "ds_read_b128 v[52:55], v239 offset:34816\n\t"
      "v_mfma_f32_16x16x128_f8f6f4 v[112:115], a[144:151], v[32:39], v[112:115] cbsz:1 blgp:1\n\t"
      "s_cbranch_vccnz XS103_%=\n\t"
      "XR104_%=:\n\t"
      "ds_add_u32 v203, v204 offset:0\n\t"
      "ds_read_b32 v253, v203 offset:256\n\t"
      "ds_read_b128 v[56:59], v238 offset:36864\n\t"
      "v_mfma_f32_16x16x128_f8f6f4 v[116:119], a[152:159], v[32:39], v[116:119] cbsz:1 blgp:1\n\t"
      "ds_read_b128 v[60:63], v239 offset:36864\n\t"
      "s_waitcnt lgkmcnt(6)\n\t"
      "v_mfma_f32_16x16x128_f8f6f4 v[104:107], a[48:55], v[40:47], v[104:107] cbsz:1 blgp:1\n\t"
      "ds_read_b128 v[64:67], v238 offset:38912\n\t"
      "s_add_u32 m0, s52, 106496\n\t"
      "s_add_u32 s80, s18, 24576\n\t"
      "buffer_load_dwordx4 v200, s[44:47], s80 offen offset:0 sc0 sc1 lds\n\t"
      "v_mfma_f32_16x16x128_f8f6f4 v[108:111], a[56:63], v[40:47], v[108:111] cbsz:1 blgp:1\n\t"
      "s_waitcnt lgkmcnt(3)\n\t"
      "v_cmp_gt_u32 vcc, 48, v253\n\t"
      "ds_read_b128 v[68:71], v239 offset:38912\n\t"
      "v_mfma_f32_16x16x128_f8f6f4 v[104:107], a[160:167], v[48:55], v[104:107] cbsz:1 blgp:1\n\t"
      "s_cbranch_vccnz AQ107_%=\n\t"
      "AR108_%=:\n\t"
      "s_waitcnt vmcnt(7)\n\t"
      "ds_read2st64_b32 v[228:229], v251 offset0:0 offset1:4\n\t"
      "ds_read_b128 v[32:35], v240 offset:8192\n\t"
      "buffer_load_dwordx4 v200, s[44:47], s80 offen offset:1024 sc0 sc1 lds\n\t"
      "v_mfma_f32_16x16x128_f8f6f4 v[108:111], a[168:175], v[48:55], v[108:111] cbsz:1 blgp:1\n\t"
      "ds_read_b128 v[36:39], v241 offset:8192\n\t"
      "global_store_dwordx4 v210, v[96:99], s[30:31] sc0 sc1\n\t"
      "s_waitcnt lgkmcnt(5)\n\t"
      "v_mfma_f32_16x16x128_f8f6f4 v[112:115], a[48:55], v[56:63], v[112:115] cbsz:1 blgp:1\n\t"
      "ds_read_b128 v[40:43], v240 offset:10240\n\t"
      "global_store_dwordx4 v211, v[100:103], s[30:31] sc0 sc1\n\t"
      "s_add_u32 s30, s30, 0x40000\n\t"
      "s_addc_u32 s31, s31, 0\n\t"
      "v_mfma_f32_16x16x128_f8f6f4 v[116:119], a[56:63], v[56:63], v[116:119] cbsz:1 blgp:1\n\t"
      "s_waitcnt lgkmcnt(3)\n\t"
      "v_min_u32 v228, v228, v229\n\t"
      "v_cmp_eq_u32 vcc, 0, v228\n\t"
      "ds_read_b128 v[44:47], v241 offset:10240\n\t"
      "v_mfma_f32_16x16x128_f8f6f4 v[112:115], a[160:167], v[64:71], v[112:115] cbsz:1 blgp:1\n\t"
      "s_cbranch_vccnz XS109_%=\n\t"
      "XR110_%=:\n\t"
      "ds_add_u32 v203, v204 offset:256\n\t"
      "ds_read_b32 v253, v203 offset:0\n\t"
      "ds_read_b128 v[48:51], v240 offset:12288\n\t"
      "v_mfma_f32_16x16x128_f8f6f4 v[116:119], a[168:175], v[64:71], v[116:119] cbsz:1 blgp:1\n\t"
      "ds_read_b128 v[52:55], v241 offset:12288\n\t"
      "s_waitcnt lgkmcnt(6)\n\t"
      "v_mfma_f32_16x16x128_f8f6f4 v[104:107], a[64:71], v[32:39], v[104:107] cbsz:1 blgp:1\n\t"
      "ds_read_b128 v[56:59], v240 offset:14336\n\t"
      "s_add_u32 m0, s52, 98304\n\t"
      "s_add_u32 s80, s19, 24576\n\t"
      "buffer_load_dwordx4 v200, s[44:47], s80 offen offset:0 sc0 sc1 lds\n\t"
      "v_mfma_f32_16x16x128_f8f6f4 v[108:111], a[72:79], v[32:39], v[108:111] cbsz:1 blgp:1\n\t"
      "s_waitcnt lgkmcnt(3)\n\t"
      "v_cmp_gt_u32 vcc, 48, v253\n\t"
      "ds_read_b128 v[60:63], v241 offset:14336\n\t"
      "v_mfma_f32_16x16x128_f8f6f4 v[104:107], a[176:183], v[40:47], v[104:107] cbsz:1 blgp:1\n\t"
      "s_cbranch_vccnz AQ113_%=\n\t"
      "AR114_%=:\n\t"
      "s_waitcnt vmcnt(9)\n\t"
      "ds_read2st64_b32 v[228:229], v251 offset0:224 offset1:228\n\t"
      "ds_read_b128 v[64:67], v242 offset:16384\n\t"
      "buffer_load_dwordx4 v200, s[44:47], s80 offen offset:1024 sc0 sc1 lds\n\t"
      "v_mfma_f32_16x16x128_f8f6f4 v[108:111], a[184:191], v[40:47], v[108:111] cbsz:1 blgp:1\n\t"
      "ds_read_b128 v[68:71], v243 offset:16384\n\t"
      "s_waitcnt lgkmcnt(5)\n\t"
      "v_mfma_f32_16x16x128_f8f6f4 v[112:115], a[64:71], v[48:55], v[112:115] cbsz:1 blgp:1\n\t"
      "ds_read_b128 v[32:35], v242 offset:18432\n\t"
      "v_mfma_f32_16x16x128_f8f6f4 v[116:119], a[72:79], v[48:55], v[116:119] cbsz:1 blgp:1\n\t"
      "s_waitcnt lgkmcnt(3)\n\t"
      "v_min_u32 v228, v228, v229\n\t"
      "v_cmp_eq_u32 vcc, 0, v228\n\t"
      "ds_read_b128 v[36:39], v243 offset:18432\n\t"
      "v_mfma_f32_16x16x128_f8f6f4 v[112:115], a[176:183], v[56:63], v[112:115] cbsz:1 blgp:1\n\t"
      "s_cbranch_vccnz XS115_%=\n\t"
      "XR116_%=:\n\t"
      "ds_add_u32 v203, v204 offset:0\n\t"
      "ds_read_b32 v253, v203 offset:256\n\t"
      "ds_read_b128 v[40:43], v242 offset:20480\n\t"
      "v_mfma_f32_16x16x128_f8f6f4 v[116:119], a[184:191], v[56:63], v[116:119] cbsz:1 blgp:1\n\t"
      "ds_read_b128 v[44:47], v243 offset:20480\n\t"
      "s_waitcnt lgkmcnt(6)\n\t"
      "v_mfma_f32_16x16x128_f8f6f4 v[104:107], a[80:87], v[64:71], v[104:107] cbsz:1 blgp:1\n\t"
      "ds_read_b128 v[48:51], v242 offset:22528\n\t"
      "s_add_u32 m0, s52, 40960\n\t"
      "s_add_u32 s80, s20, 24576\n\t"
      "buffer_load_dwordx4 v200, s[44:47], s80 offen offset:0 sc0 sc1 lds\n\t"
      "v_mfma_f32_16x16x128_f8f6f4 v[108:111], a[88:95], v[64:71], v[108:111] cbsz:1 blgp:1\n\t"
      "s_waitcnt lgkmcnt(3)\n\t"
      "v_cmp_gt_u32 vcc, 52, v253\n\t"
      "ds_read_b128 v[52:55], v243 offset:22528\n\t"
      "v_mfma_f32_16x16x128_f8f6f4 v[104:107], a[192:199], v[32:39], v[104:107] cbsz:1 blgp:1\n\t"
      "s_cbranch_vccnz AQ119_%=\n\t"
      "AR120_%=:\n\t"
      "s_waitcnt vmcnt(9)\n\t"
      "ds_read2st64_b32 v[228:229], v251 offset0:192 offset1:196\n\t"
      "ds_read_b128 v[56:59], v240 offset:0\n\t"
      "buffer_load_dwordx4 v200, s[44:47], s80 offen offset:1024 sc0 sc1 lds\n\t"
      "v_mfma_f32_16x16x128_f8f6f4 v[108:111], a[200:207], v[32:39], v[108:111] cbsz:1 blgp:1\n\t"
      "ds_read_b128 v[60:63], v241 offset:0\n\t"
      "s_waitcnt lgkmcnt(5)\n\t"
      "v_mfma_f32_16x16x128_f8f6f4 v[112:115], a[80:87], v[40:47], v[112:115] cbsz:1 blgp:1\n\t"
      "ds_read_b128 v[64:67], v240 offset:2048\n\t"
      "v_mfma_f32_16x16x128_f8f6f4 v[116:119], a[88:95], v[40:47], v[116:119] cbsz:1 blgp:1\n\t"
      "s_waitcnt lgkmcnt(3)\n\t"
      "v_min_u32 v228, v228, v229\n\t"
      "v_cmp_eq_u32 vcc, 0, v228\n\t"
      "ds_read_b128 v[68:71], v241 offset:2048\n\t"
      "v_mfma_f32_16x16x128_f8f6f4 v[112:115], a[192:199], v[48:55], v[112:115] cbsz:1 blgp:1\n\t"
      "s_cbranch_vccnz XS121_%=\n\t"
      "XR122_%=:\n\t"
      "ds_add_u32 v203, v204 offset:256\n\t"
      "ds_read_b32 v253, v203 offset:0\n\t"
      "ds_read_b128 v[32:35], v240 offset:4096\n\t"
      "v_mfma_f32_16x16x128_f8f6f4 v[116:119], a[200:207], v[48:55], v[116:119] cbsz:1 blgp:1\n\t"
      "ds_read_b128 v[36:39], v241 offset:4096\n\t"
      "s_waitcnt lgkmcnt(6)\n\t"
      "v_mfma_f32_16x16x128_f8f6f4 v[104:107], a[96:103], v[56:63], v[104:107] cbsz:1 blgp:1\n\t"
      "ds_read_b128 v[40:43], v240 offset:6144\n\t"
      "s_add_u32 m0, s52, 32768\n\t"
      "s_add_u32 s80, s21, 24576\n\t"
      "buffer_load_dwordx4 v200, s[44:47], s80 offen offset:0 sc0 sc1 lds\n\t"
      "v_mfma_f32_16x16x128_f8f6f4 v[108:111], a[104:111], v[56:63], v[108:111] cbsz:1 blgp:1\n\t"
      "s_waitcnt lgkmcnt(3)\n\t"
      "v_cmp_gt_u32 vcc, 52, v253\n\t"
      "ds_read_b128 v[44:47], v241 offset:6144\n\t"
      "v_mfma_f32_16x16x128_f8f6f4 v[104:107], a[208:215], v[64:71], v[104:107] cbsz:1 blgp:1\n\t"
      "s_cbranch_vccnz AQ125_%=\n\t"
      "AR126_%=:\n\t"
      "s_waitcnt vmcnt(7)\n\t"
      "ds_read2st64_b32 v[228:229], v251 offset0:160 offset1:164\n\t"
      "ds_read_b128 v[48:51], v240 offset:57344\n\t"
      "buffer_load_dwordx4 v200, s[44:47], s80 offen offset:1024 sc0 sc1 lds\n\t"
      "s_waitcnt vmcnt(6)\n\t"
      "v_mfma_f32_16x16x128_f8f6f4 v[108:111], a[216:223], v[64:71], v[108:111] cbsz:1 blgp:1\n\t"
      "ds_read_b128 v[52:55], v241 offset:57344\n\t"
      "v_mov_b32 v218, s64\n\t"
      "global_store_dword v212, v218, s[16:17] offset:128 sc0 sc1\n\t"
      "s_waitcnt lgkmcnt(5)\n\t"
      "v_mfma_f32_16x16x128_f8f6f4 v[112:115], a[96:103], v[32:39], v[112:115] cbsz:1 blgp:1\n\t"
      "ds_read_b128 v[56:59], v240 offset:59392\n\t"
      "v_mfma_f32_16x16x128_f8f6f4 v[116:119], a[104:111], v[32:39], v[116:119] cbsz:1 blgp:1\n\t"
      "s_waitcnt lgkmcnt(3)\n\t"
      "v_min_u32 v228, v228, v229\n\t"
      "v_cmp_eq_u32 vcc, 0, v228\n\t"
      "ds_read_b128 v[60:63], v241 offset:59392\n\t"
      "v_mfma_f32_16x16x128_f8f6f4 v[112:115], a[208:215], v[40:47], v[112:115] cbsz:1 blgp:1\n\t"
      "s_cbranch_vccnz XS127_%=\n\t"
      "XR128_%=:\n\t"
      "ds_add_u32 v203, v204 offset:0\n\t"
      "ds_read_b32 v253, v203 offset:256\n\t"
      "ds_read_b128 v[64:67], v240 offset:61440\n\t"
      "v_mfma_f32_16x16x128_f8f6f4 v[116:119], a[216:223], v[40:47], v[116:119] cbsz:1 blgp:1\n\t"
      "ds_read_b128 v[68:71], v241 offset:61440\n\t"
      "s_waitcnt lgkmcnt(6)\n\t"
      "v_mfma_f32_16x16x128_f8f6f4 v[104:107], a[112:119], v[48:55], v[104:107] cbsz:1 blgp:1\n\t"
      "ds_read_b128 v[32:35], v240 offset:63488\n\t"
      "s_add_u32 m0, s52, 73728\n\t"
      "s_add_u32 s80, s24, 24576\n\t"
      "buffer_load_dwordx4 v200, s[44:47], s80 offen offset:0 sc0 sc1 lds\n\t"
      "v_mfma_f32_16x16x128_f8f6f4 v[108:111], a[120:127], v[48:55], v[108:111] cbsz:1 blgp:1\n\t"
      "s_waitcnt lgkmcnt(3)\n\t"
      "v_cmp_gt_u32 vcc, 56, v253\n\t"
      "ds_read_b128 v[36:39], v241 offset:63488\n\t"
      "v_mfma_f32_16x16x128_f8f6f4 v[104:107], a[224:231], v[56:63], v[104:107] cbsz:1 blgp:1\n\t"
      "s_cbranch_vccnz AQ131_%=\n\t"
      "AR132_%=:\n\t"
      "s_waitcnt vmcnt(6)\n\t"
      "ds_read2st64_b32 v[228:229], v251 offset0:128 offset1:132\n\t"
      "ds_read_b128 v[40:43], v240 offset:49152\n\t"
      "buffer_load_dwordx4 v200, s[44:47], s80 offen offset:1024 sc0 sc1 lds\n\t"
      "v_mfma_f32_16x16x128_f8f6f4 v[108:111], a[232:239], v[56:63], v[108:111] cbsz:1 blgp:1\n\t"
      "ds_read_b128 v[44:47], v241 offset:49152\n\t"
      "s_waitcnt lgkmcnt(5)\n\t"
      "v_mfma_f32_16x16x128_f8f6f4 v[112:115], a[112:119], v[64:71], v[112:115] cbsz:1 blgp:1\n\t"
      "ds_read_b128 v[48:51], v240 offset:51200\n\t"
      "v_mfma_f32_16x16x128_f8f6f4 v[116:119], a[120:127], v[64:71], v[116:119] cbsz:1 blgp:1\n\t"
      "s_waitcnt lgkmcnt(3)\n\t"
      "v_min_u32 v228, v228, v229\n\t"
      "v_cmp_eq_u32 vcc, 0, v228\n\t"
      "ds_read_b128 v[52:55], v241 offset:51200\n\t"
      "v_mfma_f32_16x16x128_f8f6f4 v[112:115], a[224:231], v[32:39], v[112:115] cbsz:1 blgp:1\n\t"
      "s_cbranch_vccnz XS133_%=\n\t"
      "XR134_%=:\n\t"
      "ds_add_u32 v203, v204 offset:256\n\t"
      "ds_read_b32 v253, v203 offset:0\n\t"
      "ds_read_b128 v[56:59], v240 offset:53248\n\t"
      "v_mfma_f32_16x16x128_f8f6f4 v[116:119], a[232:239], v[32:39], v[116:119] cbsz:1 blgp:1\n\t"
      "ds_read_b128 v[60:63], v241 offset:53248\n\t"
      "s_waitcnt lgkmcnt(6)\n\t"
      "v_mfma_f32_16x16x128_f8f6f4 v[104:107], a[128:135], v[40:47], v[104:107] cbsz:1 blgp:1\n\t"
      "ds_read_b128 v[64:67], v240 offset:55296\n\t"
      "s_add_u32 m0, s52, 147456\n\t"
      "s_add_u32 s80, s25, 24576\n\t"
      "buffer_load_dwordx4 v200, s[44:47], s80 offen offset:0 sc0 sc1 lds\n\t"
      "v_mfma_f32_16x16x128_f8f6f4 v[108:111], a[136:143], v[40:47], v[108:111] cbsz:1 blgp:1\n\t"
      "s_waitcnt lgkmcnt(3)\n\t"
      "v_cmp_gt_u32 vcc, 56, v253\n\t"
      "ds_read_b128 v[68:71], v241 offset:55296\n\t"
      "v_mfma_f32_16x16x128_f8f6f4 v[104:107], a[240:247], v[48:55], v[104:107] cbsz:1 blgp:1\n\t"
      "s_cbranch_vccnz AQ137_%=\n\t"
      "AR138_%=:\n\t"
      "s_waitcnt vmcnt(6)\n\t"
      "ds_read2st64_b32 v[228:229], v250 offset0:160 offset1:164\n\t"
      "ds_read_b128 v[32:35], v240 offset:40960\n\t"
      "buffer_load_dwordx4 v200, s[44:47], s80 offen offset:1024 sc0 sc1 lds\n\t"
      "v_mfma_f32_16x16x128_f8f6f4 v[108:111], a[248:255], v[48:55], v[108:111] cbsz:1 blgp:1\n\t"
      "ds_read_b128 v[36:39], v241 offset:40960\n\t"
      "global_load_dword v185, v213, s[16:17] offset:128 sc0 sc1\n\t"
      "s_waitcnt lgkmcnt(5)\n\t"
      "v_mfma_f32_16x16x128_f8f6f4 v[112:115], a[128:135], v[56:63], v[112:115] cbsz:1 blgp:1\n\t"
      "ds_read_b128 v[40:43], v240 offset:43008\n\t"
      "v_mfma_f32_16x16x128_f8f6f4 v[116:119], a[136:143], v[56:63], v[116:119] cbsz:1 blgp:1\n\t"
      "s_waitcnt lgkmcnt(3)\n\t"
      "v_min_u32 v228, v228, v229\n\t"
      "v_cmp_eq_u32 vcc, 0, v228\n\t"
      "ds_read_b128 v[44:47], v241 offset:43008\n\t"
      "v_mfma_f32_16x16x128_f8f6f4 v[112:115], a[240:247], v[64:71], v[112:115] cbsz:1 blgp:1\n\t"
      "s_cbranch_vccnz XS139_%=\n\t"
      "XR140_%=:\n\t"
      "ds_add_u32 v203, v204 offset:0\n\t"
      "ds_read_b32 v253, v203 offset:256\n\t"
      "ds_read_b128 v[48:51], v240 offset:45056\n\t"
      "v_mfma_f32_16x16x128_f8f6f4 v[116:119], a[248:255], v[64:71], v[116:119] cbsz:1 blgp:1\n\t"
      "ds_read_b128 v[52:55], v241 offset:45056\n\t"
      "s_waitcnt lgkmcnt(6)\n\t"
      "v_mfma_f32_16x16x128_f8f6f4 v[120:123], a[32:39], v[32:39], v[120:123] cbsz:1 blgp:1\n\t"
      "ds_read_b128 v[56:59], v240 offset:47104\n\t"
      "s_add_u32 m0, s52, 65536\n\t"
      "s_add_u32 s80, s26, 24576\n\t"
      "buffer_load_dwordx4 v200, s[44:47], s80 offen offset:0 sc0 sc1 lds\n\t"
      "v_mfma_f32_16x16x128_f8f6f4 v[124:127], a[40:47], v[32:39], v[124:127] cbsz:1 blgp:1\n\t"
      "s_waitcnt lgkmcnt(3)\n\t"
      "v_cmp_gt_u32 vcc, 60, v253\n\t"
      "ds_read_b128 v[60:63], v241 offset:47104\n\t"
      "v_mfma_f32_16x16x128_f8f6f4 v[120:123], a[144:151], v[40:47], v[120:123] cbsz:1 blgp:1\n\t"
      "s_cbranch_vccnz AQ143_%=\n\t"
      "AR144_%=:\n\t"
      "s_waitcnt vmcnt(7)\n\t"
      "ds_read2st64_b32 v[228:229], v250 offset0:128 offset1:132\n\t"
      "ds_read_b128 v[64:67], v240 offset:32768\n\t"
      "buffer_load_dwordx4 v200, s[44:47], s80 offen offset:1024 sc0 sc1 lds\n\t"
      "v_mfma_f32_16x16x128_f8f6f4 v[124:127], a[152:159], v[40:47], v[124:127] cbsz:1 blgp:1\n\t"
      "ds_read_b128 v[68:71], v241 offset:32768\n\t"
      "s_waitcnt lgkmcnt(5)\n\t"
      "v_mfma_f32_16x16x128_f8f6f4 v[128:131], a[32:39], v[48:55], v[128:131] cbsz:1 blgp:1\n\t"
      "ds_read_b128 v[32:35], v240 offset:34816\n\t"
      "v_mfma_f32_16x16x128_f8f6f4 v[132:135], a[40:47], v[48:55], v[132:135] cbsz:1 blgp:1\n\t"
      "s_waitcnt lgkmcnt(3)\n\t"
      "v_min_u32 v228, v228, v229\n\t"
      "v_cmp_eq_u32 vcc, 0, v228\n\t"
      "ds_read_b128 v[36:39], v241 offset:34816\n\t"
      "v_mfma_f32_16x16x128_f8f6f4 v[128:131], a[144:151], v[56:63], v[128:131] cbsz:1 blgp:1\n\t"
      "s_cbranch_vccnz XS145_%=\n\t"
      "XR146_%=:\n\t"
      "ds_add_u32 v203, v204 offset:256\n\t"
      "ds_read_b32 v253, v203 offset:0\n\t"
      "ds_read_b128 v[40:43], v240 offset:36864\n\t"
      "v_mfma_f32_16x16x128_f8f6f4 v[132:135], a[152:159], v[56:63], v[132:135] cbsz:1 blgp:1\n\t"
      "ds_read_b128 v[44:47], v241 offset:36864\n\t"
      "s_waitcnt lgkmcnt(6)\n\t"
      "v_mfma_f32_16x16x128_f8f6f4 v[120:123], a[48:55], v[64:71], v[120:123] cbsz:1 blgp:1\n\t"
      "ds_read_b128 v[48:51], v240 offset:38912\n\t"
      "v_mfma_f32_16x16x128_f8f6f4 v[124:127], a[56:63], v[64:71], v[124:127] cbsz:1 blgp:1\n\t"
      "s_waitcnt lgkmcnt(3)\n\t"
      "v_cmp_gt_u32 vcc, 60, v253\n\t"
      "ds_read_b128 v[52:55], v241 offset:38912\n\t"
      "v_mfma_f32_16x16x128_f8f6f4 v[120:123], a[160:167], v[32:39], v[120:123] cbsz:1 blgp:1\n\t"
      "s_cbranch_vccnz AQ149_%=\n\t"
      "AR150_%=:\n\t"
      "s_waitcnt vmcnt(5)\n\t"
      "ds_read2st64_b32 v[228:229], v251 offset0:32 offset1:36\n\t"
      "ds_read_b128 v[56:59], v238 offset:40960\n\t"
      "v_mfma_f32_16x16x128_f8f6f4 v[124:127], a[168:175], v[32:39], v[124:127] cbsz:1 blgp:1\n\t"
      "ds_read_b128 v[60:63], v239 offset:40960\n\t"
      "s_waitcnt lgkmcnt(5)\n\t"
      "v_mfma_f32_16x16x128_f8f6f4 v[128:131], a[48:55], v[40:47], v[128:131] cbsz:1 blgp:1\n\t"
      "ds_read_b128 v[64:67], v238 offset:43008\n\t"
      "v_mfma_f32_16x16x128_f8f6f4 v[132:135], a[56:63], v[40:47], v[132:135] cbsz:1 blgp:1\n\t"
      "s_waitcnt lgkmcnt(3)\n\t"
      "v_min_u32 v228, v228, v229\n\t"
      "v_cmp_eq_u32 vcc, 0, v228\n\t"
      "ds_read_b128 v[68:71], v239 offset:43008\n\t"
      "v_mfma_f32_16x16x128_f8f6f4 v[128:131], a[160:167], v[48:55], v[128:131] cbsz:1 blgp:1\n\t"
      "s_cbranch_vccnz XS151_%=\n\t"
      "XR152_%=:\n\t"
      "ds_add_u32 v203, v204 offset:0\n\t"
      "ds_read_b32 v253, v203 offset:256\n\t"
      "ds_read_b128 v[32:35], v238 offset:45056\n\t"
      "v_mfma_f32_16x16x128_f8f6f4 v[132:135], a[168:175], v[48:55], v[132:135] cbsz:1 blgp:1\n\t"
      "ds_read_b128 v[36:39], v239 offset:45056\n\t"
      "s_waitcnt lgkmcnt(6)\n\t"
      "v_mfma_f32_16x16x128_f8f6f4 v[120:123], a[64:71], v[56:63], v[120:123] cbsz:1 blgp:1\n\t"
      "ds_read_b128 v[40:43], v238 offset:47104\n\t"
      "v_mfma_f32_16x16x128_f8f6f4 v[124:127], a[72:79], v[56:63], v[124:127] cbsz:1 blgp:1\n\t"
      "s_waitcnt lgkmcnt(3)\n\t"
      "v_cmp_gt_u32 vcc, 64, v253\n\t"
      "ds_read_b128 v[44:47], v239 offset:47104\n\t"
      "v_mfma_f32_16x16x128_f8f6f4 v[120:123], a[176:183], v[64:71], v[120:123] cbsz:1 blgp:1\n\t"
      "s_cbranch_vccnz AQ155_%=\n\t"
      "AR156_%=:\n\t"
      "s_waitcnt vmcnt(3)\n\t"
      "ds_read2st64_b32 v[228:229], v252 offset0:64 offset1:68\n\t"
      "ds_read_b128 v[48:51], v238 offset:32768\n\t"
      "s_waitcnt vmcnt(2)\n\t"
      "v_mfma_f32_16x16x128_f8f6f4 v[124:127], a[184:191], v[64:71], v[124:127] cbsz:1 blgp:1\n\t"
      "ds_read_b128 v[52:55], v239 offset:32768\n\t"
      "v_cmp_ne_u32 vcc, s64, v185\n\t"
      "s_cbranch_vccnz FS157_%=\n\t"
      "FR158_%=:\n\t"
      "global_load_dwordx4 v[136:139], v210, s[94:95] sc0 sc1\n\t"
      "s_waitcnt lgkmcnt(5)\n\t"
      "v_mfma_f32_16x16x128_f8f6f4 v[128:131], a[64:71], v[32:39], v[128:131] cbsz:1 blgp:1\n\t"
      "ds_read_b128 v[56:59], v238 offset:34816\n\t"
      "global_load_dwordx4 v[140:143], v211, s[94:95] sc0 sc1\n\t"
      "s_add_u32 s94, s94, 0x40000\n\t"
      "s_addc_u32 s95, s95, 0\n\t"
      "v_mfma_f32_16x16x128_f8f6f4 v[132:135], a[72:79], v[32:39], v[132:135] cbsz:1 blgp:1\n\t"
      "s_waitcnt lgkmcnt(3)\n\t"
      "v_min_u32 v228, v228, v229\n\t"
      "v_cmp_eq_u32 vcc, 0, v228\n\t"
      "ds_read_b128 v[60:63], v239 offset:34816\n\t"
      "v_mfma_f32_16x16x128_f8f6f4 v[128:131], a[176:183], v[40:47], v[128:131] cbsz:1 blgp:1\n\t"
      "s_cbranch_vccnz XS159_%=\n\t"
      "XR160_%=:\n\t"
      "ds_add_u32 v203, v204 offset:256\n\t"
      "ds_read_b32 v253, v203 offset:0\n\t"
      "ds_read_b128 v[64:67], v238 offset:36864\n\t"
      "global_load_dwordx4 v[144:147], v210, s[94:95] sc0 sc1\n\t"
      "v_mfma_f32_16x16x128_f8f6f4 v[132:135], a[184:191], v[40:47], v[132:135] cbsz:1 blgp:1\n\t"
      "ds_read_b128 v[68:71], v239 offset:36864\n\t"
      "global_load_dwordx4 v[148:151], v211, s[94:95] sc0 sc1\n\t"
      "s_add_u32 s94, s94, 0x40000\n\t"
      "s_addc_u32 s95, s95, 0\n\t"
      "s_waitcnt lgkmcnt(6)\n\t"
      "v_mfma_f32_16x16x128_f8f6f4 v[120:123], a[80:87], v[48:55], v[120:123] cbsz:1 blgp:1\n\t"
      "ds_read_b128 v[32:35], v238 offset:38912\n\t"
      "global_load_dwordx4 v[152:155], v210, s[94:95] sc0 sc1\n\t"
      "v_mfma_f32_16x16x128_f8f6f4 v[124:127], a[88:95], v[48:55], v[124:127] cbsz:1 blgp:1\n\t"
      "s_waitcnt lgkmcnt(3)\n\t"
      "v_cmp_gt_u32 vcc, 64, v253\n\t"
      "ds_read_b128 v[36:39], v239 offset:38912\n\t"
      "v_mfma_f32_16x16x128_f8f6f4 v[120:123], a[192:199], v[56:63], v[120:123] cbsz:1 blgp:1\n\t"
      "s_cbranch_vccnz AQ163_%=\n\t"
      "AR164_%=:\n\t"
      "s_waitcnt vmcnt(5)\n\t"
      "ds_read2st64_b32 v[228:229], v251 offset0:0 offset1:4\n\t"
      "ds_read_b128 v[40:43], v240 offset:8192\n\t"
      "global_load_dwordx4 v[156:159], v211, s[94:95] sc0 sc1\n\t"
      "v_mfma_f32_16x16x128_f8f6f4 v[124:127], a[200:207], v[56:63], v[124:127] cbsz:1 blgp:1\n\t"
      "ds_read_b128 v[44:47], v241 offset:8192\n\t"
      "s_add_u32 s94, s94, 0x40000\n\t"
      "s_addc_u32 s95, s95, 0\n\t"
      "global_load_dwordx4 v[160:163], v210, s[94:95] sc0 sc1\n\t"
      "s_waitcnt lgkmcnt(5)\n\t"
      "v_mfma_f32_16x16x128_f8f6f4 v[128:131], a[80:87], v[64:71], v[128:131] cbsz:1 blgp:1\n\t"
      "ds_read_b128 v[48:51], v240 offset:10240\n\t"
      "global_load_dwordx4 v[164:167], v211, s[94:95] sc0 sc1\n\t"
      "s_add_u32 s94, s94, 0x40000\n\t"
      "s_addc_u32 s95, s95, 0\n\t"
      "v_mfma_f32_16x16x128_f8f6f4 v[132:135], a[88:95], v[64:71], v[132:135] cbsz:1 blgp:1\n\t"
      "s_waitcnt lgkmcnt(3)\n\t"
      "v_min_u32 v228, v228, v229\n\t"
      "v_cmp_eq_u32 vcc, 0, v228\n\t"
      "ds_read_b128 v[52:55], v241 offset:10240\n\t"
      "v_mfma_f32_16x16x128_f8f6f4 v[128:131], a[192:199], v[32:39], v[128:131] cbsz:1 blgp:1\n\t"
      "s_cbranch_vccnz XS165_%=\n\t"
      "XR166_%=:\n\t"
      "ds_add_u32 v203, v204 offset:0\n\t"
      "ds_read_b32 v253, v203 offset:256\n\t"
      "ds_read_b128 v[56:59], v240 offset:12288\n\t"
      "v_mfma_f32_16x16x128_f8f6f4 v[132:135], a[200:207], v[32:39], v[132:135] cbsz:1 blgp:1\n\t"
      "ds_read_b128 v[60:63], v241 offset:12288\n\t"
      "s_waitcnt lgkmcnt(6)\n\t"
      "v_mfma_f32_16x16x128_f8f6f4 v[120:123], a[96:103], v[40:47], v[120:123] cbsz:1 blgp:1\n\t"
      "ds_read_b128 v[64:67], v240 offset:14336\n\t"
      "v_mfma_f32_16x16x128_f8f6f4 v[124:127], a[104:111], v[40:47], v[124:127] cbsz:1 blgp:1\n\t"
      "s_waitcnt lgkmcnt(3)\n\t"
      "v_cmp_gt_u32 vcc, 68, v253\n\t"
      "ds_read_b128 v[68:71], v241 offset:14336\n\t"
      "v_mfma_f32_16x16x128_f8f6f4 v[120:123], a[208:215], v[48:55], v[120:123] cbsz:1 blgp:1\n\t"
      "s_cbranch_vccnz AQ169_%=\n\t"
      "AR170_%=:\n\t"
      "ds_read_b128 v[32:35], v242 offset:16384\n\t"
      "v_mfma_f32_16x16x128_f8f6f4 v[124:127], a[216:223], v[48:55], v[124:127] cbsz:1 blgp:1\n\t"
      "ds_read_b128 v[36:39], v243 offset:16384\n\t"
      "s_waitcnt lgkmcnt(4)\n\t"
      "v_mfma_f32_16x16x128_f8f6f4 v[128:131], a[96:103], v[56:63], v[128:131] cbsz:1 blgp:1\n\t"
      "ds_read_b128 v[40:43], v242 offset:18432\n\t"
      "v_mfma_f32_16x16x128_f8f6f4 v[132:135], a[104:111], v[56:63], v[132:135] cbsz:1 blgp:1\n\t"
      "ds_read_b128 v[44:47], v243 offset:18432\n\t"
      "s_waitcnt lgkmcnt(4)\n\t"
      "v_mfma_f32_16x16x128_f8f6f4 v[128:131], a[208:215], v[64:71], v[128:131] cbsz:1 blgp:1\n\t"
      "ds_read_b32 v253, v203 offset:0\n\t"
      "ds_read_b128 v[48:51], v242 offset:20480\n\t"
      "v_mfma_f32_16x16x128_f8f6f4 v[132:135], a[216:223], v[64:71], v[132:135] cbsz:1 blgp:1\n\t"
      "ds_read_b128 v[52:55], v243 offset:20480\n\t"
      "s_waitcnt lgkmcnt(5)\n\t"
      "v_mfma_f32_16x16x128_f8f6f4 v[120:123], a[112:119], v[32:39], v[120:123] cbsz:1 blgp:1\n\t"
      "ds_read_b128 v[56:59], v242 offset:22528\n\t"
      "v_mfma_f32_16x16x128_f8f6f4 v[124:127], a[120:127], v[32:39], v[124:127] cbsz:1 blgp:1\n\t"
      "s_waitcnt lgkmcnt(3)\n\t"
      "v_cmp_gt_u32 vcc, 68, v253\n\t"
      "ds_read_b128 v[60:63], v243 offset:22528\n\t"
      "v_mfma_f32_16x16x128_f8f6f4 v[120:123], a[224:231], v[40:47], v[120:123] cbsz:1 blgp:1\n\t"
      "s_cbranch_vccnz AQ171_%=\n\t"
      "AR172_%=:\n\t"
      "ds_read_b128 v[64:67], v240 offset:0\n\t"
      "s_waitcnt vmcnt(6)\n\t"
      "v_mul_f32 v136, s75, v136\n\t"
      "v_mul_f32 v137, s75, v137\n\t"
      "v_mul_f32 v138, s75, v138\n\t"
      "v_mfma_f32_16x16x128_f8f6f4 v[124:127], a[232:239], v[40:47], v[124:127] cbsz:1 blgp:1\n\t"
      "ds_read_b128 v[68:71], v241 offset:0\n\t"
      "v_mul_f32 v139, s75, v139\n\t"
      "v_mul_f32 v140, s75, v140\n\t"
      "v_mul_f32 v141, s75, v141\n\t"
      "v_mul_f32 v142, s75, v142\n\t"
      "s_waitcnt lgkmcnt(4)\n\t"
      "v_mfma_f32_16x16x128_f8f6f4 v[128:131], a[112:119], v[48:55], v[128:131] cbsz:1 blgp:1\n\t"
      "ds_read_b128 v[32:35], v240 offset:2048\n\t"
      "v_mul_f32 v143, s75, v143\n\t"
      "s_waitcnt vmcnt(4)\n\t"
      "v_mul_f32 v144, s75, v144\n\t"
      "v_mul_f32 v145, s75, v145\n\t"
      "v_mul_f32 v146, s75, v146\n\t"
      "v_mfma_f32_16x16x128_f8f6f4 v[132:135], a[120:127], v[48:55], v[132:135] cbsz:1 blgp:1\n\t"
      "ds_read_b128 v[36:39], v241 offset:2048\n\t"
      "v_mul_f32 v147, s75, v147\n\t"
      "v_mul_f32 v148, s75, v148\n\t"
      "v_mul_f32 v149, s75, v149\n\t"
      "v_mul_f32 v150, s75, v150\n\t"
      "s_waitcnt lgkmcnt(4)\n\t"
      "v_mfma_f32_16x16x128_f8f6f4 v[128:131], a[224:231], v[56:63], v[128:131] cbsz:1 blgp:1\n\t"
      "ds_read_b128 v[40:43], v240 offset:4096\n\t"
      "v_mul_f32 v151, s75, v151\n\t"
      "v_fma_f32 v104, s75, v104, v136\n\t"
      "v_fma_f32 v105, s75, v105, v137\n\t"
      "v_fma_f32 v106, s75, v106, v138\n\t"
      "v_fma_f32 v107, s75, v107, v139\n\t"
      "v_mfma_f32_16x16x128_f8f6f4 v[132:135], a[232:239], v[56:63], v[132:135] cbsz:1 blgp:1\n\t"
      "ds_read_b128 v[44:47], v241 offset:4096\n\t"
      "v_maximum3_f32 v104, 0, v104, 0\n\t"
      "v_maximum3_f32 v105, 0, v105, 0\n\t"
      "v_maximum3_f32 v106, 0, v106, 0\n\t"
      "v_maximum3_f32 v107, 0, v107, 0\n\t"
      "s_waitcnt lgkmcnt(4)\n\t"
      "v_mfma_f32_16x16x128_f8f6f4 v[120:123], a[128:135], v[64:71], v[120:123] cbsz:1 blgp:1\n\t"
      "ds_read_b128 v[48:51], v240 offset:6144\n\t"
      "ds_write_b128 v236, v[104:107] offset:0\n\t"
      "v_fma_f32 v108, s75, v108, v140\n\t"
      "v_fma_f32 v109, s75, v109, v141\n\t"
      "v_fma_f32 v110, s75, v110, v142\n\t"
      "v_fma_f32 v111, s75, v111, v143\n\t"
      "v_mfma_f32_16x16x128_f8f6f4 v[124:127], a[136:143], v[64:71], v[124:127] cbsz:1 blgp:1\n\t"
      "ds_read_b128 v[52:55], v241 offset:6144\n\t"
      "v_maximum3_f32 v108, 0, v108, 0\n\t"
      "v_maximum3_f32 v109, 0, v109, 0\n\t"
      "s_waitcnt lgkmcnt(5)\n\t"
      "v_mfma_f32_16x16x128_f8f6f4 v[120:123], a[240:247], v[32:39], v[120:123] cbsz:1 blgp:1\n\t"
      "v_maximum3_f32 v110, 0, v110, 0\n\t"
      "v_maximum3_f32 v111, 0, v111, 0\n\t"
      "ds_write_b128 v237, v[108:111] offset:0\n\t"
      "ds_read_b128 v[168:171], v209 offset:0\n\t"
      "ds_read_b128 v[172:175], v209 offset:1024\n\t"
      "v_fma_f32 v112, s75, v112, v144\n\t"
      "v_mfma_f32_16x16x128_f8f6f4 v[124:127], a[248:255], v[32:39], v[124:127] cbsz:1 blgp:1\n\t"
      "v_fma_f32 v113, s75, v113, v145\n\t"
      "v_fma_f32 v114, s75, v114, v146\n\t"
      "v_fma_f32 v115, s75, v115, v147\n\t"
      "v_maximum3_f32 v112, 0, v112, 0\n\t"
      "v_maximum3_f32 v113, 0, v113, 0\n\t"
      "s_waitcnt lgkmcnt(6)\n\t"
      "v_mfma_f32_16x16x128_f8f6f4 v[128:131], a[128:135], v[40:47], v[128:131] cbsz:1 blgp:1\n\t"
      "v_maximum3_f32 v114, 0, v114, 0\n\t"
      "v_maximum3_f32 v115, 0, v115, 0\n\t"
      "ds_write_b128 v236, v[112:115] offset:2048\n\t"
      "v_fma_f32 v116, s75, v116, v148\n\t"
      "v_fma_f32 v117, s75, v117, v149\n\t"
      "v_fma_f32 v118, s75, v118, v150\n\t"
      "v_mfma_f32_16x16x128_f8f6f4 v[132:135], a[136:143], v[40:47], v[132:135] cbsz:1 blgp:1\n\t"
      "v_fma_f32 v119, s75, v119, v151\n\t"
      "v_maximum3_f32 v116, 0, v116, 0\n\t"
      "v_maximum3_f32 v117, 0, v117, 0\n\t"
      "v_maximum3_f32 v118, 0, v118, 0\n\t"
      "v_maximum3_f32 v119, 0, v119, 0\n\t"
      "s_waitcnt lgkmcnt(4)\n\t"
      "v_mfma_f32_16x16x128_f8f6f4 v[128:131], a[240:247], v[48:55], v[128:131] cbsz:1 blgp:1\n\t"
      "ds_write_b128 v237, v[116:119] offset:2048\n\t"
      "ds_read_b128 v[176:179], v209 offset:2048\n\t"
      "ds_read_b128 v[180:183], v209 offset:3072\n\t"
      "s_waitcnt lgkmcnt(4)\n\t"
      "global_store_dwordx4 v210, v[168:171], s[22:23] sc0 sc1\n\t"
      "v_mfma_f32_16x16x128_f8f6f4 v[132:135], a[248:255], v[48:55], v[132:135] cbsz:1 blgp:1\n\t"
      "global_store_dwordx4 v211, v[172:175], s[22:23] sc0 sc1\n\t"
      "s_add_u32 s22, s22, 0x40000\n\t"
      "s_addc_u32 s23, s23, 0\n\t"
      "s_nop 7\n\t"
      "s_waitcnt lgkmcnt(0)\n\t"
      "global_store_dwordx4 v210, v[176:179], s[22:23] sc0 sc1\n\t"
      "global_store_dwordx4 v211, v[180:183], s[22:23] sc0 sc1\n\t"
      "s_add_u32 s22, s22, 0x40000\n\t"
      "s_addc_u32 s23, s23, 0\n\t"
      "s_waitcnt vmcnt(6)\n\t"
      "v_mul_f32 v152, s75, v152\n\t"
      "v_mul_f32 v153, s75, v153\n\t"
      "v_mul_f32 v154, s75, v154\n\t"
      "v_mul_f32 v155, s75, v155\n\t"
      "v_mul_f32 v156, s75, v156\n\t"
      "v_mul_f32 v157, s75, v157\n\t"
      "v_mul_f32 v158, s75, v158\n\t"
      "v_mul_f32 v159, s75, v159\n\t"
      "s_waitcnt vmcnt(4)\n\t"
      "v_mul_f32 v160, s75, v160\n\t"
      "v_mul_f32 v161, s75, v161\n\t"
      "v_mul_f32 v162, s75, v162\n\t"
      "v_mul_f32 v163, s75, v163\n\t"
      "v_mul_f32 v164, s75, v164\n\t"
      "v_mul_f32 v165, s75, v165\n\t"
      "v_mul_f32 v166, s75, v166\n\t"
      "v_mul_f32 v167, s75, v167\n\t"
      "v_fma_f32 v120, s75, v120, v152\n\t"
      "v_fma_f32 v121, s75, v121, v153\n\t"
      "v_fma_f32 v122, s75, v122, v154\n\t"
      "v_fma_f32 v123, s75, v123, v155\n\t"
      "v_maximum3_f32 v120, 0, v120, 0\n\t"
      "v_maximum3_f32 v121, 0, v121, 0\n\t"
      "v_maximum3_f32 v122, 0, v122, 0\n\t"
      "v_maximum3_f32 v123, 0, v123, 0\n\t"
      "ds_write_b128 v236, v[120:123] offset:0\n\t"
      "v_fma_f32 v124, s75, v124, v156\n\t"
      "v_fma_f32 v125, s75, v125, v157\n\t"
      "v_fma_f32 v126, s75, v126, v158\n\t"
      "v_fma_f32 v127, s75, v127, v159\n\t"
      "v_maximum3_f32 v124, 0, v124, 0\n\t"
      "v_maximum3_f32 v125, 0, v125, 0\n\t"
      "v_maximum3_f32 v126, 0, v126, 0\n\t"
      "v_maximum3_f32 v127, 0, v127, 0\n\t"
      "ds_write_b128 v237, v[124:127] offset:0\n\t"
      "ds_read_b128 v[168:171], v209 offset:0\n\t"
      "ds_read_b128 v[172:175], v209 offset:1024\n\t"
      "v_fma_f32 v128, s75, v128, v160\n\t"
      "v_fma_f32 v129, s75, v129, v161\n\t"
      "v_fma_f32 v130, s75, v130, v162\n\t"
      "v_fma_f32 v131, s75, v131, v163\n\t"
      "v_maximum3_f32 v128, 0, v128, 0\n\t"
      "v_maximum3_f32 v129, 0, v129, 0\n\t"
      "v_maximum3_f32 v130, 0, v130, 0\n\t"
      "v_maximum3_f32 v131, 0, v131, 0\n\t"
      "ds_write_b128 v236, v[128:131] offset:2048\n\t"
      "v_fma_f32 v132, s75, v132, v164\n\t"
      "v_fma_f32 v133, s75, v133, v165\n\t"
      "v_fma_f32 v134, s75, v134, v166\n\t"
      "v_fma_f32 v135, s75, v135, v167\n\t"
      "v_maximum3_f32 v132, 0, v132, 0\n\t"
      "v_maximum3_f32 v133, 0, v133, 0\n\t"
      "v_maximum3_f32 v134, 0, v134, 0\n\t"
      "v_maximum3_f32 v135, 0, v135, 0\n\t"
      "ds_write_b128 v237, v[132:135] offset:2048\n\t"
      "ds_read_b128 v[176:179], v209 offset:2048\n\t"
      "ds_read_b128 v[180:183], v209 offset:3072\n\t"
      "s_waitcnt lgkmcnt(4)\n\t"
      "global_store_dwordx4 v210, v[168:171], s[22:23] sc0 sc1\n\t"
      "global_store_dwordx4 v211, v[172:175], s[22:23] sc0 sc1\n\t"
      "s_add_u32 s22, s22, 0x40000\n\t"
      "s_addc_u32 s23, s23, 0\n\t"
      "s_waitcnt lgkmcnt(0)\n\t"
      "global_store_dwordx4 v210, v[176:179], s[22:23] sc0 sc1\n\t"
      "global_store_dwordx4 v211, v[180:183], s[22:23] sc0 sc1\n\t"
      "s_branch FINISH_%=\n\t"
      "AQ1_%=:\n\t"
      "s_movk_i32 s82, 4\n\t"
      "v_add_u32 v217, 256, v203\n\t"
      "s_call_b64 s[90:91], ASPIN_%=\n\t"
      "s_branch AR2_%=\n\t"
      "AQ3_%=:\n\t"
      "s_movk_i32 s82, 4\n\t"
      "v_add_u32 v217, 0, v203\n\t"
      "s_call_b64 s[90:91], ASPIN_%=\n\t"
      "s_branch AR4_%=\n\t"
      "AQ5_%=:\n\t"
      "s_movk_i32 s82, 8\n\t"
      "v_add_u32 v217, 256, v203\n\t"
      "s_call_b64 s[90:91], ASPIN_%=\n\t"
      "s_branch AR6_%=\n\t"
      "AQ7_%=:\n\t"
      "s_movk_i32 s82, 8\n\t"
      "v_add_u32 v217, 0, v203\n\t"
      "s_call_b64 s[90:91], ASPIN_%=\n\t"
      "s_branch AR8_%=\n\t"
      "AQ9_%=:\n\t"
      "s_movk_i32 s82, 12\n\t"
      "v_add_u32 v217, 256, v203\n\t"
      "s_call_b64 s[90:91], ASPIN_%=\n\t"
      "s_branch AR10_%=\n\t"
      "AQ11_%=:\n\t"
      "s_movk_i32 s82, 12\n\t"
      "v_add_u32 v217, 0, v203\n\t"
      "s_call_b64 s[90:91], ASPIN_%=\n\t"
      "s_branch AR12_%=\n\t"
      "AQ13_%=:\n\t"
      "s_movk_i32 s82, 16\n\t"
      "v_add_u32 v217, 256, v203\n\t"
      "s_call_b64 s[90:91], ASPIN_%=\n\t"
      "s_branch AR14_%=\n\t"
      "XS15_%=:\n\t"
      "s_mov_b32 s85, m0\n\t"
      "XL17_%=:\n\t"
      "s_add_u32 m0, s52, 147456\n\t"
      "s_add_u32 s83, s18, 0\n\t"
      "buffer_load_dwordx4 v200, s[44:47], s83 offen offset:0 sc0 sc1 lds\n\t"
      "buffer_load_dwordx4 v200, s[44:47], s83 offen offset:1024 sc0 sc1 lds\n\t"
      "s_add_u32 m0, s52, 122880\n\t"
      "s_add_u32 s83, s19, 0\n\t"
      "buffer_load_dwordx4 v200, s[44:47], s83 offen offset:0 sc0 sc1 lds\n\t"
      "buffer_load_dwordx4 v200, s[44:47], s83 offen offset:1024 sc0 sc1 lds\n\t"
      "s_add_u32 m0, s52, 114688\n\t"
      "s_add_u32 s83, s20, 0\n\t"
      "buffer_load_dwordx4 v200, s[44:47], s83 offen offset:0 sc0 sc1 lds\n\t"
      "buffer_load_dwordx4 v200, s[44:47], s83 offen offset:1024 sc0 sc1 lds\n\t"
      "s_call_b64 s[88:89], TMO_%=\n\t"
      "s_cbranch_scc1 XO18_%=\n\t"
      "s_waitcnt vmcnt(2)\n\t"
      "ds_read2st64_b32 v[228:229], v252 offset0:64 offset1:68\n\t"
      "ds_read2st64_b32 v[230:231], v251 offset0:224 offset1:228\n\t"
      "s_waitcnt lgkmcnt(0)\n\t"
      "v_min3_u32 v228, v228, v229, v230\n\t"
      "v_min_u32 v228, v228, v231\n\t"
      "v_cmp_eq_u32 vcc, 0, v228\n\t"
      "s_cbranch_vccnz XL17_%=\n\t"
      "s_cmp_eq_u32 0, 1\n\t"
      "XO18_%=:\n\t"
      "s_cselect_b32 s75, 0x7fc00000, s75\n\t"
      "s_mov_b32 m0, s85\n\t"
      "s_branch XR16_%=\n\t"
      "AQ19_%=:\n\t"
      "s_movk_i32 s82, 16\n\t"
      "v_add_u32 v217, 0, v203\n\t"
      "s_call_b64 s[90:91], ASPIN_%=\n\t"
      "s_branch AR20_%=\n\t"
      "XS21_%=:\n\t"
      "s_mov_b32 s85, m0\n\t"
      "XL23_%=:\n\t"
      "s_add_u32 m0, s52, 114688\n\t"
      "s_add_u32 s83, s20, 0\n\t"
      "buffer_load_dwordx4 v200, s[44:47], s83 offen offset:0 sc0 sc1 lds\n\t"
      "buffer_load_dwordx4 v200, s[44:47], s83 offen offset:1024 sc0 sc1 lds\n\t"
      "s_add_u32 m0, s52, 106496\n\t"
      "s_add_u32 s83, s21, 0\n\t"
      "buffer_load_dwordx4 v200, s[44:47], s83 offen offset:0 sc0 sc1 lds\n\t"
      "buffer_load_dwordx4 v200, s[44:47], s83 offen offset:1024 sc0 sc1 lds\n\t"
      "s_add_u32 m0, s52, 98304\n\t"
      "s_add_u32 s83, s24, 0\n\t"
      "buffer_load_dwordx4 v200, s[44:47], s83 offen offset:0 sc0 sc1 lds\n\t"
      "buffer_load_dwordx4 v200, s[44:47], s83 offen offset:1024 sc0 sc1 lds\n\t"
      "s_add_u32 m0, s52, 40960\n\t"
      "s_add_u32 s83, s25, 0\n\t"
      "buffer_load_dwordx4 v200, s[44:47], s83 offen offset:0 sc0 sc1 lds\n\t"
      "buffer_load_dwordx4 v200, s[44:47], s83 offen offset:1024 sc0 sc1 lds\n\t"
      "s_add_u32 m0, s52, 32768\n\t"
      "s_add_u32 s83, s26, 0\n\t"
      "buffer_load_dwordx4 v200, s[44:47], s83 offen offset:0 sc0 sc1 lds\n\t"
      "buffer_load_dwordx4 v200, s[44:47], s83 offen offset:1024 sc0 sc1 lds\n\t"
      "s_call_b64 s[88:89], TMO_%=\n\t"
      "s_cbranch_scc1 XO24_%=\n\t"
      "s_waitcnt vmcnt(6)\n\t"
      "ds_read2st64_b32 v[228:229], v251 offset0:192 offset1:196\n\t"
      "ds_read2st64_b32 v[230:231], v251 offset0:160 offset1:164\n\t"
      "s_waitcnt lgkmcnt(0)\n\t"
      "v_min3_u32 v228, v228, v229, v230\n\t"
      "v_min_u32 v228, v228, v231\n\t"
      "v_cmp_eq_u32 vcc, 0, v228\n\t"
      "s_cbranch_vccnz XL23_%=\n\t"
      "s_cmp_eq_u32 0, 1\n\t"
      "XO24_%=:\n\t"
      "s_cselect_b32 s75, 0x7fc00000, s75\n\t"
      "s_mov_b32 m0, s85\n\t"
      "s_branch XR22_%=\n\t"
      "AQ25_%=:\n\t"
      "s_movk_i32 s82, 20\n\t"
      "v_add_u32 v217, 0, v203\n\t"
      "s_call_b64 s[90:91], ASPIN_%=\n\t"
      "s_branch AR26_%=\n\t"
      "XS27_%=:\n\t"
      "s_mov_b32 s85, m0\n\t"
      "XL29_%=:\n\t"
      "s_add_u32 m0, s52, 98304\n\t"
      "s_add_u32 s83, s24, 0\n\t"
      "buffer_load_dwordx4 v200, s[44:47], s83 offen offset:0 sc0 sc1 lds\n\t"
      "buffer_load_dwordx4 v200, s[44:47], s83 offen offset:1024 sc0 sc1 lds\n\t"
      "s_add_u32 m0, s52, 40960\n\t"
      "s_add_u32 s83, s25, 0\n\t"
      "buffer_load_dwordx4 v200, s[44:47], s83 offen offset:0 sc0 sc1 lds\n\t"
      "buffer_load_dwordx4 v200, s[44:47], s83 offen offset:1024 sc0 sc1 lds\n\t"
      "s_add_u32 m0, s52, 32768\n\t"
      "s_add_u32 s83, s26, 0\n\t"
      "buffer_load_dwordx4 v200, s[44:47], s83 offen offset:0 sc0 sc1 lds\n\t"
      "buffer_load_dwordx4 v200, s[44:47], s83 offen offset:1024 sc0 sc1 lds\n\t"
      "s_add_u32 m0, s52, 73728\n\t"
      "s_add_u32 s83, s18, 8192\n\t"
      "buffer_load_dwordx4 v200, s[44:47], s83 offen offset:0 sc0 sc1 lds\n\t"
      "buffer_load_dwordx4 v200, s[44:47], s83 offen offset:1024 sc0 sc1 lds\n\t"
      "s_call_b64 s[88:89], TMO_%=\n\t"
      "s_cbranch_scc1 XO30_%=\n\t"
      "s_waitcnt vmcnt(6)\n\t"
      "ds_read2st64_b32 v[228:229], v251 offset0:128 offset1:132\n\t"
      "s_waitcnt lgkmcnt(0)\n\t"
      "v_min_u32 v228, v228, v229\n\t"
      "v_cmp_eq_u32 vcc, 0, v228\n\t"
      "s_cbranch_vccnz XL29_%=\n\t"
      "s_cmp_eq_u32 0, 1\n\t"
      "XO30_%=:\n\t"
      "s_cselect_b32 s75, 0x7fc00000, s75\n\t"
      "s_mov_b32 m0, s85\n\t"
      "s_branch XR28_%=\n\t"
      "XS31_%=:\n\t"
      "s_mov_b32 s85, m0\n\t"
      "XL33_%=:\n\t"
      "s_add_u32 m0, s52, 40960\n\t"
      "s_add_u32 s83, s25, 0\n\t"
      "buffer_load_dwordx4 v200, s[44:47], s83 offen offset:0 sc0 sc1 lds\n\t"
      "buffer_load_dwordx4 v200, s[44:47], s83 offen offset:1024 sc0 sc1 lds\n\t"
      "s_add_u32 m0, s52, 32768\n\t"
      "s_add_u32 s83, s26, 0\n\t"
      "buffer_load_dwordx4 v200, s[44:47], s83 offen offset:0 sc0 sc1 lds\n\t"
      "buffer_load_dwordx4 v200, s[44:47], s83 offen offset:1024 sc0 sc1 lds\n\t"
      "s_add_u32 m0, s52, 73728\n\t"
      "s_add_u32 s83, s18, 8192\n\t"
      "buffer_load_dwordx4 v200, s[44:47], s83 offen offset:0 sc0 sc1 lds\n\t"
      "buffer_load_dwordx4 v200, s[44:47], s83 offen offset:1024 sc0 sc1 lds\n\t"
      "s_add_u32 m0, s52, 147456\n\t"
      "s_add_u32 s83, s19, 8192\n\t"
      "buffer_load_dwordx4 v200, s[44:47], s83 offen offset:0 sc0 sc1 lds\n\t"
      "buffer_load_dwordx4 v200, s[44:47], s83 offen offset:1024 sc0 sc1 lds\n\t"
      "s_call_b64 s[88:89], TMO_%=\n\t"
      "s_cbranch_scc1 XO34_%=\n\t"
      "s_waitcnt vmcnt(6)\n\t"
      "ds_read2st64_b32 v[228:229], v250 offset0:160 offset1:164\n\t"
      "s_waitcnt lgkmcnt(0)\n\t"
      "v_min_u32 v228, v228, v229\n\t"
      "v_cmp_eq_u32 vcc, 0, v228\n\t"
      "s_cbranch_vccnz XL33_%=\n\t"
      "s_cmp_eq_u32 0, 1\n\t"
      "XO34_%=:\n\t"
      "s_cselect_b32 s75, 0x7fc00000, s75\n\t"
      "s_mov_b32 m0, s85\n\t"
      "s_branch XR32_%=\n\t"
      "AQ35_%=:\n\t"
      "s_movk_i32 s82, 24\n\t"
      "v_add_u32 v217, 256, v203\n\t"
      "s_call_b64 s[90:91], ASPIN_%=\n\t"
      "s_branch AR36_%=\n\t"
      "XS37_%=:\n\t"
      "s_mov_b32 s85, m0\n\t"
      "XL39_%=:\n\t"
      "s_add_u32 m0, s52, 32768\n\t"
      "s_add_u32 s83, s26, 0\n\t"
      "buffer_load_dwordx4 v200, s[44:47], s83 offen offset:0 sc0 sc1 lds\n\t"
      "buffer_load_dwordx4 v200, s[44:47], s83 offen offset:1024 sc0 sc1 lds\n\t"
      "s_add_u32 m0, s52, 73728\n\t"
      "s_add_u32 s83, s18, 8192\n\t"
      "buffer_load_dwordx4 v200, s[44:47], s83 offen offset:0 sc0 sc1 lds\n\t"
      "buffer_load_dwordx4 v200, s[44:47], s83 offen offset:1024 sc0 sc1 lds\n\t"
      "s_add_u32 m0, s52, 147456\n\t"
      "s_add_u32 s83, s19, 8192\n\t"
      "buffer_load_dwordx4 v200, s[44:47], s83 offen offset:0 sc0 sc1 lds\n\t"
      "buffer_load_dwordx4 v200, s[44:47], s83 offen offset:1024 sc0 sc1 lds\n\t"
      "s_add_u32 m0, s52, 65536\n\t"
      "s_add_u32 s83, s20, 8192\n\t"
      "buffer_load_dwordx4 v200, s[44:47], s83 offen offset:0 sc0 sc1 lds\n\t"
      "buffer_load_dwordx4 v200, s[44:47], s83 offen offset:1024 sc0 sc1 lds\n\t"
      "s_call_b64 s[88:89], TMO_%=\n\t"
      "s_cbranch_scc1 XO40_%=\n\t"
      "s_waitcnt vmcnt(6)\n\t"
      "ds_read2st64_b32 v[228:229], v250 offset0:128 offset1:132\n\t"
      "s_waitcnt lgkmcnt(0)\n\t"
      "v_min_u32 v228, v228, v229\n\t"
      "v_cmp_eq_u32 vcc, 0, v228\n\t"
      "s_cbranch_vccnz XL39_%=\n\t"
      "s_cmp_eq_u32 0, 1\n\t"
      "XO40_%=:\n\t"
      "s_cselect_b32 s75, 0x7fc00000, s75\n\t"
      "s_mov_b32 m0, s85\n\t"
      "s_branch XR38_%=\n\t"
      "AQ41_%=:\n\t"
      "s_movk_i32 s82, 24\n\t"
      "v_add_u32 v217, 0, v203\n\t"
      "s_call_b64 s[90:91], ASPIN_%=\n\t"
      "s_branch AR42_%=\n\t"
      "XS43_%=:\n\t"
      "s_mov_b32 s85, m0\n\t"
      "XL45_%=:\n\t"
      "s_add_u32 m0, s52, 73728\n\t"
      "s_add_u32 s83, s18, 8192\n\t"
      "buffer_load_dwordx4 v200, s[44:47], s83 offen offset:0 sc0 sc1 lds\n\t"
      "buffer_load_dwordx4 v200, s[44:47], s83 offen offset:1024 sc0 sc1 lds\n\t"
      "s_add_u32 m0, s52, 147456\n\t"
      "s_add_u32 s83, s19, 8192\n\t"
      "buffer_load_dwordx4 v200, s[44:47], s83 offen offset:0 sc0 sc1 lds\n\t"
      "buffer_load_dwordx4 v200, s[44:47], s83 offen offset:1024 sc0 sc1 lds\n\t"
      "s_add_u32 m0, s52, 65536\n\t"
      "s_add_u32 s83, s20, 8192\n\t"
      "buffer_load_dwordx4 v200, s[44:47], s83 offen offset:0 sc0 sc1 lds\n\t"
      "buffer_load_dwordx4 v200, s[44:47], s83 offen offset:1024 sc0 sc1 lds\n\t"
      "s_add_u32 m0, s52, 122880\n\t"
      "s_add_u32 s83, s21, 8192\n\t"
      "buffer_load_dwordx4 v200, s[44:47], s83 offen offset:0 sc0 sc1 lds\n\t"
      "buffer_load_dwordx4 v200, s[44:47], s83 offen offset:1024 sc0 sc1 lds\n\t"
      "s_call_b64 s[88:89], TMO_%=\n\t"
      "s_cbranch_scc1 XO46_%=\n\t"
      "s_waitcnt vmcnt(6)\n\t"
      "ds_read2st64_b32 v[228:229], v251 offset0:32 offset1:36\n\t"
      "s_waitcnt lgkmcnt(0)\n\t"
      "v_min_u32 v228, v228, v229\n\t"
      "v_cmp_eq_u32 vcc, 0, v228\n\t"
      "s_cbranch_vccnz XL45_%=\n\t"
      "s_cmp_eq_u32 0, 1\n\t"
      "XO46_%=:\n\t"
      "s_cselect_b32 s75, 0x7fc00000, s75\n\t"
      "s_mov_b32 m0, s85\n\t"
      "s_branch XR44_%=\n\t"
      "AQ47_%=:\n\t"
      "s_movk_i32 s82, 28\n\t"
      "v_add_u32 v217, 256, v203\n\t"
      "s_call_b64 s[90:91], ASPIN_%=\n\t"
      "s_branch AR48_%=\n\t"
      "XS49_%=:\n\t"
      "s_mov_b32 s85, m0\n\t"
      "XL51_%=:\n\t"
      "s_add_u32 m0, s52, 147456\n\t"
      "s_add_u32 s83, s19, 8192\n\t"
      "buffer_load_dwordx4 v200, s[44:47], s83 offen offset:0 sc0 sc1 lds\n\t"
      "buffer_load_dwordx4 v200, s[44:47], s83 offen offset:1024 sc0 sc1 lds\n\t"
      "s_add_u32 m0, s52, 65536\n\t"
      "s_add_u32 s83, s20, 8192\n\t"
      "buffer_load_dwordx4 v200, s[44:47], s83 offen offset:0 sc0 sc1 lds\n\t"
      "buffer_load_dwordx4 v200, s[44:47], s83 offen offset:1024 sc0 sc1 lds\n\t"
      "s_add_u32 m0, s52, 122880\n\t"
      "s_add_u32 s83, s21, 8192\n\t"
      "buffer_load_dwordx4 v200, s[44:47], s83 offen offset:0 sc0 sc1 lds\n\t"
      "buffer_load_dwordx4 v200, s[44:47], s83 offen offset:1024 sc0 sc1 lds\n\t"
      "s_add_u32 m0, s52, 114688\n\t"
      "s_add_u32 s83, s24, 8192\n\t"
      "buffer_load_dwordx4 v200, s[44:47], s83 offen offset:0 sc0 sc1 lds\n\t"
      "buffer_load_dwordx4 v200, s[44:47], s83 offen offset:1024 sc0 sc1 lds\n\t"
      "s_call_b64 s[88:89], TMO_%=\n\t"
      "s_cbranch_scc1 XO52_%=\n\t"
      "s_waitcnt vmcnt(6)\n\t"
      "ds_read2st64_b32 v[228:229], v252 offset0:64 offset1:68\n\t"
      "s_waitcnt lgkmcnt(0)\n\t"
      "v_min_u32 v228, v228, v229\n\t"
      "v_cmp_eq_u32 vcc, 0, v228\n\t"
      "s_cbranch_vccnz XL51_%=\n\t"
      "s_cmp_eq_u32 0, 1\n\t"
      "XO52_%=:\n\t"
      "s_cselect_b32 s75, 0x7fc00000, s75\n\t"
      "s_mov_b32 m0, s85\n\t"
      "s_branch XR50_%=\n\t"
      "AQ53_%=:\n\t"
      "s_movk_i32 s82, 28\n\t"
      "v_add_u32 v217, 0, v203\n\t"
      "s_call_b64 s[90:91], ASPIN_%=\n\t"
      "s_branch AR54_%=\n\t"
      "XS55_%=:\n\t"
      "s_mov_b32 s85, m0\n\t"
      "XL57_%=:\n\t"
      "s_add_u32 m0, s52, 65536\n\t"
      "s_add_u32 s83, s20, 8192\n\t"
      "buffer_load_dwordx4 v200, s[44:47], s83 offen offset:0 sc0 sc1 lds\n\t"
      "buffer_load_dwordx4 v200, s[44:47], s83 offen offset:1024 sc0 sc1 lds\n\t"
      "s_add_u32 m0, s52, 122880\n\t"
      "s_add_u32 s83, s21, 8192\n\t"
      "buffer_load_dwordx4 v200, s[44:47], s83 offen offset:0 sc0 sc1 lds\n\t"
      "buffer_load_dwordx4 v200, s[44:47], s83 offen offset:1024 sc0 sc1 lds\n\t"
      "s_add_u32 m0, s52, 114688\n\t"
      "s_add_u32 s83, s24, 8192\n\t"
      "buffer_load_dwordx4 v200, s[44:47], s83 offen offset:0 sc0 sc1 lds\n\t"
      "buffer_load_dwordx4 v200, s[44:47], s83 offen offset:1024 sc0 sc1 lds\n\t"
      "s_add_u32 m0, s52, 106496\n\t"
      "s_add_u32 s83, s25, 8192\n\t"
      "buffer_load_dwordx4 v200, s[44:47], s83 offen offset:0 sc0 sc1 lds\n\t"
      "buffer_load_dwordx4 v200, s[44:47], s83 offen offset:1024 sc0 sc1 lds\n\t"
      "s_call_b64 s[88:89], TMO_%=\n\t"
      "s_cbranch_scc1 XO58_%=\n\t"
      "s_waitcnt vmcnt(6)\n\t"
      "ds_read2st64_b32 v[228:229], v251 offset0:0 offset1:4\n\t"
      "s_waitcnt lgkmcnt(0)\n\t"
      "v_min_u32 v228, v228, v229\n\t"
      "v_cmp_eq_u32 vcc, 0, v228\n\t"
      "s_cbranch_vccnz XL57_%=\n\t"
      "s_cmp_eq_u32 0, 1\n\t"
      "XO58_%=:\n\t"
      "s_cselect_b32 s75, 0x7fc00000, s75\n\t"
      "s_mov_b32 m0, s85\n\t"
      "s_branch XR56_%=\n\t"
      "AQ59_%=:\n\t"
      "s_movk_i32 s82, 32\n\t"
      "v_add_u32 v217, 256, v203\n\t"
      "s_call_b64 s[90:91], ASPIN_%=\n\t"
      "s_branch AR60_%=\n\t"
      "XS61_%=:\n\t"
      "s_mov_b32 s85, m0\n\t"
      "XL63_%=:\n\t"
      "s_add_u32 m0, s52, 122880\n\t"
      "s_add_u32 s83, s21, 8192\n\t"
      "buffer_load_dwordx4 v200, s[44:47], s83 offen offset:0 sc0 sc1 lds\n\t"
      "buffer_load_dwordx4 v200, s[44:47], s83 offen offset:1024 sc0 sc1 lds\n\t"
      "s_add_u32 m0, s52, 114688\n\t"
      "s_add_u32 s83, s24, 8192\n\t"
      "buffer_load_dwordx4 v200, s[44:47], s83 offen offset:0 sc0 sc1 lds\n\t"
      "buffer_load_dwordx4 v200, s[44:47], s83 offen offset:1024 sc0 sc1 lds\n\t"
      "s_add_u32 m0, s52, 106496\n\t"
      "s_add_u32 s83, s25, 8192\n\t"
      "buffer_load_dwordx4 v200, s[44:47], s83 offen offset:0 sc0 sc1 lds\n\t"
      "buffer_load_dwordx4 v200, s[44:47], s83 offen offset:1024 sc0 sc1 lds\n\t"
      "s_add_u32 m0, s52, 98304\n\t"
      "s_add_u32 s83, s26, 8192\n\t"
      "buffer_load_dwordx4 v200, s[44:47], s83 offen offset:0 sc0 sc1 lds\n\t"
      "buffer_load_dwordx4 v200, s[44:47], s83 offen offset:1024 sc0 sc1 lds\n\t"
      "s_call_b64 s[88:89], TMO_%=\n\t"
      "s_cbranch_scc1 XO64_%=\n\t"
      "s_waitcnt vmcnt(6)\n\t"
      "ds_read2st64_b32 v[228:229], v251 offset0:224 offset1:228\n\t"
      "s_waitcnt lgkmcnt(0)\n\t"
      "v_min_u32 v228, v228, v229\n\t"
      "v_cmp_eq_u32 vcc, 0, v228\n\t"
      "s_cbranch_vccnz XL63_%=\n\t"
      "s_cmp_eq_u32 0, 1\n\t"
      "XO64_%=:\n\t"
      "s_cselect_b32 s75, 0x7fc00000, s75\n\t"
      "s_mov_b32 m0, s85\n\t"
      "s_branch XR62_%=\n\t"
      "AQ65_%=:\n\t"
      "s_movk_i32 s82, 32\n\t"
      "v_add_u32 v217, 0, v203\n\t"
      "s_call_b64 s[90:91], ASPIN_%=\n\t"
      "s_branch AR66_%=\n\t"
      "XS67_%=:\n\t"
      "s_mov_b32 s85, m0\n\t"
      "XL69_%=:\n\t"
      "s_add_u32 m0, s52, 114688\n\t"
      "s_add_u32 s83, s24, 8192\n\t"
      "buffer_load_dwordx4 v200, s[44:47], s83 offen offset:0 sc0 sc1 lds\n\t"
      "buffer_load_dwordx4 v200, s[44:47], s83 offen offset:1024 sc0 sc1 lds\n\t"
      "s_add_u32 m0, s52, 106496\n\t"
      "s_add_u32 s83, s25, 8192\n\t"
      "buffer_load_dwordx4 v200, s[44:47], s83 offen offset:0 sc0 sc1 lds\n\t"
      "buffer_load_dwordx4 v200, s[44:47], s83 offen offset:1024 sc0 sc1 lds\n\t"
      "s_add_u32 m0, s52, 98304\n\t"
      "s_add_u32 s83, s26, 8192\n\t"
      "buffer_load_dwordx4 v200, s[44:47], s83 offen offset:0 sc0 sc1 lds\n\t"
      "buffer_load_dwordx4 v200, s[44:47], s83 offen offset:1024 sc0 sc1 lds\n\t"
      "s_add_u32 m0, s52, 40960\n\t"
      "s_add_u32 s83, s18, 16384\n\t"
      "buffer_load_dwordx4 v200, s[44:47], s83 offen offset:0 sc0 sc1 lds\n\t"
      "buffer_load_dwordx4 v200, s[44:47], s83 offen offset:1024 sc0 sc1 lds\n\t"
      "s_call_b64 s[88:89], TMO_%=\n\t"
      "s_cbranch_scc1 XO70_%=\n\t"
      "s_waitcnt vmcnt(6)\n\t"
      "ds_read2st64_b32 v[228:229], v251 offset0:192 offset1:196\n\t"
      "s_waitcnt lgkmcnt(0)\n\t"
      "v_min_u32 v228, v228, v229\n\t"
      "v_cmp_eq_u32 vcc, 0, v228\n\t"
      "s_cbranch_vccnz XL69_%=\n\t"
      "s_cmp_eq_u32 0, 1\n\t"
      "XO70_%=:\n\t"
      "s_cselect_b32 s75, 0x7fc00000, s75\n\t"
      "s_mov_b32 m0, s85\n\t"
      "s_branch XR68_%=\n\t"
      "AQ71_%=:\n\t"
      "s_movk_i32 s82, 36\n\t"
      "v_add_u32 v217, 256, v203\n\t"
      "s_call_b64 s[90:91], ASPIN_%=\n\t"
      "s_branch AR72_%=\n\t"
      "XS73_%=:\n\t"
      "s_mov_b32 s85, m0\n\t"
      "XL75_%=:\n\t"
      "s_add_u32 m0, s52, 106496\n\t"
      "s_add_u32 s83, s25, 8192\n\t"
      "buffer_load_dwordx4 v200, s[44:47], s83 offen offset:0 sc0 sc1 lds\n\t"
      "buffer_load_dwordx4 v200, s[44:47], s83 offen offset:1024 sc0 sc1 lds\n\t"
      "s_add_u32 m0, s52, 98304\n\t"
      "s_add_u32 s83, s26, 8192\n\t"
      "buffer_load_dwordx4 v200, s[44:47], s83 offen offset:0 sc0 sc1 lds\n\t"
      "buffer_load_dwordx4 v200, s[44:47], s83 offen offset:1024 sc0 sc1 lds\n\t"
      "s_add_u32 m0, s52, 40960\n\t"
      "s_add_u32 s83, s18, 16384\n\t"
      "buffer_load_dwordx4 v200, s[44:47], s83 offen offset:0 sc0 sc1 lds\n\t"
      "buffer_load_dwordx4 v200, s[44:47], s83 offen offset:1024 sc0 sc1 lds\n\t"
      "s_add_u32 m0, s52, 32768\n\t"
      "s_add_u32 s83, s19, 16384\n\t"
      "buffer_load_dwordx4 v200, s[44:47], s83 offen offset:0 sc0 sc1 lds\n\t"
      "buffer_load_dwordx4 v200, s[44:47], s83 offen offset:1024 sc0 sc1 lds\n\t"
      "s_call_b64 s[88:89], TMO_%=\n\t"
      "s_cbranch_scc1 XO76_%=\n\t"
      "s_waitcnt vmcnt(6)\n\t"
      "ds_read2st64_b32 v[228:229], v251 offset0:160 offset1:164\n\t"
      "s_waitcnt lgkmcnt(0)\n\t"
      "v_min_u32 v228, v228, v229\n\t"
      "v_cmp_eq_u32 vcc, 0, v228\n\t"
      "s_cbranch_vccnz XL75_%=\n\t"
      "s_cmp_eq_u32 0, 1\n\t"
      "XO76_%=:\n\t"
      "s_cselect_b32 s75, 0x7fc00000, s75\n\t"
      "s_mov_b32 m0, s85\n\t"
      "s_branch XR74_%=\n\t"
      "AQ77_%=:\n\t"
      "s_movk_i32 s82, 36\n\t"
      "v_add_u32 v217, 0, v203\n\t"
      "s_call_b64 s[90:91], ASPIN_%=\n\t"
      "s_branch AR78_%=\n\t"
      "XS79_%=:\n\t"
      "s_mov_b32 s85, m0\n\t"
      "XL81_%=:\n\t"
      "s_add_u32 m0, s52, 98304\n\t"
      "s_add_u32 s83, s26, 8192\n\t"
      "buffer_load_dwordx4 v200, s[44:47], s83 offen offset:0 sc0 sc1 lds\n\t"
      "buffer_load_dwordx4 v200, s[44:47], s83 offen offset:1024 sc0 sc1 lds\n\t"
      "s_add_u32 m0, s52, 40960\n\t"
      "s_add_u32 s83, s18, 16384\n\t"
      "buffer_load_dwordx4 v200, s[44:47], s83 offen offset:0 sc0 sc1 lds\n\t"
      "buffer_load_dwordx4 v200, s[44:47], s83 offen offset:1024 sc0 sc1 lds\n\t"
      "s_add_u32 m0, s52, 32768\n\t"
      "s_add_u32 s83, s19, 16384\n\t"
      "buffer_load_dwordx4 v200, s[44:47], s83 offen offset:0 sc0 sc1 lds\n\t"
      "buffer_load_dwordx4 v200, s[44:47], s83 offen offset:1024 sc0 sc1 lds\n\t"
      "s_add_u32 m0, s52, 73728\n\t"
      "s_add_u32 s83, s20, 16384\n\t"
      "buffer_load_dwordx4 v200, s[44:47], s83 offen offset:0 sc0 sc1 lds\n\t"
      "buffer_load_dwordx4 v200, s[44:47], s83 offen offset:1024 sc0 sc1 lds\n\t"
      "s_call_b64 s[88:89], TMO_%=\n\t"
      "s_cbranch_scc1 XO82_%=\n\t"
      "s_waitcnt vmcnt(6)\n\t"
      "ds_read2st64_b32 v[228:229], v251 offset0:128 offset1:132\n\t"
      "s_waitcnt lgkmcnt(0)\n\t"
      "v_min_u32 v228, v228, v229\n\t"
      "v_cmp_eq_u32 vcc, 0, v228\n\t"
      "s_cbranch_vccnz XL81_%=\n\t"
      "s_cmp_eq_u32 0, 1\n\t"
      "XO82_%=:\n\t"
      "s_cselect_b32 s75, 0x7fc00000, s75\n\t"
      "s_mov_b32 m0, s85\n\t"
      "s_branch XR80_%=\n\t"
      "AQ83_%=:\n\t"
      "s_movk_i32 s82, 40\n\t"
      "v_add_u32 v217, 256, v203\n\t"
      "s_call_b64 s[90:91], ASPIN_%=\n\t"
      "s_branch AR84_%=\n\t"
      "XS85_%=:\n\t"
      "s_mov_b32 s85, m0\n\t"
      "XL87_%=:\n\t"
      "s_add_u32 m0, s52, 40960\n\t"
      "s_add_u32 s83, s18, 16384\n\t"
      "buffer_load_dwordx4 v200, s[44:47], s83 offen offset:0 sc0 sc1 lds\n\t"
      "buffer_load_dwordx4 v200, s[44:47], s83 offen offset:1024 sc0 sc1 lds\n\t"
      "s_add_u32 m0, s52, 32768\n\t"
      "s_add_u32 s83, s19, 16384\n\t"
      "buffer_load_dwordx4 v200, s[44:47], s83 offen offset:0 sc0 sc1 lds\n\t"
      "buffer_load_dwordx4 v200, s[44:47], s83 offen offset:1024 sc0 sc1 lds\n\t"
      "s_add_u32 m0, s52, 73728\n\t"
      "s_add_u32 s83, s20, 16384\n\t"
      "buffer_load_dwordx4 v200, s[44:47], s83 offen offset:0 sc0 sc1 lds\n\t"
      "buffer_load_dwordx4 v200, s[44:47], s83 offen offset:1024 sc0 sc1 lds\n\t"
      "s_add_u32 m0, s52, 147456\n\t"
      "s_add_u32 s83, s21, 16384\n\t"
      "buffer_load_dwordx4 v200, s[44:47], s83 offen offset:0 sc0 sc1 lds\n\t"
      "buffer_load_dwordx4 v200, s[44:47], s83 offen offset:1024 sc0 sc1 lds\n\t"
      "s_call_b64 s[88:89], TMO_%=\n\t"
      "s_cbranch_scc1 XO88_%=\n\t"
      "s_waitcnt vmcnt(6)\n\t"
      "ds_read2st64_b32 v[228:229], v250 offset0:160 offset1:164\n\t"
      "s_waitcnt lgkmcnt(0)\n\t"
      "v_min_u32 v228, v228, v229\n\t"
      "v_cmp_eq_u32 vcc, 0, v228\n\t"
      "s_cbranch_vccnz XL87_%=\n\t"
      "s_cmp_eq_u32 0, 1\n\t"
      "XO88_%=:\n\t"
      "s_cselect_b32 s75, 0x7fc00000, s75\n\t"
      "s_mov_b32 m0, s85\n\t"
      "s_branch XR86_%=\n\t"
      "AQ89_%=:\n\t"
      "s_movk_i32 s82, 40\n\t"
      "v_add_u32 v217, 0, v203\n\t"
      "s_call_b64 s[90:91], ASPIN_%=\n\t"
      "s_branch AR90_%=\n\t"
      "XS91_%=:\n\t"
      "s_mov_b32 s85, m0\n\t"
      "XL93_%=:\n\t"
      "s_add_u32 m0, s52, 32768\n\t"
      "s_add_u32 s83, s19, 16384\n\t"
      "buffer_load_dwordx4 v200, s[44:47], s83 offen offset:0 sc0 sc1 lds\n\t"
      "buffer_load_dwordx4 v200, s[44:47], s83 offen offset:1024 sc0 sc1 lds\n\t"
      "s_add_u32 m0, s52, 73728\n\t"
      "s_add_u32 s83, s20, 16384\n\t"
      "buffer_load_dwordx4 v200, s[44:47], s83 offen offset:0 sc0 sc1 lds\n\t"
      "buffer_load_dwordx4 v200, s[44:47], s83 offen offset:1024 sc0 sc1 lds\n\t"
      "s_add_u32 m0, s52, 147456\n\t"
      "s_add_u32 s83, s21, 16384\n\t"
      "buffer_load_dwordx4 v200, s[44:47], s83 offen offset:0 sc0 sc1 lds\n\t"
      "buffer_load_dwordx4 v200, s[44:47], s83 offen offset:1024 sc0 sc1 lds\n\t"
      "s_add_u32 m0, s52, 65536\n\t"
      "s_add_u32 s83, s24, 16384\n\t"
      "buffer_load_dwordx4 v200, s[44:47], s83 offen offset:0 sc0 sc1 lds\n\t"
      "buffer_load_dwordx4 v200, s[44:47], s83 offen offset:1024 sc0 sc1 lds\n\t"
      "s_call_b64 s[88:89], TMO_%=\n\t"
      "s_cbranch_scc1 XO94_%=\n\t"
      "s_waitcnt vmcnt(6)\n\t"
      "ds_read2st64_b32 v[228:229], v250 offset0:128 offset1:132\n\t"
      "s_waitcnt lgkmcnt(0)\n\t"
      "v_min_u32 v228, v228, v229\n\t"
      "v_cmp_eq_u32 vcc, 0, v228\n\t"
      "s_cbranch_vccnz XL93_%=\n\t"
      "s_cmp_eq_u32 0, 1\n\t"
      "XO94_%=:\n\t"
      "s_cselect_b32 s75, 0x7fc00000, s75\n\t"
      "s_mov_b32 m0, s85\n\t"
      "s_branch XR92_%=\n\t"
      "AQ95_%=:\n\t"
      "s_movk_i32 s82, 44\n\t"
      "v_add_u32 v217, 256, v203\n\t"
      "s_call_b64 s[90:91], ASPIN_%=\n\t"
      "s_branch AR96_%=\n\t"
      "XS97_%=:\n\t"
      "s_mov_b32 s85, m0\n\t"
      "XL99_%=:\n\t"
      "s_add_u32 m0, s52, 73728\n\t"
      "s_add_u32 s83, s20, 16384\n\t"
      "buffer_load_dwordx4 v200, s[44:47], s83 offen offset:0 sc0 sc1 lds\n\t"
      "buffer_load_dwordx4 v200, s[44:47], s83 offen offset:1024 sc0 sc1 lds\n\t"
      "s_add_u32 m0, s52, 147456\n\t"
      "s_add_u32 s83, s21, 16384\n\t"
      "buffer_load_dwordx4 v200, s[44:47], s83 offen offset:0 sc0 sc1 lds\n\t"
      "buffer_load_dwordx4 v200, s[44:47], s83 offen offset:1024 sc0 sc1 lds\n\t"
      "s_add_u32 m0, s52, 65536\n\t"
      "s_add_u32 s83, s24, 16384\n\t"
      "buffer_load_dwordx4 v200, s[44:47], s83 offen offset:0 sc0 sc1 lds\n\t"
      "buffer_load_dwordx4 v200, s[44:47], s83 offen offset:1024 sc0 sc1 lds\n\t"
      "s_add_u32 m0, s52, 122880\n\t"
      "s_add_u32 s83, s25, 16384\n\t"
      "buffer_load_dwordx4 v200, s[44:47], s83 offen offset:0 sc0 sc1 lds\n\t"
      "buffer_load_dwordx4 v200, s[44:47], s83 offen offset:1024 sc0 sc1 lds\n\t"
      "s_call_b64 s[88:89], TMO_%=\n\t"
      "s_cbranch_scc1 XO100_%=\n\t"
      "s_waitcnt vmcnt(6)\n\t"
      "ds_read2st64_b32 v[228:229], v251 offset0:32 offset1:36\n\t"
      "s_waitcnt lgkmcnt(0)\n\t"
      "v_min_u32 v228, v228, v229\n\t"
      "v_cmp_eq_u32 vcc, 0, v228\n\t"
      "s_cbranch_vccnz XL99_%=\n\t"
      "s_cmp_eq_u32 0, 1\n\t"
      "XO100_%=:\n\t"
      "s_cselect_b32 s75, 0x7fc00000, s75\n\t"
      "s_mov_b32 m0, s85\n\t"
      "s_branch XR98_%=\n\t"
      "AQ101_%=:\n\t"
      "s_movk_i32 s82, 44\n\t"
      "v_add_u32 v217, 0, v203\n\t"
      "s_call_b64 s[90:91], ASPIN_%=\n\t"
      "s_branch AR102_%=\n\t"
      "XS103_%=:\n\t"
      "s_mov_b32 s85, m0\n\t"
      "XL105_%=:\n\t"
      "s_add_u32 m0, s52, 147456\n\t"
      "s_add_u32 s83, s21, 16384\n\t"
      "buffer_load_dwordx4 v200, s[44:47], s83 offen offset:0 sc0 sc1 lds\n\t"
      "buffer_load_dwordx4 v200, s[44:47], s83 offen offset:1024 sc0 sc1 lds\n\t"
      "s_add_u32 m0, s52, 65536\n\t"
      "s_add_u32 s83, s24, 16384\n\t"
      "buffer_load_dwordx4 v200, s[44:47], s83 offen offset:0 sc0 sc1 lds\n\t"
      "buffer_load_dwordx4 v200, s[44:47], s83 offen offset:1024 sc0 sc1 lds\n\t"
      "s_add_u32 m0, s52, 122880\n\t"
      "s_add_u32 s83, s25, 16384\n\t"
      "buffer_load_dwordx4 v200, s[44:47], s83 offen offset:0 sc0 sc1 lds\n\t"
      "buffer_load_dwordx4 v200, s[44:47], s83 offen offset:1024 sc0 sc1 lds\n\t"
      "s_add_u32 m0, s52, 114688\n\t"
      "s_add_u32 s83, s26, 16384\n\t"
      "buffer_load_dwordx4 v200, s[44:47], s83 offen offset:0 sc0 sc1 lds\n\t"
      "buffer_load_dwordx4 v200, s[44:47], s83 offen offset:1024 sc0 sc1 lds\n\t"
      "s_call_b64 s[88:89], TMO_%=\n\t"
      "s_cbranch_scc1 XO106_%=\n\t"
      "s_waitcnt vmcnt(6)\n\t"
      "ds_read2st64_b32 v[228:229], v252 offset0:64 offset1:68\n\t"
      "s_waitcnt lgkmcnt(0)\n\t"
      "v_min_u32 v228, v228, v229\n\t"
      "v_cmp_eq_u32 vcc, 0, v228\n\t"
      "s_cbranch_vccnz XL105_%=\n\t"
      "s_cmp_eq_u32 0, 1\n\t"
      "XO106_%=:\n\t"
      "s_cselect_b32 s75, 0x7fc00000, s75\n\t"
      "s_mov_b32 m0, s85\n\t"
      "s_branch XR104_%=\n\t"
      "AQ107_%=:\n\t"
      "s_movk_i32 s82, 48\n\t"
      "v_add_u32 v217, 256, v203\n\t"
      "s_call_b64 s[90:91], ASPIN_%=\n\t"
      "s_branch AR108_%=\n\t"
      "XS109_%=:\n\t"
      "s_mov_b32 s85, m0\n\t"
      "XL111_%=:\n\t"
      "s_add_u32 m0, s52, 65536\n\t"
      "s_add_u32 s83, s24, 16384\n\t"
      "buffer_load_dwordx4 v200, s[44:47], s83 offen offset:0 sc0 sc1 lds\n\t"
      "buffer_load_dwordx4 v200, s[44:47], s83 offen offset:1024 sc0 sc1 lds\n\t"
      "s_add_u32 m0, s52, 122880\n\t"
      "s_add_u32 s83, s25, 16384\n\t"
      "buffer_load_dwordx4 v200, s[44:47], s83 offen offset:0 sc0 sc1 lds\n\t"
      "buffer_load_dwordx4 v200, s[44:47], s83 offen offset:1024 sc0 sc1 lds\n\t"
      "s_add_u32 m0, s52, 114688\n\t"
      "s_add_u32 s83, s26, 16384\n\t"
      "buffer_load_dwordx4 v200, s[44:47], s83 offen offset:0 sc0 sc1 lds\n\t"
      "buffer_load_dwordx4 v200, s[44:47], s83 offen offset:1024 sc0 sc1 lds\n\t"
      "s_add_u32 m0, s52, 106496\n\t"
      "s_add_u32 s83, s18, 24576\n\t"
      "buffer_load_dwordx4 v200, s[44:47], s83 offen offset:0 sc0 sc1 lds\n\t"
      "buffer_load_dwordx4 v200, s[44:47], s83 offen offset:1024 sc0 sc1 lds\n\t"
      "s_call_b64 s[88:89], TMO_%=\n\t"
      "s_cbranch_scc1 XO112_%=\n\t"
      "s_waitcnt vmcnt(6)\n\t"
      "ds_read2st64_b32 v[228:229], v251 offset0:0 offset1:4\n\t"
      "s_waitcnt lgkmcnt(0)\n\t"
      "v_min_u32 v228, v228, v229\n\t"
      "v_cmp_eq_u32 vcc, 0, v228\n\t"
      "s_cbranch_vccnz XL111_%=\n\t"
      "s_cmp_eq_u32 0, 1\n\t"
      "XO112_%=:\n\t"
      "s_cselect_b32 s75, 0x7fc00000, s75\n\t"
      "s_mov_b32 m0, s85\n\t"
      "s_branch XR110_%=\n\t"
      "AQ113_%=:\n\t"
      "s_movk_i32 s82, 48\n\t"
      "v_add_u32 v217, 0, v203\n\t"
      "s_call_b64 s[90:91], ASPIN_%=\n\t"
      "s_branch AR114_%=\n\t"
      "XS115_%=:\n\t"
      "s_mov_b32 s85, m0\n\t"
      "XL117_%=:\n\t"
      "s_add_u32 m0, s52, 122880\n\t"
      "s_add_u32 s83, s25, 16384\n\t"
      "buffer_load_dwordx4 v200, s[44:47], s83 offen offset:0 sc0 sc1 lds\n\t"
      "buffer_load_dwordx4 v200, s[44:47], s83 offen offset:1024 sc0 sc1 lds\n\t"
      "s_add_u32 m0, s52, 114688\n\t"
      "s_add_u32 s83, s26, 16384\n\t"
      "buffer_load_dwordx4 v200, s[44:47], s83 offen offset:0 sc0 sc1 lds\n\t"
      "buffer_load_dwordx4 v200, s[44:47], s83 offen offset:1024 sc0 sc1 lds\n\t"
      "s_add_u32 m0, s52, 106496\n\t"
      "s_add_u32 s83, s18, 24576\n\t"
      "buffer_load_dwordx4 v200, s[44:47], s83 offen offset:0 sc0 sc1 lds\n\t"
      "buffer_load_dwordx4 v200, s[44:47], s83 offen offset:1024 sc0 sc1 lds\n\t"
      "s_add_u32 m0, s52, 98304\n\t"
      "s_add_u32 s83, s19, 24576\n\t"
      "buffer_load_dwordx4 v200, s[44:47], s83 offen offset:0 sc0 sc1 lds\n\t"
      "buffer_load_dwordx4 v200, s[44:47], s83 offen offset:1024 sc0 sc1 lds\n\t"
      "s_call_b64 s[88:89], TMO_%=\n\t"
      "s_cbranch_scc1 XO118_%=\n\t"
      "s_waitcnt vmcnt(6)\n\t"
      "ds_read2st64_b32 v[228:229], v251 offset0:224 offset1:228\n\t"
      "s_waitcnt lgkmcnt(0)\n\t"
      "v_min_u32 v228, v228, v229\n\t"
      "v_cmp_eq_u32 vcc, 0, v228\n\t"
      "s_cbranch_vccnz XL117_%=\n\t"
      "s_cmp_eq_u32 0, 1\n\t"
      "XO118_%=:\n\t"
      "s_cselect_b32 s75, 0x7fc00000, s75\n\t"
      "s_mov_b32 m0, s85\n\t"
      "s_branch XR116_%=\n\t"
      "AQ119_%=:\n\t"
      "s_movk_i32 s82, 52\n\t"
      "v_add_u32 v217, 256, v203\n\t"
      "s_call_b64 s[90:91], ASPIN_%=\n\t"
      "s_branch AR120_%=\n\t"
      "XS121_%=:\n\t"
      "s_mov_b32 s85, m0\n\t"
      "XL123_%=:\n\t"
      "s_add_u32 m0, s52, 114688\n\t"
      "s_add_u32 s83, s26, 16384\n\t"
      "buffer_load_dwordx4 v200, s[44:47], s83 offen offset:0 sc0 sc1 lds\n\t"
      "buffer_load_dwordx4 v200, s[44:47], s83 offen offset:1024 sc0 sc1 lds\n\t"
      "s_add_u32 m0, s52, 106496\n\t"
      "s_add_u32 s83, s18, 24576\n\t"
      "buffer_load_dwordx4 v200, s[44:47], s83 offen offset:0 sc0 sc1 lds\n\t"
      "buffer_load_dwordx4 v200, s[44:47], s83 offen offset:1024 sc0 sc1 lds\n\t"
      "s_add_u32 m0, s52, 98304\n\t"
      "s_add_u32 s83, s19, 24576\n\t"
      "buffer_load_dwordx4 v200, s[44:47], s83 offen offset:0 sc0 sc1 lds\n\t"
      "buffer_load_dwordx4 v200, s[44:47], s83 offen offset:1024 sc0 sc1 lds\n\t"
      "s_add_u32 m0, s52, 40960\n\t"
      "s_add_u32 s83, s20, 24576\n\t"
      "buffer_load_dwordx4 v200, s[44:47], s83 offen offset:0 sc0 sc1 lds\n\t"
      "buffer_load_dwordx4 v200, s[44:47], s83 offen offset:1024 sc0 sc1 lds\n\t"
      "s_call_b64 s[88:89], TMO_%=\n\t"
      "s_cbranch_scc1 XO124_%=\n\t"
      "s_waitcnt vmcnt(6)\n\t"
      "ds_read2st64_b32 v[228:229], v251 offset0:192 offset1:196\n\t"
      "s_waitcnt lgkmcnt(0)\n\t"
      "v_min_u32 v228, v228, v229\n\t"
      "v_cmp_eq_u32 vcc, 0, v228\n\t"
      "s_cbranch_vccnz XL123_%=\n\t"
      "s_cmp_eq_u32 0, 1\n\t"
      "XO124_%=:\n\t"
      "s_cselect_b32 s75, 0x7fc00000, s75\n\t"
      "s_mov_b32 m0, s85\n\t"
      "s_branch XR122_%=\n\t"
      "AQ125_%=:\n\t"
      "s_movk_i32 s82, 52\n\t"
      "v_add_u32 v217, 0, v203\n\t"
      "s_call_b64 s[90:91], ASPIN_%=\n\t"
      "s_branch AR126_%=\n\t"
      "XS127_%=:\n\t"
      "s_mov_b32 s85, m0\n\t"
      "XL129_%=:\n\t"
      "s_add_u32 m0, s52, 106496\n\t"
      "s_add_u32 s83, s18, 24576\n\t"
      "buffer_load_dwordx4 v200, s[44:47], s83 offen offset:0 sc0 sc1 lds\n\t"
      "buffer_load_dwordx4 v200, s[44:47], s83 offen offset:1024 sc0 sc1 lds\n\t"
      "s_add_u32 m0, s52, 98304\n\t"
      "s_add_u32 s83, s19, 24576\n\t"
      "buffer_load_dwordx4 v200, s[44:47], s83 offen offset:0 sc0 sc1 lds\n\t"
      "buffer_load_dwordx4 v200, s[44:47], s83 offen offset:1024 sc0 sc1 lds\n\t"
      "s_add_u32 m0, s52, 40960\n\t"
      "s_add_u32 s83, s20, 24576\n\t"
      "buffer_load_dwordx4 v200, s[44:47], s83 offen offset:0 sc0 sc1 lds\n\t"
      "buffer_load_dwordx4 v200, s[44:47], s83 offen offset:1024 sc0 sc1 lds\n\t"
      "s_add_u32 m0, s52, 32768\n\t"
      "s_add_u32 s83, s21, 24576\n\t"
      "buffer_load_dwordx4 v200, s[44:47], s83 offen offset:0 sc0 sc1 lds\n\t"
      "buffer_load_dwordx4 v200, s[44:47], s83 offen offset:1024 sc0 sc1 lds\n\t"
      "s_call_b64 s[88:89], TMO_%=\n\t"
      "s_cbranch_scc1 XO130_%=\n\t"
      "s_waitcnt vmcnt(6)\n\t"
      "ds_read2st64_b32 v[228:229], v251 offset0:160 offset1:164\n\t"
      "s_waitcnt lgkmcnt(0)\n\t"
      "v_min_u32 v228, v228, v229\n\t"
      "v_cmp_eq_u32 vcc, 0, v228\n\t"
      "s_cbranch_vccnz XL129_%=\n\t"
      "s_cmp_eq_u32 0, 1\n\t"
      "XO130_%=:\n\t"
      "s_cselect_b32 s75, 0x7fc00000, s75\n\t"
      "s_mov_b32 m0, s85\n\t"
      "s_branch XR128_%=\n\t"
      "AQ131_%=:\n\t"
      "s_movk_i32 s82, 56\n\t"
      "v_add_u32 v217, 256, v203\n\t"
      "s_call_b64 s[90:91], ASPIN_%=\n\t"
      "s_branch AR132_%=\n\t"
      "XS133_%=:\n\t"
      "s_mov_b32 s85, m0\n\t"
      "XL135_%=:\n\t"
      "s_add_u32 m0, s52, 98304\n\t"
      "s_add_u32 s83, s19, 24576\n\t"
      "buffer_load_dwordx4 v200, s[44:47], s83 offen offset:0 sc0 sc1 lds\n\t"
      "buffer_load_dwordx4 v200, s[44:47], s83 offen offset:1024 sc0 sc1 lds\n\t"
      "s_add_u32 m0, s52, 40960\n\t"
      "s_add_u32 s83, s20, 24576\n\t"
      "buffer_load_dwordx4 v200, s[44:47], s83 offen offset:0 sc0 sc1 lds\n\t"
      "buffer_load_dwordx4 v200, s[44:47], s83 offen offset:1024 sc0 sc1 lds\n\t"
      "s_add_u32 m0, s52, 32768\n\t"
      "s_add_u32 s83, s21, 24576\n\t"
      "buffer_load_dwordx4 v200, s[44:47], s83 offen offset:0 sc0 sc1 lds\n\t"
      "buffer_load_dwordx4 v200, s[44:47], s83 offen offset:1024 sc0 sc1 lds\n\t"
      "s_add_u32 m0, s52, 73728\n\t"
      "s_add_u32 s83, s24, 24576\n\t"
      "buffer_load_dwordx4 v200, s[44:47], s83 offen offset:0 sc0 sc1 lds\n\t"
      "buffer_load_dwordx4 v200, s[44:47], s83 offen offset:1024 sc0 sc1 lds\n\t"
      "s_call_b64 s[88:89], TMO_%=\n\t"
      "s_cbranch_scc1 XO136_%=\n\t"
      "s_waitcnt vmcnt(6)\n\t"
      "ds_read2st64_b32 v[228:229], v251 offset0:128 offset1:132\n\t"
      "s_waitcnt lgkmcnt(0)\n\t"
      "v_min_u32 v228, v228, v229\n\t"
      "v_cmp_eq_u32 vcc, 0, v228\n\t"
      "s_cbranch_vccnz XL135_%=\n\t"
      "s_cmp_eq_u32 0, 1\n\t"
      "XO136_%=:\n\t"
      "s_cselect_b32 s75, 0x7fc00000, s75\n\t"
      "s_mov_b32 m0, s85\n\t"
      "s_branch XR134_%=\n\t"
      "AQ137_%=:\n\t"
      "s_movk_i32 s82, 56\n\t"
      "v_add_u32 v217, 0, v203\n\t"
      "s_call_b64 s[90:91], ASPIN_%=\n\t"
      "s_branch AR138_%=\n\t"
      "XS139_%=:\n\t"
      "s_mov_b32 s85, m0\n\t"
      "XL141_%=:\n\t"
      "s_add_u32 m0, s52, 40960\n\t"
      "s_add_u32 s83, s20, 24576\n\t"
      "buffer_load_dwordx4 v200, s[44:47], s83 offen offset:0 sc0 sc1 lds\n\t"
      "buffer_load_dwordx4 v200, s[44:47], s83 offen offset:1024 sc0 sc1 lds\n\t"
      "s_add_u32 m0, s52, 32768\n\t"
      "s_add_u32 s83, s21, 24576\n\t"
      "buffer_load_dwordx4 v200, s[44:47], s83 offen offset:0 sc0 sc1 lds\n\t"
      "buffer_load_dwordx4 v200, s[44:47], s83 offen offset:1024 sc0 sc1 lds\n\t"
      "s_add_u32 m0, s52, 73728\n\t"
      "s_add_u32 s83, s24, 24576\n\t"
      "buffer_load_dwordx4 v200, s[44:47], s83 offen offset:0 sc0 sc1 lds\n\t"
      "buffer_load_dwordx4 v200, s[44:47], s83 offen offset:1024 sc0 sc1 lds\n\t"
      "s_add_u32 m0, s52, 147456\n\t"
      "s_add_u32 s83, s25, 24576\n\t"
      "buffer_load_dwordx4 v200, s[44:47], s83 offen offset:0 sc0 sc1 lds\n\t"
      "buffer_load_dwordx4 v200, s[44:47], s83 offen offset:1024 sc0 sc1 lds\n\t"
      "s_call_b64 s[88:89], TMO_%=\n\t"
      "s_cbranch_scc1 XO142_%=\n\t"
      "s_waitcnt vmcnt(6)\n\t"
      "ds_read2st64_b32 v[228:229], v250 offset0:160 offset1:164\n\t"
      "s_waitcnt lgkmcnt(0)\n\t"
      "v_min_u32 v228, v228, v229\n\t"
      "v_cmp_eq_u32 vcc, 0, v228\n\t"
      "s_cbranch_vccnz XL141_%=\n\t"
      "s_cmp_eq_u32 0, 1\n\t"
      "XO142_%=:\n\t"
      "s_cselect_b32 s75, 0x7fc00000, s75\n\t"
      "s_mov_b32 m0, s85\n\t"
      "s_branch XR140_%=\n\t"
      "AQ143_%=:\n\t"
      "s_movk_i32 s82, 60\n\t"
      "v_add_u32 v217, 256, v203\n\t"
      "s_call_b64 s[90:91], ASPIN_%=\n\t"
      "s_branch AR144_%=\n\t"
      "XS145_%=:\n\t"
      "s_mov_b32 s85, m0\n\t"
      "XL147_%=:\n\t"
      "s_add_u32 m0, s52, 32768\n\t"
      "s_add_u32 s83, s21, 24576\n\t"
      "buffer_load_dwordx4 v200, s[44:47], s83 offen offset:0 sc0 sc1 lds\n\t"
      "buffer_load_dwordx4 v200, s[44:47], s83 offen offset:1024 sc0 sc1 lds\n\t"
      "s_add_u32 m0, s52, 73728\n\t"
      "s_add_u32 s83, s24, 24576\n\t"
      "buffer_load_dwordx4 v200, s[44:47], s83 offen offset:0 sc0 sc1 lds\n\t"
      "buffer_load_dwordx4 v200, s[44:47], s83 offen offset:1024 sc0 sc1 lds\n\t"
      "s_add_u32 m0, s52, 147456\n\t"
      "s_add_u32 s83, s25, 24576\n\t"
      "buffer_load_dwordx4 v200, s[44:47], s83 offen offset:0 sc0 sc1 lds\n\t"
      "buffer_load_dwordx4 v200, s[44:47], s83 offen offset:1024 sc0 sc1 lds\n\t"
      "s_add_u32 m0, s52, 65536\n\t"
      "s_add_u32 s83, s26, 24576\n\t"
      "buffer_load_dwordx4 v200, s[44:47], s83 offen offset:0 sc0 sc1 lds\n\t"
      "buffer_load_dwordx4 v200, s[44:47], s83 offen offset:1024 sc0 sc1 lds\n\t"
      "s_call_b64 s[88:89], TMO_%=\n\t"
      "s_cbranch_scc1 XO148_%=\n\t"
      "s_waitcnt vmcnt(6)\n\t"
      "ds_read2st64_b32 v[228:229], v250 offset0:128 offset1:132\n\t"
      "s_waitcnt lgkmcnt(0)\n\t"
      "v_min_u32 v228, v228, v229\n\t"
      "v_cmp_eq_u32 vcc, 0, v228\n\t"
      "s_cbranch_vccnz XL147_%=\n\t"
      "s_cmp_eq_u32 0, 1\n\t"
      "XO148_%=:\n\t"
      "s_cselect_b32 s75, 0x7fc00000, s75\n\t"
      "s_mov_b32 m0, s85\n\t"
      "s_branch XR146_%=\n\t"
      "AQ149_%=:\n\t"
      "s_movk_i32 s82, 60\n\t"
      "v_add_u32 v217, 0, v203\n\t"
      "s_call_b64 s[90:91], ASPIN_%=\n\t"
      "s_branch AR150_%=\n\t"
      "XS151_%=:\n\t"
      "s_mov_b32 s85, m0\n\t"
      "XL153_%=:\n\t"
      "s_add_u32 m0, s52, 73728\n\t"
      "s_add_u32 s83, s24, 24576\n\t"
      "buffer_load_dwordx4 v200, s[44:47], s83 offen offset:0 sc0 sc1 lds\n\t"
      "buffer_load_dwordx4 v200, s[44:47], s83 offen offset:1024 sc0 sc1 lds\n\t"
      "s_add_u32 m0, s52, 147456\n\t"
      "s_add_u32 s83, s25, 24576\n\t"
      "buffer_load_dwordx4 v200, s[44:47], s83 offen offset:0 sc0 sc1 lds\n\t"
      "buffer_load_dwordx4 v200, s[44:47], s83 offen offset:1024 sc0 sc1 lds\n\t"
      "s_add_u32 m0, s52, 65536\n\t"
      "s_add_u32 s83, s26, 24576\n\t"
      "buffer_load_dwordx4 v200, s[44:47], s83 offen offset:0 sc0 sc1 lds\n\t"
      "buffer_load_dwordx4 v200, s[44:47], s83 offen offset:1024 sc0 sc1 lds\n\t"
      "s_call_b64 s[88:89], TMO_%=\n\t"
      "s_cbranch_scc1 XO154_%=\n\t"
      "s_waitcnt vmcnt(4)\n\t"
      "ds_read2st64_b32 v[228:229], v251 offset0:32 offset1:36\n\t"
      "s_waitcnt lgkmcnt(0)\n\t"
      "v_min_u32 v228, v228, v229\n\t"
      "v_cmp_eq_u32 vcc, 0, v228\n\t"
      "s_cbranch_vccnz XL153_%=\n\t"
      "s_cmp_eq_u32 0, 1\n\t"
      "XO154_%=:\n\t"
      "s_cselect_b32 s75, 0x7fc00000, s75\n\t"
      "s_mov_b32 m0, s85\n\t"
      "s_branch XR152_%=\n\t"
      "AQ155_%=:\n\t"
      "s_movk_i32 s82, 64\n\t"
      "v_add_u32 v217, 256, v203\n\t"
      "s_call_b64 s[90:91], ASPIN_%=\n\t"
      "s_branch AR156_%=\n\t"
      "FS157_%=:\n\t"
      "v_add_u32 v217, 128, v213\n\t"
      "s_call_b64 s[90:91], FSPIN_%=\n\t"
      "s_branch FR158_%=\n\t"
      "XS159_%=:\n\t"
      "s_mov_b32 s85, m0\n\t"
      "XL161_%=:\n\t"
      "s_add_u32 m0, s52, 147456\n\t"
      "s_add_u32 s83, s25, 24576\n\t"
      "buffer_load_dwordx4 v200, s[44:47], s83 offen offset:0 sc0 sc1 lds\n\t"
      "buffer_load_dwordx4 v200, s[44:47], s83 offen offset:1024 sc0 sc1 lds\n\t"
      "s_add_u32 m0, s52, 65536\n\t"
      "s_add_u32 s83, s26, 24576\n\t"
      "buffer_load_dwordx4 v200, s[44:47], s83 offen offset:0 sc0 sc1 lds\n\t"
      "buffer_load_dwordx4 v200, s[44:47], s83 offen offset:1024 sc0 sc1 lds\n\t"
      "s_call_b64 s[88:89], TMO_%=\n\t"
      "s_cbranch_scc1 XO162_%=\n\t"
      "s_waitcnt vmcnt(2)\n\t"
      "ds_read2st64_b32 v[228:229], v252 offset0:64 offset1:68\n\t"
      "s_waitcnt lgkmcnt(0)\n\t"
      "v_min_u32 v228, v228, v229\n\t"
      "v_cmp_eq_u32 vcc, 0, v228\n\t"
      "s_cbranch_vccnz XL161_%=\n\t"
      "s_cmp_eq_u32 0, 1\n\t"
      "XO162_%=:\n\t"
      "s_cselect_b32 s75, 0x7fc00000, s75\n\t"
      "s_mov_b32 m0, s85\n\t"
      "s_branch XR160_%=\n\t"
      "AQ163_%=:\n\t"
      "s_movk_i32 s82, 64\n\t"
      "v_add_u32 v217, 0, v203\n\t"
      "s_call_b64 s[90:91], ASPIN_%=\n\t"
      "s_branch AR164_%=\n\t"
      "XS165_%=:\n\t"
      "s_mov_b32 s85, m0\n\t"
      "XL167_%=:\n\t"
      "s_add_u32 m0, s52, 65536\n\t"
      "s_add_u32 s83, s26, 24576\n\t"
      "buffer_load_dwordx4 v200, s[44:47], s83 offen offset:0 sc0 sc1 lds\n\t"
      "buffer_load_dwordx4 v200, s[44:47], s83 offen offset:1024 sc0 sc1 lds\n\t"
      "s_call_b64 s[88:89], TMO_%=\n\t"
      "s_cbranch_scc1 XO168_%=\n\t"
      "s_waitcnt vmcnt(0)\n\t"
      "ds_read2st64_b32 v[228:229], v251 offset0:0 offset1:4\n\t"
      "s_waitcnt lgkmcnt(0)\n\t"
      "v_min_u32 v228, v228, v229\n\t"
      "v_cmp_eq_u32 vcc, 0, v228\n\t"
      "s_cbranch_vccnz XL167_%=\n\t"
      "s_cmp_eq_u32 0, 1\n\t"
      "XO168_%=:\n\t"
      "s_cselect_b32 s75, 0x7fc00000, s75\n\t"
      "s_mov_b32 m0, s85\n\t"
      "s_branch XR166_%=\n\t"
      "AQ169_%=:\n\t"
      "s_movk_i32 s82, 68\n\t"
      "v_add_u32 v217, 256, v203\n\t"
      "s_call_b64 s[90:91], ASPIN_%=\n\t"
      "s_branch AR170_%=\n\t"
      "AQ171_%=:\n\t"
      "s_movk_i32 s82, 68\n\t"
      "v_add_u32 v217, 0, v203\n\t"
      "s_call_b64 s[90:91], ASPIN_%=\n\t"
      "s_branch AR172_%=\n\t"
      "AQ179_%=:\n\t"
      "s_movk_i32 s82, 4\n\t"
      "v_add_u32 v217, 256, v203\n\t"
      "s_call_b64 s[90:91], ASPIN_%=\n\t"
      "s_branch AR180_%=\n\t"
      "AQ185_%=:\n\t"
      "s_movk_i32 s82, 4\n\t"
      "v_add_u32 v217, 0, v203\n\t"
      "s_call_b64 s[90:91], ASPIN_%=\n\t"
      "s_branch AR186_%=\n\t"
      "AQ191_%=:\n\t"
      "s_movk_i32 s82, 8\n\t"
      "v_add_u32 v217, 256, v203\n\t"
      "s_call_b64 s[90:91], ASPIN_%=\n\t"
      "s_branch AR192_%=\n\t"
      "AQ197_%=:\n\t"
      "s_movk_i32 s82, 8\n\t"
      "v_add_u32 v217, 0, v203\n\t"
      "s_call_b64 s[90:91], ASPIN_%=\n\t"
      "s_branch AR198_%=\n\t"
      "AQ203_%=:\n\t"
      "s_movk_i32 s82, 12\n\t"
      "v_add_u32 v217, 256, v203\n\t"
      "s_call_b64 s[90:91], ASPIN_%=\n\t"
      "s_branch AR204_%=\n\t"
      "AQ209_%=:\n\t"
      "s_movk_i32 s82, 12\n\t"
      "v_add_u32 v217, 0, v203\n\t"
      "s_call_b64 s[90:91], ASPIN_%=\n\t"
      "s_branch AR210_%=\n\t"
      "AQ215_%=:\n\t"
      "s_movk_i32 s82, 16\n\t"
      "v_add_u32 v217, 256, v203\n\t"
      "s_call_b64 s[90:91], ASPIN_%=\n\t"
      "s_branch AR216_%=\n\t"
      "XS219_%=:\n\t"
      "s_mov_b32 s85, m0\n\t"
      "XL221_%=:\n\t"
      "s_add_u32 m0, s52, 147456\n\t"
      "s_add_u32 s83, s18, 0\n\t"
      "buffer_load_dwordx4 v200, s[44:47], s83 offen offset:0 sc0 sc1 lds\n\t"
      "buffer_load_dwordx4 v200, s[44:47], s83 offen offset:1024 sc0 sc1 lds\n\t"
      "s_add_u32 m0, s52, 122880\n\t"
      "s_add_u32 s83, s19, 0\n\t"
      "buffer_load_dwordx4 v200, s[44:47], s83 offen offset:0 sc0 sc1 lds\n\t"
      "buffer_load_dwordx4 v200, s[44:47], s83 offen offset:1024 sc0 sc1 lds\n\t"
      "s_add_u32 m0, s52, 114688\n\t"
      "s_add_u32 s83, s20, 0\n\t"
      "buffer_load_dwordx4 v200, s[44:47], s83 offen offset:0 sc0 sc1 lds\n\t"
      "buffer_load_dwordx4 v200, s[44:47], s83 offen offset:1024 sc0 sc1 lds\n\t"
      "s_call_b64 s[88:89], TMO_%=\n\t"
      "s_cbranch_scc1 XO222_%=\n\t"
      "s_waitcnt vmcnt(2)\n\t"
      "ds_read2st64_b32 v[228:229], v252 offset0:64 offset1:68\n\t"
      "ds_read2st64_b32 v[230:231], v251 offset0:224 offset1:228\n\t"
      "s_cmp_lg_u32 s65, 0\n\t"
      "s_cbranch_scc1 Lp223_%=\n\t"
      "Lr224_%=:\n\t"
      "s_waitcnt lgkmcnt(0)\n\t"
      "v_min3_u32 v228, v228, v229, v230\n\t"
      "v_min_u32 v228, v228, v231\n\t"
      "v_cmp_eq_u32 vcc, 0, v228\n\t"
      "s_cbranch_vccnz XL221_%=\n\t"
      "s_cmp_eq_u32 0, 1\n\t"
      "XO222_%=:\n\t"
      "s_cselect_b32 s75, 0x7fc00000, s75\n\t"
      "s_mov_b32 m0, s85\n\t"
      "s_branch XR220_%=\n\t"
      "AQ227_%=:\n\t"
      "s_movk_i32 s82, 16\n\t"
      "v_add_u32 v217, 0, v203\n\t"
      "s_call_b64 s[90:91], ASPIN_%=\n\t"
      "s_branch AR228_%=\n\t"
      "XS235_%=:\n\t"
      "s_mov_b32 s85, m0\n\t"
      "XL237_%=:\n\t"
      "s_add_u32 m0, s52, 114688\n\t"
      "s_add_u32 s83, s20, 0\n\t"
      "buffer_load_dwordx4 v200, s[44:47], s83 offen offset:0 sc0 sc1 lds\n\t"
      "buffer_load_dwordx4 v200, s[44:47], s83 offen offset:1024 sc0 sc1 lds\n\t"
      "s_add_u32 m0, s52, 106496\n\t"
      "s_add_u32 s83, s21, 0\n\t"
      "buffer_load_dwordx4 v200, s[44:47], s83 offen offset:0 sc0 sc1 lds\n\t"
      "buffer_load_dwordx4 v200, s[44:47], s83 offen offset:1024 sc0 sc1 lds\n\t"
      "s_add_u32 m0, s52, 98304\n\t"
      "s_add_u32 s83, s24, 0\n\t"
      "buffer_load_dwordx4 v200, s[44:47], s83 offen offset:0 sc0 sc1 lds\n\t"
      "buffer_load_dwordx4 v200, s[44:47], s83 offen offset:1024 sc0 sc1 lds\n\t"
      "s_add_u32 m0, s52, 40960\n\t"
      "s_add_u32 s83, s25, 0\n\t"
      "buffer_load_dwordx4 v200, s[44:47], s83 offen offset:0 sc0 sc1 lds\n\t"
      "buffer_load_dwordx4 v200, s[44:47], s83 offen offset:1024 sc0 sc1 lds\n\t"
      "s_add_u32 m0, s52, 32768\n\t"
      "s_add_u32 s83, s26, 0\n\t"
      "buffer_load_dwordx4 v200, s[44:47], s83 offen offset:0 sc0 sc1 lds\n\t"
      "buffer_load_dwordx4 v200, s[44:47], s83 offen offset:1024 sc0 sc1 lds\n\t"
      "s_call_b64 s[88:89], TMO_%=\n\t"
      "s_cbranch_scc1 XO238_%=\n\t"
      "s_waitcnt vmcnt(6)\n\t"
      "ds_read2st64_b32 v[228:229], v251 offset0:192 offset1:196\n\t"
      "ds_read2st64_b32 v[230:231], v251 offset0:160 offset1:164\n\t"
      "s_cmp_lg_u32 s65, 0\n\t"
      "s_cbranch_scc1 Lp239_%=\n\t"
      "Lr240_%=:\n\t"
      "s_waitcnt lgkmcnt(0)\n\t"
      "v_min3_u32 v228, v228, v229, v230\n\t"
      "v_min_u32 v228, v228, v231\n\t"
      "v_cmp_eq_u32 vcc, 0, v228\n\t"
      "s_cbranch_vccnz XL237_%=\n\t"
      "s_cmp_eq_u32 0, 1\n\t"
      "XO238_%=:\n\t"
      "s_cselect_b32 s75, 0x7fc00000, s75\n\t"
      "s_mov_b32 m0, s85\n\t"
      "s_branch XR236_%=\n\t"
      "AQ243_%=:\n\t"
      "s_movk_i32 s82, 20\n\t"
      "v_add_u32 v217, 0, v203\n\t"
      "s_call_b64 s[90:91], ASPIN_%=\n\t"
      "s_branch AR244_%=\n\t"
      "XS245_%=:\n\t"
      "s_mov_b32 s85, m0\n\t"
      "XL247_%=:\n\t"
      "s_add_u32 m0, s52, 98304\n\t"
      "s_add_u32 s83, s24, 0\n\t"
      "buffer_load_dwordx4 v200, s[44:47], s83 offen offset:0 sc0 sc1 lds\n\t"
      "buffer_load_dwordx4 v200, s[44:47], s83 offen offset:1024 sc0 sc1 lds\n\t"
      "s_add_u32 m0, s52, 40960\n\t"
      "s_add_u32 s83, s25, 0\n\t"
      "buffer_load_dwordx4 v200, s[44:47], s83 offen offset:0 sc0 sc1 lds\n\t"
      "buffer_load_dwordx4 v200, s[44:47], s83 offen offset:1024 sc0 sc1 lds\n\t"
      "s_add_u32 m0, s52, 32768\n\t"
      "s_add_u32 s83, s26, 0\n\t"
      "buffer_load_dwordx4 v200, s[44:47], s83 offen offset:0 sc0 sc1 lds\n\t"
      "buffer_load_dwordx4 v200, s[44:47], s83 offen offset:1024 sc0 sc1 lds\n\t"
      "s_add_u32 m0, s52, 73728\n\t"
      "s_add_u32 s83, s18, 8192\n\t"
      "buffer_load_dwordx4 v200, s[44:47], s83 offen offset:0 sc0 sc1 lds\n\t"
      "buffer_load_dwordx4 v200, s[44:47], s83 offen offset:1024 sc0 sc1 lds\n\t"
      "s_call_b64 s[88:89], TMO_%=\n\t"
      "s_cbranch_scc1 XO248_%=\n\t"
      "s_waitcnt vmcnt(6)\n\t"
      "ds_read2st64_b32 v[228:229], v251 offset0:128 offset1:132\n\t"
      "s_cmp_lg_u32 s65, 0\n\t"
      "s_cbranch_scc1 Lp249_%=\n\t"
      "Lr250_%=:\n\t"
      "s_waitcnt lgkmcnt(0)\n\t"
      "v_min_u32 v228, v228, v229\n\t"
      "v_cmp_eq_u32 vcc, 0, v228\n\t"
      "s_cbranch_vccnz XL247_%=\n\t"
      "s_cmp_eq_u32 0, 1\n\t"
      "XO248_%=:\n\t"
      "s_cselect_b32 s75, 0x7fc00000, s75\n\t"
      "s_mov_b32 m0, s85\n\t"
      "s_branch XR246_%=\n\t"
      "XS253_%=:\n\t"
      "s_mov_b32 s85, m0\n\t"
      "XL255_%=:\n\t"
      "s_add_u32 m0, s52, 40960\n\t"
      "s_add_u32 s83, s25, 0\n\t"
      "buffer_load_dwordx4 v200, s[44:47], s83 offen offset:0 sc0 sc1 lds\n\t"
      "buffer_load_dwordx4 v200, s[44:47], s83 offen offset:1024 sc0 sc1 lds\n\t"
      "s_add_u32 m0, s52, 32768\n\t"
      "s_add_u32 s83, s26, 0\n\t"
      "buffer_load_dwordx4 v200, s[44:47], s83 offen offset:0 sc0 sc1 lds\n\t"
      "buffer_load_dwordx4 v200, s[44:47], s83 offen offset:1024 sc0 sc1 lds\n\t"
      "s_add_u32 m0, s52, 73728\n\t"
      "s_add_u32 s83, s18, 8192\n\t"
      "buffer_load_dwordx4 v200, s[44:47], s83 offen offset:0 sc0 sc1 lds\n\t"
      "buffer_load_dwordx4 v200, s[44:47], s83 offen offset:1024 sc0 sc1 lds\n\t"
      "s_add_u32 m0, s52, 147456\n\t"
      "s_add_u32 s83, s19, 8192\n\t"
      "buffer_load_dwordx4 v200, s[44:47], s83 offen offset:0 sc0 sc1 lds\n\t"
      "buffer_load_dwordx4 v200, s[44:47], s83 offen offset:1024 sc0 sc1 lds\n\t"
      "s_call_b64 s[88:89], TMO_%=\n\t"
      "s_cbranch_scc1 XO256_%=\n\t"
      "s_waitcnt vmcnt(6)\n\t"
      "ds_read2st64_b32 v[228:229], v250 offset0:160 offset1:164\n\t"
      "s_cmp_lg_u32 s65, 0\n\t"
      "s_cbranch_scc1 Lp257_%=\n\t"
      "Lr258_%=:\n\t"
      "s_waitcnt lgkmcnt(0)\n\t"
      "v_min_u32 v228, v228, v229\n\t"
      "v_cmp_eq_u32 vcc, 0, v228\n\t"
      "s_cbranch_vccnz XL255_%=\n\t"
      "s_cmp_eq_u32 0, 1\n\t"
      "XO256_%=:\n\t"
      "s_cselect_b32 s75, 0x7fc00000, s75\n\t"
      "s_mov_b32 m0, s85\n\t"
      "s_branch XR254_%=\n\t"
      "AQ261_%=:\n\t"
      "s_movk_i32 s82, 24\n\t"
      "v_add_u32 v217, 256, v203\n\t"
      "s_call_b64 s[90:91], ASPIN_%=\n\t"
      "s_branch AR262_%=\n\t"
      "XS263_%=:\n\t"
      "s_mov_b32 s85, m0\n\t"
      "XL265_%=:\n\t"
      "s_add_u32 m0, s52, 32768\n\t"
      "s_add_u32 s83, s26, 0\n\t"
      "buffer_load_dwordx4 v200, s[44:47], s83 offen offset:0 sc0 sc1 lds\n\t"
      "buffer_load_dwordx4 v200, s[44:47], s83 offen offset:1024 sc0 sc1 lds\n\t"
      "s_add_u32 m0, s52, 73728\n\t"
      "s_add_u32 s83, s18, 8192\n\t"
      "buffer_load_dwordx4 v200, s[44:47], s83 offen offset:0 sc0 sc1 lds\n\t"
      "buffer_load_dwordx4 v200, s[44:47], s83 offen offset:1024 sc0 sc1 lds\n\t"
      "s_add_u32 m0, s52, 147456\n\t"
      "s_add_u32 s83, s19, 8192\n\t"
      "buffer_load_dwordx4 v200, s[44:47], s83 offen offset:0 sc0 sc1 lds\n\t"
      "buffer_load_dwordx4 v200, s[44:47], s83 offen offset:1024 sc0 sc1 lds\n\t"
      "s_add_u32 m0, s52, 65536\n\t"
      "s_add_u32 s83, s20, 8192\n\t"
      "buffer_load_dwordx4 v200, s[44:47], s83 offen offset:0 sc0 sc1 lds\n\t"
      "buffer_load_dwordx4 v200, s[44:47], s83 offen offset:1024 sc0 sc1 lds\n\t"
      "s_call_b64 s[88:89], TMO_%=\n\t"
      "s_cbranch_scc1 XO266_%=\n\t"
      "s_waitcnt vmcnt(6)\n\t"
      "ds_read2st64_b32 v[228:229], v250 offset0:128 offset1:132\n\t"
      "s_cmp_lg_u32 s65, 0\n\t"
      "s_cbranch_scc1 Lp267_%=\n\t"
      "Lr268_%=:\n\t"
      "s_waitcnt lgkmcnt(0)\n\t"
      "v_min_u32 v228, v228, v229\n\t"
      "v_cmp_eq_u32 vcc, 0, v228\n\t"
      "s_cbranch_vccnz XL265_%=\n\t"
      "s_cmp_eq_u32 0, 1\n\t"
      "XO266_%=:\n\t"
      "s_cselect_b32 s75, 0x7fc00000, s75\n\t"
      "s_mov_b32 m0, s85\n\t"
      "s_branch XR264_%=\n\t"
      "AQ271_%=:\n\t"
      "s_movk_i32 s82, 24\n\t"
      "v_add_u32 v217, 0, v203\n\t"
      "s_call_b64 s[90:91], ASPIN_%=\n\t"
      "s_branch AR272_%=\n\t"
      "XS273_%=:\n\t"
      "s_mov_b32 s85, m0\n\t"
      "XL275_%=:\n\t"
      "s_add_u32 m0, s52, 73728\n\t"
      "s_add_u32 s83, s18, 8192\n\t"
      "buffer_load_dwordx4 v200, s[44:47], s83 offen offset:0 sc0 sc1 lds\n\t"
      "buffer_load_dwordx4 v200, s[44:47], s83 offen offset:1024 sc0 sc1 lds\n\t"
      "s_add_u32 m0, s52, 147456\n\t"
      "s_add_u32 s83, s19, 8192\n\t"
      "buffer_load_dwordx4 v200, s[44:47], s83 offen offset:0 sc0 sc1 lds\n\t"
      "buffer_load_dwordx4 v200, s[44:47], s83 offen offset:1024 sc0 sc1 lds\n\t"
      "s_add_u32 m0, s52, 65536\n\t"
      "s_add_u32 s83, s20, 8192\n\t"
      "buffer_load_dwordx4 v200, s[44:47], s83 offen offset:0 sc0 sc1 lds\n\t"
      "buffer_load_dwordx4 v200, s[44:47], s83 offen offset:1024 sc0 sc1 lds\n\t"
      "s_add_u32 m0, s52, 122880\n\t"
      "s_add_u32 s83, s21, 8192\n\t"
      "buffer_load_dwordx4 v200, s[44:47], s83 offen offset:0 sc0 sc1 lds\n\t"
      "buffer_load_dwordx4 v200, s[44:47], s83 offen offset:1024 sc0 sc1 lds\n\t"
      "s_call_b64 s[88:89], TMO_%=\n\t"
      "s_cbranch_scc1 XO276_%=\n\t"
      "s_waitcnt vmcnt(6)\n\t"
      "ds_read2st64_b32 v[228:229], v251 offset0:32 offset1:36\n\t"
      "s_cmp_lg_u32 s65, 0\n\t"
      "s_cbranch_scc1 Lp277_%=\n\t"
      "Lr278_%=:\n\t"
      "s_waitcnt lgkmcnt(0)\n\t"
      "v_min_u32 v228, v228, v229\n\t"
      "v_cmp_eq_u32 vcc, 0, v228\n\t"
      "s_cbranch_vccnz XL275_%=\n\t"
      "s_cmp_eq_u32 0, 1\n\t"
      "XO276_%=:\n\t"
      "s_cselect_b32 s75, 0x7fc00000, s75\n\t"
      "s_mov_b32 m0, s85\n\t"
      "s_branch XR274_%=\n\t"
      "AQ281_%=:\n\t"
      "s_movk_i32 s82, 28\n\t"
      "v_add_u32 v217, 256, v203\n\t"
      "s_call_b64 s[90:91], ASPIN_%=\n\t"
      "s_branch AR282_%=\n\t"
      "XS283_%=:\n\t"
      "s_mov_b32 s85, m0\n\t"
      "XL285_%=:\n\t"
      "s_add_u32 m0, s52, 147456\n\t"
      "s_add_u32 s83, s19, 8192\n\t"
      "buffer_load_dwordx4 v200, s[44:47], s83 offen offset:0 sc0 sc1 lds\n\t"
      "buffer_load_dwordx4 v200, s[44:47], s83 offen offset:1024 sc0 sc1 lds\n\t"
      "s_add_u32 m0, s52, 65536\n\t"
      "s_add_u32 s83, s20, 8192\n\t"
      "buffer_load_dwordx4 v200, s[44:47], s83 offen offset:0 sc0 sc1 lds\n\t"
      "buffer_load_dwordx4 v200, s[44:47], s83 offen offset:1024 sc0 sc1 lds\n\t"
      "s_add_u32 m0, s52, 122880\n\t"
      "s_add_u32 s83, s21, 8192\n\t"
      "buffer_load_dwordx4 v200, s[44:47], s83 offen offset:0 sc0 sc1 lds\n\t"
      "buffer_load_dwordx4 v200, s[44:47], s83 offen offset:1024 sc0 sc1 lds\n\t"
      "s_add_u32 m0, s52, 114688\n\t"
      "s_add_u32 s83, s24, 8192\n\t"
      "buffer_load_dwordx4 v200, s[44:47], s83 offen offset:0 sc0 sc1 lds\n\t"
      "buffer_load_dwordx4 v200, s[44:47], s83 offen offset:1024 sc0 sc1 lds\n\t"
      "s_call_b64 s[88:89], TMO_%=\n\t"
      "s_cbranch_scc1 XO286_%=\n\t"
      "s_waitcnt vmcnt(6)\n\t"
      "ds_read2st64_b32 v[228:229], v252 offset0:64 offset1:68\n\t"
      "s_cmp_lg_u32 s65, 0\n\t"
      "s_cbranch_scc1 Lp287_%=\n\t"
      "Lr288_%=:\n\t"
      "s_waitcnt lgkmcnt(0)\n\t"
      "v_min_u32 v228, v228, v229\n\t"
      "v_cmp_eq_u32 vcc, 0, v228\n\t"
      "s_cbranch_vccnz XL285_%=\n\t"
      "s_cmp_eq_u32 0, 1\n\t"
      "XO286_%=:\n\t"
      "s_cselect_b32 s75, 0x7fc00000, s75\n\t"
      "s_mov_b32 m0, s85\n\t"
      "s_branch XR284_%=\n\t"
      "AQ291_%=:\n\t"
      "s_movk_i32 s82, 28\n\t"
      "v_add_u32 v217, 0, v203\n\t"
      "s_call_b64 s[90:91], ASPIN_%=\n\t"
      "s_branch AR292_%=\n\t"
      "XS293_%=:\n\t"
      "s_mov_b32 s85, m0\n\t"
      "XL295_%=:\n\t"
      "s_add_u32 m0, s52, 65536\n\t"
      "s_add_u32 s83, s20, 8192\n\t"
      "buffer_load_dwordx4 v200, s[44:47], s83 offen offset:0 sc0 sc1 lds\n\t"
      "buffer_load_dwordx4 v200, s[44:47], s83 offen offset:1024 sc0 sc1 lds\n\t"
      "s_add_u32 m0, s52, 122880\n\t"
      "s_add_u32 s83, s21, 8192\n\t"
      "buffer_load_dwordx4 v200, s[44:47], s83 offen offset:0 sc0 sc1 lds\n\t"
      "buffer_load_dwordx4 v200, s[44:47], s83 offen offset:1024 sc0 sc1 lds\n\t"
      "s_add_u32 m0, s52, 114688\n\t"
      "s_add_u32 s83, s24, 8192\n\t"
      "buffer_load_dwordx4 v200, s[44:47], s83 offen offset:0 sc0 sc1 lds\n\t"
      "buffer_load_dwordx4 v200, s[44:47], s83 offen offset:1024 sc0 sc1 lds\n\t"
      "s_add_u32 m0, s52, 106496\n\t"
      "s_add_u32 s83, s25, 8192\n\t"
      "buffer_load_dwordx4 v200, s[44:47], s83 offen offset:0 sc0 sc1 lds\n\t"
      "buffer_load_dwordx4 v200, s[44:47], s83 offen offset:1024 sc0 sc1 lds\n\t"
      "s_call_b64 s[88:89], TMO_%=\n\t"
      "s_cbranch_scc1 XO296_%=\n\t"
      "s_waitcnt vmcnt(6)\n\t"
      "ds_read2st64_b32 v[228:229], v251 offset0:0 offset1:4\n\t"
      "s_cmp_lg_u32 s65, 0\n\t"
      "s_cbranch_scc1 Lp297_%=\n\t"
      "Lr298_%=:\n\t"
      "s_waitcnt lgkmcnt(0)\n\t"
      "v_min_u32 v228, v228, v229\n\t"
      "v_cmp_eq_u32 vcc, 0, v228\n\t"
      "s_cbranch_vccnz XL295_%=\n\t"
      "s_cmp_eq_u32 0, 1\n\t"
      "XO296_%=:\n\t"
      "s_cselect_b32 s75, 0x7fc00000, s75\n\t"
      "s_mov_b32 m0, s85\n\t"
      "s_branch XR294_%=\n\t"
      "AQ301_%=:\n\t"
      "s_movk_i32 s82, 32\n\t"
      "v_add_u32 v217, 256, v203\n\t"
      "s_call_b64 s[90:91], ASPIN_%=\n\t"
      "s_branch AR302_%=\n\t"
      "XS303_%=:\n\t"
      "s_mov_b32 s85, m0\n\t"
      "XL305_%=:\n\t"
      "s_add_u32 m0, s52, 122880\n\t"
      "s_add_u32 s83, s21, 8192\n\t"
      "buffer_load_dwordx4 v200, s[44:47], s83 offen offset:0 sc0 sc1 lds\n\t"
      "buffer_load_dwordx4 v200, s[44:47], s83 offen offset:1024 sc0 sc1 lds\n\t"
      "s_add_u32 m0, s52, 114688\n\t"
      "s_add_u32 s83, s24, 8192\n\t"
      "buffer_load_dwordx4 v200, s[44:47], s83 offen offset:0 sc0 sc1 lds\n\t"
      "buffer_load_dwordx4 v200, s[44:47], s83 offen offset:1024 sc0 sc1 lds\n\t"
      "s_add_u32 m0, s52, 106496\n\t"
      "s_add_u32 s83, s25, 8192\n\t"
      "buffer_load_dwordx4 v200, s[44:47], s83 offen offset:0 sc0 sc1 lds\n\t"
      "buffer_load_dwordx4 v200, s[44:47], s83 offen offset:1024 sc0 sc1 lds\n\t"
      "s_add_u32 m0, s52, 98304\n\t"
      "s_add_u32 s83, s26, 8192\n\t"
      "buffer_load_dwordx4 v200, s[44:47], s83 offen offset:0 sc0 sc1 lds\n\t"
      "buffer_load_dwordx4 v200, s[44:47], s83 offen offset:1024 sc0 sc1 lds\n\t"
      "s_call_b64 s[88:89], TMO_%=\n\t"
      "s_cbranch_scc1 XO306_%=\n\t"
      "s_waitcnt vmcnt(6)\n\t"
      "ds_read2st64_b32 v[228:229], v251 offset0:224 offset1:228\n\t"
      "s_cmp_lg_u32 s65, 0\n\t"
      "s_cbranch_scc1 Lp307_%=\n\t"
      "Lr308_%=:\n\t"
      "s_waitcnt lgkmcnt(0)\n\t"
      "v_min_u32 v228, v228, v229\n\t"
      "v_cmp_eq_u32 vcc, 0, v228\n\t"
      "s_cbranch_vccnz XL305_%=\n\t"
      "s_cmp_eq_u32 0, 1\n\t"
      "XO306_%=:\n\t"
      "s_cselect_b32 s75, 0x7fc00000, s75\n\t"
      "s_mov_b32 m0, s85\n\t"
      "s_branch XR304_%=\n\t"
      "AQ311_%=:\n\t"
      "s_movk_i32 s82, 32\n\t"
      "v_add_u32 v217, 0, v203\n\t"
      "s_call_b64 s[90:91], ASPIN_%=\n\t"
      "s_branch AR312_%=\n\t"
      "XS313_%=:\n\t"
      "s_mov_b32 s85, m0\n\t"
      "XL315_%=:\n\t"
      "s_add_u32 m0, s52, 114688\n\t"
      "s_add_u32 s83, s24, 8192\n\t"
      "buffer_load_dwordx4 v200, s[44:47], s83 offen offset:0 sc0 sc1 lds\n\t"
      "buffer_load_dwordx4 v200, s[44:47], s83 offen offset:1024 sc0 sc1 lds\n\t"
      "s_add_u32 m0, s52, 106496\n\t"
      "s_add_u32 s83, s25, 8192\n\t"
      "buffer_load_dwordx4 v200, s[44:47], s83 offen offset:0 sc0 sc1 lds\n\t"
      "buffer_load_dwordx4 v200, s[44:47], s83 offen offset:1024 sc0 sc1 lds\n\t"
      "s_add_u32 m0, s52, 98304\n\t"
      "s_add_u32 s83, s26, 8192\n\t"
      "buffer_load_dwordx4 v200, s[44:47], s83 offen offset:0 sc0 sc1 lds\n\t"
      "buffer_load_dwordx4 v200, s[44:47], s83 offen offset:1024 sc0 sc1 lds\n\t"
      "s_add_u32 m0, s52, 40960\n\t"
      "s_add_u32 s83, s18, 16384\n\t"
      "buffer_load_dwordx4 v200, s[44:47], s83 offen offset:0 sc0 sc1 lds\n\t"
      "buffer_load_dwordx4 v200, s[44:47], s83 offen offset:1024 sc0 sc1 lds\n\t"
      "s_call_b64 s[88:89], TMO_%=\n\t"
      "s_cbranch_scc1 XO316_%=\n\t"
      "s_waitcnt vmcnt(6)\n\t"
      "ds_read2st64_b32 v[228:229], v251 offset0:192 offset1:196\n\t"
      "s_cmp_lg_u32 s65, 0\n\t"
      "s_cbranch_scc1 Lp317_%=\n\t"
      "Lr318_%=:\n\t"
      "s_waitcnt lgkmcnt(0)\n\t"
      "v_min_u32 v228, v228, v229\n\t"
      "v_cmp_eq_u32 vcc, 0, v228\n\t"
      "s_cbranch_vccnz XL315_%=\n\t"
      "s_cmp_eq_u32 0, 1\n\t"
      "XO316_%=:\n\t"
      "s_cselect_b32 s75, 0x7fc00000, s75\n\t"
      "s_mov_b32 m0, s85\n\t"
      "s_branch XR314_%=\n\t"
      "AQ321_%=:\n\t"
      "s_movk_i32 s82, 36\n\t"
      "v_add_u32 v217, 256, v203\n\t"
      "s_call_b64 s[90:91], ASPIN_%=\n\t"
      "s_branch AR322_%=\n\t"
      "XS323_%=:\n\t"
      "s_mov_b32 s85, m0\n\t"
      "XL325_%=:\n\t"
      "s_add_u32 m0, s52, 106496\n\t"
      "s_add_u32 s83, s25, 8192\n\t"
      "buffer_load_dwordx4 v200, s[44:47], s83 offen offset:0 sc0 sc1 lds\n\t"
      "buffer_load_dwordx4 v200, s[44:47], s83 offen offset:1024 sc0 sc1 lds\n\t"
      "s_add_u32 m0, s52, 98304\n\t"
      "s_add_u32 s83, s26, 8192\n\t"
      "buffer_load_dwordx4 v200, s[44:47], s83 offen offset:0 sc0 sc1 lds\n\t"
      "buffer_load_dwordx4 v200, s[44:47], s83 offen offset:1024 sc0 sc1 lds\n\t"
      "s_add_u32 m0, s52, 40960\n\t"
      "s_add_u32 s83, s18, 16384\n\t"
      "buffer_load_dwordx4 v200, s[44:47], s83 offen offset:0 sc0 sc1 lds\n\t"
      "buffer_load_dwordx4 v200, s[44:47], s83 offen offset:1024 sc0 sc1 lds\n\t"
      "s_add_u32 m0, s52, 32768\n\t"
      "s_add_u32 s83, s19, 16384\n\t"
      "buffer_load_dwordx4 v200, s[44:47], s83 offen offset:0 sc0 sc1 lds\n\t"
      "buffer_load_dwordx4 v200, s[44:47], s83 offen offset:1024 sc0 sc1 lds\n\t"
      "s_call_b64 s[88:89], TMO_%=\n\t"
      "s_cbranch_scc1 XO326_%=\n\t"
      "s_waitcnt vmcnt(6)\n\t"
      "ds_read2st64_b32 v[228:229], v251 offset0:160 offset1:164\n\t"
      "s_cmp_lg_u32 s65, 0\n\t"
      "s_cbranch_scc1 Lp327_%=\n\t"
      "Lr328_%=:\n\t"
      "s_waitcnt lgkmcnt(0)\n\t"
      "v_min_u32 v228, v228, v229\n\t"
      "v_cmp_eq_u32 vcc, 0, v228\n\t"
      "s_cbranch_vccnz XL325_%=\n\t"
      "s_cmp_eq_u32 0, 1\n\t"
      "XO326_%=:\n\t"
      "s_cselect_b32 s75, 0x7fc00000, s75\n\t"
      "s_mov_b32 m0, s85\n\t"
      "s_branch XR324_%=\n\t"
      "AQ331_%=:\n\t"
      "s_movk_i32 s82, 36\n\t"
      "v_add_u32 v217, 0, v203\n\t"
      "s_call_b64 s[90:91], ASPIN_%=\n\t"
      "s_branch AR332_%=\n\t"
      "XS333_%=:\n\t"
      "s_mov_b32 s85, m0\n\t"
      "XL335_%=:\n\t"
      "s_add_u32 m0, s52, 98304\n\t"
      "s_add_u32 s83, s26, 8192\n\t"
      "buffer_load_dwordx4 v200, s[44:47], s83 offen offset:0 sc0 sc1 lds\n\t"
      "buffer_load_dwordx4 v200, s[44:47], s83 offen offset:1024 sc0 sc1 lds\n\t"
      "s_add_u32 m0, s52, 40960\n\t"
      "s_add_u32 s83, s18, 16384\n\t"
      "buffer_load_dwordx4 v200, s[44:47], s83 offen offset:0 sc0 sc1 lds\n\t"
      "buffer_load_dwordx4 v200, s[44:47], s83 offen offset:1024 sc0 sc1 lds\n\t"
      "s_add_u32 m0, s52, 32768\n\t"
      "s_add_u32 s83, s19, 16384\n\t"
      "buffer_load_dwordx4 v200, s[44:47], s83 offen offset:0 sc0 sc1 lds\n\t"
      "buffer_load_dwordx4 v200, s[44:47], s83 offen offset:1024 sc0 sc1 lds\n\t"
      "s_add_u32 m0, s52, 73728\n\t"
      "s_add_u32 s83, s20, 16384\n\t"
      "buffer_load_dwordx4 v200, s[44:47], s83 offen offset:0 sc0 sc1 lds\n\t"
      "buffer_load_dwordx4 v200, s[44:47], s83 offen offset:1024 sc0 sc1 lds\n\t"
      "s_call_b64 s[88:89], TMO_%=\n\t"
      "s_cbranch_scc1 XO336_%=\n\t"
      "s_waitcnt vmcnt(6)\n\t"
      "ds_read2st64_b32 v[228:229], v251 offset0:128 offset1:132\n\t"
      "s_cmp_lg_u32 s65, 0\n\t"
      "s_cbranch_scc1 Lp337_%=\n\t"
      "Lr338_%=:\n\t"
      "s_waitcnt lgkmcnt(0)\n\t"
      "v_min_u32 v228, v228, v229\n\t"
      "v_cmp_eq_u32 vcc, 0, v228\n\t"
      "s_cbranch_vccnz XL335_%=\n\t"
      "s_cmp_eq_u32 0, 1\n\t"
      "XO336_%=:\n\t"
      "s_cselect_b32 s75, 0x7fc00000, s75\n\t"
      "s_mov_b32 m0, s85\n\t"
      "s_branch XR334_%=\n\t"
      "AQ341_%=:\n\t"
      "s_movk_i32 s82, 40\n\t"
      "v_add_u32 v217, 256, v203\n\t"
      "s_call_b64 s[90:91], ASPIN_%=\n\t"
      "s_branch AR342_%=\n\t"
      "XS343_%=:\n\t"
      "s_mov_b32 s85, m0\n\t"
      "XL345_%=:\n\t"
      "s_add_u32 m0, s52, 40960\n\t"
      "s_add_u32 s83, s18, 16384\n\t"
      "buffer_load_dwordx4 v200, s[44:47], s83 offen offset:0 sc0 sc1 lds\n\t"
      "buffer_load_dwordx4 v200, s[44:47], s83 offen offset:1024 sc0 sc1 lds\n\t"
      "s_add_u32 m0, s52, 32768\n\t"
      "s_add_u32 s83, s19, 16384\n\t"
      "buffer_load_dwordx4 v200, s[44:47], s83 offen offset:0 sc0 sc1 lds\n\t"
      "buffer_load_dwordx4 v200, s[44:47], s83 offen offset:1024 sc0 sc1 lds\n\t"
      "s_add_u32 m0, s52, 73728\n\t"
      "s_add_u32 s83, s20, 16384\n\t"
      "buffer_load_dwordx4 v200, s[44:47], s83 offen offset:0 sc0 sc1 lds\n\t"
      "buffer_load_dwordx4 v200, s[44:47], s83 offen offset:1024 sc0 sc1 lds\n\t"
      "s_add_u32 m0, s52, 147456\n\t"
      "s_add_u32 s83, s21, 16384\n\t"
      "buffer_load_dwordx4 v200, s[44:47], s83 offen offset:0 sc0 sc1 lds\n\t"
      "buffer_load_dwordx4 v200, s[44:47], s83 offen offset:1024 sc0 sc1 lds\n\t"
      "s_call_b64 s[88:89], TMO_%=\n\t"
      "s_cbranch_scc1 XO346_%=\n\t"
      "s_waitcnt vmcnt(6)\n\t"
      "ds_read2st64_b32 v[228:229], v250 offset0:160 offset1:164\n\t"
      "s_cmp_lg_u32 s65, 0\n\t"
      "s_cbranch_scc1 Lp347_%=\n\t"
      "Lr348_%=:\n\t"
      "s_waitcnt lgkmcnt(0)\n\t"
      "v_min_u32 v228, v228, v229\n\t"
      "v_cmp_eq_u32 vcc, 0, v228\n\t"
      "s_cbranch_vccnz XL345_%=\n\t"
      "s_cmp_eq_u32 0, 1\n\t"
      "XO346_%=:\n\t"
      "s_cselect_b32 s75, 0x7fc00000, s75\n\t"
      "s_mov_b32 m0, s85\n\t"
      "s_branch XR344_%=\n\t"
      "AQ351_%=:\n\t"
      "s_movk_i32 s82, 40\n\t"
      "v_add_u32 v217, 0, v203\n\t"
      "s_call_b64 s[90:91], ASPIN_%=\n\t"
      "s_branch AR352_%=\n\t"
      "XS353_%=:\n\t"
      "s_mov_b32 s85, m0\n\t"
      "XL355_%=:\n\t"
      "s_add_u32 m0, s52, 32768\n\t"
      "s_add_u32 s83, s19, 16384\n\t"
      "buffer_load_dwordx4 v200, s[44:47], s83 offen offset:0 sc0 sc1 lds\n\t"
      "buffer_load_dwordx4 v200, s[44:47], s83 offen offset:1024 sc0 sc1 lds\n\t"
      "s_add_u32 m0, s52, 73728\n\t"
      "s_add_u32 s83, s20, 16384\n\t"
      "buffer_load_dwordx4 v200, s[44:47], s83 offen offset:0 sc0 sc1 lds\n\t"
      "buffer_load_dwordx4 v200, s[44:47], s83 offen offset:1024 sc0 sc1 lds\n\t"
      "s_add_u32 m0, s52, 147456\n\t"
      "s_add_u32 s83, s21, 16384\n\t"
      "buffer_load_dwordx4 v200, s[44:47], s83 offen offset:0 sc0 sc1 lds\n\t"
      "buffer_load_dwordx4 v200, s[44:47], s83 offen offset:1024 sc0 sc1 lds\n\t"
      "s_add_u32 m0, s52, 65536\n\t"
      "s_add_u32 s83, s24, 16384\n\t"
      "buffer_load_dwordx4 v200, s[44:47], s83 offen offset:0 sc0 sc1 lds\n\t"
      "buffer_load_dwordx4 v200, s[44:47], s83 offen offset:1024 sc0 sc1 lds\n\t"
      "s_call_b64 s[88:89], TMO_%=\n\t"
      "s_cbranch_scc1 XO356_%=\n\t"
      "s_waitcnt vmcnt(6)\n\t"
      "ds_read2st64_b32 v[228:229], v250 offset0:128 offset1:132\n\t"
      "s_cmp_lg_u32 s65, 0\n\t"
      "s_cbranch_scc1 Lp357_%=\n\t"
      "Lr358_%=:\n\t"
      "s_waitcnt lgkmcnt(0)\n\t"
      "v_min_u32 v228, v228, v229\n\t"
      "v_cmp_eq_u32 vcc, 0, v228\n\t"
      "s_cbranch_vccnz XL355_%=\n\t"
      "s_cmp_eq_u32 0, 1\n\t"
      "XO356_%=:\n\t"
      "s_cselect_b32 s75, 0x7fc00000, s75\n\t"
      "s_mov_b32 m0, s85\n\t"
      "s_branch XR354_%=\n\t"
      "AQ361_%=:\n\t"
      "s_movk_i32 s82, 44\n\t"
      "v_add_u32 v217, 256, v203\n\t"
      "s_call_b64 s[90:91], ASPIN_%=\n\t"
      "s_branch AR362_%=\n\t"
      "XS363_%=:\n\t"
      "s_mov_b32 s85, m0\n\t"
      "XL365_%=:\n\t"
      "s_add_u32 m0, s52, 73728\n\t"
      "s_add_u32 s83, s20, 16384\n\t"
      "buffer_load_dwordx4 v200, s[44:47], s83 offen offset:0 sc0 sc1 lds\n\t"
      "buffer_load_dwordx4 v200, s[44:47], s83 offen offset:1024 sc0 sc1 lds\n\t"
      "s_add_u32 m0, s52, 147456\n\t"
      "s_add_u32 s83, s21, 16384\n\t"
      "buffer_load_dwordx4 v200, s[44:47], s83 offen offset:0 sc0 sc1 lds\n\t"
      "buffer_load_dwordx4 v200, s[44:47], s83 offen offset:1024 sc0 sc1 lds\n\t"
      "s_add_u32 m0, s52, 65536\n\t"
      "s_add_u32 s83, s24, 16384\n\t"
      "buffer_load_dwordx4 v200, s[44:47], s83 offen offset:0 sc0 sc1 lds\n\t"
      "buffer_load_dwordx4 v200, s[44:47], s83 offen offset:1024 sc0 sc1 lds\n\t"
      "s_add_u32 m0, s52, 122880\n\t"
      "s_add_u32 s83, s25, 16384\n\t"
      "buffer_load_dwordx4 v200, s[44:47], s83 offen offset:0 sc0 sc1 lds\n\t"
      "buffer_load_dwordx4 v200, s[44:47], s83 offen offset:1024 sc0 sc1 lds\n\t"
      "s_call_b64 s[88:89], TMO_%=\n\t"
      "s_cbranch_scc1 XO366_%=\n\t"
      "s_waitcnt vmcnt(6)\n\t"
      "ds_read2st64_b32 v[228:229], v251 offset0:32 offset1:36\n\t"
      "s_cmp_lg_u32 s65, 0\n\t"
      "s_cbranch_scc1 Lp367_%=\n\t"
      "Lr368_%=:\n\t"
      "s_waitcnt lgkmcnt(0)\n\t"
      "v_min_u32 v228, v228, v229\n\t"
      "v_cmp_eq_u32 vcc, 0, v228\n\t"
      "s_cbranch_vccnz XL365_%=\n\t"
      "s_cmp_eq_u32 0, 1\n\t"
      "XO366_%=:\n\t"
      "s_cselect_b32 s75, 0x7fc00000, s75\n\t"
      "s_mov_b32 m0, s85\n\t"
      "s_branch XR364_%=\n\t"
      "AQ371_%=:\n\t"
      "s_movk_i32 s82, 44\n\t"
      "v_add_u32 v217, 0, v203\n\t"
      "s_call_b64 s[90:91], ASPIN_%=\n\t"
      "s_branch AR372_%=\n\t"
      "XS373_%=:\n\t"
      "s_mov_b32 s85, m0\n\t"
      "XL375_%=:\n\t"
      "s_add_u32 m0, s52, 147456\n\t"
      "s_add_u32 s83, s21, 16384\n\t"
      "buffer_load_dwordx4 v200, s[44:47], s83 offen offset:0 sc0 sc1 lds\n\t"
      "buffer_load_dwordx4 v200, s[44:47], s83 offen offset:1024 sc0 sc1 lds\n\t"
      "s_add_u32 m0, s52, 65536\n\t"
      "s_add_u32 s83, s24, 16384\n\t"
      "buffer_load_dwordx4 v200, s[44:47], s83 offen offset:0 sc0 sc1 lds\n\t"
      "buffer_load_dwordx4 v200, s[44:47], s83 offen offset:1024 sc0 sc1 lds\n\t"
      "s_add_u32 m0, s52, 122880\n\t"
      "s_add_u32 s83, s25, 16384\n\t"
      "buffer_load_dwordx4 v200, s[44:47], s83 offen offset:0 sc0 sc1 lds\n\t"
      "buffer_load_dwordx4 v200, s[44:47], s83 offen offset:1024 sc0 sc1 lds\n\t"
      "s_add_u32 m0, s52, 114688\n\t"
      "s_add_u32 s83, s26, 16384\n\t"
      "buffer_load_dwordx4 v200, s[44:47], s83 offen offset:0 sc0 sc1 lds\n\t"
      "buffer_load_dwordx4 v200, s[44:47], s83 offen offset:1024 sc0 sc1 lds\n\t"
      "s_call_b64 s[88:89], TMO_%=\n\t"
      "s_cbranch_scc1 XO376_%=\n\t"
      "s_waitcnt vmcnt(6)\n\t"
      "ds_read2st64_b32 v[228:229], v252 offset0:64 offset1:68\n\t"
      "s_cmp_lg_u32 s65, 0\n\t"
      "s_cbranch_scc1 Lp377_%=\n\t"
      "Lr378_%=:\n\t"
      "s_waitcnt lgkmcnt(0)\n\t"
      "v_min_u32 v228, v228, v229\n\t"
      "v_cmp_eq_u32 vcc, 0, v228\n\t"
      "s_cbranch_vccnz XL375_%=\n\t"
      "s_cmp_eq_u32 0, 1\n\t"
      "XO376_%=:\n\t"
      "s_cselect_b32 s75, 0x7fc00000, s75\n\t"
      "s_mov_b32 m0, s85\n\t"
      "s_branch XR374_%=\n\t"
      "AQ381_%=:\n\t"
      "s_movk_i32 s82, 48\n\t"
      "v_add_u32 v217, 256, v203\n\t"
      "s_call_b64 s[90:91], ASPIN_%=\n\t"
      "s_branch AR382_%=\n\t"
      "XS383_%=:\n\t"
      "s_mov_b32 s85, m0\n\t"
      "XL385_%=:\n\t"
      "s_add_u32 m0, s52, 65536\n\t"
      "s_add_u32 s83, s24, 16384\n\t"
      "buffer_load_dwordx4 v200, s[44:47], s83 offen offset:0 sc0 sc1 lds\n\t"
      "buffer_load_dwordx4 v200, s[44:47], s83 offen offset:1024 sc0 sc1 lds\n\t"
      "s_add_u32 m0, s52, 122880\n\t"
      "s_add_u32 s83, s25, 16384\n\t"
      "buffer_load_dwordx4 v200, s[44:47], s83 offen offset:0 sc0 sc1 lds\n\t"
      "buffer_load_dwordx4 v200, s[44:47], s83 offen offset:1024 sc0 sc1 lds\n\t"
      "s_add_u32 m0, s52, 114688\n\t"
      "s_add_u32 s83, s26, 16384\n\t"
      "buffer_load_dwordx4 v200, s[44:47], s83 offen offset:0 sc0 sc1 lds\n\t"
      "buffer_load_dwordx4 v200, s[44:47], s83 offen offset:1024 sc0 sc1 lds\n\t"
      "s_add_u32 m0, s52, 106496\n\t"
      "s_add_u32 s83, s18, 24576\n\t"
      "buffer_load_dwordx4 v200, s[44:47], s83 offen offset:0 sc0 sc1 lds\n\t"
      "buffer_load_dwordx4 v200, s[44:47], s83 offen offset:1024 sc0 sc1 lds\n\t"
      "s_call_b64 s[88:89], TMO_%=\n\t"
      "s_cbranch_scc1 XO386_%=\n\t"
      "s_waitcnt vmcnt(6)\n\t"
      "ds_read2st64_b32 v[228:229], v251 offset0:0 offset1:4\n\t"
      "s_cmp_lg_u32 s65, 0\n\t"
      "s_cbranch_scc1 Lp387_%=\n\t"
      "Lr388_%=:\n\t"
      "s_waitcnt lgkmcnt(0)\n\t"
      "v_min_u32 v228, v228, v229\n\t"
      "v_cmp_eq_u32 vcc, 0, v228\n\t"
      "s_cbranch_vccnz XL385_%=\n\t"
      "s_cmp_eq_u32 0, 1\n\t"
      "XO386_%=:\n\t"
      "s_cselect_b32 s75, 0x7fc00000, s75\n\t"
      "s_mov_b32 m0, s85\n\t"
      "s_branch XR384_%=\n\t"
      "AQ391_%=:\n\t"
      "s_movk_i32 s82, 48\n\t"
      "v_add_u32 v217, 0, v203\n\t"
      "s_call_b64 s[90:91], ASPIN_%=\n\t"
      "s_branch AR392_%=\n\t"
      "XS393_%=:\n\t"
      "s_mov_b32 s85, m0\n\t"
      "XL395_%=:\n\t"
      "s_add_u32 m0, s52, 122880\n\t"
      "s_add_u32 s83, s25, 16384\n\t"
      "buffer_load_dwordx4 v200, s[44:47], s83 offen offset:0 sc0 sc1 lds\n\t"
      "buffer_load_dwordx4 v200, s[44:47], s83 offen offset:1024 sc0 sc1 lds\n\t"
      "s_add_u32 m0, s52, 114688\n\t"
      "s_add_u32 s83, s26, 16384\n\t"
      "buffer_load_dwordx4 v200, s[44:47], s83 offen offset:0 sc0 sc1 lds\n\t"
      "buffer_load_dwordx4 v200, s[44:47], s83 offen offset:1024 sc0 sc1 lds\n\t"
      "s_add_u32 m0, s52, 106496\n\t"
      "s_add_u32 s83, s18, 24576\n\t"
      "buffer_load_dwordx4 v200, s[44:47], s83 offen offset:0 sc0 sc1 lds\n\t"
      "buffer_load_dwordx4 v200, s[44:47], s83 offen offset:1024 sc0 sc1 lds\n\t"
      "s_add_u32 m0, s52, 98304\n\t"
      "s_add_u32 s83, s19, 24576\n\t"
      "buffer_load_dwordx4 v200, s[44:47], s83 offen offset:0 sc0 sc1 lds\n\t"
      "buffer_load_dwordx4 v200, s[44:47], s83 offen offset:1024 sc0 sc1 lds\n\t"
      "s_call_b64 s[88:89], TMO_%=\n\t"
      "s_cbranch_scc1 XO396_%=\n\t"
      "s_waitcnt vmcnt(6)\n\t"
      "ds_read2st64_b32 v[228:229], v251 offset0:224 offset1:228\n\t"
      "s_cmp_lg_u32 s65, 0\n\t"
      "s_cbranch_scc1 Lp397_%=\n\t"
      "Lr398_%=:\n\t"
      "s_waitcnt lgkmcnt(0)\n\t"
      "v_min_u32 v228, v228, v229\n\t"
      "v_cmp_eq_u32 vcc, 0, v228\n\t"
      "s_cbranch_vccnz XL395_%=\n\t"
      "s_cmp_eq_u32 0, 1\n\t"
      "XO396_%=:\n\t"
      "s_cselect_b32 s75, 0x7fc00000, s75\n\t"
      "s_mov_b32 m0, s85\n\t"
      "s_branch XR394_%=\n\t"
      "AQ401_%=:\n\t"
      "s_movk_i32 s82, 52\n\t"
      "v_add_u32 v217, 256, v203\n\t"
      "s_call_b64 s[90:91], ASPIN_%=\n\t"
      "s_branch AR402_%=\n\t"
      "XS403_%=:\n\t"
      "s_mov_b32 s85, m0\n\t"
      "XL405_%=:\n\t"
      "s_add_u32 m0, s52, 114688\n\t"
      "s_add_u32 s83, s26, 16384\n\t"
      "buffer_load_dwordx4 v200, s[44:47], s83 offen offset:0 sc0 sc1 lds\n\t"
      "buffer_load_dwordx4 v200, s[44:47], s83 offen offset:1024 sc0 sc1 lds\n\t"
      "s_add_u32 m0, s52, 106496\n\t"
      "s_add_u32 s83, s18, 24576\n\t"
      "buffer_load_dwordx4 v200, s[44:47], s83 offen offset:0 sc0 sc1 lds\n\t"
      "buffer_load_dwordx4 v200, s[44:47], s83 offen offset:1024 sc0 sc1 lds\n\t"
      "s_add_u32 m0, s52, 98304\n\t"
      "s_add_u32 s83, s19, 24576\n\t"
      "buffer_load_dwordx4 v200, s[44:47], s83 offen offset:0 sc0 sc1 lds\n\t"
      "buffer_load_dwordx4 v200, s[44:47], s83 offen offset:1024 sc0 sc1 lds\n\t"
      "s_add_u32 m0, s52, 40960\n\t"
      "s_add_u32 s83, s20, 24576\n\t"
      "buffer_load_dwordx4 v200, s[44:47], s83 offen offset:0 sc0 sc1 lds\n\t"
      "buffer_load_dwordx4 v200, s[44:47], s83 offen offset:1024 sc0 sc1 lds\n\t"
      "s_call_b64 s[88:89], TMO_%=\n\t"
      "s_cbranch_scc1 XO406_%=\n\t"
      "s_waitcnt vmcnt(6)\n\t"
      "ds_read2st64_b32 v[228:229], v251 offset0:192 offset1:196\n\t"
      "s_cmp_lg_u32 s65, 0\n\t"
      "s_cbranch_scc1 Lp407_%=\n\t"
      "Lr408_%=:\n\t"
      "s_waitcnt lgkmcnt(0)\n\t"
      "v_min_u32 v228, v228, v229\n\t"
      "v_cmp_eq_u32 vcc, 0, v228\n\t"
      "s_cbranch_vccnz XL405_%=\n\t"
      "s_cmp_eq_u32 0, 1\n\t"
      "XO406_%=:\n\t"
      "s_cselect_b32 s75, 0x7fc00000, s75\n\t"
      "s_mov_b32 m0, s85\n\t"
      "s_branch XR404_%=\n\t"
      "AQ411_%=:\n\t"
      "s_movk_i32 s82, 52\n\t"
      "v_add_u32 v217, 0, v203\n\t"
      "s_call_b64 s[90:91], ASPIN_%=\n\t"
      "s_branch AR412_%=\n\t"
      "XS413_%=:\n\t"
      "s_mov_b32 s85, m0\n\t"
      "XL415_%=:\n\t"
      "s_add_u32 m0, s52, 106496\n\t"
      "s_add_u32 s83, s18, 24576\n\t"
      "buffer_load_dwordx4 v200, s[44:47], s83 offen offset:0 sc0 sc1 lds\n\t"
      "buffer_load_dwordx4 v200, s[44:47], s83 offen offset:1024 sc0 sc1 lds\n\t"
      "s_add_u32 m0, s52, 98304\n\t"
      "s_add_u32 s83, s19, 24576\n\t"
      "buffer_load_dwordx4 v200, s[44:47], s83 offen offset:0 sc0 sc1 lds\n\t"
      "buffer_load_dwordx4 v200, s[44:47], s83 offen offset:1024 sc0 sc1 lds\n\t"
      "s_add_u32 m0, s52, 40960\n\t"
      "s_add_u32 s83, s20, 24576\n\t"
      "buffer_load_dwordx4 v200, s[44:47], s83 offen offset:0 sc0 sc1 lds\n\t"
      "buffer_load_dwordx4 v200, s[44:47], s83 offen offset:1024 sc0 sc1 lds\n\t"
      "s_add_u32 m0, s52, 32768\n\t"
      "s_add_u32 s83, s21, 24576\n\t"
      "buffer_load_dwordx4 v200, s[44:47], s83 offen offset:0 sc0 sc1 lds\n\t"
      "buffer_load_dwordx4 v200, s[44:47], s83 offen offset:1024 sc0 sc1 lds\n\t"
      "s_call_b64 s[88:89], TMO_%=\n\t"
      "s_cbranch_scc1 XO416_%=\n\t"
      "s_waitcnt vmcnt(6)\n\t"
      "ds_read2st64_b32 v[228:229], v251 offset0:160 offset1:164\n\t"
      "s_cmp_lg_u32 s65, 0\n\t"
      "s_cbranch_scc1 Lp417_%=\n\t"
      "Lr418_%=:\n\t"
      "s_waitcnt lgkmcnt(0)\n\t"
      "v_min_u32 v228, v228, v229\n\t"
      "v_cmp_eq_u32 vcc, 0, v228\n\t"
      "s_cbranch_vccnz XL415_%=\n\t"
      "s_cmp_eq_u32 0, 1\n\t"
      "XO416_%=:\n\t"
      "s_cselect_b32 s75, 0x7fc00000, s75\n\t"
      "s_mov_b32 m0, s85\n\t"
      "s_branch XR414_%=\n\t"
      "AQ421_%=:\n\t"
      "s_movk_i32 s82, 56\n\t"
      "v_add_u32 v217, 256, v203\n\t"
      "s_call_b64 s[90:91], ASPIN_%=\n\t"
      "s_branch AR422_%=\n\t"
      "XS423_%=:\n\t"
      "s_mov_b32 s85, m0\n\t"
      "XL425_%=:\n\t"
      "s_add_u32 m0, s52, 98304\n\t"
      "s_add_u32 s83, s19, 24576\n\t"
      "buffer_load_dwordx4 v200, s[44:47], s83 offen offset:0 sc0 sc1 lds\n\t"
      "buffer_load_dwordx4 v200, s[44:47], s83 offen offset:1024 sc0 sc1 lds\n\t"
      "s_add_u32 m0, s52, 40960\n\t"
      "s_add_u32 s83, s20, 24576\n\t"
      "buffer_load_dwordx4 v200, s[44:47], s83 offen offset:0 sc0 sc1 lds\n\t"
      "buffer_load_dwordx4 v200, s[44:47], s83 offen offset:1024 sc0 sc1 lds\n\t"
      "s_add_u32 m0, s52, 32768\n\t"
      "s_add_u32 s83, s21, 24576\n\t"
      "buffer_load_dwordx4 v200, s[44:47], s83 offen offset:0 sc0 sc1 lds\n\t"
      "buffer_load_dwordx4 v200, s[44:47], s83 offen offset:1024 sc0 sc1 lds\n\t"
      "s_add_u32 m0, s52, 73728\n\t"
      "s_add_u32 s83, s24, 24576\n\t"
      "buffer_load_dwordx4 v200, s[44:47], s83 offen offset:0 sc0 sc1 lds\n\t"
      "buffer_load_dwordx4 v200, s[44:47], s83 offen offset:1024 sc0 sc1 lds\n\t"
      "s_call_b64 s[88:89], TMO_%=\n\t"
      "s_cbranch_scc1 XO426_%=\n\t"
      "s_waitcnt vmcnt(6)\n\t"
      "ds_read2st64_b32 v[228:229], v251 offset0:128 offset1:132\n\t"
      "s_cmp_lg_u32 s65, 0\n\t"
      "s_cbranch_scc1 Lp427_%=\n\t"
      "Lr428_%=:\n\t"
      "s_waitcnt lgkmcnt(0)\n\t"
      "v_min_u32 v228, v228, v229\n\t"
      "v_cmp_eq_u32 vcc, 0, v228\n\t"
      "s_cbranch_vccnz XL425_%=\n\t"
      "s_cmp_eq_u32 0, 1\n\t"
      "XO426_%=:\n\t"
      "s_cselect_b32 s75, 0x7fc00000, s75\n\t"
      "s_mov_b32 m0, s85\n\t"
      "s_branch XR424_%=\n\t"
      "AQ431_%=:\n\t"
      "s_movk_i32 s82, 56\n\t"
      "v_add_u32 v217, 0, v203\n\t"
      "s_call_b64 s[90:91], ASPIN_%=\n\t"
      "s_branch AR432_%=\n\t"
      "XS433_%=:\n\t"
      "s_mov_b32 s85, m0\n\t"
      "XL435_%=:\n\t"
      "s_add_u32 m0, s52, 40960\n\t"
      "s_add_u32 s83, s20, 24576\n\t"
      "buffer_load_dwordx4 v200, s[44:47], s83 offen offset:0 sc0 sc1 lds\n\t"
      "buffer_load_dwordx4 v200, s[44:47], s83 offen offset:1024 sc0 sc1 lds\n\t"
      "s_add_u32 m0, s52, 32768\n\t"
      "s_add_u32 s83, s21, 24576\n\t"
      "buffer_load_dwordx4 v200, s[44:47], s83 offen offset:0 sc0 sc1 lds\n\t"
      "buffer_load_dwordx4 v200, s[44:47], s83 offen offset:1024 sc0 sc1 lds\n\t"
      "s_add_u32 m0, s52, 73728\n\t"
      "s_add_u32 s83, s24, 24576\n\t"
      "buffer_load_dwordx4 v200, s[44:47], s83 offen offset:0 sc0 sc1 lds\n\t"
      "buffer_load_dwordx4 v200, s[44:47], s83 offen offset:1024 sc0 sc1 lds\n\t"
      "s_add_u32 m0, s52, 147456\n\t"
      "s_add_u32 s83, s25, 24576\n\t"
      "buffer_load_dwordx4 v200, s[44:47], s83 offen offset:0 sc0 sc1 lds\n\t"
      "buffer_load_dwordx4 v200, s[44:47], s83 offen offset:1024 sc0 sc1 lds\n\t"
      "s_call_b64 s[88:89], TMO_%=\n\t"
      "s_cbranch_scc1 XO436_%=\n\t"
      "s_waitcnt vmcnt(6)\n\t"
      "ds_read2st64_b32 v[228:229], v250 offset0:160 offset1:164\n\t"
      "s_cmp_lg_u32 s65, 0\n\t"
      "s_cbranch_scc1 Lp437_%=\n\t"
      "Lr438_%=:\n\t"
      "s_waitcnt lgkmcnt(0)\n\t"
      "v_min_u32 v228, v228, v229\n\t"
      "v_cmp_eq_u32 vcc, 0, v228\n\t"
      "s_cbranch_vccnz XL435_%=\n\t"
      "s_cmp_eq_u32 0, 1\n\t"
      "XO436_%=:\n\t"
      "s_cselect_b32 s75, 0x7fc00000, s75\n\t"
      "s_mov_b32 m0, s85\n\t"
      "s_branch XR434_%=\n\t"
      "AQ441_%=:\n\t"
      "s_movk_i32 s82, 60\n\t"
      "v_add_u32 v217, 256, v203\n\t"
      "s_call_b64 s[90:91], ASPIN_%=\n\t"
      "s_branch AR442_%=\n\t"
      "XS443_%=:\n\t"
      "s_mov_b32 s85, m0\n\t"
      "XL445_%=:\n\t"
      "s_add_u32 m0, s52, 32768\n\t"
      "s_add_u32 s83, s21, 24576\n\t"
      "buffer_load_dwordx4 v200, s[44:47], s83 offen offset:0 sc0 sc1 lds\n\t"
      "buffer_load_dwordx4 v200, s[44:47], s83 offen offset:1024 sc0 sc1 lds\n\t"
      "s_add_u32 m0, s52, 73728\n\t"
      "s_add_u32 s83, s24, 24576\n\t"
      "buffer_load_dwordx4 v200, s[44:47], s83 offen offset:0 sc0 sc1 lds\n\t"
      "buffer_load_dwordx4 v200, s[44:47], s83 offen offset:1024 sc0 sc1 lds\n\t"
      "s_add_u32 m0, s52, 147456\n\t"
      "s_add_u32 s83, s25, 24576\n\t"
      "buffer_load_dwordx4 v200, s[44:47], s83 offen offset:0 sc0 sc1 lds\n\t"
      "buffer_load_dwordx4 v200, s[44:47], s83 offen offset:1024 sc0 sc1 lds\n\t"
      "s_add_u32 m0, s52, 65536\n\t"
      "s_add_u32 s83, s26, 24576\n\t"
      "buffer_load_dwordx4 v200, s[44:47], s83 offen offset:0 sc0 sc1 lds\n\t"
      "buffer_load_dwordx4 v200, s[44:47], s83 offen offset:1024 sc0 sc1 lds\n\t"
      "s_call_b64 s[88:89], TMO_%=\n\t"
      "s_cbranch_scc1 XO446_%=\n\t"
      "s_waitcnt vmcnt(6)\n\t"
      "ds_read2st64_b32 v[228:229], v250 offset0:128 offset1:132\n\t"
      "s_cmp_lg_u32 s65, 0\n\t"
      "s_cbranch_scc1 Lp447_%=\n\t"
      "Lr448_%=:\n\t"
      "s_waitcnt lgkmcnt(0)\n\t"
      "v_min_u32 v228, v228, v229\n\t"
      "v_cmp_eq_u32 vcc, 0, v228\n\t"
      "s_cbranch_vccnz XL445_%=\n\t"
      "s_cmp_eq_u32 0, 1\n\t"
      "XO446_%=:\n\t"
      "s_cselect_b32 s75, 0x7fc00000, s75\n\t"
      "s_mov_b32 m0, s85\n\t"
      "s_branch XR444_%=\n\t"
      "AQ451_%=:\n\t"
      "s_movk_i32 s82, 60\n\t"
      "v_add_u32 v217, 0, v203\n\t"
      "s_call_b64 s[90:91], ASPIN_%=\n\t"
      "s_branch AR452_%=\n\t"
      "XS453_%=:\n\t"
      "s_mov_b32 s85, m0\n\t"
      "XL455_%=:\n\t"
      "s_add_u32 m0, s52, 73728\n\t"
      "s_add_u32 s83, s24, 24576\n\t"
      "buffer_load_dwordx4 v200, s[44:47], s83 offen offset:0 sc0 sc1 lds\n\t"
      "buffer_load_dwordx4 v200, s[44:47], s83 offen offset:1024 sc0 sc1 lds\n\t"
      "s_add_u32 m0, s52, 147456\n\t"
      "s_add_u32 s83, s25, 24576\n\t"
      "buffer_load_dwordx4 v200, s[44:47], s83 offen offset:0 sc0 sc1 lds\n\t"
      "buffer_load_dwordx4 v200, s[44:47], s83 offen offset:1024 sc0 sc1 lds\n\t"
      "s_add_u32 m0, s52, 65536\n\t"
      "s_add_u32 s83, s26, 24576\n\t"
      "buffer_load_dwordx4 v200, s[44:47], s83 offen offset:0 sc0 sc1 lds\n\t"
      "buffer_load_dwordx4 v200, s[44:47], s83 offen offset:1024 sc0 sc1 lds\n\t"
      "s_call_b64 s[88:89], TMO_%=\n\t"
      "s_cbranch_scc1 XO456_%=\n\t"
      "s_waitcnt vmcnt(4)\n\t"
      "ds_read2st64_b32 v[228:229], v251 offset0:32 offset1:36\n\t"
      "s_cmp_lg_u32 s65, 0\n\t"
      "s_cbranch_scc1 Lp457_%=\n\t"
      "Lr458_%=:\n\t"
      "s_waitcnt lgkmcnt(0)\n\t"
      "v_min_u32 v228, v228, v229\n\t"
      "v_cmp_eq_u32 vcc, 0, v228\n\t"
      "s_cbranch_vccnz XL455_%=\n\t"
      "s_cmp_eq_u32 0, 1\n\t"
      "XO456_%=:\n\t"
      "s_cselect_b32 s75, 0x7fc00000, s75\n\t"
      "s_mov_b32 m0, s85\n\t"
      "s_branch XR454_%=\n\t"
      "AQ461_%=:\n\t"
      "s_movk_i32 s82, 64\n\t"
      "v_add_u32 v217, 256, v203\n\t"
      "s_call_b64 s[90:91], ASPIN_%=\n\t"
      "s_branch AR462_%=\n\t"
      "FS465_%=:\n\t"
      "v_add_u32 v217, 128, v213\n\t"
      "s_call_b64 s[90:91], FSPIN_%=\n\t"
      "s_branch FR466_%=\n\t"
      "XS467_%=:\n\t"
      "s_mov_b32 s85, m0\n\t"
      "XL469_%=:\n\t"
      "s_add_u32 m0, s52, 147456\n\t"
      "s_add_u32 s83, s25, 24576\n\t"
      "buffer_load_dwordx4 v200, s[44:47], s83 offen offset:0 sc0 sc1 lds\n\t"
      "buffer_load_dwordx4 v200, s[44:47], s83 offen offset:1024 sc0 sc1 lds\n\t"
      "s_add_u32 m0, s52, 65536\n\t"
      "s_add_u32 s83, s26, 24576\n\t"
      "buffer_load_dwordx4 v200, s[44:47], s83 offen offset:0 sc0 sc1 lds\n\t"
      "buffer_load_dwordx4 v200, s[44:47], s83 offen offset:1024 sc0 sc1 lds\n\t"
      "s_call_b64 s[88:89], TMO_%=\n\t"
      "s_cbranch_scc1 XO470_%=\n\t"
      "s_waitcnt vmcnt(2)\n\t"
      "ds_read2st64_b32 v[228:229], v252 offset0:64 offset1:68\n\t"
      "s_cmp_lg_u32 s65, 0\n\t"
      "s_cbranch_scc1 Lp471_%=\n\t"
      "Lr472_%=:\n\t"
      "s_waitcnt lgkmcnt(0)\n\t"
      "v_min_u32 v228, v228, v229\n\t"
      "v_cmp_eq_u32 vcc, 0, v228\n\t"
      "s_cbranch_vccnz XL469_%=\n\t"
      "s_cmp_eq_u32 0, 1\n\t"
      "XO470_%=:\n\t"
      "s_cselect_b32 s75, 0x7fc00000, s75\n\t"
      "s_mov_b32 m0, s85\n\t"
      "s_branch XR468_%=\n\t"
      "AQ475_%=:\n\t"
      "s_movk_i32 s82, 64\n\t"
      "v_add_u32 v217, 0, v203\n\t"
      "s_call_b64 s[90:91], ASPIN_%=\n\t"
      "s_branch AR476_%=\n\t"
      "XS477_%=:\n\t"
      "s_mov_b32 s85, m0\n\t"
      "XL479_%=:\n\t"
      "s_add_u32 m0, s52, 65536\n\t"
      "s_add_u32 s83, s26, 24576\n\t"
      "buffer_load_dwordx4 v200, s[44:47], s83 offen offset:0 sc0 sc1 lds\n\t"
      "buffer_load_dwordx4 v200, s[44:47], s83 offen offset:1024 sc0 sc1 lds\n\t"
      "s_call_b64 s[88:89], TMO_%=\n\t"
      "s_cbranch_scc1 XO480_%=\n\t"
      "s_waitcnt vmcnt(0)\n\t"
      "ds_read2st64_b32 v[228:229], v251 offset0:0 offset1:4\n\t"
      "s_cmp_lg_u32 s65, 0\n\t"
      "s_cbranch_scc1 Lp481_%=\n\t"
      "Lr482_%=:\n\t"
      "s_waitcnt lgkmcnt(0)\n\t"
      "v_min_u32 v228, v228, v229\n\t"
      "v_cmp_eq_u32 vcc, 0, v228\n\t"
      "s_cbranch_vccnz XL479_%=\n\t"
      "s_cmp_eq_u32 0, 1\n\t"
      "XO480_%=:\n\t"
      "s_cselect_b32 s75, 0x7fc00000, s75\n\t"
      "s_mov_b32 m0, s85\n\t"
      "s_branch XR478_%=\n\t"
      "AQ485_%=:\n\t"
      "s_movk_i32 s82, 68\n\t"
      "v_add_u32 v217, 256, v203\n\t"
      "s_call_b64 s[90:91], ASPIN_%=\n\t"
      "s_branch AR486_%=\n\t"
      "AQ489_%=:\n\t"
      "s_movk_i32 s82, 68\n\t"
      "v_add_u32 v217, 0, v203\n\t"
      "s_call_b64 s[90:91], ASPIN_%=\n\t"
      "s_branch AR490_%=\n\t"
      "XGU_%=:\n\t"
      "s_mov_b32 s75, 0x7fc00000\n\t"
      "s_waitcnt vmcnt(0)\n\t"
      "s_setpc_b64 s[90:91]\n\t"
      "ASPIN_%=:\n\t"
      "s_mov_b32 s83, 2\n\t"
      "ASP2_%=:\n\t"
      "ds_read_b32 v253, v217\n\t"
      "s_sub_u32 s83, s83, 1\n\t"
      "s_waitcnt lgkmcnt(0)\n\t"
      "v_cmp_gt_u32 vcc, s82, v253\n\t"
      "s_cbranch_vccz AOK_%=\n\t"
      "s_sleep 1\n\t"
      "s_cbranch_scc0 ASP2_%=\n\t"
      "s_cmp_lg_u32 s65, 0\n\t"
      "s_cbranch_scc1 Lp494_%=\n\t"
      "Lr495_%=:\n\t"
      "s_call_b64 s[88:89], TMO_%=\n\t"
      "s_cbranch_scc0 ASPIN_%=\n\t"
      "s_mov_b32 s75, 0x7fc00000\n\t"
      "AOK_%=:\n\t"
      "s_setpc_b64 s[90:91]\n\t"
      "FSPIN_%=:\n\t"
      "global_load_dword v216, v217, s[16:17] sc0 sc1\n\t"
      "s_call_b64 s[88:89], TMO_%=\n\t"
      "s_cbranch_scc1 XGU_%=\n\t"
      "s_waitcnt vmcnt(0)\n\t"
      "s_cmp_lg_u32 s65, 0\n\t"
      "s_cbranch_scc1 Lp496_%=\n\t"
      "Lr497_%=:\n\t"
      "v_cmp_ne_u32 vcc, s64, v216\n\t"
      "s_cbranch_vccnz FSPIN_%=\n\t"
      "s_setpc_b64 s[90:91]\n\t"
      "PCHK_%=:\n\t"
      "s_memrealtime s[86:87]\n\t"
      "s_waitcnt lgkmcnt(0)\n\t"
      "s_sub_u32 s87, s86, s66\n\t"
      "s_cmp_lt_i32 s87, 0\n\t"
      "s_cbranch_scc0 PUSH_%=\n\t"
      "s_setpc_b64 s[88:89]\n\t"
      "PUSH_%=:\n\t"
      "s_setprio 3\n\t"
      "global_store_dwordx4 v200, v[136:139], s[98:99] offset:0 sc0 sc1\n\t"
      "global_store_dwordx4 v200, v[140:143], s[98:99] offset:1024 sc0 sc1\n\t"
      "global_store_dwordx4 v200, v[144:147], s[98:99] offset:2048 sc0 sc1\n\t"
      "global_store_dwordx4 v200, v[148:151], s[98:99] offset:3072 sc0 sc1\n\t"
      "s_setprio 0\n\t"
      "s_mov_b32 s65, 0\n\t"
      "s_setpc_b64 s[88:89]\n\t"
      "TMO_%=:\n\t"
      "s_memrealtime s[86:87]\n\t"
      "s_waitcnt lgkmcnt(0)\n\t"
      "s_sub_u32 s86, s86, s67\n\t"
      "s_cmp_lt_u32 s86, 20000\n\t"
      "s_cbranch_scc1 TNO_%=\n\t"
      "v_mov_b32 v207, 6815744\n\t"
      "global_load_dword v205, v207, s[16:17] sc0 sc1\n\t"
      "s_waitcnt vmcnt(0)\n\t"
      "v_cmp_eq_u32 vcc, s64, v205\n\t"
      "s_cbranch_vccnz TYES_%=\n\t"
      "s_cmp_gt_u32 s86, 40000000\n\t"
      "s_cbranch_scc1 TYES_%=\n\t"
      "s_sleep 16\n\t"
      "TNO_%=:\n\t"
      "s_cmp_eq_u32 0, 1\n\t"
      "s_setpc_b64 s[88:89]\n\t"
      "TYES_%=:\n\t"
      "v_mov_b32 v205, s64\n\t"
      "global_store_dword v207, v205, s[16:17] sc0 sc1\n\t"
      "s_cmp_eq_u32 0, 0\n\t"
      "s_setpc_b64 s[88:89]\n\t"
      "Lp175_%=:\n\t"
      "s_call_b64 s[88:89], PCHK_%=\n\t"
      "s_branch Lr176_%=\n\t"
      "Lp177_%=:\n\t"
      "s_call_b64 s[88:89], PCHK_%=\n\t"
      "s_branch Lr178_%=\n\t"
      "Lp181_%=:\n\t"
      "s_call_b64 s[88:89], PCHK_%=\n\t"
      "s_branch Lr182_%=\n\t"
      "Lp183_%=:\n\t"
      "s_call_b64 s[88:89], PCHK_%=\n\t"
      "s_branch Lr184_%=\n\t"
      "Lp187_%=:\n\t"
      "s_call_b64 s[88:89], PCHK_%=\n\t"
      "s_branch Lr188_%=\n\t"
      "Lp189_%=:\n\t"
      "s_call_b64 s[88:89], PCHK_%=\n\t"
      "s_branch Lr190_%=\n\t"
      "Lp193_%=:\n\t"
      "s_call_b64 s[88:89], PCHK_%=\n\t"
      "s_branch Lr194_%=\n\t"
      "Lp195_%=:\n\t"
      "s_call_b64 s[88:89], PCHK_%=\n\t"
      "s_branch Lr196_%=\n\t"
      "Lp199_%=:\n\t"
      "s_call_b64 s[88:89], PCHK_%=\n\t"
      "s_branch Lr200_%=\n\t"
      "Lp201_%=:\n\t"
      "s_call_b64 s[88:89], PCHK_%=\n\t"
      "s_branch Lr202_%=\n\t"
      "Lp205_%=:\n\t"
      "s_call_b64 s[88:89], PCHK_%=\n\t"
      "s_branch Lr206_%=\n\t"
      "Lp207_%=:\n\t"
      "s_call_b64 s[88:89], PCHK_%=\n\t"
      "s_branch Lr208_%=\n\t"
      "Lp211_%=:\n\t"
      "s_call_b64 s[88:89], PCHK_%=\n\t"
      "s_branch Lr212_%=\n\t"
      "Lp213_%=:\n\t"
      "s_call_b64 s[88:89], PCHK_%=\n\t"
      "s_branch Lr214_%=\n\t"
      "Lp217_%=:\n\t"
      "s_call_b64 s[88:89], PCHK_%=\n\t"
      "s_branch Lr218_%=\n\t"
      "Lp223_%=:\n\t"
      "s_call_b64 s[88:89], PCHK_%=\n\t"
      "s_branch Lr224_%=\n\t"
      "Lp225_%=:\n\t"
      "s_call_b64 s[88:89], PCHK_%=\n\t"
      "s_branch Lr226_%=\n\t"
      "Lp229_%=:\n\t"
      "s_call_b64 s[88:89], PCHK_%=\n\t"
      "s_branch Lr230_%=\n\t"
      "Lp231_%=:\n\t"
      "s_call_b64 s[88:89], PCHK_%=\n\t"
      "s_branch Lr232_%=\n\t"
      "Lp233_%=:\n\t"
      "s_call_b64 s[88:89], PCHK_%=\n\t"
      "s_branch Lr234_%=\n\t"
      "Lp239_%=:\n\t"
      "s_call_b64 s[88:89], PCHK_%=\n\t"
      "s_branch Lr240_%=\n\t"
      "Lp241_%=:\n\t"
      "s_call_b64 s[88:89], PCHK_%=\n\t"
      "s_branch Lr242_%=\n\t"
      "Lp249_%=:\n\t"
      "s_call_b64 s[88:89], PCHK_%=\n\t"
      "s_branch Lr250_%=\n\t"
      "Lp251_%=:\n\t"
      "s_call_b64 s[88:89], PCHK_%=\n\t"
      "s_branch Lr252_%=\n\t"
      "Lp257_%=:\n\t"
      "s_call_b64 s[88:89], PCHK_%=\n\t"
      "s_branch Lr258_%=\n\t"
      "Lp259_%=:\n\t"
      "s_call_b64 s[88:89], PCHK_%=\n\t"
      "s_branch Lr260_%=\n\t"
      "Lp267_%=:\n\t"
      "s_call_b64 s[88:89], PCHK_%=\n\t"
      "s_branch Lr268_%=\n\t"
      "Lp269_%=:\n\t"
      "s_call_b64 s[88:89], PCHK_%=\n\t"
      "s_branch Lr270_%=\n\t"
      "Lp277_%=:\n\t"
      "s_call_b64 s[88:89], PCHK_%=\n\t"
      "s_branch Lr278_%=\n\t"
      "Lp279_%=:\n\t"
      "s_call_b64 s[88:89], PCHK_%=\n\t"
      "s_branch Lr280_%=\n\t"
      "Lp287_%=:\n\t"
      "s_call_b64 s[88:89], PCHK_%=\n\t"
      "s_branch Lr288_%=\n\t"
      "Lp289_%=:\n\t"
      "s_call_b64 s[88:89], PCHK_%=\n\t"
      "s_branch Lr290_%=\n\t"
      "Lp297_%=:\n\t"
      "s_call_b64 s[88:89], PCHK_%=\n\t"
      "s_branch Lr298_%=\n\t"
      "Lp299_%=:\n\t"
      "s_call_b64 s[88:89], PCHK_%=\n\t"
      "s_branch Lr300_%=\n\t"
      "Lp307_%=:\n\t"
      "s_call_b64 s[88:89], PCHK_%=\n\t"
      "s_branch Lr308_%=\n\t"
      "Lp309_%=:\n\t"
      "s_call_b64 s[88:89], PCHK_%=\n\t"
      "s_branch Lr310_%=\n\t"
      "Lp317_%=:\n\t"
      "s_call_b64 s[88:89], PCHK_%=\n\t"
      "s_branch Lr318_%=\n\t"
      "Lp319_%=:\n\t"
      "s_call_b64 s[88:89], PCHK_%=\n\t"
      "s_branch Lr320_%=\n\t"
      "Lp327_%=:\n\t"
      "s_call_b64 s[88:89], PCHK_%=\n\t"
      "s_branch Lr328_%=\n\t"
      "Lp329_%=:\n\t"
      "s_call_b64 s[88:89], PCHK_%=\n\t"
      "s_branch Lr330_%=\n\t"
      "Lp337_%=:\n\t"
      "s_call_b64 s[88:89], PCHK_%=\n\t"
      "s_branch Lr338_%=\n\t"
      "Lp339_%=:\n\t"
      "s_call_b64 s[88:89], PCHK_%=\n\t"
      "s_branch Lr340_%=\n\t"
      "Lp347_%=:\n\t"
      "s_call_b64 s[88:89], PCHK_%=\n\t"
      "s_branch Lr348_%=\n\t"
      "Lp349_%=:\n\t"
      "s_call_b64 s[88:89], PCHK_%=\n\t"
      "s_branch Lr350_%=\n\t"
      "Lp357_%=:\n\t"
      "s_call_b64 s[88:89], PCHK_%=\n\t"
      "s_branch Lr358_%=\n\t"
      "Lp359_%=:\n\t"
      "s_call_b64 s[88:89], PCHK_%=\n\t"
      "s_branch Lr360_%=\n\t"
      "Lp367_%=:\n\t"
      "s_call_b64 s[88:89], PCHK_%=\n\t"
      "s_branch Lr368_%=\n\t"
      "Lp369_%=:\n\t"
      "s_call_b64 s[88:89], PCHK_%=\n\t"
      "s_branch Lr370_%=\n\t"
      "Lp377_%=:\n\t"
      "s_call_b64 s[88:89], PCHK_%=\n\t"
      "s_branch Lr378_%=\n\t"
      "Lp379_%=:\n\t"
      "s_call_b64 s[88:89], PCHK_%=\n\t"
      "s_branch Lr380_%=\n\t"
      "Lp387_%=:\n\t"
      "s_call_b64 s[88:89], PCHK_%=\n\t"
      "s_branch Lr388_%=\n\t"
      "Lp389_%=:\n\t"
      "s_call_b64 s[88:89], PCHK_%=\n\t"
      "s_branch Lr390_%=\n\t"
      "Lp397_%=:\n\t"
      "s_call_b64 s[88:89], PCHK_%=\n\t"
      "s_branch Lr398_%=\n\t"
      "Lp399_%=:\n\t"
      "s_call_b64 s[88:89], PCHK_%=\n\t"
      "s_branch Lr400_%=\n\t"
      "Lp407_%=:\n\t"
      "s_call_b64 s[88:89], PCHK_%=\n\t"
      "s_branch Lr408_%=\n\t"
      "Lp409_%=:\n\t"
      "s_call_b64 s[88:89], PCHK_%=\n\t"
      "s_branch Lr410_%=\n\t"
      "Lp417_%=:\n\t"
      "s_call_b64 s[88:89], PCHK_%=\n\t"
      "s_branch Lr418_%=\n\t"
      "Lp419_%=:\n\t"
      "s_call_b64 s[88:89], PCHK_%=\n\t"
      "s_branch Lr420_%=\n\t"
      "Lp427_%=:\n\t"
      "s_call_b64 s[88:89], PCHK_%=\n\t"
      "s_branch Lr428_%=\n\t"
      "Lp429_%=:\n\t"
      "s_call_b64 s[88:89], PCHK_%=\n\t"
      "s_branch Lr430_%=\n\t"
      "Lp437_%=:\n\t"
      "s_call_b64 s[88:89], PCHK_%=\n\t"
      "s_branch Lr438_%=\n\t"
      "Lp439_%=:\n\t"
      "s_call_b64 s[88:89], PCHK_%=\n\t"
      "s_branch Lr440_%=\n\t"
      "Lp447_%=:\n\t"
      "s_call_b64 s[88:89], PCHK_%=\n\t"
      "s_branch Lr448_%=\n\t"
      "Lp449_%=:\n\t"
      "s_call_b64 s[88:89], PCHK_%=\n\t"
      "s_branch Lr450_%=\n\t"
      "Lp457_%=:\n\t"
      "s_call_b64 s[88:89], PCHK_%=\n\t"
      "s_branch Lr458_%=\n\t"
      "Lp459_%=:\n\t"
      "s_call_b64 s[88:89], PCHK_%=\n\t"
      "s_branch Lr460_%=\n\t"
      "Lf463_%=:\n\t"
      "s_call_b64 s[88:89], PUSH_%=\n\t"
      "s_branch Lg464_%=\n\t"
      "Lp471_%=:\n\t"
      "s_call_b64 s[88:89], PCHK_%=\n\t"
      "s_branch Lr472_%=\n\t"
      "Lp473_%=:\n\t"
      "s_call_b64 s[88:89], PCHK_%=\n\t"
      "s_branch Lr474_%=\n\t"
      "Lp481_%=:\n\t"
      "s_call_b64 s[88:89], PCHK_%=\n\t"
      "s_branch Lr482_%=\n\t"
      "Lp483_%=:\n\t"
      "s_call_b64 s[88:89], PCHK_%=\n\t"
      "s_branch Lr484_%=\n\t"
      "Lp487_%=:\n\t"
      "s_call_b64 s[88:89], PCHK_%=\n\t"
      "s_branch Lr488_%=\n\t"
      "Lp491_%=:\n\t"
      "s_call_b64 s[88:89], PCHK_%=\n\t"
      "s_branch Lr492_%=\n\t"
      "Lp494_%=:\n\t"
      "s_call_b64 s[88:89], PCHK_%=\n\t"
      "s_branch Lr495_%=\n\t"
      "Lp496_%=:\n\t"
      "s_call_b64 s[88:89], PCHK_%=\n\t"
      "s_branch Lr497_%=\n\t"
      "END_%=:\n\t"
      :: [karg] "s"(__builtin_amdgcn_kernarg_segment_ptr()), [bid] "s"(blockIdx.x), [tid] "v"(threadIdx.x), "s"(ldsb)
      : "v32", "v33", "v34", "v35", "v36", "v37", "v38", "v39", "v40", "v41", "v42", "v43", "v44", "v45", "v46", "v47", "v48", "v49", "v50", "v51", "v52", "v53", "v54", "v55",
        "v56", "v57", "v58", "v59", "v60", "v61", "v62", "v63", "v64", "v65", "v66", "v67", "v68", "v69", "v70", "v71", "v72", "v73", "v74", "v75", "v76", "v77", "v78", "v79",
        "v80", "v81", "v82", "v83", "v84", "v85", "v86", "v87", "v88", "v89", "v90", "v91", "v92", "v93", "v94", "v95", "v96", "v97", "v98", "v99", "v100", "v101", "v102", "v103",
        "v104", "v105", "v106", "v107", "v108", "v109", "v110", "v111", "v112", "v113", "v114", "v115", "v116", "v117", "v118", "v119", "v120", "v121", "v122", "v123", "v124", "v125", "v126", "v127",
        "v128", "v129", "v130", "v131", "v132", "v133", "v134", "v135", "v136", "v137", "v138", "v139", "v140", "v141", "v142", "v143", "v144", "v145", "v146", "v147", "v148", "v149", "v150", "v151",
        "v152", "v153", "v154", "v155", "v156", "v157", "v158", "v159", "v160", "v161", "v162", "v163", "v164", "v165", "v166", "v167", "v168", "v169", "v170", "v171", "v172", "v173", "v174", "v175",
        "v176", "v177", "v178", "v179", "v180", "v181", "v182", "v183", "v184", "v185", "v186", "v187", "v188", "v189", "v190", "v191", "v192", "v193", "v194", "v195", "v196", "v197", "v198", "v199",
        "v200", "v201", "v202", "v203", "v204", "v205", "v206", "v207", "v208", "v209", "v210", "v211", "v212", "v213", "v214", "v215", "v216", "v217", "v218", "v219", "v220", "v221", "v222", "v223",
        "v224", "v225", "v226", "v227", "v228", "v229", "v230", "v231", "v232", "v233", "v234", "v235", "v236", "v237", "v238", "v239", "v240", "v241", "v242", "v243", "v244", "v245", "v246", "v247",
        "v248", "v249", "v250", "v251", "v252", "v253", "v254", "v255", "a0", "a1", "a2", "a3", "a4", "a5", "a6", "a7", "a8", "a9", "a10", "a11", "a12", "a13", "a14", "a15",
        "a16", "a17", "a18", "a19", "a20", "a21", "a22", "a23", "a24", "a25", "a26", "a27", "a28", "a29", "a30", "a31", "a32", "a33", "a34", "a35", "a36", "a37", "a38", "a39",
        "a40", "a41", "a42", "a43", "a44", "a45", "a46", "a47", "a48", "a49", "a50", "a51", "a52", "a53", "a54", "a55", "a56", "a57", "a58", "a59", "a60", "a61", "a62", "a63",
        "a64", "a65", "a66", "a67", "a68", "a69", "a70", "a71", "a72", "a73", "a74", "a75", "a76", "a77", "a78", "a79", "a80", "a81", "a82", "a83", "a84", "a85", "a86", "a87",
        "a88", "a89", "a90", "a91", "a92", "a93", "a94", "a95", "a96", "a97", "a98", "a99", "a100", "a101", "a102", "a103", "a104", "a105", "a106", "a107", "a108", "a109", "a110", "a111",
        "a112", "a113", "a114", "a115", "a116", "a117", "a118", "a119", "a120", "a121", "a122", "a123", "a124", "a125", "a126", "a127", "a128", "a129", "a130", "a131", "a132", "a133", "a134", "a135",
        "a136", "a137", "a138", "a139", "a140", "a141", "a142", "a143", "a144", "a145", "a146", "a147", "a148", "a149", "a150", "a151", "a152", "a153", "a154", "a155", "a156", "a157", "a158", "a159",
        "a160", "a161", "a162", "a163", "a164", "a165", "a166", "a167", "a168", "a169", "a170", "a171", "a172", "a173", "a174", "a175", "a176", "a177", "a178", "a179", "a180", "a181", "a182", "a183",
        "a184", "a185", "a186", "a187", "a188", "a189", "a190", "a191", "a192", "a193", "a194", "a195", "a196", "a197", "a198", "a199", "a200", "a201", "a202", "a203", "a204", "a205", "a206", "a207",
        "a208", "a209", "a210", "a211", "a212", "a213", "a214", "a215", "a216", "a217", "a218", "a219", "a220", "a221", "a222", "a223", "a224", "a225", "a226", "a227", "a228", "a229", "a230", "a231",
        "a232", "a233", "a234", "a235", "a236", "a237", "a238", "a239", "a240", "a241", "a242", "a243", "a244", "a245", "a246", "a247", "a248", "a249", "a250", "a251", "a252", "a253", "a254", "a255",
        "s16", "s17", "s18", "s19", "s20", "s21", "s22", "s23", "s24", "s25", "s26", "s27", "s28", "s29", "s30", "s31", "s34", "s35", "s36", "s37", "s38", "s39", "s40", "s41",
        "s42", "s43", "s44", "s45", "s46", "s47", "s48", "s49", "s50", "s51", "s52", "s53", "s54", "s55", "s56", "s57", "s58", "s59", "s60", "s61", "s62", "s63", "s64", "s65",
        "s66", "s67", "s68", "s69", "s70", "s71", "s72", "s73", "s74", "s75", "s76", "s77", "s78", "s79", "s80", "s81", "s82", "s83", "s84", "s85", "s86", "s87", "s88", "s89",
        "s90", "s91", "s92", "s93", "s94", "s95", "s96", "s97", "s98", "s99", "vcc", "scc", "m0", "memory");
}

}

extern "C" void kernel_launch(int rank, int nranks, ncclComm_t comm,
                              hipStream_t stream,
                              void* const* d_in, const int* in_sizes, int n_in,
                              void* d_out, int out_size,
                              const int* in_dtypes, int out_dtype,
                              void* const* d_out_all,
                              void* const* d_ws_all, size_t ws_bytes) {
  (void)comm; (void)in_sizes; (void)in_dtypes; (void)out_dtype; (void)d_out_all;
  if (nranks != NR || n_in != 4 || out_size != 512 * 4096 || ws_bytes < WS_NEED || (unsigned)rank >= (unsigned)NR) {
    fprintf(stderr, "unexpected problem shape\n");
    return;
  }
  Params p;
  p.myws = (unsigned char*)d_ws_all[rank];
  p.x = (const unsigned char*)d_in[0];
  p.w = (const unsigned char*)d_in[1];
  p.sx = (const float*)d_in[2];
  p.sw = (const float*)d_in[3];
  p.out = (float*)d_out;
  p.rank = rank;
  p.pad[0] = p.pad[1] = p.pad[2] = 0;
  for (int d = 0; d < NR; ++d) {
    const int dst = (rank + d) % NR;
    p.rel[d].src = p.x + (size_t)dst * 262144;
    p.rel[d].dst = (unsigned char*)d_ws_all[dst] + (size_t)d * 262144;
  }
  for (u32 bi = 0; bi < 32; ++bi) {
    const u32 H = bi >> 4, pt = (bi >> 2) & 3u, c = 2u * pt + H;
    for (u32 i = 0; i < 8; ++i) p.cls[bi][i] = 0;
    p.cls[bi][1] = ((u32)rank * 512u + 128u * pt + 64u * H) * 512u + 256u * H;
    p.cls[bi][8] = ((u32)rank * 512u + 256u * H) * 4096u;
    const u32 o = c;
    for (u32 i = 1; i < 8; ++i) {
      const u32 tau = (i - 1u) + ((i - 1u) >= o ? 1u : 0u), del = (tau - o) & 7u, src = ((u32)rank - del) & 7u;
      p.cls[bi][0] |= del << (4u * (i - 1u));
      p.cls[bi][8 + i] = (src * 512u + 256u * H) * 4096u;
    }
  }
  a2a_gemm_kernel<<<dim3(NBLK), dim3(256), 0, stream>>>(p);
  HIP_CHECK(hipGetLastError());
}
```
